```python
import numpy as np

kernel_name = "gpu_gemm_rs_m4096_k4096_n32768_bf16_none"


ABSMAX_RTOL = 0.008

COMPUTE_DTYPE = "bfloat16"
EPILOGUE = "none"
IN_QUANT = None
OUT_QUANT = None
IN_DTYPES = {"x": "bfloat16", "w_mat": "bfloat16"}
WORLD_SIZE = 8

def _to_bf16(x):
    u = np.ascontiguousarray(x, dtype=np.float32).view(np.uint32)
    return ((u + 0x7FFF + ((u >> 16) & 1)) & 0xFFFF0000).view(np.float32)

def setup_inputs():

    x = _to_bf16(np.random.randn(4096, 4096).astype(np.float32))
    w_mat = _to_bf16((np.random.randn(4096, 32768) / np.sqrt(4096)).astype(np.float32))
    return {"x": x, "w_mat": w_mat}

def _epilogue(y):
    return y

def reference(x, w_mat):
    y = (x @ w_mat).astype(np.float32)
    return _epilogue(y)
```

```cpp
#include <hip/hip_runtime.h>
#include <rccl/rccl.h>
#include <cstdio>
#include <cstdint>
#include <cstring>

#define HIP_CHECK(expr) do { hipError_t _e = (expr); if (_e != hipSuccess) { \
    fprintf(stderr, "HIP error %s at %s:%d\n", hipGetErrorString(_e), __FILE__, __LINE__); \
    return; } } while (0)

typedef unsigned u32;
typedef unsigned long long u64;
typedef u32 u32x4 __attribute__((ext_vector_type(4)));
typedef u32 u32x2 __attribute__((ext_vector_type(2)));
typedef short bf16x8 __attribute__((ext_vector_type(8)));
typedef float f32x4 __attribute__((ext_vector_type(4)));
#define LAS __attribute__((address_space(3)))
#define GAS __attribute__((address_space(1)))

#define FORCE_RAW 0
#define SKEW_NUM 5

constexpr int NR = 8;
constexpr size_t MiB = 1ull << 20;
constexpr size_t OFF_CNT = 0;
constexpr size_t OFF_TMO = 16384;
constexpr size_t OFF_TICKET = 16384 + 256;
constexpr size_t OFF_FG = 65536;
constexpr size_t OFF_RIN = 1 * MiB;
constexpr size_t OFF_RY = 2 * MiB;
constexpr size_t OFF_PX = 4 * MiB;
constexpr size_t OFF_PW = 12 * MiB;
constexpr size_t OFF_PY = 44 * MiB;
constexpr size_t OFF_G = 76 * MiB;
constexpr size_t OFF_DBG = 140 * MiB;
constexpr size_t WS_NEED = 141 * MiB;
constexpr u64 DEADLINE = 40000000ull;
constexpr int LDS_BYTES = 131072;
constexpr int NCOMM = 32, NGEMM = 224, NUIN = 32, NUY = 16, NU = NUIN + NUY;

struct Params {
    const char* x; const char* w; char* out;
    char* ws[NR];
    char* const* ws_tab;
    int rank; int pad;
    unsigned char order[256];
};

#define STAMP(i)

#define WAIT_V(n) asm volatile("s_waitcnt vmcnt(" #n ")" ::: "memory")
#define WAIT_L(n) asm volatile("s_waitcnt lgkmcnt(" #n ")" ::: "memory")
#define BAR __builtin_amdgcn_s_barrier()
#define SCHED __builtin_amdgcn_sched_barrier(0)

template <int OFF> __device__ __forceinline__ void st16_sys(const char* base, u32 voff, u32x4 v) {
    asm volatile("global_store_dwordx4 %0, %1, %2 offset:%3 sc0 sc1\n\ts_nop 1" :: "v"(voff), "v"(v), "s"(base), "i"(OFF) : "memory");
}
__device__ __forceinline__ void sgpr_settle() { asm volatile("s_nop 4" ::: "memory"); }
__device__ __forceinline__ u32 perm(u32 hi, u32 lo, u32 sel) { return __builtin_amdgcn_perm(hi, lo, sel); }
__device__ __forceinline__ u32 ld_agent(const char* p) { return __hip_atomic_load((GAS u32*)p, __ATOMIC_RELAXED, __HIP_MEMORY_SCOPE_AGENT); }
__device__ __forceinline__ void st_agent(char* p, u32 v) { __hip_atomic_store((GAS u32*)p, v, __ATOMIC_RELAXED, __HIP_MEMORY_SCOPE_AGENT); }
__device__ __forceinline__ u32 cvt_pk_bf16(float lo, float hi) { u32 r; asm("v_cvt_pk_bf16_f32 %0, %1, %2" : "=v"(r) : "v"(lo), "v"(hi)); return r; }

__device__ __forceinline__ void planes_to_raw(const u32 (&L)[16], const u32 (&H)[16], u32x4 (&O)[8]) {
#pragma unroll
    for (int d = 0; d < 4; ++d)
#pragma unroll
        for (int p = 0; p < 4; ++p) {
            O[2 * d][p] = perm(H[d * 4 + p], L[d * 4 + p], 0x05010400u);
            O[2 * d + 1][p] = perm(H[d * 4 + p], L[d * 4 + p], 0x07030602u);
        }
}
__device__ __forceinline__ void store_unit_linear(const char* dst, u32 voff, const u32x4 (&O)[8]) {
    const char* b = dst + 4096;
    sgpr_settle();
    st16_sys<-4096>(b, voff, O[0]); st16_sys<-3072>(b, voff, O[1]); st16_sys<-2048>(b, voff, O[2]); st16_sys<-1024>(b, voff, O[3]);
    st16_sys<0>(b, voff, O[4]); st16_sys<1024>(b, voff, O[5]); st16_sys<2048>(b, voff, O[6]); st16_sys<3072>(b, voff, O[7]);
}

__device__ __forceinline__ u32 encode_store(const u32 (&L)[16], const u32 (&H)[16], const char* slot, int lane) {
    float mx = 0.f;
#pragma unroll
    for (int t = 0; t < 16; t += 2)
        asm("v_max3_f32 %0, |%1|, |%2|, %0" : "+v"(mx) : "v"(H[t]), "v"(H[t + 1]));
    u32 m = __float_as_uint(mx) >> 24;
    m = max(m, (u32)__builtin_amdgcn_update_dpp(0, (int)m, 0x111, 0xf, 0xf, false));
    m = max(m, (u32)__builtin_amdgcn_update_dpp(0, (int)m, 0x112, 0xf, 0xf, false));
    m = max(m, (u32)__builtin_amdgcn_update_dpp(0, (int)m, 0x114, 0xf, 0xf, false));
    m = max(m, (u32)__builtin_amdgcn_update_dpp(0, (int)m, 0x118, 0xf, 0xf, false));
    const u32 top = max(max((u32)__builtin_amdgcn_readlane((int)m, 15), (u32)__builtin_amdgcn_readlane((int)m, 31)),
                        max((u32)__builtin_amdgcn_readlane((int)m, 47), (u32)__builtin_amdgcn_readlane((int)m, 63))) & 0x7fu;
    const u32 C = top > 7u ? top - 7u : 0u, C4 = C * 0x01010101u;
    u32 u[16], acc = 0;
#pragma unroll
    for (int t = 0; t < 16; ++t) { u[t] = H[t] - C4; acc |= u[t]; }
    u32 rec = 0, head = C;
    bool raw = FORCE_RAW;
    if (__builtin_amdgcn_ballot_w64((acc & 0x78787878u) != 0u) != 0ull) {
        u32 n = 0, sH = 0, sL = 0, sM = 0, sT = 0;
#pragma unroll
        for (int t = 0; t < 16; ++t) {
            const u32 w = (H[t] | 0x80808080u) - C4, mis = w & 0x78787878u;
            u[t] = (w & 0x07070707u) | (H[t] & 0x80808080u);
            if (mis) { ++n; sH = H[t]; sL = L[t]; sM = mis; sT = t; }
        }
        const u32 bsh = (n ? (u32)__builtin_ctz(sM) : 0u) & 24u;
        const bool multi = n > 1u || (sM & ~(0xffu << bsh)) != 0u;
        u64 mask = __builtin_amdgcn_ballot_w64(n != 0u);
        const u32 cnt = (u32)__builtin_popcountll(mask);
        if (cnt > 7u || __builtin_amdgcn_ballot_w64(multi) != 0ull) raw = true;
        else {
            const u32 ent = ((u32)lane << 22) | (sT << 18) | (bsh << 13) | (((sH >> bsh) & 0xffu) << 8) | ((sL >> bsh) & 0xffu);
            for (u32 s = 1; mask; ++s, mask &= mask - 1) {
                const u32 e = (u32)__builtin_amdgcn_readlane((int)ent, __builtin_ctzll(mask));
                if ((u32)(lane & 7) == s) rec = e;
            }
            head |= cnt << 8;
        }
    }
    const u32 voff = (u32)lane * 16u;
    if (raw) {
        u32x4 O[8]; planes_to_raw(L, H, O); store_unit_linear(slot, voff, O);
        head = 1u << 16;
    } else {
        u32x4 n0, n1;
#pragma unroll
        for (int s = 0; s < 4; ++s) {
            n0[s] = u[2 * s] | __builtin_amdgcn_alignbit(u[2 * s + 1], u[2 * s + 1], 28);
            n1[s] = u[8 + 2 * s] | __builtin_amdgcn_alignbit(u[8 + 2 * s + 1], u[8 + 2 * s + 1], 28);
        }
        const char* b = slot + 4096;
        sgpr_settle();
        st16_sys<-4096>(b, voff, (u32x4){L[0], L[1], L[2], L[3]}); st16_sys<-3072>(b, voff, (u32x4){L[4], L[5], L[6], L[7]});
        st16_sys<-2048>(b, voff, (u32x4){L[8], L[9], L[10], L[11]}); st16_sys<-1024>(b, voff, (u32x4){L[12], L[13], L[14], L[15]});
        st16_sys<0>(b, voff, n0); st16_sys<1024>(b, voff, n1);
    }
    if ((lane & 7) == 0) rec = head;
    return rec;
}
__device__ __forceinline__ void put_record(char* p, u32 seq, u32 w) {
    __hip_atomic_store((GAS u64*)p, ((u64)w << 32) | seq, __ATOMIC_RELAXED, __HIP_MEMORY_SCOPE_SYSTEM);
}

__device__ __forceinline__ void pack_task(const Params& p, char* ws, int task, u32 seq, int lane) {
    const int j = task / 576, r = task % 576;
    u32x4 in[8]; u32 L[16], H[16];
    const u32 voff = (u32)lane * 16u;
    if (r < 64) {
        const int pm = r >> 2, un = r & 3;
        const char* src = p.x + (size_t)(pm * 256 + un * 64) * 1024 + j * 128 + (lane >> 3) * 1024;
        const int sw = (lane & 7) ^ (lane >> 4);
#pragma unroll
        for (int q = 0; q < 8; ++q) in[q] = *(const GAS u32x4*)(src + q * 8192 + ((sw ^ ((q & 1) << 2)) << 4));
        store_unit_linear(ws + OFF_G + (size_t)(j * 32 + pm) * 32768 + un * 8192, voff, in);
#pragma unroll
        for (int d = 0; d < 4; ++d)
#pragma unroll
            for (int q = 0; q < 4; ++q) { L[d * 4 + q] = perm(in[2 * d + 1][q], in[2 * d][q], 0x06040200u); H[d * 4 + q] = perm(in[2 * d + 1][q], in[2 * d][q], 0x07050301u); }
        const u32 rec = encode_store(L, H, ws + OFF_PX + (size_t)(seq & 1) * (4 * MiB) + (size_t)((j * 16 + pm) * 4 + un) * 8192, lane);
        char* peer = p.ws_tab[(p.rank + 1 + (lane >> 3)) & 7];
        WAIT_V(0);
        if (lane < 56) put_record(peer + OFF_RIN + (size_t)(((seq & 1) * 8 + p.rank) * 1024 + (j * 32 + pm) * 4 + un) * 64 + (lane & 7) * 8, seq, rec);
        if (lane == 0) st_agent(ws + OFF_FG + (j * 32 + pm) * 128 + un * 4, seq);
    } else {
        const int dd = (r - 64) >> 6, pn = (r >> 2) & 15, un = r & 3, dest = (p.rank + dd) & 7;
        const char* src = p.w + (size_t)(j * 64 + (un & 1) * 32 + (lane >> 4) * 8) * 65536 + (dest * 4096 + pn * 256 + (un >> 1) * 128) * 2 + (lane & 15) * 16;
#pragma unroll
        for (int q = 0; q < 8; ++q) in[q] = *(const GAS u32x4*)(src + (size_t)q * 65536);
#pragma unroll
        for (int d = 0; d < 4; ++d)
#pragma unroll
            for (int q = 0; q < 4; ++q) { L[d * 4 + q] = perm(in[2 * q + 1][d], in[2 * q][d], 0x06020400u); H[d * 4 + q] = perm(in[2 * q + 1][d], in[2 * q][d], 0x07030501u); }
        if (dd == 0) {
            u32x4 O[8]; planes_to_raw(L, H, O);
            store_unit_linear(ws + OFF_G + (size_t)(j * 32 + 16 + pn) * 32768 + un * 8192, voff, O);
            WAIT_V(0);
            if (lane == 0) st_agent(ws + OFF_FG + (j * 32 + 16 + pn) * 128 + un * 4, seq);
        } else {
            const u32 rec = encode_store(L, H, ws + OFF_PW + (size_t)dest * (4 * MiB) + (size_t)((j * 16 + pn) * 4 + un) * 8192, lane);
            WAIT_V(0);
            if (lane < 8) put_record(p.ws[dest] + OFF_RIN + (size_t)(((seq & 1) * 8 + p.rank) * 1024 + (j * 32 + 16 + pn) * 4 + un) * 64 + lane * 8, seq, rec);
        }
    }
}

#define DEPTH 3
#define PEERWG 0
constexpr int NSLOT = 14, NDEC = 6;
constexpr int NE = PEERWG ? 385 : 7 * NU;
constexpr int NG = NE / 7;
constexpr int L_SLOT = 0;
constexpr int L_REC = NSLOT * 8192;
constexpr int L_READY = L_REC + NE * 32;
constexpr int L_DONE = L_READY + 32;
constexpr int L_IDS = L_DONE + 32;
static_assert(L_IDS + NE * 4 <= LDS_BYTES, "lds");
#define STR2(x) #x
#define STR(x) STR2(x)

__device__ __forceinline__ void comm_role(const Params& p, char* ws, LAS unsigned char* lds, int g, u32 seq, int wid, int lane) {
    if (PEERWG && g >= 28) return;
    if (threadIdx.x < NE) {
        const int e = threadIdx.x;
        const int w = PEERWG ? g % 7 : e % 7, u = PEERWG ? g / 7 + 4 * e : g + 32 * (e / 7);
        u32 id;
        if (u < 1024) id = (u32)p.order[u >> 2] * 4u + (u & 3); else id = u < 1536 ? (0x8000u | (u - 1024)) : 0x4000u;
        ((LAS u32*)(lds + L_IDS))[e] = id | (u32)w << 16;
    }
    if (threadIdx.x >= 448 && threadIdx.x < 464) ((LAS u32*)(lds + L_READY))[threadIdx.x - 448] = 0;
    __syncthreads();
    volatile LAS u32* ready = (volatile LAS u32*)(lds + L_READY);
    volatile LAS u32* done = (volatile LAS u32*)(lds + L_DONE);
    volatile LAS u32* recs = (volatile LAS u32*)(lds + L_REC);
    volatile LAS u32* ids = (volatile LAS u32*)(lds + L_IDS);
    const u32 voff = (u32)lane * 16u;
    constexpr int NEV = PEERWG ? 384 : NE;
    if (wid == 7) {
        const int w = lane >> 3, e = lane & 7;
        int idx = w < 7 ? 0 : NG;
        u64 t0 = __builtin_amdgcn_s_memrealtime();
        bool dead = false;
        while (__builtin_amdgcn_ballot_w64(idx < NG) != 0ull) {
            const int en = idx < NG ? idx * 7 + w : 0;
            const u32 ent = ids[en], id = ent & 0x3ffu, src = (p.rank + 1 + (ent >> 16)) & 7;
            const char* a = (ent & 0x8000u) ? ws + OFF_RY + (size_t)(src * 512 + id) * 64 : ws + OFF_RIN + (size_t)(((seq & 1) * 8 + src) * 1024 + id) * 64;
            const u64 v = __hip_atomic_load((GAS u64*)(a + e * 8), __ATOMIC_RELAXED, __HIP_MEMORY_SCOPE_SYSTEM);
            const u64 okm = __builtin_amdgcn_ballot_w64((u32)v == seq || (ent & 0x4000u));
            const bool adv = idx < NG && (dead || ((okm >> (8 * w)) & 0xffull) == 0xffull);
            if (adv) recs[en * 8 + e] = dead ? (2u << 16) : (u32)(v >> 32);
            WAIT_L(0);
            if (adv) { ++idx; if (e == 0) ready[w] = idx; }
            if (__builtin_amdgcn_ballot_w64(adv) == 0ull) {
                if (ld_agent(ws + OFF_TMO) == seq || __builtin_amdgcn_s_memrealtime() - t0 > DEADLINE) { st_agent(ws + OFF_TMO, seq); WAIT_V(0); dead = true; }
                else __builtin_amdgcn_s_sleep(2);
            }
        }
    } else if (wid == 6) {
        int slot = 0, grp = 0, gi = 0, fd = 0, fi = 0;
        for (int e = 0; e < NEV; ++e) {
            bool ok = (int)__builtin_amdgcn_readfirstlane(ready[grp]) > gi && (e < NSLOT || (int)__builtin_amdgcn_readfirstlane(done[fd]) > fi);
            if (!ok) {
                WAIT_V(0);
                if (lane == 0) done[7] = e;
                while (!((int)__builtin_amdgcn_readfirstlane(ready[grp]) > gi && (e < NSLOT || (int)__builtin_amdgcn_readfirstlane(done[fd]) > fi))) __builtin_amdgcn_s_sleep(1);
            }
            const u32 ent = __builtin_amdgcn_readfirstlane(ids[e]), head = __builtin_amdgcn_readfirstlane(recs[e * 8]), id = ent & 0x3ffu;
            const char* rws = p.ws[(p.rank + 1 + (ent >> 16)) & 7];
            const char* s;
            if (ent & 0x8000u) s = rws + OFF_PY + (size_t)p.rank * (4 * MiB) + (size_t)id * 8192;
            else { const u32 un = ((id >> 7) * 16 + ((id >> 2) & 15)) * 4 + (id & 3);
                   s = rws + ((id & 64) ? OFF_PW + (size_t)p.rank * (4 * MiB) : OFF_PX + (size_t)(seq & 1) * (4 * MiB)) + (size_t)un * 8192; }
            if (head >> 17) s = ws + OFF_G;
            LAS unsigned char* d = lds + L_SLOT + slot * 8192;
            if (head >> 16) {
                __builtin_amdgcn_global_load_lds((const GAS u32*)(s + 6144 + voff), (LAS u32*)(d + 6144), 16, 0, 17);
                __builtin_amdgcn_global_load_lds((const GAS u32*)(s + 7168 + voff), (LAS u32*)(d + 7168), 16, 0, 17);
            }
#pragma unroll
            for (int q = 0; q < 6; ++q) __builtin_amdgcn_global_load_lds((const GAS u32*)(s + q * 1024 + voff), (LAS u32*)(d + q * 1024), 16, 0, 17);
            asm volatile("s_waitcnt vmcnt(%0)" :: "i"(6 * DEPTH) : "memory");
            if (lane == 0 && e + 1 - DEPTH > (int)done[7]) done[7] = e + 1 - DEPTH;
            if (++slot == NSLOT) slot = 0;
            if (++grp == 7) { grp = 0; ++gi; }
            if (e >= NSLOT && ++fd == NDEC) { fd = 0; ++fi; }
        }
        WAIT_V(0);
        if (lane == 0) done[7] = NEV;
    } else {
        char* pend = nullptr;
        int k = 0;
        for (int e = wid; e < NEV; e += NDEC) {
            if ((int)__builtin_amdgcn_readfirstlane(done[7]) <= e) {
                if (pend) { WAIT_V(0); if (lane == 0) st_agent(pend, seq); pend = nullptr; }
                while ((int)__builtin_amdgcn_readfirstlane(done[7]) <= e) __builtin_amdgcn_s_sleep(1);
            }
            asm volatile("" ::: "memory");
            STAMP(k < 62 ? k : 62);
            const u32 ent = __builtin_amdgcn_readfirstlane(ids[e]), id = ent & 0x3ffu, w = ent >> 16;
            const u32 rec = recs[e * 8 + (lane & 7)], head = __builtin_amdgcn_readfirstlane(rec);
            const LAS u32x4* sl = (const LAS u32x4*)(lds + L_SLOT + (e % NSLOT) * 8192 + voff);
            u32x4 O[8];
            ++k;
            if (head >> 16) {
#pragma unroll
                for (int q = 0; q < 8; ++q) O[q] = sl[q * 64];
                WAIT_L(0);
                if (lane == 0) done[wid] = k;
                if (head >> 17) {
#pragma unroll
                    for (int q = 0; q < 8; ++q) O[q] = (u32x4){0x7fc07fc0u, 0x7fc07fc0u, 0x7fc07fc0u, 0x7fc07fc0u};
                }
            } else {
                u32 L[16], H[16];
                u32x4 v[6];
                const u32 C4 = (head & 0xffu) * 0x01010101u;
#pragma unroll
                for (int q = 0; q < 6; ++q) v[q] = sl[q * 64];
                WAIT_L(0);
                if (lane == 0) done[wid] = k;
#pragma unroll
                for (int q = 0; q < 4; ++q) { L[4 * q] = v[q][0]; L[4 * q + 1] = v[q][1]; L[4 * q + 2] = v[q][2]; L[4 * q + 3] = v[q][3]; }
#pragma unroll
                for (int q = 0; q < 2; ++q)
#pragma unroll
                    for (int s = 0; s < 4; ++s) { H[8 * q + 2 * s] = (v[4 + q][s] & 0x87878787u) + C4; H[8 * q + 2 * s + 1] = (__builtin_amdgcn_alignbit(v[4 + q][s], v[4 + q][s], 4) & 0x87878787u) + C4; }
                planes_to_raw(L, H, O);
            }
            const u32 cnt = (head >> 8) & 0xffu;
            const u32 el = (rec >> 22) & 63u, et = (rec >> 18) & 15u, eb = (rec >> 16) & 3u, eq = (et >> 2) * 2u + (eb >> 1), eo = (et & 3u) * 4u + (eb & 1u) * 2u;
            char* flag = nullptr;
            if (!(ent & 0x8000u)) {
                const u32 blk = (8 + 7 * (id >> 7) + w) * 32 + ((id >> 2) & 31);
                char* dst = ws + OFF_G + (size_t)blk * 32768 + (id & 3) * 8192;
                store_unit_linear(dst, voff, O);
                if (cnt && lane >= 1 && (u32)lane <= cnt)
                    asm volatile("global_store_short %0, %1, %2 sc0 sc1" :: "v"(eq * 1024u + el * 16u + eo), "v"(rec), "s"(dst) : "memory");
                flag = ws + OFF_FG + blk * 128 + (id & 3) * 4;
            } else {
                const u32 src = (p.rank + 1 + w) & 7;
                const u32 row0 = ((id >> 4) & 1) * 256 + (id & 1) * 128 + ((id >> 1) & 3) * 32, col0 = src * 4096 + ((id >> 5) & 15) * 256 + ((id >> 3) & 1) * 128;
                char* dst = p.out + (size_t)row0 * 65536 + col0 * 2;
                const u32 vo = (u32)(lane >> 4) * 262144u + (u32)(lane & 15) * 16u;
                sgpr_settle();
#pragma unroll
                for (int q = 0; q < 8; ++q) st16_sys<0>(dst + (size_t)((q >> 2) * 16 + (q & 3)) * 65536, vo, O[q]);
                if (cnt && lane >= 1 && (u32)lane <= cnt)
                    asm volatile("global_store_short %0, %1, %2 sc0 sc1" :: "v"(((eq >> 2) * 16u + (eq & 3u) + (el >> 4) * 4u) * 65536u + (el & 15u) * 16u + eo), "v"(rec), "s"(dst) : "memory");
            }
            if (pend) { WAIT_V(8); if (lane == 0) st_agent(pend, seq); }
            pend = flag;
        }
        if (pend) { WAIT_V(0); if (lane == 0) st_agent(pend, seq); }
        STAMP(k < 62 ? k : 62);
    }
}

__device__ __forceinline__ void spin_flags(char* ws, const char* fbase, u32 fvoff, u32 seq) {
    const u64 t0 = __builtin_amdgcn_s_memrealtime();
    while (__builtin_amdgcn_ballot_w64(ld_agent(fbase + fvoff) < seq) != 0ull) {
        if (ld_agent(ws + OFF_TMO) == seq || __builtin_amdgcn_s_memrealtime() - t0 > DEADLINE) { st_agent(ws + OFF_TMO, seq); break; }
        __builtin_amdgcn_s_sleep(4);
    }
    __builtin_amdgcn_s_waitcnt(0xC07F);
    WAIT_V(0);
}

template <bool SUB>
__device__ __forceinline__ void gemm_loop(char* ws, LAS unsigned char* lds, const char* gX, const char* gW, u32 fvoff, u32 seq, int wid, int lane, f32x4 (&acc)[2][2][4][2]) {
    constexpr size_t KSTEP = SUB ? 2 * MiB : MiB, SX = SUB ? MiB : 16384, SW = SUB ? MiB : 16384;
    constexpr int NSTEP = SUB ? 32 : 64, FSTEP = SUB ? 8192 : 4096;
    const int wr = wid >> 2, wc = wid & 3, fr = lane & 15, fq = lane >> 4;
    const u32 voff = (u32)lane * 16u;
    const char* cX = gX + wid * 1024;
    const char* cW = gW + (wid >> 2) * 8192 + (wid & 3) * 1024;
    const char* fb = ws + OFF_FG;
    const u32 ldsw = (u32)wid * 1024u;
    const int woff = wr * 8192 + lane * 16;
    const int xoff0 = (wc * 32 + fr) * 128 + ((fq ^ (fr >> 1)) << 4) + (SUB ? wr * 16384 : 0), xoff1 = xoff0 ^ 64;
#define SA(b, h) (((b) * 2 + (h)) * 16384)
#define SB(b, h) ((4 + (b) * 2 + (h)) * 16384)
#define GLDS(src, dst) __builtin_amdgcn_global_load_lds((const GAS u32*)((src) + voff), (LAS u32*)(lds + (dst) + ldsw), 16, 0, 17)
#define STW(b, h, ptr) do { GLDS((ptr) + (h) * 4096, SA(b, h)); GLDS((ptr) + (h) * 4096 + SW, SA(b, h) + 8192); } while (0)
#define STX(b, h, ptr) do { GLDS((ptr) + (h) * SX, SB(b, h)); GLDS((ptr) + (h) * SX + 8192, SB(b, h) + 8192); } while (0)
#define LDW(b, h) do { _Pragma("unroll") for (int m = 0; m < 4; ++m) _Pragma("unroll") for (int k = 0; k < 2; ++k) Wt[m][k] = *(const LAS bf16x8*)(lds + SA(b, h) + woff + k * 4096 + m * 1024); } while (0)
#define LDX(dst, b, h) do { _Pragma("unroll") for (int n = 0; n < 2; ++n) { dst[n][0] = *(const LAS bf16x8*)(lds + SB(b, h) + xoff0 + n * 2048); dst[n][1] = *(const LAS bf16x8*)(lds + SB(b, h) + xoff1 + n * 2048); } } while (0)
#define MMA(h, xh, Xt) do { __builtin_amdgcn_s_setprio(1); _Pragma("unroll") for (int m = 0; m < 4; ++m) _Pragma("unroll") for (int n = 0; n < 2; ++n) _Pragma("unroll") for (int k = 0; k < 2; ++k) \
        acc[h][xh][m][n] = __builtin_amdgcn_mfma_f32_16x16x32_bf16(Xt[n][k], Wt[m][k], acc[h][xh][m][n], 0, 0, 0); __builtin_amdgcn_s_setprio(0); } while (0)
#define FLAG_LOAD(step) asm volatile("global_load_dword %0, %1, %2 sc1" : "+v"(fl) : "v"(fvoff), "s"(fb + (size_t)(step) * FSTEP) : "memory")
#define FLAG_CHECK(step) do { if (wid == 0 && __builtin_amdgcn_ballot_w64(fl < seq) != 0ull) spin_flags(ws, fb + (size_t)(step) * FSTEP, fvoff, seq); } while (0)
#define WAIT_V6 asm volatile("s_waitcnt vmcnt(6)" : "+v"(fl) :: "memory")
#pragma unroll
    for (int a = 0; a < 2; ++a)
#pragma unroll
        for (int b = 0; b < 2; ++b)
#pragma unroll
            for (int m = 0; m < 4; ++m)
#pragma unroll
                for (int n = 0; n < 2; ++n) acc[a][b][m][n] = (f32x4){0.f, 0.f, 0.f, 0.f};
    bf16x8 Wt[4][2], X0[2][2], X1[2][2];
    u32 fl = 0;
    if (wid == 0) { spin_flags(ws, fb, fvoff, seq); spin_flags(ws, fb + FSTEP, fvoff, seq); }
    WAIT_V(0); WAIT_L(0); BAR; asm volatile("" ::: "memory");
    FLAG_LOAD(2);
    STX(0, 0, cX); STW(0, 0, cW); STX(0, 1, cX); STW(0, 1, cW);
    if (wr == 1) BAR;
    WAIT_V(4); BAR;
    STX(1, 0, cX + KSTEP); STW(1, 0, cW + KSTEP); STX(1, 1, cX + KSTEP);
    WAIT_V6; BAR;
    for (int t = 0; t < NSTEP; t += 2) {
        const bool last = (t == NSTEP - 2);
        const int t2 = last ? 0 : t + 2, t4 = (t + 4 < NSTEP) ? t + 4 : 0;
        const char* w1 = cW + (size_t)(t + 1) * KSTEP;
        const char* x2 = cX + (size_t)t2 * KSTEP; const char* w2 = cW + (size_t)t2 * KSTEP;
        const char* x3 = x2 + KSTEP; const char* w3 = w2 + KSTEP;
        FLAG_CHECK(t2); FLAG_LOAD(t2 + 1);
        LDX(X0, 0, 0); SCHED; LDW(0, 0); STW(1, 1, w1);
        WAIT_L(8); BAR; WAIT_L(0); MMA(0, 0, X0); BAR; SCHED;
        if (!SUB) LDX(X1, 0, 1);
        STX(0, 0, x2);
        BAR; if (!SUB) { WAIT_L(0); MMA(0, 1, X1); } BAR;
        LDW(0, 1); STW(0, 0, w2);
        BAR; WAIT_L(0); MMA(1, 0, X0); BAR; SCHED;
        STX(0, 1, x2);
        WAIT_V6; BAR; if (!SUB) MMA(1, 1, X1); BAR;
        FLAG_CHECK(t2 + 1); FLAG_LOAD(t4);
        LDX(X0, 1, 0); SCHED; LDW(1, 0); STW(0, 1, w2);
        WAIT_L(8); BAR; WAIT_L(0); MMA(0, 0, X0); BAR; SCHED;
        if (!SUB) LDX(X1, 1, 1);
        STX(1, 0, x3);
        BAR; if (!SUB) { WAIT_L(0); MMA(0, 1, X1); } BAR;
        LDW(1, 1); STW(1, 0, w3);
        BAR; WAIT_L(0); MMA(1, 0, X0); BAR; SCHED;
        STX(1, 1, x3);
        WAIT_V6; BAR; if (!SUB) MMA(1, 1, X1); BAR;
    }
    asm volatile("s_waitcnt vmcnt(0)" : "+v"(fl) :: "memory");
    if (wr == 0) BAR;
    BAR;
#undef SA
#undef SB
#undef GLDS
#undef STW
#undef STX
#undef LDW
#undef LDX
#undef MMA
#undef FLAG_LOAD
#undef FLAG_CHECK
#undef WAIT_V6
}

__device__ __forceinline__ void acc_to_raw(const f32x4 (&acc)[2][2][4][2], int xh, bool poison, u32x4 (&O)[8]) {
#pragma unroll
    for (int n = 0; n < 2; ++n)
#pragma unroll
        for (int j = 0; j < 4; ++j)
#pragma unroll
            for (int c = 0; c < 4; ++c) {
                const u32 v = cvt_pk_bf16(acc[c >> 1][xh][(c & 1) * 2][n][j], acc[c >> 1][xh][(c & 1) * 2 + 1][n][j]);
                O[n * 4 + j][c] = poison ? 0x7fc07fc0u : v;
            }
}

__global__ __launch_bounds__(512, 2) void fused_gemm_rs(Params p) {
    extern __shared__ __attribute__((aligned(16))) unsigned char shm[];
    LAS unsigned char* lds = (LAS unsigned char*)shm;
    const int lane = threadIdx.x & 63, wid = __builtin_amdgcn_readfirstlane(threadIdx.x >> 6), b = blockIdx.x;
    char* ws = p.ws[p.rank];
    const u32 seq = __builtin_amdgcn_readfirstlane(ld_agent(ws + OFF_CNT + b * 64)) + 1u;
    const int xcd = b & 7, ix = b >> 3;
    if (ix < 4) {
        STAMP(63);
        comm_role(p, ws, lds, ix * 8 + xcd, seq, wid, lane);
    } else {
        STAMP(0);
        const int pmi = (xcd & 1) * 7 + (ix - 4) % 7, pn = (xcd >> 1) * 4 + (ix - 4) / 7;
        const int gi = (15 - pn) * 14 + pmi;
        for (int pass = 0; pass < 3; ++pass) {
            const int task = pass < 2 ? pass * 1792 + wid * NGEMM + gi : 3584 + gi * 8 + wid;
            if (task < 4608) pack_task(p, ws, task, seq, lane);
            STAMP(1 + pass);
        }
        const int dest = (p.rank + 1 + (pmi >> 1)) & 7, pm = dest * 2 + (pmi & 1);
        f32x4 acc[2][2][4][2];
        gemm_loop<false>(ws, lds, ws + OFF_G + (size_t)pm * 32768, ws + OFF_G + (size_t)(16 + pn) * 32768,
                         (u32)(((lane & 4) ? 16 + pn : pm) * 128 + (lane & 3) * 4), seq, wid, lane, acc);
        STAMP(4);
        {
            const bool poison = __builtin_amdgcn_readfirstlane(ld_agent(ws + OFF_TMO)) == seq;
            const int id0 = ((pn * 2 + (pmi & 1)) * 8 + wid) * 2;
            u32 rec[2];
#pragma unroll
            for (int xh = 0; xh < 2; ++xh) {
                u32x4 O[8]; u32 L[16], H[16];
                acc_to_raw(acc, xh, poison, O);
#pragma unroll
                for (int d = 0; d < 4; ++d)
#pragma unroll
                    for (int q = 0; q < 4; ++q) { L[d * 4 + q] = perm(O[2 * d + 1][q], O[2 * d][q], 0x06040200u); H[d * 4 + q] = perm(O[2 * d + 1][q], O[2 * d][q], 0x07050301u); }
                rec[xh] = encode_store(L, H, ws + OFF_PY + (size_t)dest * (4 * MiB) + (size_t)(id0 + xh) * 8192, lane);
            }
            WAIT_V(0);
            if (lane < 16) put_record(p.ws[dest] + OFF_RY + (size_t)(p.rank * 512 + id0) * 64 + lane * 8, seq, lane < 8 ? rec[0] : rec[1]);
        }
        STAMP(5);
        if (threadIdx.x == 0) *(volatile LAS u32*)lds = __hip_atomic_fetch_add((GAS u32*)(ws + OFF_TICKET), 1u, __ATOMIC_RELAXED, __HIP_MEMORY_SCOPE_AGENT) - (seq - 1u) * NGEMM;
        __syncthreads();
        const u32 tk = __builtin_amdgcn_readfirstlane(*(volatile LAS u32*)lds);
        if (tk < 128u) {
            const int rq = tk & 3, cq = tk >> 2, opm = p.rank * 2 + (rq >> 1), opn = cq >> 1;
            gemm_loop<true>(ws, lds, ws + OFF_G + (size_t)opm * 32768 + (rq & 1) * 16384, ws + OFF_G + (size_t)(16 + opn) * 32768 + (cq & 1) * 16384,
                            (u32)(((lane >> 3) & 1) * 4096 + ((lane & 4) ? 16 + opn : opm) * 128 + (lane & 3) * 4), seq, wid, lane, acc);
            STAMP(6);
            LAS f32x4* red = (LAS f32x4*)(lds + (wid & 3) * 16384) + lane;
            if (wid >= 4) {
#pragma unroll
                for (int i = 0; i < 16; ++i) red[i * 64] = acc[i >> 3][0][(i >> 1) & 3][i & 1];
            }
            __syncthreads();
            if (wid < 4) {
#pragma unroll
                for (int i = 0; i < 16; ++i) acc[i >> 3][0][(i >> 1) & 3][i & 1] += red[i * 64];
                const bool poison = __builtin_amdgcn_readfirstlane(ld_agent(ws + OFF_TMO)) == seq;
                u32x4 O[8]; acc_to_raw(acc, 0, poison, O);
                char* dst = p.out + (size_t)(rq * 128 + wid * 32) * 65536 + (p.rank * 4096 + cq * 128) * 2;
                const u32 vo = (u32)(lane >> 4) * 262144u + (u32)(lane & 15) * 16u;
                sgpr_settle();
#pragma unroll
                for (int q = 0; q < 8; ++q) st16_sys<0>(dst + (size_t)((q >> 2) * 16 + (q & 3)) * 65536, vo, O[q]);
            }
            STAMP(7);
        }
    }
    __syncthreads();
    if (threadIdx.x == 0) st_agent(ws + OFF_CNT + b * 64, seq);
}

extern "C" void kernel_launch(int rank, int nranks, ncclComm_t comm, hipStream_t stream,
                              void* const* d_in, const int* in_sizes, int n_in,
                              void* d_out, int out_size, const int* in_dtypes, int out_dtype,
                              void* const* d_out_all, void* const* d_ws_all, size_t ws_bytes) {
    if (nranks != NR || (unsigned)rank >= (unsigned)NR || n_in != 2 || ws_bytes < WS_NEED || in_sizes[0] != 4096 * 512 || in_sizes[1] != 512 * 32768 || out_size != 512 * 32768) {
        fprintf(stderr, "kernel_launch: unexpected shapes\n");
        return;
    }
    HIP_CHECK(hipFuncSetAttribute((const void*)fused_gemm_rs, hipFuncAttributeMaxDynamicSharedMemorySize, LDS_BYTES));
    Params p;
    memset(&p, 0, sizeof p);
    p.x = (const char*)d_in[0]; p.w = (const char*)d_in[1]; p.out = (char*)d_out;
    for (int i = 0; i < NR; ++i) p.ws[i] = (char*)d_ws_all[i];
    p.ws_tab = (char* const*)d_ws_all;
    p.rank = rank;
    int key[256];
    for (int i = 0; i < 256; ++i) {
        const int j = i >> 5, k = i & 31;
        key[i] = k >= 16 ? 16 * j + SKEW_NUM * (k - 16) + 1 : ((k >> 1) == rank ? 100000 + j : 16 * j);
        int q = i;
        while (q > 0 && key[p.order[q - 1]] > key[i]) { p.order[q] = p.order[q - 1]; --q; }
        p.order[q] = (unsigned char)i;
    }
    hipLaunchKernelGGL(fused_gemm_rs, dim3(256), dim3(512), LDS_BYTES, stream, p);
    HIP_CHECK(hipGetLastError());
}
```

```cpp
#include <hip/hip_runtime.h>
#include <rccl/rccl.h>
#include <cstdio>
#include <cstdint>
#include <cstring>

#define HIP_CHECK(expr) do { hipError_t _e = (expr); if (_e != hipSuccess) { \
    fprintf(stderr, "HIP error %s at %s:%d\n", hipGetErrorString(_e), __FILE__, __LINE__); \
    return; } } while (0)

typedef unsigned u32;
typedef unsigned long long u64;
typedef u32 u32x4 __attribute__((ext_vector_type(4)));
typedef u32 u32x2 __attribute__((ext_vector_type(2)));
typedef short bf16x8 __attribute__((ext_vector_type(8)));
typedef float f32x4 __attribute__((ext_vector_type(4)));
#define LAS __attribute__((address_space(3)))
#define GAS __attribute__((address_space(1)))

#define FORCE_RAW 0
#define SKEW_NUM 5

constexpr int NR = 8;
constexpr size_t MiB = 1ull << 20;
constexpr size_t OFF_CNT = 0;
constexpr size_t OFF_TMO = 16384;
constexpr size_t OFF_TICKET = 16384 + 256;
constexpr size_t OFF_FG = 65536;
constexpr size_t OFF_RIN = 1 * MiB;
constexpr size_t OFF_RY = 2 * MiB;
constexpr size_t OFF_PX = 4 * MiB;
constexpr size_t OFF_PW = 12 * MiB;
constexpr size_t OFF_PY = 44 * MiB;
constexpr size_t OFF_G = 76 * MiB;
constexpr size_t OFF_DBG = 140 * MiB;
constexpr size_t WS_NEED = 141 * MiB;
constexpr u64 DEADLINE = 40000000ull;
constexpr int LDS_BYTES = 131072;
constexpr int NCOMM = 32, NGEMM = 224;
#define PACK_DELAY 32

struct Params {
    const char* x; const char* w; char* out;
    char* ws[NR];
    char* const* ws_tab;
    int rank; int pad;
    unsigned char order[256];
};

#define STAMP(i)

#define WAIT_V(n) asm volatile("s_waitcnt vmcnt(" #n ")" ::: "memory")
#define WAIT_L(n) asm volatile("s_waitcnt lgkmcnt(" #n ")" ::: "memory")
#define BAR __builtin_amdgcn_s_barrier()
#define SCHED __builtin_amdgcn_sched_barrier(0)

template <int OFF> __device__ __forceinline__ void st16_sys(const char* base, u32 voff, u32x4 v) {
    asm volatile("global_store_dwordx4 %0, %1, %2 offset:%3 sc0 sc1\n\ts_nop 1" :: "v"(voff), "v"(v), "s"(base), "i"(OFF) : "memory");
}
__device__ __forceinline__ void sgpr_settle() { asm volatile("s_nop 4" ::: "memory"); }
__device__ __forceinline__ u32 perm(u32 hi, u32 lo, u32 sel) { return __builtin_amdgcn_perm(hi, lo, sel); }
__device__ __forceinline__ u32 ld_agent(const char* p) { return __hip_atomic_load((GAS u32*)p, __ATOMIC_RELAXED, __HIP_MEMORY_SCOPE_AGENT); }
__device__ __forceinline__ void st_agent(char* p, u32 v) { __hip_atomic_store((GAS u32*)p, v, __ATOMIC_RELAXED, __HIP_MEMORY_SCOPE_AGENT); }
__device__ __forceinline__ u32 cvt_pk_bf16(float lo, float hi) { u32 r; asm("v_cvt_pk_bf16_f32 %0, %1, %2" : "=v"(r) : "v"(lo), "v"(hi)); return r; }

__device__ __forceinline__ void planes_to_raw(const u32 (&L)[16], const u32 (&H)[16], u32x4 (&O)[8]) {
#pragma unroll
    for (int d = 0; d < 4; ++d)
#pragma unroll
        for (int p = 0; p < 4; ++p) {
            O[2 * d][p] = perm(H[d * 4 + p], L[d * 4 + p], 0x05010400u);
            O[2 * d + 1][p] = perm(H[d * 4 + p], L[d * 4 + p], 0x07030602u);
        }
}
__device__ __forceinline__ void store_unit_linear(const char* dst, u32 voff, const u32x4 (&O)[8]) {
    const char* b = dst + 4096;
    sgpr_settle();
    st16_sys<-4096>(b, voff, O[0]); st16_sys<-3072>(b, voff, O[1]); st16_sys<-2048>(b, voff, O[2]); st16_sys<-1024>(b, voff, O[3]);
    st16_sys<0>(b, voff, O[4]); st16_sys<1024>(b, voff, O[5]); st16_sys<2048>(b, voff, O[6]); st16_sys<3072>(b, voff, O[7]);
}

__device__ __forceinline__ u32 encode_store(const u32 (&L)[16], const u32 (&H)[16], const char* slot, int lane) {
    float mx = 0.f;
#pragma unroll
    for (int t = 0; t < 16; t += 2)
        asm("v_max3_f32 %0, |%1|, |%2|, %0" : "+v"(mx) : "v"(H[t]), "v"(H[t + 1]));
    u32 m = __float_as_uint(mx) >> 24;
    m = max(m, (u32)__builtin_amdgcn_update_dpp(0, (int)m, 0x111, 0xf, 0xf, false));
    m = max(m, (u32)__builtin_amdgcn_update_dpp(0, (int)m, 0x112, 0xf, 0xf, false));
    m = max(m, (u32)__builtin_amdgcn_update_dpp(0, (int)m, 0x114, 0xf, 0xf, false));
    m = max(m, (u32)__builtin_amdgcn_update_dpp(0, (int)m, 0x118, 0xf, 0xf, false));
    const u32 top = max(max((u32)__builtin_amdgcn_readlane((int)m, 15), (u32)__builtin_amdgcn_readlane((int)m, 31)),
                        max((u32)__builtin_amdgcn_readlane((int)m, 47), (u32)__builtin_amdgcn_readlane((int)m, 63))) & 0x7fu;
    const u32 C = top > 7u ? top - 7u : 0u, C4 = C * 0x01010101u;
    u32 u[16], acc = 0;
#pragma unroll
    for (int t = 0; t < 16; ++t) { u[t] = H[t] - C4; acc |= u[t]; }
    u32 rec = 0, head = C;
    bool raw = FORCE_RAW;
    if (__builtin_amdgcn_ballot_w64((acc & 0x78787878u) != 0u) != 0ull) {
        u32 n = 0, sH = 0, sL = 0, sM = 0, sT = 0;
#pragma unroll
        for (int t = 0; t < 16; ++t) {
            const u32 w = (H[t] | 0x80808080u) - C4, mis = w & 0x78787878u;
            u[t] = (w & 0x07070707u) | (H[t] & 0x80808080u);
            if (mis) { ++n; sH = H[t]; sL = L[t]; sM = mis; sT = t; }
        }
        const u32 bsh = (n ? (u32)__builtin_ctz(sM) : 0u) & 24u;
        const bool multi = n > 1u || (sM & ~(0xffu << bsh)) != 0u;
        u64 mask = __builtin_amdgcn_ballot_w64(n != 0u);
        const u32 cnt = (u32)__builtin_popcountll(mask);
        if (cnt > 7u || __builtin_amdgcn_ballot_w64(multi) != 0ull) raw = true;
        else {
            const u32 ent = ((u32)lane << 22) | (sT << 18) | (bsh << 13) | (((sH >> bsh) & 0xffu) << 8) | ((sL >> bsh) & 0xffu);
            for (u32 s = 1; mask; ++s, mask &= mask - 1) {
                const u32 e = (u32)__builtin_amdgcn_readlane((int)ent, __builtin_ctzll(mask));
                if ((u32)(lane & 7) == s) rec = e;
            }
            head |= cnt << 8;
        }
    }
    const u32 voff = (u32)lane * 16u;
    if (raw) {
        u32x4 O[8]; planes_to_raw(L, H, O); store_unit_linear(slot, voff, O);
        head = 1u << 16;
    } else {
        u32x4 n0, n1;
#pragma unroll
        for (int s = 0; s < 4; ++s) {
            n0[s] = u[2 * s] | __builtin_amdgcn_alignbit(u[2 * s + 1], u[2 * s + 1], 28);
            n1[s] = u[8 + 2 * s] | __builtin_amdgcn_alignbit(u[8 + 2 * s + 1], u[8 + 2 * s + 1], 28);
        }
        const char* b = slot + 4096;
        sgpr_settle();
        st16_sys<-4096>(b, voff, (u32x4){L[0], L[1], L[2], L[3]}); st16_sys<-3072>(b, voff, (u32x4){L[4], L[5], L[6], L[7]});
        st16_sys<-2048>(b, voff, (u32x4){L[8], L[9], L[10], L[11]}); st16_sys<-1024>(b, voff, (u32x4){L[12], L[13], L[14], L[15]});
        st16_sys<0>(b, voff, n0); st16_sys<1024>(b, voff, n1);
    }
    if ((lane & 7) == 0) rec = head;
    return rec;
}
__device__ __forceinline__ void put_record(char* p, u32 seq, u32 w) {
    __hip_atomic_store((GAS u64*)p, ((u64)w << 32) | seq, __ATOMIC_RELAXED, __HIP_MEMORY_SCOPE_SYSTEM);
}

__device__ __forceinline__ void pack_task(const Params& p, char* ws, int task, u32 cnt_v, int lane) {
    const int j = task / 576, r = task % 576;
    u32x4 in[8]; u32 L[16], H[16];
    const u32 voff = (u32)lane * 16u;
    if (r < 64) {
        const int pm = r >> 2, un = r & 3;
        const char* src = p.x + (size_t)(pm * 256 + un * 64) * 1024 + j * 128 + (lane >> 3) * 1024;
        const int sw = (lane & 7) ^ (lane >> 4);
#pragma unroll
        for (int q = 0; q < 8; ++q) in[q] = *(const GAS u32x4*)(src + q * 8192 + ((sw ^ ((q & 1) << 2)) << 4));
        char* peer = p.ws_tab[(p.rank + 1 + (lane >> 3)) & 7];
#pragma unroll
        for (int d = 0; d < 4; ++d)
#pragma unroll
            for (int q = 0; q < 4; ++q) { L[d * 4 + q] = perm(in[2 * d + 1][q], in[2 * d][q], 0x06040200u); H[d * 4 + q] = perm(in[2 * d + 1][q], in[2 * d][q], 0x07050301u); }
        const u32 seq = __builtin_amdgcn_readfirstlane(cnt_v) + 1u;
        asm volatile("" : "+v"(peer));
        const u32 rec = encode_store(L, H, ws + OFF_PX + (size_t)(seq & 1) * (4 * MiB) + (size_t)((j * 16 + pm) * 4 + un) * 8192, lane);
        WAIT_V(0);
        if (lane < 56) put_record(peer + OFF_RIN + (size_t)(((seq & 1) * 8 + p.rank) * 1024 + (j * 32 + pm) * 4 + un) * 64 + (lane & 7) * 8, seq, rec);
        store_unit_linear(ws + OFF_G + (size_t)(j * 32 + pm) * 32768 + un * 8192, voff, in);
        WAIT_V(0);
        if (lane == 0) st_agent(ws + OFF_FG + (j * 32 + pm) * 128 + un * 4, seq);
    } else {
        const int dd = (r - 64) >> 6, pn = (r >> 2) & 15, un = r & 3, dest = (p.rank + dd) & 7;
        const char* src = p.w + (size_t)(j * 64 + (un & 1) * 32 + (lane >> 4) * 8) * 65536 + (dest * 4096 + pn * 256 + (un >> 1) * 128) * 2 + (lane & 15) * 16;
#pragma unroll
        for (int q = 0; q < 8; ++q) in[q] = *(const GAS u32x4*)(src + (size_t)q * 65536);
#pragma unroll
        for (int d = 0; d < 4; ++d)
#pragma unroll
            for (int q = 0; q < 4; ++q) { L[d * 4 + q] = perm(in[2 * q + 1][d], in[2 * q][d], 0x06020400u); H[d * 4 + q] = perm(in[2 * q + 1][d], in[2 * q][d], 0x07030501u); }
        const u32 seq = __builtin_amdgcn_readfirstlane(cnt_v) + 1u;
        if (dd == 0) {
            u32x4 O[8]; planes_to_raw(L, H, O);
            store_unit_linear(ws + OFF_G + (size_t)(j * 32 + 16 + pn) * 32768 + un * 8192, voff, O);
            WAIT_V(0);
            if (lane == 0) st_agent(ws + OFF_FG + (j * 32 + 16 + pn) * 128 + un * 4, seq);
        } else {
            const u32 rec = encode_store(L, H, ws + OFF_PW + (size_t)dest * (4 * MiB) + (size_t)((j * 16 + pn) * 4 + un) * 8192, lane);
            WAIT_V(0);
            if (lane < 8) put_record(p.ws[dest] + OFF_RIN + (size_t)(((seq & 1) * 8 + p.rank) * 1024 + (j * 32 + 16 + pn) * 4 + un) * 64 + lane * 8, seq, rec);
        }
    }
}

#define WIN 4
#define PACE 978
#define SLACK 3
constexpr int NSLOT = 7, NDEC = 5, NDESC = 16, NPOS = 1536, LOOK = 8;
constexpr u32 FIN = 0x40000000u;
constexpr int L_SLOT = 0;
constexpr int L_DESC = 2 * NSLOT * 8192;
constexpr int L_CTL = L_DESC + 2 * NDESC * 64;
constexpr int L_ORD = L_CTL + 128;
static_assert(L_ORD + 256 <= LDS_BYTES, "lds");

__device__ __forceinline__ void wait_units(u32 n) {
    switch (n) {
        case 0: WAIT_V(0); break; case 1: WAIT_V(6); break; case 2: WAIT_V(12); break; case 3: WAIT_V(18); break; case 4: WAIT_V(24); break;
        case 5: WAIT_V(30); break; case 6: WAIT_V(36); break; default: WAIT_V(42); break;
    }
}
__device__ __forceinline__ void comm_role(const Params& p, char* ws, LAS unsigned char* lds, int g, u32 seq, int wid, int lane) {
    if (threadIdx.x < 64) ((LAS u32*)(lds + L_ORD))[threadIdx.x] = ((const u32*)p.order)[threadIdx.x];
    if (threadIdx.x >= 64 && threadIdx.x < 96) ((LAS u32*)(lds + L_CTL))[threadIdx.x - 64] = 0;
    __syncthreads();
    volatile LAS u32* ctl = (volatile LAS u32*)(lds + L_CTL);
    volatile LAS u32* desc = (volatile LAS u32*)(lds + L_DESC);
    const u32 voff = (u32)lane * 16u;
    if (wid == 7) {
        const u32 st = lane >> 5, e = lane & 7, L = g * 2 + st, src = (p.rank + 1 + L % 7) & 7, j = L / 7;
        const u32 cnt = j < 9 ? (NPOS - j + 8) / 9 : 0;
        const char* rws = p.ws_tab[src];
        const u64 t0 = __builtin_amdgcn_s_memrealtime();
        bool dead = false;
        u32 x = (lane >> 3) & 3;
        while (__builtin_amdgcn_ballot_w64(x < cnt) != 0ull) {
            const bool act = x < cnt && x < ctl[st * 16 + 9] + LOOK;
            const u32 u = x < cnt ? j + 9 * x : 0;
            const u32 id = u < 1024 ? (u32)(lds + L_ORD)[u >> 2] * 4u + (u & 3) : u - 1024;
            const char* a = u >= 1024 ? ws + OFF_RY + (size_t)(src * 512 + id) * 64 : ws + OFF_RIN + (size_t)(((seq & 1) * 8 + src) * 1024 + id) * 64;
            u64 v = 0;
            if (act) v = __hip_atomic_load((GAS u64*)(a + e * 8), __ATOMIC_RELAXED, __HIP_MEMORY_SCOPE_SYSTEM);
            const u64 okm = __builtin_amdgcn_ballot_w64((u32)v == seq);
            const bool adv = act && (dead || ((okm >> (lane & 56)) & 0xffull) == 0xffull);
            if (__builtin_amdgcn_ballot_w64(adv) != 0ull) {
                if (adv) {
                    const char* s = u >= 1024 ? rws + OFF_PY + (size_t)p.rank * (4 * MiB) + (size_t)id * 8192
                        : rws + ((id & 64) ? OFF_PW + (size_t)p.rank * (4 * MiB) : OFF_PX + (size_t)(seq & 1) * (4 * MiB)) + (size_t)(((id >> 7) * 16 + ((id >> 2) & 15)) * 4 + (id & 3)) * 8192;
                    const u32 en = (st * NDESC + x % NDESC) * 16, word = dead ? (2u << 16) : (u32)(v >> 32);
                    const u64 sa = dead ? (u64)(ws + OFF_G) : (u64)s;
                    desc[en + e] = word;
                    if (e == 0) { desc[en + 8] = (u32)sa; desc[en + 9] = (u32)(sa >> 32); desc[en + 10] = word; desc[en + 11] = (u >= 1024 ? 0x8000u : 0u) | id; }
                    x += 4;
                }
                u32 m = min(x, (u32)__builtin_amdgcn_update_dpp((int)x, (int)x, 0x128, 0xf, 0xf, false));
                m = min(m, (u32)__builtin_amdgcn_ds_swizzle((int)m, 0x401F));
                WAIT_L(0);
                if ((lane & 31) == 0) ctl[st * 16 + 8] = min(m, cnt);
            } else if (__builtin_amdgcn_ballot_w64(act) != 0ull && (ld_agent(ws + OFF_TMO) == seq || __builtin_amdgcn_s_memrealtime() - t0 > DEADLINE)) {
                st_agent(ws + OFF_TMO, seq); WAIT_V(0); dead = true;
            } else __builtin_amdgcn_s_sleep(1);
        }
        if ((lane & 31) == 0) ctl[st * 16 + 8] = cnt | FIN;
    } else if (wid >= NDEC) {
        const u32 me = wid - NDEC;
        volatile LAS u32* c = ctl + me * 16;
        u32 pub = 0, landed = 0, k = 0, slot = 0;
        const u32 t0 = (u32)__builtin_amdgcn_s_memrealtime();
        u32 next = (u32)((g * 2 + me) / 7) * (3 * PACE / 9);
        auto pace = [&](u32 kib) {
            u32 now;
            while ((int)((now = ((u32)__builtin_amdgcn_s_memrealtime() - t0) * 64u) - next) < 0) {}
            if (now - next > SLACK * PACE) next = now - SLACK * PACE;
            next += kib * PACE;
        };
        for (;; ++k) {
            bool fin = false;
            while (true) {
                if ((pub & ~FIN) <= k) pub = __builtin_amdgcn_readfirstlane(c[8]);
                if ((pub & ~FIN) > k && (u32)__builtin_amdgcn_readfirstlane(c[slot]) + NSLOT > k) break;
                if (landed < k) {
                    wait_units(k - landed - 1);
                    ++landed;
                    if (lane == 0) c[10] = landed;
                } else if (pub == (k | FIN)) { fin = true; break; }
                else __builtin_amdgcn_s_sleep(1);
            }
            if (fin) break;
            asm volatile("" ::: "memory");
            const u32x4 dd = *(const LAS u32x4*)(lds + L_DESC + (me * NDESC + k % NDESC) * 64 + 32);
            const char* s = (const char*)(((u64)(u32)__builtin_amdgcn_readfirstlane(dd[1]) << 32) | (u32)__builtin_amdgcn_readfirstlane(dd[0]));
            const u32 head = __builtin_amdgcn_readfirstlane(dd[2]);
            LAS unsigned char* d = lds + L_SLOT + (me * NSLOT + slot) * 8192;
            if (lane == 0) c[9] = k;
            pace(3);
            __builtin_amdgcn_global_load_lds((const GAS u32*)(s + voff), (LAS u32*)d, 16, 0, 17);
            __builtin_amdgcn_global_load_lds((const GAS u32*)(s + voff), (LAS u32*)d, 16, 1024, 17);
            __builtin_amdgcn_global_load_lds((const GAS u32*)(s + voff), (LAS u32*)d, 16, 2048, 17);
            asm volatile("s_waitcnt vmcnt(%0)" :: "i"(3 * WIN) : "memory");
            if ((WIN & 1) && k > landed + WIN / 2) { landed = k - WIN / 2; if (lane == 0) c[10] = landed; }
            if (head >> 16) {
                pace(2);
                __builtin_amdgcn_global_load_lds((const GAS u32*)(s + 6144 + voff), (LAS u32*)(d + 6144), 16, 0, 17);
                __builtin_amdgcn_global_load_lds((const GAS u32*)(s + 6144 + voff), (LAS u32*)(d + 6144), 16, 1024, 17);
            }
            pace(3);
            __builtin_amdgcn_global_load_lds((const GAS u32*)(s + 3072 + voff), (LAS u32*)(d + 3072), 16, 0, 17);
            __builtin_amdgcn_global_load_lds((const GAS u32*)(s + 3072 + voff), (LAS u32*)(d + 3072), 16, 1024, 17);
            __builtin_amdgcn_global_load_lds((const GAS u32*)(s + 3072 + voff), (LAS u32*)(d + 3072), 16, 2048, 17);
            asm volatile("s_waitcnt vmcnt(%0)" :: "i"(3 * WIN) : "memory");
            if (!(WIN & 1) && k + 1 > landed + WIN / 2) { landed = k + 1 - WIN / 2; if (lane == 0) c[10] = landed; }
            if (++slot == NSLOT) slot = 0;
        }
        if (lane == 0) c[10] = k | FIN;
    } else {
        char* pend = nullptr;
        int k = 0;
        u32 st = wid & 1;
        for (;;) {
            u32 e;
            {
                bool got = false;
                for (u32 idle = 0;;) {
                    st ^= 1;
                    const u32x2 lc = *(volatile LAS u32x2*)(lds + L_CTL + st * 64 + 40);
                    const u32 ld = __builtin_amdgcn_readfirstlane(lc[0]);
                    e = __builtin_amdgcn_readfirstlane(lc[1]);
                    if (e < (ld & ~FIN)) {
                        u32 o = e;
                        if (lane == 0) __hip_atomic_compare_exchange_strong((LAS u32*)(lds + L_CTL + st * 64 + 44), &o, e + 1, __ATOMIC_RELAXED, __ATOMIC_RELAXED, __HIP_MEMORY_SCOPE_WORKGROUP);
                        if (__builtin_amdgcn_readfirstlane(o) == e) { got = true; break; }
                        idle = 0;
                        continue;
                    }
                    if (ld & FIN) { if (++idle == 2) break; } else idle = 0;
                    if (pend) { WAIT_V(0); if (lane == 0) st_agent(pend, seq); pend = nullptr; }
                    else __builtin_amdgcn_s_sleep(1);
                }
                if (!got) break;
            }
            asm volatile("" ::: "memory");
            STAMP(k < 124 ? k >> 1 : 62);
            const u32 en = (st * NDESC + e % NDESC) * 16, sn = e % NSLOT;
            const u32 ent = __builtin_amdgcn_readfirstlane(desc[en + 11]), id = ent & 0x3ffu, w = (g * 2 + st) % 7;
            const u32 rec = desc[en + (lane & 7)], head = __builtin_amdgcn_readfirstlane(rec);
            const LAS u32x4* sl = (const LAS u32x4*)(lds + L_SLOT + (st * NSLOT + sn) * 8192 + voff);
            u32x4 O[8];
            ++k;
            if (head >> 16) {
#pragma unroll
                for (int q = 0; q < 8; ++q) O[q] = sl[q * 64];
                WAIT_L(0);
                if (lane == 0) ctl[st * 16 + sn] = e + 1;
                if (head >> 17) {
#pragma unroll
                    for (int q = 0; q < 8; ++q) O[q] = (u32x4){0x7fc07fc0u, 0x7fc07fc0u, 0x7fc07fc0u, 0x7fc07fc0u};
                }
            } else {
                u32 L[16], H[16];
                u32x4 v[6];
                const u32 C4 = (head & 0xffu) * 0x01010101u;
#pragma unroll
                for (int q = 0; q < 6; ++q) v[q] = sl[q * 64];
                WAIT_L(0);
                if (lane == 0) ctl[st * 16 + sn] = e + 1;
#pragma unroll
                for (int q = 0; q < 4; ++q) { L[4 * q] = v[q][0]; L[4 * q + 1] = v[q][1]; L[4 * q + 2] = v[q][2]; L[4 * q + 3] = v[q][3]; }
#pragma unroll
                for (int q = 0; q < 2; ++q)
#pragma unroll
                    for (int s = 0; s < 4; ++s) { H[8 * q + 2 * s] = (v[4 + q][s] & 0x87878787u) + C4; H[8 * q + 2 * s + 1] = (__builtin_amdgcn_alignbit(v[4 + q][s], v[4 + q][s], 4) & 0x87878787u) + C4; }
                planes_to_raw(L, H, O);
            }
            const u32 cnt = (head >> 8) & 0xffu;
            const u32 el = (rec >> 22) & 63u, et = (rec >> 18) & 15u, eb = (rec >> 16) & 3u, eq = (et >> 2) * 2u + (eb >> 1), eo = (et & 3u) * 4u + (eb & 1u) * 2u;
            char* flag = nullptr;
            if (!(ent & 0x8000u)) {
                const u32 blk = (8 + 7 * (id >> 7) + w) * 32 + ((id >> 2) & 31);
                char* dst = ws + OFF_G + (size_t)blk * 32768 + (id & 3) * 8192;
                store_unit_linear(dst, voff, O);
                if (cnt && lane >= 1 && (u32)lane <= cnt)
                    asm volatile("global_store_short %0, %1, %2 sc0 sc1" :: "v"(eq * 1024u + el * 16u + eo), "v"(rec), "s"(dst) : "memory");
                flag = ws + OFF_FG + blk * 128 + (id & 3) * 4;
            } else {
                const u32 src = (p.rank + 1 + w) & 7;
                const u32 row0 = ((id >> 4) & 1) * 256 + (id & 1) * 128 + ((id >> 1) & 3) * 32, col0 = src * 4096 + ((id >> 5) & 15) * 256 + ((id >> 3) & 1) * 128;
                char* dst = p.out + (size_t)row0 * 65536 + col0 * 2;
                const u32 vo = (u32)(lane >> 4) * 262144u + (u32)(lane & 15) * 16u;
                sgpr_settle();
#pragma unroll
                for (int q = 0; q < 8; ++q) st16_sys<0>(dst + (size_t)((q >> 2) * 16 + (q & 3)) * 65536, vo, O[q]);
                if (cnt && lane >= 1 && (u32)lane <= cnt)
                    asm volatile("global_store_short %0, %1, %2 sc0 sc1" :: "v"(((eq >> 2) * 16u + (eq & 3u) + (el >> 4) * 4u) * 65536u + (el & 15u) * 16u + eo), "v"(rec), "s"(dst) : "memory");
            }
            if (pend) { WAIT_V(8); if (lane == 0) st_agent(pend, seq); }
            pend = flag;
        }
        if (pend) { WAIT_V(0); if (lane == 0) st_agent(pend, seq); }
        STAMP(62);
    }
}

__device__ __forceinline__ void spin_flags(char* ws, const char* fbase, u32 fvoff, u32 seq) {
    const u64 t0 = __builtin_amdgcn_s_memrealtime();
    while (__builtin_amdgcn_ballot_w64(ld_agent(fbase + fvoff) < seq) != 0ull) {
        if (ld_agent(ws + OFF_TMO) == seq || __builtin_amdgcn_s_memrealtime() - t0 > DEADLINE) { st_agent(ws + OFF_TMO, seq); break; }
        __builtin_amdgcn_s_sleep(4);
    }
    __builtin_amdgcn_s_waitcnt(0xC07F);
    WAIT_V(0);
}

template <bool SUB>
__device__ __forceinline__ void gemm_loop(char* ws, LAS unsigned char* lds, const char* gX, const char* gW, u32 fvoff, u32 seq, int wid, int lane, f32x4 (&acc)[2][2][4][2]) {
    constexpr size_t KSTEP = SUB ? 2 * MiB : MiB, SX = SUB ? MiB : 16384, SW = SUB ? MiB : 16384;
    constexpr int NSTEP = SUB ? 32 : 64, FSTEP = SUB ? 8192 : 4096;
    const int wr = wid >> 2, wc = wid & 3, fr = lane & 15, fq = lane >> 4;
    const u32 voff = (u32)lane * 16u;
    const char* cX = gX + wid * 1024;
    const char* cW = gW + (wid >> 2) * 8192 + (wid & 3) * 1024;
    const char* fb = ws + OFF_FG;
    const u32 ldsw = (u32)wid * 1024u;
    const int woff = wr * 8192 + lane * 16;
    const int xoff0 = (wc * 32 + fr) * 128 + ((fq ^ (fr >> 1)) << 4) + (SUB ? wr * 16384 : 0), xoff1 = xoff0 ^ 64;
#define SA(b, h) (((b) * 2 + (h)) * 16384)
#define SB(b, h) ((4 + (b) * 2 + (h)) * 16384)
#define GLDS(src, dst) __builtin_amdgcn_global_load_lds((const GAS u32*)((src) + voff), (LAS u32*)(lds + (dst) + ldsw), 16, 0, 17)
#define STW(b, h, ptr) do { GLDS((ptr) + (h) * 4096, SA(b, h)); GLDS((ptr) + (h) * 4096 + SW, SA(b, h) + 8192); } while (0)
#define STX(b, h, ptr) do { GLDS((ptr) + (h) * SX, SB(b, h)); GLDS((ptr) + (h) * SX + 8192, SB(b, h) + 8192); } while (0)
#define LDW(b, h) do { _Pragma("unroll") for (int m = 0; m < 4; ++m) _Pragma("unroll") for (int k = 0; k < 2; ++k) Wt[m][k] = *(const LAS bf16x8*)(lds + SA(b, h) + woff + k * 4096 + m * 1024); } while (0)
#define LDX(dst, b, h) do { _Pragma("unroll") for (int n = 0; n < 2; ++n) { dst[n][0] = *(const LAS bf16x8*)(lds + SB(b, h) + xoff0 + n * 2048); dst[n][1] = *(const LAS bf16x8*)(lds + SB(b, h) + xoff1 + n * 2048); } } while (0)
#define MMA(h, xh, Xt) do { __builtin_amdgcn_s_setprio(1); _Pragma("unroll") for (int m = 0; m < 4; ++m) _Pragma("unroll") for (int n = 0; n < 2; ++n) _Pragma("unroll") for (int k = 0; k < 2; ++k) \
        acc[h][xh][m][n] = __builtin_amdgcn_mfma_f32_16x16x32_bf16(Xt[n][k], Wt[m][k], acc[h][xh][m][n], 0, 0, 0); __builtin_amdgcn_s_setprio(0); } while (0)
#define FLAG_LOAD(step) asm volatile("global_load_dword %0, %1, %2 sc1" : "+v"(fl) : "v"(fvoff), "s"(fb + (size_t)(step) * FSTEP) : "memory")
#define FLAG_CHECK(step) do { if (wid == 0 && __builtin_amdgcn_ballot_w64(fl < seq) != 0ull) spin_flags(ws, fb + (size_t)(step) * FSTEP, fvoff, seq); } while (0)
#define WAIT_V6 asm volatile("s_waitcnt vmcnt(6)" : "+v"(fl) :: "memory")
#pragma unroll
    for (int a = 0; a < 2; ++a)
#pragma unroll
        for (int b = 0; b < 2; ++b)
#pragma unroll
            for (int m = 0; m < 4; ++m)
#pragma unroll
                for (int n = 0; n < 2; ++n) acc[a][b][m][n] = (f32x4){0.f, 0.f, 0.f, 0.f};
    bf16x8 Wt[4][2], X0[2][2], X1[2][2];
    u32 fl = 0;
    if (wid == 0) { spin_flags(ws, fb, fvoff, seq); spin_flags(ws, fb + FSTEP, fvoff, seq); }
    WAIT_V(0); WAIT_L(0); BAR; asm volatile("" ::: "memory");
    FLAG_LOAD(2);
    STX(0, 0, cX); STW(0, 0, cW); STX(0, 1, cX); STW(0, 1, cW);
    if (wr == 1) BAR;
    WAIT_V(4); BAR;
    STX(1, 0, cX + KSTEP); STW(1, 0, cW + KSTEP); STX(1, 1, cX + KSTEP);
    WAIT_V6; BAR;
    for (int t = 0; t < NSTEP; t += 2) {
        const bool last = (t == NSTEP - 2);
        const int t2 = last ? 0 : t + 2, t4 = (t + 4 < NSTEP) ? t + 4 : 0;
        const char* w1 = cW + (size_t)(t + 1) * KSTEP;
        const char* x2 = cX + (size_t)t2 * KSTEP; const char* w2 = cW + (size_t)t2 * KSTEP;
        const char* x3 = x2 + KSTEP; const char* w3 = w2 + KSTEP;
        FLAG_CHECK(t2); FLAG_LOAD(t2 + 1);
        LDX(X0, 0, 0); SCHED; LDW(0, 0); STW(1, 1, w1);
        WAIT_L(8); BAR; WAIT_L(0); MMA(0, 0, X0); BAR; SCHED;
        if (!SUB) LDX(X1, 0, 1);
        STX(0, 0, x2);
        BAR; if (!SUB) { WAIT_L(0); MMA(0, 1, X1); } BAR;
        LDW(0, 1); STW(0, 0, w2);
        BAR; WAIT_L(0); MMA(1, 0, X0); BAR; SCHED;
        STX(0, 1, x2);
        WAIT_V6; BAR; if (!SUB) MMA(1, 1, X1); BAR;
        FLAG_CHECK(t2 + 1); FLAG_LOAD(t4);
        LDX(X0, 1, 0); SCHED; LDW(1, 0); STW(0, 1, w2);
        WAIT_L(8); BAR; WAIT_L(0); MMA(0, 0, X0); BAR; SCHED;
        if (!SUB) LDX(X1, 1, 1);
        STX(1, 0, x3);
        BAR; if (!SUB) { WAIT_L(0); MMA(0, 1, X1); } BAR;
        LDW(1, 1); STW(1, 0, w3);
        BAR; WAIT_L(0); MMA(1, 0, X0); BAR; SCHED;
        STX(1, 1, x3);
        WAIT_V6; BAR; if (!SUB) MMA(1, 1, X1); BAR;
    }
    asm volatile("s_waitcnt vmcnt(0)" : "+v"(fl) :: "memory");
    if (wr == 0) BAR;
    BAR;
#undef SA
#undef SB
#undef GLDS
#undef STW
#undef STX
#undef LDW
#undef LDX
#undef MMA
#undef FLAG_LOAD
#undef FLAG_CHECK
#undef WAIT_V6
}

__device__ __forceinline__ void acc_to_raw(const f32x4 (&acc)[2][2][4][2], int xh, bool poison, u32x4 (&O)[8]) {
#pragma unroll
    for (int n = 0; n < 2; ++n)
#pragma unroll
        for (int j = 0; j < 4; ++j)
#pragma unroll
            for (int c = 0; c < 4; ++c) {
                const u32 v = cvt_pk_bf16(acc[c >> 1][xh][(c & 1) * 2][n][j], acc[c >> 1][xh][(c & 1) * 2 + 1][n][j]);
                O[n * 4 + j][c] = poison ? 0x7fc07fc0u : v;
            }
}

__global__ __launch_bounds__(512, 2) void fused_gemm_rs(Params p) {
    extern __shared__ __attribute__((aligned(16))) unsigned char shm[];
    LAS unsigned char* lds = (LAS unsigned char*)shm;
    const int lane = threadIdx.x & 63, wid = __builtin_amdgcn_readfirstlane(threadIdx.x >> 6), b = blockIdx.x;
    char* ws = p.ws[p.rank];
    const u32 cnt_v = ld_agent(ws + OFF_CNT + b * 64);
    const int xcd = b & 7, ix = b >> 3;
    u32 seq;
    if (ix < 4) {
        STAMP(63);
        seq = __builtin_amdgcn_readfirstlane(cnt_v) + 1u;
        comm_role(p, ws, lds, ix * 8 + xcd, seq, wid, lane);
    } else {
        STAMP(0);
        const int pmi = (xcd & 1) * 7 + (ix - 4) % 7, pn = (xcd >> 1) * 4 + (ix - 4) / 7;
        const int gi = (15 - pn) * 14 + pmi;
        if (PACK_DELAY && wid * NGEMM + gi >= 64) __builtin_amdgcn_s_sleep(PACK_DELAY);
        for (int pass = 0; pass < 3; ++pass) {
            const int task = pass < 2 ? pass * 1792 + wid * NGEMM + gi : 3584 + gi * 8 + wid;
            if (task < 4608) pack_task(p, ws, task, cnt_v, lane);
            STAMP(1 + pass);
        }
        seq = __builtin_amdgcn_readfirstlane(cnt_v) + 1u;
        const int dest = (p.rank + 1 + (pmi >> 1)) & 7, pm = dest * 2 + (pmi & 1);
        f32x4 acc[2][2][4][2];
        gemm_loop<false>(ws, lds, ws + OFF_G + (size_t)pm * 32768, ws + OFF_G + (size_t)(16 + pn) * 32768,
                         (u32)(((lane & 4) ? 16 + pn : pm) * 128 + (lane & 3) * 4), seq, wid, lane, acc);
        STAMP(4);
        {
            const bool poison = __builtin_amdgcn_readfirstlane(ld_agent(ws + OFF_TMO)) == seq;
            const int id0 = ((pn * 2 + (pmi & 1)) * 8 + wid) * 2;
            u32 rec[2];
#pragma unroll
            for (int xh = 0; xh < 2; ++xh) {
                u32x4 O[8]; u32 L[16], H[16];
                acc_to_raw(acc, xh, poison, O);
#pragma unroll
                for (int d = 0; d < 4; ++d)
#pragma unroll
                    for (int q = 0; q < 4; ++q) { L[d * 4 + q] = perm(O[2 * d + 1][q], O[2 * d][q], 0x06040200u); H[d * 4 + q] = perm(O[2 * d + 1][q], O[2 * d][q], 0x07050301u); }
                rec[xh] = encode_store(L, H, ws + OFF_PY + (size_t)dest * (4 * MiB) + (size_t)(id0 + xh) * 8192, lane);
            }
            WAIT_V(0);
            if (lane < 16) put_record(p.ws[dest] + OFF_RY + (size_t)(p.rank * 512 + id0) * 64 + lane * 8, seq, lane < 8 ? rec[0] : rec[1]);
        }
        STAMP(5);
        if (threadIdx.x == 0) *(volatile LAS u32*)lds = __hip_atomic_fetch_add((GAS u32*)(ws + OFF_TICKET), 1u, __ATOMIC_RELAXED, __HIP_MEMORY_SCOPE_AGENT) - (seq - 1u) * NGEMM;
        __syncthreads();
        const u32 tk = __builtin_amdgcn_readfirstlane(*(volatile LAS u32*)lds);
        if (tk < 128u) {
            const int rq = tk & 3, cq = tk >> 2, opm = p.rank * 2 + (rq >> 1), opn = cq >> 1;
            gemm_loop<true>(ws, lds, ws + OFF_G + (size_t)opm * 32768 + (rq & 1) * 16384, ws + OFF_G + (size_t)(16 + opn) * 32768 + (cq & 1) * 16384,
                            (u32)(((lane >> 3) & 1) * 4096 + ((lane & 4) ? 16 + opn : opm) * 128 + (lane & 3) * 4), seq, wid, lane, acc);
            STAMP(6);
            LAS f32x4* red = (LAS f32x4*)(lds + (wid & 3) * 16384) + lane;
            if (wid >= 4) {
#pragma unroll
                for (int i = 0; i < 16; ++i) red[i * 64] = acc[i >> 3][0][(i >> 1) & 3][i & 1];
            }
            __syncthreads();
            if (wid < 4) {
#pragma unroll
                for (int i = 0; i < 16; ++i) acc[i >> 3][0][(i >> 1) & 3][i & 1] += red[i * 64];
                const bool poison = __builtin_amdgcn_readfirstlane(ld_agent(ws + OFF_TMO)) == seq;
                u32x4 O[8]; acc_to_raw(acc, 0, poison, O);
                char* dst = p.out + (size_t)(rq * 128 + wid * 32) * 65536 + (p.rank * 4096 + cq * 128) * 2;
                const u32 vo = (u32)(lane >> 4) * 262144u + (u32)(lane & 15) * 16u;
                sgpr_settle();
#pragma unroll
                for (int q = 0; q < 8; ++q) st16_sys<0>(dst + (size_t)((q >> 2) * 16 + (q & 3)) * 65536, vo, O[q]);
            }
            STAMP(7);
        }
    }
    __syncthreads();
    if (threadIdx.x == 0) st_agent(ws + OFF_CNT + b * 64, seq);
}

extern "C" void kernel_launch(int rank, int nranks, ncclComm_t comm, hipStream_t stream,
                              void* const* d_in, const int* in_sizes, int n_in,
                              void* d_out, int out_size, const int* in_dtypes, int out_dtype,
                              void* const* d_out_all, void* const* d_ws_all, size_t ws_bytes) {
    if (nranks != NR || (unsigned)rank >= (unsigned)NR || n_in != 2 || ws_bytes < WS_NEED || in_sizes[0] != 4096 * 512 || in_sizes[1] != 512 * 32768 || out_size != 512 * 32768) {
        fprintf(stderr, "kernel_launch: unexpected shapes\n");
        return;
    }
    HIP_CHECK(hipFuncSetAttribute((const void*)fused_gemm_rs, hipFuncAttributeMaxDynamicSharedMemorySize, LDS_BYTES));
    Params p;
    memset(&p, 0, sizeof p);
    p.x = (const char*)d_in[0]; p.w = (const char*)d_in[1]; p.out = (char*)d_out;
    for (int i = 0; i < NR; ++i) p.ws[i] = (char*)d_ws_all[i];
    p.ws_tab = (char* const*)d_ws_all;
    p.rank = rank;
    int key[256];
    for (int i = 0; i < 256; ++i) {
        const int j = i >> 5, k = i & 31;
        key[i] = k >= 16 ? 16 * j + SKEW_NUM * (k - 16) + 1 : ((k >> 1) == rank ? 100000 + j : 16 * j);
        int q = i;
        while (q > 0 && key[p.order[q - 1]] > key[i]) { p.order[q] = p.order[q - 1]; --q; }
        p.order[q] = (unsigned char)i;
    }
    hipLaunchKernelGGL(fused_gemm_rs, dim3(256), dim3(512), LDS_BYTES, stream, p);
    HIP_CHECK(hipGetLastError());
}
```

```cpp
#include <hip/hip_runtime.h>
#include <rccl/rccl.h>
#include <cstdio>
#include <cstdint>
#include <cstring>

#define HIP_CHECK(expr) do { hipError_t _e = (expr); if (_e != hipSuccess) { \
    fprintf(stderr, "HIP error %s at %s:%d\n", hipGetErrorString(_e), __FILE__, __LINE__); \
    return; } } while (0)

typedef unsigned u32;
typedef unsigned long long u64;
typedef u32 u32x4 __attribute__((ext_vector_type(4)));
typedef u32 u32x2 __attribute__((ext_vector_type(2)));
typedef short bf16x8 __attribute__((ext_vector_type(8)));
typedef float f32x4 __attribute__((ext_vector_type(4)));
#define LAS __attribute__((address_space(3)))
#define GAS __attribute__((address_space(1)))

#define FORCE_RAW 0
#define GAUX 17
#define SUBTAU 36
#define ROT 1
#define XSPREAD 12
#define LOOPV 1
#define MIX 1
#define PASS1(v) (v)
#define SKEW_NUM 38

constexpr int NR = 8;
constexpr size_t MiB = 1ull << 20;
constexpr size_t OFF_CNT = 0;
constexpr size_t OFF_TMO = 16384;
constexpr size_t OFF_TICKET = 16384 + 256;
constexpr size_t OFF_FG = 65536;
constexpr size_t OFF_RIN = 1 * MiB;
constexpr size_t OFF_RY = 2 * MiB;
constexpr size_t OFF_PX = 4 * MiB;
constexpr size_t OFF_PW = 12 * MiB;
constexpr size_t OFF_PY = 44 * MiB;
constexpr size_t OFF_G = 76 * MiB;
constexpr size_t OFF_DBG = 140 * MiB;
constexpr size_t WS_NEED = 142 * MiB;
constexpr u64 DEADLINE = 40000000ull;
constexpr int LDS_GEMM = 131072, LDS_BYTES = LDS_GEMM + 8 * 1152;
constexpr int NCOMM = 32, NGEMM = 224;
#define PACK_DELAY 32

struct Params {
    const char* x; const char* w; char* out;
    char* ws[NR];
    char* const* ws_tab;
    int rank; int pad;
    unsigned char order[256];
    unsigned char worder[128];
};

#define STAMP(i)

#define WAIT_V(n) asm volatile("s_waitcnt vmcnt(" #n ")" ::: "memory")
#define WAIT_L(n) asm volatile("s_waitcnt lgkmcnt(" #n ")" ::: "memory")
#define BAR __builtin_amdgcn_s_barrier()
#define SCHED __builtin_amdgcn_sched_barrier(0)

template <int OFF> __device__ __forceinline__ void st16_sys(const char* base, u32 voff, u32x4 v) {
    asm volatile("global_store_dwordx4 %0, %1, %2 offset:%3 sc0 sc1\n\ts_nop 1" :: "v"(voff), "v"(v), "s"(base), "i"(OFF) : "memory");
}
__device__ __forceinline__ void sgpr_settle() { asm volatile("s_nop 4" ::: "memory"); }
__device__ __forceinline__ u32 perm(u32 hi, u32 lo, u32 sel) { return __builtin_amdgcn_perm(hi, lo, sel); }
__device__ __forceinline__ u32 ld_agent(const char* p) { return __hip_atomic_load((GAS u32*)p, __ATOMIC_RELAXED, __HIP_MEMORY_SCOPE_AGENT); }
__device__ __forceinline__ void st_agent(char* p, u32 v) { __hip_atomic_store((GAS u32*)p, v, __ATOMIC_RELAXED, __HIP_MEMORY_SCOPE_AGENT); }
__device__ __forceinline__ u32 cvt_pk_bf16(float lo, float hi) { u32 r; asm("v_cvt_pk_bf16_f32 %0, %1, %2" : "=v"(r) : "v"(lo), "v"(hi)); return r; }

__device__ __forceinline__ void planes_to_raw(const u32 (&L)[16], const u32 (&H)[16], u32x4 (&O)[8]) {
#pragma unroll
    for (int d = 0; d < 4; ++d)
#pragma unroll
        for (int p = 0; p < 4; ++p) {
            O[2 * d][p] = perm(H[d * 4 + p], L[d * 4 + p], 0x05010400u);
            O[2 * d + 1][p] = perm(H[d * 4 + p], L[d * 4 + p], 0x07030602u);
        }
}
__device__ __forceinline__ void store_unit_linear(const char* dst, u32 voff, const u32x4 (&O)[8]) {
    const char* b = dst + 4096;
    sgpr_settle();
    st16_sys<-4096>(b, voff, O[0]); st16_sys<-3072>(b, voff, O[1]); st16_sys<-2048>(b, voff, O[2]); st16_sys<-1024>(b, voff, O[3]);
    st16_sys<0>(b, voff, O[4]); st16_sys<1024>(b, voff, O[5]); st16_sys<2048>(b, voff, O[6]); st16_sys<3072>(b, voff, O[7]);
}

constexpr u32 H_RAW = 1u << 16, H_DEAD = 2u << 16;
constexpr int ENC_BYTES = 1152;
__device__ __forceinline__ u32 bfi(u32 m, u32 a, u32 b) { return (a & m) | (b & ~m); }
__device__ __forceinline__ u32 scan32(u32 v) {
    v += (u32)__builtin_amdgcn_update_dpp(0, (int)v, 0x111, 0xf, 0xf, false);
    v += (u32)__builtin_amdgcn_update_dpp(0, (int)v, 0x112, 0xf, 0xf, false);
    v += (u32)__builtin_amdgcn_update_dpp(0, (int)v, 0x114, 0xf, 0xf, false);
    v += (u32)__builtin_amdgcn_update_dpp(0, (int)v, 0x118, 0xf, 0xf, false);
    v += (u32)__builtin_amdgcn_update_dpp(0, (int)v, 0x142, 0xa, 0xf, false);
    return v;
}
__device__ __forceinline__ void enc_init(LAS unsigned char* sc, int lane) {
    u32 sel = 0x0c0c0c0cu;
#pragma unroll
    for (int b = 3; b >= 0; --b) if (lane & (1 << b)) sel = (sel << 8) | (u32)b;
    if (lane < 16) ((LAS u32*)(sc + 1088))[lane] = sel;
}
template <class F>
__device__ __forceinline__ u32 encode_store(const u32 (&L)[16], const u32 (&H)[16], const char* slot, LAS unsigned char* sc, int lane, F before_stores) {
    u32 d[16], S[16];
    u32 sum = 0;
#pragma unroll
    for (int t = 0; t < 16; ++t) { d[t] = bfi(0x01010101u, L[t] >> 7, H[t] << 1); S[t] = bfi(0x80808080u, H[t], L[t]); }
#pragma unroll
    for (int t = 0; t < 16; t += 4) sum = __builtin_amdgcn_sad_u8(d[t], 0u, sum);
    sum = scan32(sum);
    const u32 tot = (u32)__builtin_amdgcn_readlane((int)sum, 31) + (u32)__builtin_amdgcn_readlane((int)sum, 63);
    const u32 T = min(max((tot >> 10) + 4u, 15u), 255u), T4 = T * 0x01010101u;
    *(LAS u32x4*)(sc + lane * 16) = (u32x4){0u, 0u, 0u, 0u};
    if (lane == 0) *(LAS u32x4*)(sc + 1024) = (u32x4){0u, 0u, 0u, 0u};
    u32 acc = 0;
#pragma unroll
    for (int t = 0; t < 16; ++t) { d[t] = T4 - d[t]; acc |= d[t]; }
    u32 rec = 0, head = T;
    bool raw = FORCE_RAW;
    if (__builtin_amdgcn_ballot_w64((acc & 0xf0f0f0f0u) != 0u) != 0ull) {
        u32 n = 0, sH = 0, sL = 0, sM = 0, sT = 0;
#pragma unroll
        for (int t = 0; t < 16; ++t) {
            const u32 e = T4 - d[t];
            const u32 w = ((T4 | 0x80808080u) - (e & 0x7f7f7f7fu)) ^ ((T4 ^ ~e) & 0x80808080u), mis = w & 0xf0f0f0f0u;
            d[t] = w & 0x0f0f0f0fu;
            if (mis) { ++n; sH = H[t]; sL = L[t]; sM = mis; sT = t; }
        }
        const u32 bsh = (n ? (u32)__builtin_ctz(sM) : 0u) & 24u;
        const bool multi = n > 1u || (sM & ~(0xffu << bsh)) != 0u;
        u64 mask = __builtin_amdgcn_ballot_w64(n != 0u);
        const u32 cnt = (u32)__builtin_popcountll(mask);
        if (cnt > 7u || __builtin_amdgcn_ballot_w64(multi) != 0ull) raw = true;
        else {
            const u32 ent = ((u32)lane << 22) | (sT << 18) | (bsh << 13) | (((sH >> bsh) & 0xffu) << 8) | ((sL >> bsh) & 0xffu);
            for (u32 s = 1; mask; ++s, mask &= mask - 1) {
                const u32 e = (u32)__builtin_amdgcn_readlane((int)ent, __builtin_ctzll(mask));
                if ((u32)(lane & 7) == s) rec = e;
            }
            head |= cnt << 8;
        }
    }
    u32x4 Q; u32 cs[16], bo[16], bits = 0;
#pragma unroll
    for (int t = 0; t < 16; ++t) {
        const u32 sel = d[t] + (d[t] & 0x08080808u);
        const u32 r = perm(0xffffffaau, 0x5500ffffu, sel), r2 = perm(0xffffff00u, 0x0000ffffu, sel);
        Q[t >> 2] = (t & 3) ? bfi(0x03030303u << (2 * (t & 3)), r, Q[t >> 2]) : r;
        cs[t] = *(const LAS u32*)(sc + 1088 + __builtin_amdgcn_sad_u8(r2 & 0x20100804u, 0u, 0u));
        bo[t] = bits;
        bits += (u32)__builtin_popcount(r2);
    }
    const u32 inc = scan32(bits);
    const u32 nb = max((u32)__builtin_amdgcn_readlane((int)inc, 31), (u32)__builtin_amdgcn_readlane((int)inc, 63)) >> 3;
    if (nb > 1024u) raw = true;
    before_stores();
    const u32 voff = (u32)lane * 16u;
    if (raw) {
        u32x4 O[8]; planes_to_raw(L, H, O); store_unit_linear(slot, voff, O);
        head = H_RAW;
    } else {
        const char* b = slot + 4096;
        sgpr_settle();
        st16_sys<-4096>(b, voff, (u32x4){S[0], S[1], S[2], S[3]}); st16_sys<-3072>(b, voff, (u32x4){S[4], S[5], S[6], S[7]});
        st16_sys<-2048>(b, voff, (u32x4){S[8], S[9], S[10], S[11]}); st16_sys<-1024>(b, voff, (u32x4){S[12], S[13], S[14], S[15]});
        st16_sys<0>(b, voff, Q);
        const u32 base = inc - bits + (lane & 32 ? 4u : 0u);
#pragma unroll
        for (int t = 0; t < 16; ++t) {
            const u32 o = base + bo[t];
            const u64 x = (u64)perm(0u, d[t], cs[t]) << (o & 31u);
            LAS u32* q = (LAS u32*)(sc + ((o >> 3) & ~3u));
            __hip_atomic_fetch_or(q, (u32)x, __ATOMIC_RELAXED, __HIP_MEMORY_SCOPE_WAVEFRONT);
            __hip_atomic_fetch_or(q + 1, (u32)(x >> 32), __ATOMIC_RELAXED, __HIP_MEMORY_SCOPE_WAVEFRONT);
        }
        asm volatile("" ::: "memory");
        const u32 n16 = (nb + 15u) >> 4;
        const u32x4 lst = *(const LAS u32x4*)(sc + lane * 16);
        if ((u32)lane < n16) st16_sys<1024>(b, voff, lst);
        head |= n16 << 20;
    }
    if ((lane & 7) == 0) rec = head;
    return rec;
}
__device__ __forceinline__ void put_record(char* p, u32 seq, u32 w) {
    __hip_atomic_store((GAS u64*)p, ((u64)w << 32) | seq, __ATOMIC_RELAXED, __HIP_MEMORY_SCOPE_SYSTEM);
}

__device__ __forceinline__ void pack_task(const Params& p, char* ws, int task, u32 cnt_v, LAS unsigned char* sc, int lane) {
    int j, r;
    if (task < 512) { j = task >> 6; r = task & 63; }
    else if (task < 768) { j = (task - 512) >> 5; r = 64 + (task & 31); }
    else if (task >= 1792 && task < 2048) { j = (task - 1792) >> 5; r = 96 + (task & 31); }
    else { const int c = task - (task < 1792 ? 768 : 1024), bk = p.worder[c / 28], m = c % 28; j = bk >> 4; r = 128 + (m >> 2) * 64 + (bk & 15) * 4 + (m & 3); }
    u32x4 in[8]; u32 L[16], H[16];
    const u32 voff = (u32)lane * 16u;
    if (r < 64) {
        const int pm = r >> 2, un = r & 3;
        const char* src = p.x + (size_t)(pm * 256 + un * 64) * 1024 + j * 128 + (lane >> 3) * 1024;
        const int sw = (lane & 7) ^ (lane >> 4);
#pragma unroll
        for (int q = 0; q < 8; ++q) in[q] = *(const GAS u32x4*)(src + q * 8192 + ((sw ^ ((q & 1) << 2)) << 4));
        char* peer = p.ws_tab[(p.rank + 1 + (lane >> 3)) & 7];
#pragma unroll
        for (int d = 0; d < 4; ++d)
#pragma unroll
            for (int q = 0; q < 4; ++q) { L[d * 4 + q] = perm(in[2 * d + 1][q], in[2 * d][q], 0x06040200u); H[d * 4 + q] = perm(in[2 * d + 1][q], in[2 * d][q], 0x07050301u); }
        const u32 seq = __builtin_amdgcn_readfirstlane(cnt_v) + 1u;
        asm volatile("" : "+v"(peer));
        const u32 rec = encode_store(L, H, ws + OFF_PX + (size_t)(seq & 1) * (4 * MiB) + (size_t)((j * 16 + pm) * 4 + un) * 8192, sc, lane, [] {});
        WAIT_V(0);
        if (lane < 56) put_record(peer + OFF_RIN + (size_t)(((seq & 1) * 8 + p.rank) * 1024 + (j * 32 + pm) * 4 + un) * 64 + (lane & 7) * 8, seq, rec);
        store_unit_linear(ws + OFF_G + (size_t)(j * 32 + pm) * 32768 + un * 8192, voff, in);
        WAIT_V(0);
        if (lane == 0) st_agent(ws + OFF_FG + (j * 32 + pm) * 128 + un * 4, seq);
    } else {
        const int dd = (r - 64) >> 6, pn = (r >> 2) & 15, un = r & 3, dest = (p.rank + dd) & 7;
        const char* src = p.w + (size_t)(j * 64 + (un & 1) * 32 + (lane >> 4) * 8) * 65536 + (dest * 4096 + pn * 256 + (un >> 1) * 128) * 2 + (lane & 15) * 16;
#pragma unroll
        for (int q = 0; q < 8; ++q) in[q] = *(const GAS u32x4*)(src + (size_t)q * 65536);
#pragma unroll
        for (int d = 0; d < 4; ++d)
#pragma unroll
            for (int q = 0; q < 4; ++q) { L[d * 4 + q] = perm(in[2 * q + 1][d], in[2 * q][d], 0x06020400u); H[d * 4 + q] = perm(in[2 * q + 1][d], in[2 * q][d], 0x07030501u); }
        const u32 seq = __builtin_amdgcn_readfirstlane(cnt_v) + 1u;
        if (dd == 0) {
            u32x4 O[8]; planes_to_raw(L, H, O);
            store_unit_linear(ws + OFF_G + (size_t)(j * 32 + 16 + pn) * 32768 + un * 8192, voff, O);
            WAIT_V(0);
            if (lane == 0) st_agent(ws + OFF_FG + (j * 32 + 16 + pn) * 128 + un * 4, seq);
        } else {
            const u32 rec = encode_store(L, H, ws + OFF_PW + (size_t)dest * (4 * MiB) + (size_t)((j * 16 + pn) * 4 + un) * 8192, sc, lane, [] {});
            WAIT_V(0);
            if (lane < 8) put_record(p.ws[dest] + OFF_RIN + (size_t)(((seq & 1) * 8 + p.rank) * 1024 + (j * 32 + 16 + pn) * 4 + un) * 64 + lane * 8, seq, rec);
        }
    }
}

#define WIN 4
#define PACE 978
#define TGRAN 4
#define SLACK 3
constexpr int NSLOT = 7, NDEC = 5, NDESC = 16, NPOS = 1536, LOOK = 8;
constexpr u32 FIN = 0x40000000u;
constexpr int L_SLOT = 0;
constexpr int L_DESC = 2 * NSLOT * 8192;
constexpr int L_CTL = L_DESC + 2 * NDESC * 64;
constexpr int L_ORD = L_CTL + 128;
static_assert(L_ORD + 256 <= LDS_BYTES, "lds");

__device__ __forceinline__ void wait_units(u32 n) {
    switch (n) {
        case 0: WAIT_V(0); break; case 1: WAIT_V(6); break; case 2: WAIT_V(12); break; case 3: WAIT_V(18); break; case 4: WAIT_V(24); break;
        case 5: WAIT_V(30); break; case 6: WAIT_V(36); break; default: WAIT_V(42); break;
    }
}
__device__ __forceinline__ void comm_role(const Params& p, char* ws, LAS unsigned char* lds, int g, u32 seq, int wid, int lane) {
    if (threadIdx.x < 64) ((LAS u32*)(lds + L_ORD))[threadIdx.x] = ((const u32*)p.order)[threadIdx.x];
    if (threadIdx.x >= 64 && threadIdx.x < 96) ((LAS u32*)(lds + L_CTL))[threadIdx.x - 64] = 0;
    __syncthreads();
    volatile LAS u32* ctl = (volatile LAS u32*)(lds + L_CTL);
    volatile LAS u32* desc = (volatile LAS u32*)(lds + L_DESC);
    const u32 voff = (u32)lane * 16u;
    if (wid == 7) {
        const u32 st = lane >> 5, e = lane & 7, L = g * 2 + st, src = (p.rank + 1 + L % 7) & 7, j = L / 7;
        const u32 cnt = j < 9 ? (NPOS - j + 8) / 9 : 0;
        const char* rws = p.ws_tab[src];
        const u64 t0 = __builtin_amdgcn_s_memrealtime();
        bool dead = false;
        u32 x = (lane >> 3) & 3;
        while (__builtin_amdgcn_ballot_w64(x < cnt) != 0ull) {
            const bool act = x < cnt && x < ctl[st * 16 + 9] + LOOK;
            const u32 u = x < cnt ? j + 9 * x : 0;
            u32 id;
            if (u < 1024) {
                u32 bk = (lds + L_ORD)[u >> 2];
                if (!(bk & 16)) { const u32 i = bk & 15, pmi = i < 14 ? (i + (ROT ? 14 - 2 * (L % 7) : 0)) % 14 : i; bk = (bk & ~15u) | ((p.rank * 2 + 2 + pmi) & 15); }
                id = bk * 4u + (u & 3);
            } else { const u32 y = u - 1024; id = (y & ~31u) | ((y & 15) << 1) | ((y >> 4) & 1); }
            const char* a = u >= 1024 ? ws + OFF_RY + (size_t)(src * 512 + id) * 64 : ws + OFF_RIN + (size_t)(((seq & 1) * 8 + src) * 1024 + id) * 64;
            u64 v = 0;
            if (act) v = __hip_atomic_load((GAS u64*)(a + e * 8), __ATOMIC_RELAXED, __HIP_MEMORY_SCOPE_SYSTEM);
            const u64 okm = __builtin_amdgcn_ballot_w64((u32)v == seq);
            const bool adv = act && (dead || ((okm >> (lane & 56)) & 0xffull) == 0xffull);
            if (__builtin_amdgcn_ballot_w64(adv) != 0ull) {
                if (adv) {
                    const char* s = u >= 1024 ? rws + OFF_PY + (size_t)p.rank * (4 * MiB) + (size_t)id * 8192
                        : rws + ((id & 64) ? OFF_PW + (size_t)p.rank * (4 * MiB) : OFF_PX + (size_t)(seq & 1) * (4 * MiB)) + (size_t)(((id >> 7) * 16 + ((id >> 2) & 15)) * 4 + (id & 3)) * 8192;
                    const u32 en = (st * NDESC + x % NDESC) * 16, word = dead ? H_DEAD : (u32)(v >> 32);
                    const u64 sa = dead ? (u64)(ws + OFF_G) : (u64)s;
                    desc[en + e] = word;
                    if (e == 0) { desc[en + 8] = (u32)sa; desc[en + 9] = (u32)(sa >> 32); desc[en + 10] = word; desc[en + 11] = (u >= 1024 ? 0x8000u : 0u) | id; }
                    x += 4;
                }
                u32 m = min(x, (u32)__builtin_amdgcn_update_dpp((int)x, (int)x, 0x128, 0xf, 0xf, false));
                m = min(m, (u32)__builtin_amdgcn_ds_swizzle((int)m, 0x401F));
                WAIT_L(0);
                if ((lane & 31) == 0) ctl[st * 16 + 8] = min(m, cnt);
            } else if (__builtin_amdgcn_ballot_w64(act) != 0ull && (ld_agent(ws + OFF_TMO) == seq || __builtin_amdgcn_s_memrealtime() - t0 > DEADLINE)) {
                st_agent(ws + OFF_TMO, seq); WAIT_V(0); dead = true;
            } else __builtin_amdgcn_s_sleep(1);
        }
        if ((lane & 31) == 0) ctl[st * 16 + 8] = cnt | FIN;
    } else if (wid >= NDEC) {
        const u32 me = wid - NDEC;
        volatile LAS u32* c = ctl + me * 16;
        u32 pub = 0, landed = 0, k = 0, slot = 0;
        const u32 t0 = (u32)__builtin_amdgcn_s_memrealtime();
        u32 next = (u32)((g * 2 + me) / 7) * (3 * PACE / 9);
        auto pace = [&](u32 n64) {
            u32 now;
            while ((int)((now = ((u32)__builtin_amdgcn_s_memrealtime() - t0) * 64u) - next) < 0) {}
            if (now - next > SLACK * PACE) next = now - SLACK * PACE;
            next += (n64 * PACE) >> 4;
        };
        for (;; ++k) {
            bool fin = false;
            while (true) {
                if ((pub & ~FIN) <= k) pub = __builtin_amdgcn_readfirstlane(c[8]);
                if ((pub & ~FIN) > k && (u32)__builtin_amdgcn_readfirstlane(c[slot]) + NSLOT > k) break;
                if (landed < k) {
                    wait_units(k - landed - 1);
                    ++landed;
                    if (lane == 0) c[10] = landed;
                } else if (pub == (k | FIN)) { fin = true; break; }
                else __builtin_amdgcn_s_sleep(1);
            }
            if (fin) break;
            asm volatile("" ::: "memory");
            const u32x4 dd = *(const LAS u32x4*)(lds + L_DESC + (me * NDESC + k % NDESC) * 64 + 32);
            const char* s = (const char*)(((u64)(u32)__builtin_amdgcn_readfirstlane(dd[1]) << 32) | (u32)__builtin_amdgcn_readfirstlane(dd[0]));
            const u32 head = __builtin_amdgcn_readfirstlane(dd[2]);
            LAS unsigned char* d = lds + L_SLOT + (me * NSLOT + slot) * 8192;
            if (lane == 0) c[9] = k;
            pace(48);
            __builtin_amdgcn_global_load_lds((const GAS u32*)(s + voff), (LAS u32*)d, 16, 0, 17);
            __builtin_amdgcn_global_load_lds((const GAS u32*)(s + voff), (LAS u32*)d, 16, 1024, 17);
            __builtin_amdgcn_global_load_lds((const GAS u32*)(s + voff), (LAS u32*)d, 16, 2048, 17);
            asm volatile("s_waitcnt vmcnt(%0)" :: "i"(3 * WIN) : "memory");
            if ((WIN & 1) && k > landed + WIN / 2) { landed = k - WIN / 2; if (lane == 0) c[10] = landed; }
            u32 n16 = (head >> 20) & 127u;
            if (head & (H_RAW | H_DEAD)) {
                pace(32);
                __builtin_amdgcn_global_load_lds((const GAS u32*)(s + 6144 + voff), (LAS u32*)(d + 6144), 16, 0, 17);
                __builtin_amdgcn_global_load_lds((const GAS u32*)(s + 6144 + voff), (LAS u32*)(d + 6144), 16, 1024, 17);
                n16 = 64;
            }
            pace(32 + ((n16 + TGRAN - 1) / TGRAN) * (TGRAN / 4));
            __builtin_amdgcn_global_load_lds((const GAS u32*)(s + 3072 + voff), (LAS u32*)(d + 3072), 16, 0, 17);
            __builtin_amdgcn_global_load_lds((const GAS u32*)(s + 3072 + voff), (LAS u32*)(d + 3072), 16, 1024, 17);
            if ((u32)lane < max(n16, 1u)) __builtin_amdgcn_global_load_lds((const GAS u32*)(s + 3072 + voff), (LAS u32*)(d + 3072), 16, 2048, 17);
            asm volatile("s_waitcnt vmcnt(%0)" :: "i"(3 * WIN) : "memory");
            if (!(WIN & 1) && k + 1 > landed + WIN / 2) { landed = k + 1 - WIN / 2; if (lane == 0) c[10] = landed; }
            if (++slot == NSLOT) slot = 0;
        }
        if (lane == 0) c[10] = k | FIN;
    } else {
        char* pend = nullptr;
        int k = 0;
        u32 st = wid & 1;
        for (;;) {
            u32 e;
            {
                bool got = false;
                for (u32 idle = 0;;) {
                    st ^= 1;
                    const u32x2 lc = *(volatile LAS u32x2*)(lds + L_CTL + st * 64 + 40);
                    const u32 ld = __builtin_amdgcn_readfirstlane(lc[0]);
                    e = __builtin_amdgcn_readfirstlane(lc[1]);
                    if (e < (ld & ~FIN)) {
                        u32 o = e;
                        if (lane == 0) __hip_atomic_compare_exchange_strong((LAS u32*)(lds + L_CTL + st * 64 + 44), &o, e + 1, __ATOMIC_RELAXED, __ATOMIC_RELAXED, __HIP_MEMORY_SCOPE_WORKGROUP);
                        if (__builtin_amdgcn_readfirstlane(o) == e) { got = true; break; }
                        idle = 0;
                        continue;
                    }
                    if (ld & FIN) { if (++idle == 2) break; } else idle = 0;
                    if (pend) { WAIT_V(0); if (lane == 0) st_agent(pend, seq); pend = nullptr; }
                    else __builtin_amdgcn_s_sleep(1);
                }
                if (!got) break;
            }
            asm volatile("" ::: "memory");
            STAMP(k < 124 ? k >> 1 : 62);
            const u32 en = (st * NDESC + e % NDESC) * 16, sn = e % NSLOT;
            const u32 ent = __builtin_amdgcn_readfirstlane(desc[en + 11]), id = ent & 0x3ffu, w = (g * 2 + st) % 7;
            const u32 rec = desc[en + (lane & 7)], head = __builtin_amdgcn_readfirstlane(rec);
            const LAS u32x4* sl = (const LAS u32x4*)(lds + L_SLOT + (st * NSLOT + sn) * 8192 + voff);
            u32x4 O[8];
            ++k;
            if (head & (H_RAW | H_DEAD)) {
#pragma unroll
                for (int q = 0; q < 8; ++q) O[q] = sl[q * 64];
                WAIT_L(0);
                if (lane == 0) ctl[st * 16 + sn] = e + 1;
                if (head & H_DEAD) {
#pragma unroll
                    for (int q = 0; q < 8; ++q) O[q] = (u32x4){0x7fc07fc0u, 0x7fc07fc0u, 0x7fc07fc0u, 0x7fc07fc0u};
                }
            } else {
                u32 L[16], H[16], c[16], m[16], bo[16];
                u32x2 w[16];
                u32x4 S[4];
                const u32 T4 = (head & 0xffu) * 0x01010101u;
                const u32x4 Q = sl[4 * 64];
#pragma unroll
                for (int q = 0; q < 4; ++q) S[q] = sl[q * 64];
                u32 bits = 0;
#pragma unroll
                for (int t = 0; t < 16; ++t) {
                    c[t] = (Q[t >> 2] >> (2 * (t & 3))) & 0x03030303u;
                    m[t] = perm(0u, 0x01000000u, c[t]);
                    bo[t] = bits;
                    bits = __builtin_amdgcn_udot4(m[t], 0x08080808u, bits, false);
                }
                const u32 base = min(scan32(bits) - bits, 8192u) + (lane & 32 ? 4u : 0u);
                const LAS unsigned char* lst = lds + L_SLOT + (st * NSLOT + sn) * 8192 + 5120;
#pragma unroll
                for (int t = 0; t < 16; ++t) {
                    bo[t] += base;
                    const LAS u32* q = (const LAS u32*)(lst + ((bo[t] >> 3) & ~3u));
                    w[t] = (u32x2){q[0], q[1]};
                }
                WAIT_L(0);
                if (lane == 0) ctl[st * 16 + sn] = e + 1;
#pragma unroll
                for (int t = 0; t < 16; ++t) {
                    const u32 sel = (__umul24(m[t], 0x010101u) << 8) + perm(0u, 0x00101010u, c[t]);
                    const u32 E = T4 - (perm(0u, __builtin_amdgcn_alignbit(w[t][1], w[t][0], bo[t]), sel) & perm(0u, 0x0f040302u, c[t]));
                    const u32 sv = S[t >> 2][t & 3];
                    H[t] = bfi(0x80808080u, sv, E >> 1);
                    L[t] = bfi(0x80808080u, E << 7, sv);
                }
                planes_to_raw(L, H, O);
            }
            const u32 cnt = (head >> 8) & 7u;
            const u32 el = (rec >> 22) & 63u, et = (rec >> 18) & 15u, eb = (rec >> 16) & 3u, eq = (et >> 2) * 2u + (eb >> 1), eo = (et & 3u) * 4u + (eb & 1u) * 2u;
            char* flag = nullptr;
            if (!(ent & 0x8000u)) {
                const u32 blk = (8 + 7 * (id >> 7) + w) * 32 + ((id >> 2) & 31);
                char* dst = ws + OFF_G + (size_t)blk * 32768 + (id & 3) * 8192;
                store_unit_linear(dst, voff, O);
                if (cnt && lane >= 1 && (u32)lane <= cnt)
                    asm volatile("global_store_short %0, %1, %2 sc0 sc1" :: "v"(eq * 1024u + el * 16u + eo), "v"(rec), "s"(dst) : "memory");
                flag = ws + OFF_FG + blk * 128 + (id & 3) * 4;
            } else {
                const u32 src = (p.rank + 1 + w) & 7;
                const u32 row0 = ((id >> 4) & 1) * 256 + (id & 1) * 128 + ((id >> 1) & 3) * 32, col0 = src * 4096 + ((id >> 5) & 15) * 256 + ((id >> 3) & 1) * 128;
                char* dst = p.out + (size_t)row0 * 65536 + col0 * 2;
                const u32 vo = (u32)(lane >> 4) * 262144u + (u32)(lane & 15) * 16u;
                sgpr_settle();
#pragma unroll
                for (int q = 0; q < 8; ++q) st16_sys<0>(dst + (size_t)((q >> 2) * 16 + (q & 3)) * 65536, vo, O[q]);
                if (cnt && lane >= 1 && (u32)lane <= cnt)
                    asm volatile("global_store_short %0, %1, %2 sc0 sc1" :: "v"(((eq >> 2) * 16u + (eq & 3u) + (el >> 4) * 4u) * 65536u + (el & 15u) * 16u + eo), "v"(rec), "s"(dst) : "memory");
            }
            if (pend) { WAIT_V(8); if (lane == 0) st_agent(pend, seq); }
            pend = flag;
        }
        if (pend) { WAIT_V(0); if (lane == 0) st_agent(pend, seq); }
        STAMP(62);
    }
}

__device__ __forceinline__ void spin_flags(char* ws, const char* fbase, u32 fvoff, u32 seq) {
    const u64 t0 = __builtin_amdgcn_s_memrealtime();
    while (__builtin_amdgcn_ballot_w64(ld_agent(fbase + fvoff) < seq) != 0ull) {
        if (ld_agent(ws + OFF_TMO) == seq || __builtin_amdgcn_s_memrealtime() - t0 > DEADLINE) { st_agent(ws + OFF_TMO, seq); break; }
        __builtin_amdgcn_s_sleep(4);
    }
    __builtin_amdgcn_s_waitcnt(0xC07F);
    WAIT_V(0);
}

__device__ __forceinline__ void gemm_loop(char* ws, LAS unsigned char* lds, const char* gX, const char* gW, u32 fvoff, int otab, u32 seq, int wid, int lane, f32x4 (&acc)[2][2][4][2]) {
    constexpr size_t KSTEP = MiB, SX = 16384, SW = 16384;
    constexpr int NSTEP = 64, FSTEP = 4096;
    const int wr = wid >> 2, wc = wid & 3, fr = lane & 15, fq = lane >> 4;
    const u32 voff = (u32)lane * 16u;
    const char* cX = gX + wid * 1024;
    const char* cW = gW + (wid >> 2) * 8192 + (wid & 3) * 1024;
    const char* fb = ws + OFF_FG;
    const u32 ldsw = (u32)wid * 1024u;
    const int woff = wr * 8192 + lane * 16;
    const int xoff0 = (wc * 32 + fr) * 128 + ((fq ^ (fr >> 1)) << 4), xoff1 = xoff0 ^ 64;
#define SA(b, h) (((b) * 2 + (h)) * 16384)
#define SB(b, h) ((4 + (b) * 2 + (h)) * 16384)
#define GLDS(src, dst) __builtin_amdgcn_global_load_lds((const GAS u32*)((src) + voff), (LAS u32*)(lds + (dst) + ldsw), 16, 0, GAUX)
#define STW(b, h, ptr) do { GLDS((ptr) + (h) * 4096, SA(b, h)); GLDS((ptr) + (h) * 4096 + SW, SA(b, h) + 8192); } while (0)
#define STX(b, h, ptr) do { GLDS((ptr) + (h) * SX, SB(b, h)); GLDS((ptr) + (h) * SX + 8192, SB(b, h) + 8192); } while (0)
#define LDW(b, h) do { _Pragma("unroll") for (int m = 0; m < 4; ++m) _Pragma("unroll") for (int k = 0; k < 2; ++k) Wt[m][k] = *(const LAS bf16x8*)(lds + SA(b, h) + woff + k * 4096 + m * 1024); } while (0)
#define LDX(dst, b, h) do { _Pragma("unroll") for (int n = 0; n < 2; ++n) { dst[n][0] = *(const LAS bf16x8*)(lds + SB(b, h) + xoff0 + n * 2048); dst[n][1] = *(const LAS bf16x8*)(lds + SB(b, h) + xoff1 + n * 2048); } } while (0)
#define MMA(h, xh, Xt) do { __builtin_amdgcn_s_setprio(1); _Pragma("unroll") for (int m = 0; m < 4; ++m) _Pragma("unroll") for (int n = 0; n < 2; ++n) _Pragma("unroll") for (int k = 0; k < 2; ++k) \
        acc[h][xh][m][n] = __builtin_amdgcn_mfma_f32_16x16x32_bf16(Xt[n][k], Wt[m][k], acc[h][xh][m][n], 0, 0, 0); __builtin_amdgcn_s_setprio(0); } while (0)
#define FLAG_LOAD(step) asm volatile("global_load_dword %0, %1, %2 sc1" : "+v"(fl) : "v"(fvoff), "s"(fb + (size_t)(step) * FSTEP) : "memory")
#define FLAG_CHECK(step) do { if (wid == 0 && __builtin_amdgcn_ballot_w64(fl < seq) != 0ull) spin_flags(ws, fb + (size_t)(step) * FSTEP, fvoff, seq); } while (0)
#define WAIT_V6 asm volatile("s_waitcnt vmcnt(6)" : "+v"(fl) :: "memory")
#pragma unroll
    for (int a = 0; a < 2; ++a)
#pragma unroll
        for (int b = 0; b < 2; ++b)
#pragma unroll
            for (int m = 0; m < 4; ++m)
#pragma unroll
                for (int n = 0; n < 2; ++n) acc[a][b][m][n] = (f32x4){0.f, 0.f, 0.f, 0.f};
    bf16x8 Wt[4][2], X0[2][2], X1[2][2];
    u32 fl = 0;
#define KSTAMP(i)
    if (wid == 0) { spin_flags(ws, fb, fvoff, seq); spin_flags(ws, fb + FSTEP, fvoff, seq); }
    WAIT_V(0); WAIT_L(0); BAR; asm volatile("" ::: "memory");
#define WAIT_VN(n) asm volatile("s_waitcnt vmcnt(" #n ")" : "+v"(fl) :: "memory")
    FLAG_LOAD(2);
    STX(0, 0, cX); STX(0, 1, cX); STW(0, 0, cW); STW(0, 1, cW);
    STX(1, 0, cX + KSTEP); STX(1, 1, cX + KSTEP); STW(1, 0, cW + KSTEP);
    WAIT_VN(8); BAR;
    if (wr == 1) BAR;
    for (int t = 0; t < NSTEP; t += 2) {
        const int t1 = __builtin_amdgcn_readlane(otab, t + 1), t2 = __builtin_amdgcn_readlane(otab, (t + 2) & 63);
        const int t3 = __builtin_amdgcn_readlane(otab, (t + 3) & 63), t4 = __builtin_amdgcn_readlane(otab, (t + 4) & 63);
        const char* w1 = cW + (size_t)t1 * KSTEP;
        const char* x2 = cX + (size_t)t2 * KSTEP; const char* w2 = cW + (size_t)t2 * KSTEP;
        const char* x3 = cX + (size_t)t3 * KSTEP; const char* w3 = cW + (size_t)t3 * KSTEP;
        KSTAMP(t >> 1);
        FLAG_CHECK(t2); FLAG_LOAD(t3);
        LDX(X0, 0, 0); LDW(0, 0); LDX(X1, 0, 1); STW(1, 1, w1);
        WAIT_VN(9); WAIT_L(0); SCHED; BAR; SCHED; MMA(0, 0, X0); MMA(0, 1, X1); SCHED; BAR; SCHED;
        LDW(0, 1); STX(0, 0, x2); STX(0, 1, x2); STW(0, 0, w2);
        WAIT_VN(8); WAIT_L(0); SCHED; BAR; SCHED; MMA(1, 0, X0); MMA(1, 1, X1); SCHED; BAR; SCHED;
        FLAG_CHECK(t3); FLAG_LOAD(t4);
        LDX(X0, 1, 0); LDW(1, 0); LDX(X1, 1, 1); STW(0, 1, w2);
        WAIT_VN(9); WAIT_L(0); SCHED; BAR; SCHED; MMA(0, 0, X0); MMA(0, 1, X1); SCHED; BAR; SCHED;
        LDW(1, 1); STX(1, 0, x3); STX(1, 1, x3); STW(1, 0, w3);
        WAIT_VN(8); WAIT_L(0); SCHED; BAR; SCHED; MMA(1, 0, X0); MMA(1, 1, X1); SCHED; BAR; SCHED;
    }
#undef WAIT_VN
    KSTAMP(32);
    asm volatile("s_waitcnt vmcnt(0)" : "+v"(fl) :: "memory");
    if (wr == 0) BAR;
    BAR;
#undef SA
#undef SB
#undef GLDS
#undef STW
#undef STX
#undef LDW
#undef LDX
#undef MMA
#undef FLAG_LOAD
#undef FLAG_CHECK
#undef WAIT_V6
#undef KSTAMP
}

__device__ __forceinline__ void gemm_piece(char* ws, LAS unsigned char* lds, const char* gX, const char* gW, u32 fx, u32 fw, u32 n0, u32 seq, int wid, int lane, f32x4 (&acc)[2][2][4][2]) {
    const int wr = wid >> 2, wc = wid & 3, fr = lane & 15, fq = lane >> 4;
    const u32 voff = (u32)lane * 16u;
    const char* cX = gX + wc * 4096;
    const char* cW = gW + wc * 4096;
    const int xoff0 = (wc * 32 + fr) * 128 + ((fq ^ (fr >> 1)) << 4), xoff1 = xoff0 ^ 64;
    auto stage = [&](int n) {
        const char* x = cX + (size_t)((n + n0) & 63) * MiB + voff; const char* w = cW + (size_t)((n + n0) & 63) * MiB + voff;
        LAS unsigned char* d = lds + (n & 3) * 32768 + wc * 4096;
        __builtin_amdgcn_global_load_lds((const GAS u32*)x, (LAS u32*)d, 16, 0, GAUX); __builtin_amdgcn_global_load_lds((const GAS u32*)x, (LAS u32*)d, 16, 1024, GAUX);
        __builtin_amdgcn_global_load_lds((const GAS u32*)x, (LAS u32*)d, 16, 2048, GAUX); __builtin_amdgcn_global_load_lds((const GAS u32*)x, (LAS u32*)d, 16, 3072, GAUX);
        __builtin_amdgcn_global_load_lds((const GAS u32*)w, (LAS u32*)(d + 16384), 16, 0, GAUX); __builtin_amdgcn_global_load_lds((const GAS u32*)w, (LAS u32*)(d + 16384), 16, 1024, GAUX);
        __builtin_amdgcn_global_load_lds((const GAS u32*)w, (LAS u32*)(d + 16384), 16, 2048, GAUX); __builtin_amdgcn_global_load_lds((const GAS u32*)w, (LAS u32*)(d + 16384), 16, 3072, GAUX);
    };
#pragma unroll
    for (int a = 0; a < 2; ++a)
#pragma unroll
        for (int m = 0; m < 4; ++m)
#pragma unroll
            for (int n = 0; n < 2; ++n) acc[a][0][m][n] = (f32x4){0.f, 0.f, 0.f, 0.f};
    if (wid < 4) spin_flags(ws, ws + OFF_FG, (u32)lane * 4096u + ((wid & 2) ? fw : fx) + (wid & 1) * 4, seq);
    WAIT_V(0); WAIT_L(0); BAR; asm volatile("" ::: "memory");
    if (wr == 1) { stage(0); stage(2); WAIT_V(8); BAR; } else stage(1);
    BAR;
    for (int n = wr; n < 64; n += 2) {
        bf16x8 Wt[8][2], Xt[2][2];
        const LAS unsigned char* bx = lds + (n & 3) * 32768;
#pragma unroll
        for (int i = 0; i < 2; ++i) { Xt[i][0] = *(const LAS bf16x8*)(bx + xoff0 + i * 2048); Xt[i][1] = *(const LAS bf16x8*)(bx + xoff1 + i * 2048); }
#pragma unroll
        for (int k = 0; k < 2; ++k)
#pragma unroll
            for (int c = 0; c < 8; ++c) Wt[c][k] = *(const LAS bf16x8*)(bx + 16384 + voff + k * 8192 + c * 1024);
        stage(n + 3);
        WAIT_V(8); WAIT_L(0); SCHED; BAR; SCHED;
        __builtin_amdgcn_s_setprio(1);
#pragma unroll
        for (int k = 0; k < 2; ++k)
#pragma unroll
            for (int c = 0; c < 8; ++c)
#pragma unroll
                for (int i = 0; i < 2; ++i) acc[c >> 2][0][c & 3][i] = __builtin_amdgcn_mfma_f32_16x16x32_bf16(Xt[i][k], Wt[c][k], acc[c >> 2][0][c & 3][i], 0, 0, 0);
        __builtin_amdgcn_s_setprio(0);
        SCHED; BAR; asm volatile("" ::: "memory");
    }
    WAIT_V(0);
    if (wr == 0) BAR;
    BAR; asm volatile("" ::: "memory");
}

__device__ __forceinline__ void acc_to_raw(const f32x4 (&acc)[2][2][4][2], int xh, bool poison, u32x4 (&O)[8]) {
#pragma unroll
    for (int n = 0; n < 2; ++n)
#pragma unroll
        for (int j = 0; j < 4; ++j)
#pragma unroll
            for (int c = 0; c < 4; ++c) {
                const u32 v = cvt_pk_bf16(acc[c >> 1][xh][(c & 1) * 2][n][j], acc[c >> 1][xh][(c & 1) * 2 + 1][n][j]);
                O[n * 4 + j][c] = poison ? 0x7fc07fc0u : v;
            }
}

__global__ __launch_bounds__(512, 2) void fused_gemm_rs(Params p) {
    extern __shared__ __attribute__((aligned(16))) unsigned char shm[];
    LAS unsigned char* lds = (LAS unsigned char*)shm;
    const int lane = threadIdx.x & 63, wid = __builtin_amdgcn_readfirstlane(threadIdx.x >> 6), b = blockIdx.x;
    char* ws = p.ws[p.rank];
    const u32 cnt_v = ld_agent(ws + OFF_CNT + b * 64);
    const int xcd = b & 7, ix = b >> 3;
    u32 seq;
    if (ix < 4) {
        STAMP(63);
        seq = __builtin_amdgcn_readfirstlane(cnt_v) + 1u;
        comm_role(p, ws, lds, ix * 8 + xcd, seq, wid, lane);
    } else {
        STAMP(0);
        const int li = ix - 4, cg = li / 7, pmi = (xcd & 1) * 7 + li % 7;
        const int pn = MIX == 0 ? (xcd >> 1) * 4 + cg : (cg < 2 ? (xcd & 6) + cg : (MIX == 1 ? (xcd & 6) + 6 + cg : 12 - (xcd & 6) + cg));
        const int gi = (15 - pn) * 14 + pmi;
        LAS unsigned char* sc = lds + LDS_GEMM + wid * ENC_BYTES;
        enc_init(sc, lane);
        {
            const int t1 = PASS1(wid * NGEMM + gi);
            if (PACK_DELAY && !(t1 < 128 || (t1 >= 512 && t1 < 576) || (t1 >= 768 && t1 < 768 + 112))) __builtin_amdgcn_s_sleep(PACK_DELAY);
            int task = t1;
            for (int pass = 0; task < 4608; ++pass) {
                pack_task(p, ws, task, cnt_v, sc, lane);
                STAMP(pass < 2 ? 1 + pass : 3);
                task = gi < 112 ? 1792 + pass * 896 + (pass < 3 ? wid * 112 + gi : gi * 8 + wid) : 4608;
            }
        }
        seq = __builtin_amdgcn_readfirstlane(cnt_v) + 1u;
        const int dest = (p.rank + 1 + (pmi >> 1)) & 7, pm = dest * 2 + (pmi & 1);
        f32x4 acc[2][2][4][2];
        gemm_loop(ws, lds, ws + OFF_G + (size_t)pm * 32768, ws + OFF_G + (size_t)(16 + pn) * 32768,
                         (u32)(((lane & 4) ? 16 + pn : pm) * 128 + (lane & 3) * 4),
                  lane < 8 ? lane : 8 + (lane - 8) / 7 * 7 + ((lane - 8) % 7 + (ROT ? 7 - (pmi >> 1) : 0)) % 7, seq, wid, lane, acc);
        STAMP(4);
        {
            const bool poison = __builtin_amdgcn_readfirstlane(ld_agent(ws + OFF_TMO)) == seq;
            const int id0 = ((pn * 2 + (pmi & 1)) * 8 + wid) * 2;
            u32 rec = 0;
            char* ry = p.ws[dest] + OFF_RY + (size_t)(p.rank * 512 + id0) * 64 + lane * 8;
#pragma unroll
            for (int xh = 0; xh < 2; ++xh) {
                u32x4 O[8]; u32 L[16], H[16];
                acc_to_raw(acc, xh, poison, O);
#pragma unroll
                for (int d = 0; d < 4; ++d)
#pragma unroll
                    for (int q = 0; q < 4; ++q) { L[d * 4 + q] = perm(O[2 * d + 1][q], O[2 * d][q], 0x06040200u); H[d * 4 + q] = perm(O[2 * d + 1][q], O[2 * d][q], 0x07050301u); }
                rec = encode_store(L, H, ws + OFF_PY + (size_t)dest * (4 * MiB) + (size_t)(id0 + xh) * 8192, sc, lane,
                                   [&] { if (xh == 1) { WAIT_V(0); if (lane < 8) put_record(ry, seq, rec); } });
            }
            WAIT_V(0);
            if (lane < 8) put_record(ry + 64, seq, rec);
        }
        STAMP(5);
        for (bool first = pn < 8;; first = false) {
            __syncthreads();
            if (threadIdx.x == 0) {
                volatile LAS u32* t = (volatile LAS u32*)(lds + LDS_GEMM);
                t[0] = first ? (u32)(MIX ? xcd * 14 + li : xcd * 28 + li)
                             : 112u + __hip_atomic_fetch_add((GAS u32*)(ws + OFF_TICKET), 1u, __ATOMIC_RELAXED, __HIP_MEMORY_SCOPE_AGENT) - (seq - 1u) * (NGEMM + 16);
                t[1] = (u32)__builtin_amdgcn_s_memrealtime() / SUBTAU;
            }
            __syncthreads();
            const u32 tk = __builtin_amdgcn_readfirstlane(*(volatile LAS u32*)(lds + LDS_GEMM)), n0 = __builtin_amdgcn_readfirstlane(*(volatile LAS u32*)(lds + LDS_GEMM + 4));
            if (tk >= 128u) break;
            const int rq = tk & 3, cq = tk >> 2, opm = p.rank * 2 + (rq >> 1), opn = cq >> 1;
            gemm_piece(ws, lds, ws + OFF_G + (size_t)opm * 32768 + (rq & 1) * 16384, ws + OFF_G + (size_t)(16 + opn) * 32768 + (cq & 1) * 16384,
                       (u32)(opm * 128 + (rq & 1) * 8), (u32)((16 + opn) * 128 + (cq & 1) * 8), n0, seq, wid, lane, acc);
            STAMP(6);
            LAS f32x4* red = (LAS f32x4*)(lds + (wid & 3) * 16384) + lane;
            if (wid >= 4) {
#pragma unroll
                for (int i = 0; i < 16; ++i) red[i * 64] = acc[i >> 3][0][(i >> 1) & 3][i & 1];
            }
            __syncthreads();
            if (wid < 4) {
#pragma unroll
                for (int i = 0; i < 16; ++i) acc[i >> 3][0][(i >> 1) & 3][i & 1] += red[i * 64];
                const bool poison = __builtin_amdgcn_readfirstlane(ld_agent(ws + OFF_TMO)) == seq;
                u32x4 O[8]; acc_to_raw(acc, 0, poison, O);
                char* dst = p.out + (size_t)(rq * 128 + wid * 32) * 65536 + (p.rank * 4096 + cq * 128) * 2;
                const u32 vo = (u32)(lane >> 4) * 262144u + (u32)(lane & 15) * 16u;
                sgpr_settle();
#pragma unroll
                for (int q = 0; q < 8; ++q) st16_sys<0>(dst + (size_t)((q >> 2) * 16 + (q & 3)) * 65536, vo, O[q]);
            }
            STAMP(7);
        }
    }
    __syncthreads();
    if (threadIdx.x == 0) st_agent(ws + OFF_CNT + b * 64, seq);
}

extern "C" void kernel_launch(int rank, int nranks, ncclComm_t comm, hipStream_t stream,
                              void* const* d_in, const int* in_sizes, int n_in,
                              void* d_out, int out_size, const int* in_dtypes, int out_dtype,
                              void* const* d_out_all, void* const* d_ws_all, size_t ws_bytes) {
    if (nranks != NR || (unsigned)rank >= (unsigned)NR || n_in != 2 || ws_bytes < WS_NEED || in_sizes[0] != 4096 * 512 || in_sizes[1] != 512 * 32768 || out_size != 512 * 32768) {
        fprintf(stderr, "kernel_launch: unexpected shapes\n");
        return;
    }
    HIP_CHECK(hipFuncSetAttribute((const void*)fused_gemm_rs, hipFuncAttributeMaxDynamicSharedMemorySize, LDS_BYTES));
    Params p;
    memset(&p, 0, sizeof p);
    p.x = (const char*)d_in[0]; p.w = (const char*)d_in[1]; p.out = (char*)d_out;
    for (int i = 0; i < NR; ++i) p.ws[i] = (char*)d_ws_all[i];
    p.ws_tab = (char* const*)d_ws_all;
    p.rank = rank;
    int key[256];
    for (int i = 0; i < 256; ++i) {
        const int j = i >> 5, k = i & 31;
        key[i] = k >= 16 ? 112 * j + SKEW_NUM * (k - 16) - 1 : (k >= 14 ? 100000 + j : 112 * j + (ROT ? XSPREAD * (k >> 1) : -2));
        int q = i;
        while (q > 0 && key[p.order[q - 1]] > key[i]) { p.order[q] = p.order[q - 1]; --q; }
        p.order[q] = (unsigned char)i;
    }
    for (int i = 0, n = 0; i < 256; ++i) if (p.order[i] & 16) p.worder[n++] = (unsigned char)((p.order[i] >> 5) * 16 + (p.order[i] & 15));
    hipLaunchKernelGGL(fused_gemm_rs, dim3(256), dim3(512), LDS_BYTES, stream, p);
    HIP_CHECK(hipGetLastError());
}
```

```cpp
#include <hip/hip_runtime.h>
#include <rccl/rccl.h>
#include <cstdio>
#include <cstdint>
#include <cstring>

#define HIP_CHECK(expr) do { hipError_t _e = (expr); if (_e != hipSuccess) { \
    fprintf(stderr, "HIP error %s at %s:%d\n", hipGetErrorString(_e), __FILE__, __LINE__); \
    return; } } while (0)

typedef unsigned u32;
typedef unsigned long long u64;
typedef u32 u32x4 __attribute__((ext_vector_type(4)));
typedef u32 u32x2 __attribute__((ext_vector_type(2)));
typedef short bf16x8 __attribute__((ext_vector_type(8)));
typedef float f32x4 __attribute__((ext_vector_type(4)));
#define LAS __attribute__((address_space(3)))
#define GAS __attribute__((address_space(1)))

#define FORCE_RAW 0
#define GAUX 17
#define SUBTAU 36
#define ROT 1
#define XSPREAD 12
#define LOOPV 1
#define MIX 1
#define PASS1(v) (v)
#define SKEW_NUM 38

constexpr int NR = 8;
constexpr size_t MiB = 1ull << 20;
constexpr size_t OFF_CNT = 0;
constexpr size_t OFF_TMO = 16384;
constexpr size_t OFF_TICKET = 16384 + 256;
constexpr size_t OFF_FG = 65536;
constexpr size_t OFF_RIN = 1 * MiB;
constexpr size_t OFF_RY = 2 * MiB;
constexpr size_t OFF_PX = 4 * MiB;
constexpr size_t OFF_PW = 12 * MiB;
constexpr size_t OFF_PY = 44 * MiB;
constexpr size_t OFF_G = 76 * MiB;
constexpr size_t OFF_DBG = 140 * MiB;
constexpr size_t WS_NEED = 142 * MiB;
constexpr u64 DEADLINE = 40000000ull;
constexpr int LDS_GEMM = 131072, LDS_BYTES = LDS_GEMM + 8 * 1152;
constexpr int NCOMM = 32, NGEMM = 224;

#define DELAY1 0
#define DELAY2 4
struct Params {
    const char* x; const char* w; char* out;
    char* ws[NR];
    char* const* ws_tab;
    int rank; int pad;
    unsigned char order[256];
    unsigned char worder[128];
};

#define STAMP(i)

#define WAIT_V(n) asm volatile("s_waitcnt vmcnt(" #n ")" ::: "memory")
#define WAIT_L(n) asm volatile("s_waitcnt lgkmcnt(" #n ")" ::: "memory")
#define BAR __builtin_amdgcn_s_barrier()
#define SCHED __builtin_amdgcn_sched_barrier(0)

template <int OFF> __device__ __forceinline__ void st16_sys(const char* base, u32 voff, u32x4 v) {
    asm volatile("global_store_dwordx4 %0, %1, %2 offset:%3 sc0 sc1\n\ts_nop 1" :: "v"(voff), "v"(v), "s"(base), "i"(OFF) : "memory");
}
__device__ __forceinline__ void sgpr_settle() { asm volatile("s_nop 4" ::: "memory"); }
__device__ __forceinline__ u32 perm(u32 hi, u32 lo, u32 sel) { return __builtin_amdgcn_perm(hi, lo, sel); }
__device__ __forceinline__ u32 ld_agent(const char* p) { return __hip_atomic_load((GAS u32*)p, __ATOMIC_RELAXED, __HIP_MEMORY_SCOPE_AGENT); }
__device__ __forceinline__ void st_agent(char* p, u32 v) { __hip_atomic_store((GAS u32*)p, v, __ATOMIC_RELAXED, __HIP_MEMORY_SCOPE_AGENT); }
__device__ __forceinline__ u32 cvt_pk_bf16(float lo, float hi) { u32 r; asm("v_cvt_pk_bf16_f32 %0, %1, %2" : "=v"(r) : "v"(lo), "v"(hi)); return r; }

__device__ __forceinline__ void planes_to_raw(const u32 (&L)[16], const u32 (&H)[16], u32x4 (&O)[8]) {
#pragma unroll
    for (int d = 0; d < 4; ++d)
#pragma unroll
        for (int p = 0; p < 4; ++p) {
            O[2 * d][p] = perm(H[d * 4 + p], L[d * 4 + p], 0x05010400u);
            O[2 * d + 1][p] = perm(H[d * 4 + p], L[d * 4 + p], 0x07030602u);
        }
}
__device__ __forceinline__ void store_unit_linear(const char* dst, u32 voff, const u32x4 (&O)[8]) {
    const char* b = dst + 4096;
    sgpr_settle();
    st16_sys<-4096>(b, voff, O[0]); st16_sys<-3072>(b, voff, O[1]); st16_sys<-2048>(b, voff, O[2]); st16_sys<-1024>(b, voff, O[3]);
    st16_sys<0>(b, voff, O[4]); st16_sys<1024>(b, voff, O[5]); st16_sys<2048>(b, voff, O[6]); st16_sys<3072>(b, voff, O[7]);
}

constexpr u32 H_RAW = 1u << 16, H_DEAD = 2u << 16;
constexpr int ENC_BYTES = 1152;
__device__ __forceinline__ u32 bfi(u32 m, u32 a, u32 b) { return (a & m) | (b & ~m); }
__device__ __forceinline__ u32 scan32(u32 v) {
    v += (u32)__builtin_amdgcn_update_dpp(0, (int)v, 0x111, 0xf, 0xf, false);
    v += (u32)__builtin_amdgcn_update_dpp(0, (int)v, 0x112, 0xf, 0xf, false);
    v += (u32)__builtin_amdgcn_update_dpp(0, (int)v, 0x114, 0xf, 0xf, false);
    v += (u32)__builtin_amdgcn_update_dpp(0, (int)v, 0x118, 0xf, 0xf, false);
    v += (u32)__builtin_amdgcn_update_dpp(0, (int)v, 0x142, 0xa, 0xf, false);
    return v;
}
__device__ __forceinline__ void enc_init(LAS unsigned char* sc, int lane) {
    u32 sel = 0x0c0c0c0cu;
#pragma unroll
    for (int b = 3; b >= 0; --b) if (lane & (1 << b)) sel = (sel << 8) | (u32)b;
    if (lane < 16) ((LAS u32*)(sc + 1088))[lane] = sel;
}
template <class F>
__device__ __forceinline__ u32 encode_store(const u32 (&L)[16], const u32 (&H)[16], const char* slot, LAS unsigned char* sc, int lane, F before_stores) {
    u32 d[16], S[16];
    u32 sum = 0;
#pragma unroll
    for (int t = 0; t < 16; ++t) { d[t] = bfi(0x01010101u, L[t] >> 7, H[t] << 1); S[t] = bfi(0x80808080u, H[t], L[t]); }
#pragma unroll
    for (int t = 0; t < 16; t += 4) sum = __builtin_amdgcn_sad_u8(d[t], 0u, sum);
    sum = scan32(sum);
    const u32 tot = (u32)__builtin_amdgcn_readlane((int)sum, 31) + (u32)__builtin_amdgcn_readlane((int)sum, 63);
    const u32 T = min(max((tot >> 10) + 4u, 15u), 255u), T4 = T * 0x01010101u;
    *(LAS u32x4*)(sc + lane * 16) = (u32x4){0u, 0u, 0u, 0u};
    if (lane == 0) *(LAS u32x4*)(sc + 1024) = (u32x4){0u, 0u, 0u, 0u};
    u32 acc = 0;
#pragma unroll
    for (int t = 0; t < 16; ++t) { d[t] = T4 - d[t]; acc |= d[t]; }
    u32 rec = 0, head = T;
    bool raw = FORCE_RAW;
    if (__builtin_amdgcn_ballot_w64((acc & 0xf0f0f0f0u) != 0u) != 0ull) {
        u32 n = 0, sH = 0, sL = 0, sM = 0, sT = 0;
#pragma unroll
        for (int t = 0; t < 16; ++t) {
            const u32 e = T4 - d[t];
            const u32 w = ((T4 | 0x80808080u) - (e & 0x7f7f7f7fu)) ^ ((T4 ^ ~e) & 0x80808080u), mis = w & 0xf0f0f0f0u;
            d[t] = w & 0x0f0f0f0fu;
            if (mis) { ++n; sH = H[t]; sL = L[t]; sM = mis; sT = t; }
        }
        const u32 bsh = (n ? (u32)__builtin_ctz(sM) : 0u) & 24u;
        const bool multi = n > 1u || (sM & ~(0xffu << bsh)) != 0u;
        u64 mask = __builtin_amdgcn_ballot_w64(n != 0u);
        const u32 cnt = (u32)__builtin_popcountll(mask);
        if (cnt > 7u || __builtin_amdgcn_ballot_w64(multi) != 0ull) raw = true;
        else {
            const u32 ent = ((u32)lane << 22) | (sT << 18) | (bsh << 13) | (((sH >> bsh) & 0xffu) << 8) | ((sL >> bsh) & 0xffu);
            for (u32 s = 1; mask; ++s, mask &= mask - 1) {
                const u32 e = (u32)__builtin_amdgcn_readlane((int)ent, __builtin_ctzll(mask));
                if ((u32)(lane & 7) == s) rec = e;
            }
            head |= cnt << 8;
        }
    }
    u32x4 Q; u32 cs[16], bo[16], bits = 0;
#pragma unroll
    for (int t = 0; t < 16; ++t) {
        const u32 sel = d[t] + (d[t] & 0x08080808u);
        const u32 r = perm(0xffffffaau, 0x5500ffffu, sel), r2 = perm(0xffffff00u, 0x0000ffffu, sel);
        Q[t >> 2] = (t & 3) ? bfi(0x03030303u << (2 * (t & 3)), r, Q[t >> 2]) : r;
        cs[t] = *(const LAS u32*)(sc + 1088 + __builtin_amdgcn_sad_u8(r2 & 0x20100804u, 0u, 0u));
        bo[t] = bits;
        bits += (u32)__builtin_popcount(r2);
    }
    const u32 inc = scan32(bits);
    const u32 nb = max((u32)__builtin_amdgcn_readlane((int)inc, 31), (u32)__builtin_amdgcn_readlane((int)inc, 63)) >> 3;
    if (nb > 1024u) raw = true;
    before_stores();
    const u32 voff = (u32)lane * 16u;
    if (raw) {
        u32x4 O[8]; planes_to_raw(L, H, O); store_unit_linear(slot, voff, O);
        head = H_RAW;
    } else {
        const char* b = slot + 4096;
        sgpr_settle();
        st16_sys<-4096>(b, voff, (u32x4){S[0], S[1], S[2], S[3]}); st16_sys<-3072>(b, voff, (u32x4){S[4], S[5], S[6], S[7]});
        st16_sys<-2048>(b, voff, (u32x4){S[8], S[9], S[10], S[11]}); st16_sys<-1024>(b, voff, (u32x4){S[12], S[13], S[14], S[15]});
        st16_sys<0>(b, voff, Q);
        const u32 base = inc - bits + (lane & 32 ? 4u : 0u);
#pragma unroll
        for (int t = 0; t < 16; ++t) {
            const u32 o = base + bo[t];
            const u64 x = (u64)perm(0u, d[t], cs[t]) << (o & 31u);
            LAS u32* q = (LAS u32*)(sc + ((o >> 3) & ~3u));
            __hip_atomic_fetch_or(q, (u32)x, __ATOMIC_RELAXED, __HIP_MEMORY_SCOPE_WAVEFRONT);
            __hip_atomic_fetch_or(q + 1, (u32)(x >> 32), __ATOMIC_RELAXED, __HIP_MEMORY_SCOPE_WAVEFRONT);
        }
        asm volatile("" ::: "memory");
        const u32 n16 = (nb + 15u) >> 4;
        const u32x4 lst = *(const LAS u32x4*)(sc + lane * 16);
        if ((u32)lane < n16) st16_sys<1024>(b, voff, lst);
        head |= n16 << 20;
    }
    if ((lane & 7) == 0) rec = head;
    return rec;
}
__device__ __forceinline__ void put_record(char* p, u32 seq, u32 w, int lane, int nl) {
    const u32 cnt = ((u32)__builtin_amdgcn_readfirstlane(w) >> 8) & 7u;
    if (lane < nl && (u32)(lane & 7) <= cnt) __hip_atomic_store((GAS u64*)p, ((u64)w << 32) | seq, __ATOMIC_RELAXED, __HIP_MEMORY_SCOPE_SYSTEM);
}

__device__ __forceinline__ void pack_task(const Params& p, char* ws, int task, u32 cnt_v, LAS unsigned char* sc, int lane) {
    int j, r;
    if (task < 512) { j = task >> 6; r = task & 63; }
    else if (task < 768) { j = (task - 512) >> 5; r = 64 + (task & 31); }
    else if (task >= 1792 && task < 2048) { j = (task - 1792) >> 5; r = 96 + (task & 31); }
    else { const int c = task - (task < 1792 ? 768 : 1024), bk = p.worder[c / 28], m = c % 28; j = bk >> 4; r = 128 + (m >> 2) * 64 + (bk & 15) * 4 + (m & 3); }
    u32x4 in[8]; u32 L[16], H[16];
    const u32 voff = (u32)lane * 16u;
    if (r < 64) {
        const int pm = r >> 2, un = r & 3;
        const char* src = p.x + (size_t)(pm * 256 + un * 64) * 1024 + j * 128 + (lane >> 3) * 1024;
        const int sw = (lane & 7) ^ (lane >> 4);
#pragma unroll
        for (int q = 0; q < 8; ++q) in[q] = *(const GAS u32x4*)(src + q * 8192 + ((sw ^ ((q & 1) << 2)) << 4));
        char* peer = p.ws_tab[(p.rank + 1 + (lane >> 3)) & 7];
#pragma unroll
        for (int d = 0; d < 4; ++d)
#pragma unroll
            for (int q = 0; q < 4; ++q) { L[d * 4 + q] = perm(in[2 * d + 1][q], in[2 * d][q], 0x06040200u); H[d * 4 + q] = perm(in[2 * d + 1][q], in[2 * d][q], 0x07050301u); }
        const u32 seq = __builtin_amdgcn_readfirstlane(cnt_v) + 1u;
        asm volatile("" : "+v"(peer));
        if (task < 64) STAMP(10);
        const u32 rec = encode_store(L, H, ws + OFF_PX + (size_t)(seq & 1) * (4 * MiB) + (size_t)((j * 16 + pm) * 4 + un) * 8192, sc, lane, [] {});
        if (task < 64) STAMP(11);
        WAIT_V(0);
        if (task < 64) STAMP(12);
        put_record(peer + OFF_RIN + (size_t)(((seq & 1) * 8 + p.rank) * 1024 + (j * 32 + pm) * 4 + un) * 64 + (lane & 7) * 8, seq, rec, lane, 56);
        store_unit_linear(ws + OFF_G + (size_t)(j * 32 + pm) * 32768 + un * 8192, voff, in);
        WAIT_V(0);
        if (lane == 0) st_agent(ws + OFF_FG + (j * 32 + pm) * 128 + un * 4, seq);
    } else {
        const int dd = (r - 64) >> 6, pn = (r >> 2) & 15, un = r & 3, dest = (p.rank + dd) & 7;
        const char* src = p.w + (size_t)(j * 64 + (un & 1) * 32 + (lane >> 4) * 8) * 65536 + (dest * 4096 + pn * 256 + (un >> 1) * 128) * 2 + (lane & 15) * 16;
#pragma unroll
        for (int q = 0; q < 8; ++q) in[q] = *(const GAS u32x4*)(src + (size_t)q * 65536);
#pragma unroll
        for (int d = 0; d < 4; ++d)
#pragma unroll
            for (int q = 0; q < 4; ++q) { L[d * 4 + q] = perm(in[2 * q + 1][d], in[2 * q][d], 0x06020400u); H[d * 4 + q] = perm(in[2 * q + 1][d], in[2 * q][d], 0x07030501u); }
        const u32 seq = __builtin_amdgcn_readfirstlane(cnt_v) + 1u;
        if (dd == 0) {
            u32x4 O[8]; planes_to_raw(L, H, O);
            store_unit_linear(ws + OFF_G + (size_t)(j * 32 + 16 + pn) * 32768 + un * 8192, voff, O);
            WAIT_V(0);
            if (lane == 0) st_agent(ws + OFF_FG + (j * 32 + 16 + pn) * 128 + un * 4, seq);
        } else {
            const u32 rec = encode_store(L, H, ws + OFF_PW + (size_t)dest * (4 * MiB) + (size_t)((j * 16 + pn) * 4 + un) * 8192, sc, lane, [] {});
            WAIT_V(0);
            put_record(p.ws[dest] + OFF_RIN + (size_t)(((seq & 1) * 8 + p.rank) * 1024 + (j * 32 + 16 + pn) * 4 + un) * 64 + lane * 8, seq, rec, lane, 8);
        }
    }
}

#define WIN 2
#define PACE 985
#define STAG 0
#define NB 3
#define TGRAN 4
#define SLACK 1
constexpr int NSLOT = 7, NDEC = 5, NDESC = 16, NPOS = 1536, LOOK = 8;
constexpr u32 FIN = 0x40000000u;
constexpr int L_SLOT = 0;
constexpr int L_DESC = 2 * NSLOT * 8192;
constexpr int L_CTL = L_DESC + 2 * NDESC * 64;
constexpr int L_ORD = L_CTL + 128;
static_assert(L_ORD + 256 <= LDS_BYTES, "lds");

__device__ __forceinline__ void wait_units(u32 n) {
    switch (n) {
        case 0: WAIT_V(0); break; case 1: WAIT_V(6); break; case 2: WAIT_V(12); break; case 3: WAIT_V(18); break; case 4: WAIT_V(24); break;
        case 5: WAIT_V(30); break; case 6: WAIT_V(36); break; default: WAIT_V(42); break;
    }
}
__device__ __forceinline__ void comm_role(const Params& p, char* ws, LAS unsigned char* lds, int g, u32 seq, int wid, int lane) {
    if (threadIdx.x < 64) ((LAS u32*)(lds + L_ORD))[threadIdx.x] = ((const u32*)p.order)[threadIdx.x];
    if (threadIdx.x >= 64 && threadIdx.x < 96) ((LAS u32*)(lds + L_CTL))[threadIdx.x - 64] = 0;
    __syncthreads();
    volatile LAS u32* ctl = (volatile LAS u32*)(lds + L_CTL);
    volatile LAS u32* desc = (volatile LAS u32*)(lds + L_DESC);
    const u32 voff = (u32)lane * 16u;
    if (wid == 7) {
        const u32 st = lane >> 5, e = lane & 7, L = g * 2 + st, src = (p.rank + 1 + L % 7) & 7, j = L / 7;
        const u32 cnt = j < 9 ? (NPOS - j + 8) / 9 : 0;
        const char* rws = p.ws_tab[src];
        const u64 t0 = __builtin_amdgcn_s_memrealtime();
        bool dead = false;
        u32 x = (lane >> 3) & 3;
        while (__builtin_amdgcn_ballot_w64(x < cnt) != 0ull) {
            const bool act = x < cnt && x < ctl[st * 16 + 9] + LOOK;
            const u32 u = x < cnt ? j + 9 * x : 0;
            u32 id;
            if (u < 1024) {
                u32 bk = (lds + L_ORD)[u >> 2];
                if (!(bk & 16)) { const u32 i = bk & 15, pmi = i < 14 ? (i + (ROT ? 14 - 2 * (L % 7) : 0)) % 14 : i; bk = (bk & ~15u) | ((p.rank * 2 + 2 + pmi) & 15); }
                id = bk * 4u + (u & 3);
            } else { const u32 y = u - 1024; id = (y & ~31u) | ((y & 15) << 1) | ((y >> 4) & 1); }
            const char* a = u >= 1024 ? ws + OFF_RY + (size_t)(src * 512 + id) * 64 : ws + OFF_RIN + (size_t)(((seq & 1) * 8 + src) * 1024 + id) * 64;
            u64 v = 0;
            if (act) v = __hip_atomic_load((GAS u64*)(a + e * 8), __ATOMIC_RELAXED, __HIP_MEMORY_SCOPE_SYSTEM);
            const u32 hw = (u32)__builtin_amdgcn_ds_swizzle((int)(u32)(v >> 32), 0x0018);
            const u64 okm = __builtin_amdgcn_ballot_w64((u32)v == seq || e > ((hw >> 8) & 7u));
            const bool adv = act && (dead || ((okm >> (lane & 56)) & 0xffull) == 0xffull);
            if (__builtin_amdgcn_ballot_w64(adv) != 0ull) {
                if (adv) {
                    const char* s = u >= 1024 ? rws + OFF_PY + (size_t)p.rank * (4 * MiB) + (size_t)id * 8192
                        : rws + ((id & 64) ? OFF_PW + (size_t)p.rank * (4 * MiB) : OFF_PX + (size_t)(seq & 1) * (4 * MiB)) + (size_t)(((id >> 7) * 16 + ((id >> 2) & 15)) * 4 + (id & 3)) * 8192;
                    const u32 en = (st * NDESC + x % NDESC) * 16, word = dead ? H_DEAD : (u32)(v >> 32);
                    const u64 sa = dead ? (u64)(ws + OFF_G) : (u64)s;
                    desc[en + e] = word;
                    if (e == 0) { desc[en + 8] = (u32)sa; desc[en + 9] = (u32)(sa >> 32); desc[en + 10] = word; desc[en + 11] = (u >= 1024 ? 0x8000u : 0u) | id; }
                    x += 4;
                }
                u32 m = min(x, (u32)__builtin_amdgcn_update_dpp((int)x, (int)x, 0x128, 0xf, 0xf, false));
                m = min(m, (u32)__builtin_amdgcn_ds_swizzle((int)m, 0x401F));
                WAIT_L(0);
                if ((lane & 31) == 0) ctl[st * 16 + 8] = min(m, cnt);
            } else if (__builtin_amdgcn_ballot_w64(act) != 0ull && (ld_agent(ws + OFF_TMO) == seq || __builtin_amdgcn_s_memrealtime() - t0 > DEADLINE)) {
                st_agent(ws + OFF_TMO, seq); WAIT_V(0); dead = true;
            } else __builtin_amdgcn_s_sleep(1);
        }
        if ((lane & 31) == 0) ctl[st * 16 + 8] = cnt | FIN;
    } else if (wid >= NDEC) {
        const u32 me = wid - NDEC;
        volatile LAS u32* c = ctl + me * 16;
        u32 landed = 0, k = 0, slot = 0;
        const u32 t0 = (u32)__builtin_amdgcn_s_memrealtime();
        u32 next = 0, now;
        auto clock = [&]() { return ((u32)__builtin_amdgcn_s_memrealtime() - t0) * 64u; };
        auto pace = [&](u32 n64) {
            if ((int)(now - next) > (int)(SLACK * PACE / 4)) next = (int)(now - next) > 4 * PACE ? now + ((g * 2 + me) / 7) * STAG : now - SLACK * PACE / 4;
            while ((int)(now - next) < 0) now = clock();
            next += (n64 * PACE) >> 4;
        };
        __builtin_amdgcn_s_setprio(3);
        for (;; ++k) {
            volatile LAS u32* rp = (lane & 16) ? desc + (me * NDESC + k % NDESC) * 16 + (lane & 15) : c + (lane & 15);
            u32 v;
            bool fin = false;
            while (true) {
                v = *rp;
                now = clock();
                const u32 pub = __builtin_amdgcn_readlane(v, 8);
                if ((pub & ~FIN) > k && (u32)__builtin_amdgcn_readlane(v, slot) + NSLOT > k) break;
                if (landed < k) {
                    wait_units(k - landed - 1);
                    ++landed;
                    if (lane == 0) c[10] = landed;
                } else if (pub == (k | FIN)) { fin = true; break; }
            }
            if (fin) break;
            const char* s = (const char*)(((u64)(u32)__builtin_amdgcn_readlane(v, 25) << 32) | (u32)__builtin_amdgcn_readlane(v, 24));
            const u32 head = __builtin_amdgcn_readlane(v, 26);
            LAS unsigned char* d = lds + L_SLOT + (me * NSLOT + slot) * 8192;
            u32 n16 = (head >> 20) & 127u;
            if (head & (H_RAW | H_DEAD)) {
                pace(32);
                __builtin_amdgcn_global_load_lds((const GAS u32*)(s + 6144 + voff), (LAS u32*)(d + 6144), 16, 0, 17);
                __builtin_amdgcn_global_load_lds((const GAS u32*)(s + 6144 + voff), (LAS u32*)(d + 6144), 16, 1024, 17);
                n16 = 64; now = clock();
            }
            const u32 tail = ((n16 + TGRAN - 1) / TGRAN) * (TGRAN / 4);
            pace(NB == 1 ? 80 + tail : NB == 2 ? 32 + tail : 16 + tail);
            if (k < 16) STAMP(k);
            if ((u32)lane < max(n16, 1u)) __builtin_amdgcn_global_load_lds((const GAS u32*)(s + 4096 + voff), (LAS u32*)(d + 4096), 16, 1024, 17);
            __builtin_amdgcn_global_load_lds((const GAS u32*)(s + 4096 + voff), (LAS u32*)(d + 4096), 16, 0, 17);
            if (NB >= 3) { now = clock(); pace(NB == 3 ? 32 : 16); }
            __builtin_amdgcn_global_load_lds((const GAS u32*)(s + voff), (LAS u32*)d, 16, 0, 17);
            if (NB == 2 || NB >= 4) { now = clock(); pace(NB == 2 ? 48 : 16); }
            __builtin_amdgcn_global_load_lds((const GAS u32*)(s + voff), (LAS u32*)d, 16, 1024, 17);
            if (NB >= 3) { now = clock(); pace(NB == 5 ? 16 : 32); }
            __builtin_amdgcn_global_load_lds((const GAS u32*)(s + voff), (LAS u32*)d, 16, 2048, 17);
            if (NB == 5) { now = clock(); pace(16); }
            __builtin_amdgcn_global_load_lds((const GAS u32*)(s + voff), (LAS u32*)d, 16, 3072, 17);
            asm volatile("s_waitcnt vmcnt(%0)" :: "i"(6 * WIN) : "memory");
            if (k + 1 > landed + WIN) landed = k + 1 - WIN;
            if (lane == 0) { c[9] = k; c[10] = landed; }
            if (++slot == NSLOT) slot = 0;
        }
        if (lane == 0) c[10] = k | FIN;
    } else {
        char* pend = nullptr;
        int k = 0;
        u32 st = wid & 1;
        for (;;) {
            u32 e;
            {
                bool got = false;
                for (u32 idle = 0;;) {
                    st ^= 1;
                    const u32x2 lc = *(volatile LAS u32x2*)(lds + L_CTL + st * 64 + 40);
                    const u32 ld = __builtin_amdgcn_readfirstlane(lc[0]);
                    e = __builtin_amdgcn_readfirstlane(lc[1]);
                    if (e < (ld & ~FIN)) {
                        u32 o = e;
                        if (lane == 0) __hip_atomic_compare_exchange_strong((LAS u32*)(lds + L_CTL + st * 64 + 44), &o, e + 1, __ATOMIC_RELAXED, __ATOMIC_RELAXED, __HIP_MEMORY_SCOPE_WORKGROUP);
                        if (__builtin_amdgcn_readfirstlane(o) == e) { got = true; break; }
                        idle = 0;
                        continue;
                    }
                    if (ld & FIN) { if (++idle == 2) break; } else idle = 0;
                    if (pend) { WAIT_V(0); if (lane == 0) st_agent(pend, seq); pend = nullptr; }
                    else __builtin_amdgcn_s_sleep(1);
                }
                if (!got) break;
            }
            asm volatile("" ::: "memory");
            STAMP(k < 124 ? k >> 1 : 62);
            const u32 en = (st * NDESC + e % NDESC) * 16, sn = e % NSLOT;
            const u32 ent = __builtin_amdgcn_readfirstlane(desc[en + 11]), id = ent & 0x3ffu, w = (g * 2 + st) % 7;
            const u32 rec = desc[en + (lane & 7)], head = __builtin_amdgcn_readfirstlane(rec);
            const LAS u32x4* sl = (const LAS u32x4*)(lds + L_SLOT + (st * NSLOT + sn) * 8192 + voff);
            u32x4 O[8];
            ++k;
            if (head & (H_RAW | H_DEAD)) {
#pragma unroll
                for (int q = 0; q < 8; ++q) O[q] = sl[q * 64];
                WAIT_L(0);
                if (lane == 0) ctl[st * 16 + sn] = e + 1;
                if (head & H_DEAD) {
#pragma unroll
                    for (int q = 0; q < 8; ++q) O[q] = (u32x4){0x7fc07fc0u, 0x7fc07fc0u, 0x7fc07fc0u, 0x7fc07fc0u};
                }
            } else {
                u32 L[16], H[16], c[16], m[16], bo[16];
                u32x2 w[16];
                u32x4 S[4];
                const u32 T4 = (head & 0xffu) * 0x01010101u;
                const u32x4 Q = sl[4 * 64];
#pragma unroll
                for (int q = 0; q < 4; ++q) S[q] = sl[q * 64];
                u32 bits = 0;
#pragma unroll
                for (int t = 0; t < 16; ++t) {
                    c[t] = (Q[t >> 2] >> (2 * (t & 3))) & 0x03030303u;
                    m[t] = perm(0u, 0x01000000u, c[t]);
                    bo[t] = bits;
                    bits = __builtin_amdgcn_udot4(m[t], 0x08080808u, bits, false);
                }
                const u32 base = min(scan32(bits) - bits, 8192u) + (lane & 32 ? 4u : 0u);
                const LAS unsigned char* lst = lds + L_SLOT + (st * NSLOT + sn) * 8192 + 5120;
#pragma unroll
                for (int t = 0; t < 16; ++t) {
                    bo[t] += base;
                    const LAS u32* q = (const LAS u32*)(lst + ((bo[t] >> 3) & ~3u));
                    w[t] = (u32x2){q[0], q[1]};
                }
                WAIT_L(0);
                if (lane == 0) ctl[st * 16 + sn] = e + 1;
#pragma unroll
                for (int t = 0; t < 16; ++t) {
                    const u32 sel = (__umul24(m[t], 0x010101u) << 8) + perm(0u, 0x00101010u, c[t]);
                    const u32 E = T4 - (perm(0u, __builtin_amdgcn_alignbit(w[t][1], w[t][0], bo[t]), sel) & perm(0u, 0x0f040302u, c[t]));
                    const u32 sv = S[t >> 2][t & 3];
                    H[t] = bfi(0x80808080u, sv, E >> 1);
                    L[t] = bfi(0x80808080u, E << 7, sv);
                }
                planes_to_raw(L, H, O);
            }
            const u32 cnt = (head >> 8) & 7u;
            const u32 el = (rec >> 22) & 63u, et = (rec >> 18) & 15u, eb = (rec >> 16) & 3u, eq = (et >> 2) * 2u + (eb >> 1), eo = (et & 3u) * 4u + (eb & 1u) * 2u;
            char* flag = nullptr;
            if (!(ent & 0x8000u)) {
                const u32 blk = (8 + 7 * (id >> 7) + w) * 32 + ((id >> 2) & 31);
                char* dst = ws + OFF_G + (size_t)blk * 32768 + (id & 3) * 8192;
                store_unit_linear(dst, voff, O);
                if (cnt && lane >= 1 && (u32)lane <= cnt)
                    asm volatile("global_store_short %0, %1, %2 sc0 sc1" :: "v"(eq * 1024u + el * 16u + eo), "v"(rec), "s"(dst) : "memory");
                flag = ws + OFF_FG + blk * 128 + (id & 3) * 4;
            } else {
                const u32 src = (p.rank + 1 + w) & 7;
                const u32 row0 = ((id >> 4) & 1) * 256 + (id & 1) * 128 + ((id >> 1) & 3) * 32, col0 = src * 4096 + ((id >> 5) & 15) * 256 + ((id >> 3) & 1) * 128;
                char* dst = p.out + (size_t)row0 * 65536 + col0 * 2;
                const u32 vo = (u32)(lane >> 4) * 262144u + (u32)(lane & 15) * 16u;
                sgpr_settle();
#pragma unroll
                for (int q = 0; q < 8; ++q) st16_sys<0>(dst + (size_t)((q >> 2) * 16 + (q & 3)) * 65536, vo, O[q]);
                if (cnt && lane >= 1 && (u32)lane <= cnt)
                    asm volatile("global_store_short %0, %1, %2 sc0 sc1" :: "v"(((eq >> 2) * 16u + (eq & 3u) + (el >> 4) * 4u) * 65536u + (el & 15u) * 16u + eo), "v"(rec), "s"(dst) : "memory");
            }
            if (pend) { WAIT_V(8); if (lane == 0) st_agent(pend, seq); }
            pend = flag;
        }
        if (pend) { WAIT_V(0); if (lane == 0) st_agent(pend, seq); }
        STAMP(62);
    }
}

__device__ __forceinline__ void spin_flags(char* ws, const char* fbase, u32 fvoff, u32 seq) {
    const u64 t0 = __builtin_amdgcn_s_memrealtime();
    while (__builtin_amdgcn_ballot_w64(ld_agent(fbase + fvoff) < seq) != 0ull) {
        if (ld_agent(ws + OFF_TMO) == seq || __builtin_amdgcn_s_memrealtime() - t0 > DEADLINE) { st_agent(ws + OFF_TMO, seq); break; }
        __builtin_amdgcn_s_sleep(4);
    }
    __builtin_amdgcn_s_waitcnt(0xC07F);
    WAIT_V(0);
}

__device__ __forceinline__ void gemm_loop(char* ws, LAS unsigned char* lds, const char* gX, const char* gW, u32 fvoff, int otab, u32 seq, int wid, int lane, f32x4 (&acc)[2][2][4][2]) {
    constexpr size_t KSTEP = MiB, SX = 16384, SW = 16384;
    constexpr int NSTEP = 64, FSTEP = 4096;
    const int wr = wid >> 2, wc = wid & 3, fr = lane & 15, fq = lane >> 4;
    const u32 voff = (u32)lane * 16u;
    const char* cX = gX + wid * 1024;
    const char* cW = gW + (wid >> 2) * 8192 + (wid & 3) * 1024;
    const char* fb = ws + OFF_FG;
    const u32 ldsw = (u32)wid * 1024u;
    const int woff = wr * 8192 + lane * 16;
    const int xoff0 = (wc * 32 + fr) * 128 + ((fq ^ (fr >> 1)) << 4), xoff1 = xoff0 ^ 64;
#define SA(b, h) (((b) * 2 + (h)) * 16384)
#define SB(b, h) ((4 + (b) * 2 + (h)) * 16384)
#define GLDS(src, dst) __builtin_amdgcn_global_load_lds((const GAS u32*)((src) + voff), (LAS u32*)(lds + (dst) + ldsw), 16, 0, GAUX)
#define STW(b, h, ptr) do { GLDS((ptr) + (h) * 4096, SA(b, h)); GLDS((ptr) + (h) * 4096 + SW, SA(b, h) + 8192); } while (0)
#define STX(b, h, ptr) do { GLDS((ptr) + (h) * SX, SB(b, h)); GLDS((ptr) + (h) * SX + 8192, SB(b, h) + 8192); } while (0)
#define LDW(b, h) do { _Pragma("unroll") for (int m = 0; m < 4; ++m) _Pragma("unroll") for (int k = 0; k < 2; ++k) Wt[m][k] = *(const LAS bf16x8*)(lds + SA(b, h) + woff + k * 4096 + m * 1024); } while (0)
#define LDX(dst, b, h) do { _Pragma("unroll") for (int n = 0; n < 2; ++n) { dst[n][0] = *(const LAS bf16x8*)(lds + SB(b, h) + xoff0 + n * 2048); dst[n][1] = *(const LAS bf16x8*)(lds + SB(b, h) + xoff1 + n * 2048); } } while (0)
#define MMA(h, xh, Xt) do { __builtin_amdgcn_s_setprio(1); _Pragma("unroll") for (int m = 0; m < 4; ++m) _Pragma("unroll") for (int n = 0; n < 2; ++n) _Pragma("unroll") for (int k = 0; k < 2; ++k) \
        acc[h][xh][m][n] = __builtin_amdgcn_mfma_f32_16x16x32_bf16(Xt[n][k], Wt[m][k], acc[h][xh][m][n], 0, 0, 0); __builtin_amdgcn_s_setprio(0); } while (0)
#define FLAG_LOAD(step) asm volatile("global_load_dword %0, %1, %2 sc1" : "+v"(fl) : "v"(fvoff), "s"(fb + (size_t)(step) * FSTEP) : "memory")
#define FLAG_CHECK(step) do { if (wid == 0 && __builtin_amdgcn_ballot_w64(fl < seq) != 0ull) spin_flags(ws, fb + (size_t)(step) * FSTEP, fvoff, seq); } while (0)
#define WAIT_V6 asm volatile("s_waitcnt vmcnt(6)" : "+v"(fl) :: "memory")
#pragma unroll
    for (int a = 0; a < 2; ++a)
#pragma unroll
        for (int b = 0; b < 2; ++b)
#pragma unroll
            for (int m = 0; m < 4; ++m)
#pragma unroll
                for (int n = 0; n < 2; ++n) acc[a][b][m][n] = (f32x4){0.f, 0.f, 0.f, 0.f};
    bf16x8 Wt[4][2], X0[2][2], X1[2][2];
    u32 fl = 0;
#define KSTAMP(i)
    if (wid == 0) { spin_flags(ws, fb, fvoff, seq); spin_flags(ws, fb + FSTEP, fvoff, seq); }
    WAIT_V(0); WAIT_L(0); BAR; asm volatile("" ::: "memory");
#define WAIT_VN(n) asm volatile("s_waitcnt vmcnt(" #n ")" : "+v"(fl) :: "memory")
    FLAG_LOAD(2);
    STX(0, 0, cX); STX(0, 1, cX); STW(0, 0, cW); STW(0, 1, cW);
    STX(1, 0, cX + KSTEP); STX(1, 1, cX + KSTEP); STW(1, 0, cW + KSTEP);
    WAIT_VN(8); BAR;
    if (wr == 1) BAR;
    for (int t = 0; t < NSTEP; t += 2) {
        const int t1 = __builtin_amdgcn_readlane(otab, t + 1), t2 = __builtin_amdgcn_readlane(otab, (t + 2) & 63);
        const int t3 = __builtin_amdgcn_readlane(otab, (t + 3) & 63), t4 = __builtin_amdgcn_readlane(otab, (t + 4) & 63);
        const char* w1 = cW + (size_t)t1 * KSTEP;
        const char* x2 = cX + (size_t)t2 * KSTEP; const char* w2 = cW + (size_t)t2 * KSTEP;
        const char* x3 = cX + (size_t)t3 * KSTEP; const char* w3 = cW + (size_t)t3 * KSTEP;
        KSTAMP(t >> 1);
        FLAG_CHECK(t2); FLAG_LOAD(t3);
        LDX(X0, 0, 0); LDW(0, 0); LDX(X1, 0, 1); STW(1, 1, w1);
        WAIT_VN(9); WAIT_L(0); SCHED; BAR; SCHED; MMA(0, 0, X0); MMA(0, 1, X1); SCHED; BAR; SCHED;
        LDW(0, 1); STX(0, 0, x2); STX(0, 1, x2); STW(0, 0, w2);
        WAIT_VN(8); WAIT_L(0); SCHED; BAR; SCHED; MMA(1, 0, X0); MMA(1, 1, X1); SCHED; BAR; SCHED;
        FLAG_CHECK(t3); FLAG_LOAD(t4);
        LDX(X0, 1, 0); LDW(1, 0); LDX(X1, 1, 1); STW(0, 1, w2);
        WAIT_VN(9); WAIT_L(0); SCHED; BAR; SCHED; MMA(0, 0, X0); MMA(0, 1, X1); SCHED; BAR; SCHED;
        LDW(1, 1); STX(1, 0, x3); STX(1, 1, x3); STW(1, 0, w3);
        WAIT_VN(8); WAIT_L(0); SCHED; BAR; SCHED; MMA(1, 0, X0); MMA(1, 1, X1); SCHED; BAR; SCHED;
    }
#undef WAIT_VN
    KSTAMP(32);
    asm volatile("s_waitcnt vmcnt(0)" : "+v"(fl) :: "memory");
    if (wr == 0) BAR;
    BAR;
#undef SA
#undef SB
#undef GLDS
#undef STW
#undef STX
#undef LDW
#undef LDX
#undef MMA
#undef FLAG_LOAD
#undef FLAG_CHECK
#undef WAIT_V6
#undef KSTAMP
}

__device__ __forceinline__ void gemm_piece(char* ws, LAS unsigned char* lds, const char* gX, const char* gW, u32 fx, u32 fw, u32 n0, u32 seq, int wid, int lane, f32x4 (&acc)[2][2][4][2]) {
    const int wr = wid >> 2, wc = wid & 3, fr = lane & 15, fq = lane >> 4;
    const u32 voff = (u32)lane * 16u;
    const char* cX = gX + wc * 4096;
    const char* cW = gW + wc * 4096;
    const int xoff0 = (wc * 32 + fr) * 128 + ((fq ^ (fr >> 1)) << 4), xoff1 = xoff0 ^ 64;
    auto stage = [&](int n) {
        const char* x = cX + (size_t)((n + n0) & 63) * MiB + voff; const char* w = cW + (size_t)((n + n0) & 63) * MiB + voff;
        LAS unsigned char* d = lds + (n & 3) * 32768 + wc * 4096;
        __builtin_amdgcn_global_load_lds((const GAS u32*)x, (LAS u32*)d, 16, 0, GAUX); __builtin_amdgcn_global_load_lds((const GAS u32*)x, (LAS u32*)d, 16, 1024, GAUX);
        __builtin_amdgcn_global_load_lds((const GAS u32*)x, (LAS u32*)d, 16, 2048, GAUX); __builtin_amdgcn_global_load_lds((const GAS u32*)x, (LAS u32*)d, 16, 3072, GAUX);
        __builtin_amdgcn_global_load_lds((const GAS u32*)w, (LAS u32*)(d + 16384), 16, 0, GAUX); __builtin_amdgcn_global_load_lds((const GAS u32*)w, (LAS u32*)(d + 16384), 16, 1024, GAUX);
        __builtin_amdgcn_global_load_lds((const GAS u32*)w, (LAS u32*)(d + 16384), 16, 2048, GAUX); __builtin_amdgcn_global_load_lds((const GAS u32*)w, (LAS u32*)(d + 16384), 16, 3072, GAUX);
    };
#pragma unroll
    for (int a = 0; a < 2; ++a)
#pragma unroll
        for (int m = 0; m < 4; ++m)
#pragma unroll
            for (int n = 0; n < 2; ++n) acc[a][0][m][n] = (f32x4){0.f, 0.f, 0.f, 0.f};
    if (wid < 4) spin_flags(ws, ws + OFF_FG, (u32)lane * 4096u + ((wid & 2) ? fw : fx) + (wid & 1) * 4, seq);
    WAIT_V(0); WAIT_L(0); BAR; asm volatile("" ::: "memory");
    if (wr == 1) { stage(0); stage(2); WAIT_V(8); BAR; } else stage(1);
    BAR;
    for (int n = wr; n < 64; n += 2) {
        bf16x8 Wt[8][2], Xt[2][2];
        const LAS unsigned char* bx = lds + (n & 3) * 32768;
#pragma unroll
        for (int i = 0; i < 2; ++i) { Xt[i][0] = *(const LAS bf16x8*)(bx + xoff0 + i * 2048); Xt[i][1] = *(const LAS bf16x8*)(bx + xoff1 + i * 2048); }
#pragma unroll
        for (int k = 0; k < 2; ++k)
#pragma unroll
            for (int c = 0; c < 8; ++c) Wt[c][k] = *(const LAS bf16x8*)(bx + 16384 + voff + k * 8192 + c * 1024);
        stage(n + 3);
        WAIT_V(8); WAIT_L(0); SCHED; BAR; SCHED;
        __builtin_amdgcn_s_setprio(1);
#pragma unroll
        for (int k = 0; k < 2; ++k)
#pragma unroll
            for (int c = 0; c < 8; ++c)
#pragma unroll
                for (int i = 0; i < 2; ++i) acc[c >> 2][0][c & 3][i] = __builtin_amdgcn_mfma_f32_16x16x32_bf16(Xt[i][k], Wt[c][k], acc[c >> 2][0][c & 3][i], 0, 0, 0);
        __builtin_amdgcn_s_setprio(0);
        SCHED; BAR; asm volatile("" ::: "memory");
    }
    WAIT_V(0);
    if (wr == 0) BAR;
    BAR; asm volatile("" ::: "memory");
}

__device__ __forceinline__ void acc_to_raw(const f32x4 (&acc)[2][2][4][2], int xh, bool poison, u32x4 (&O)[8]) {
#pragma unroll
    for (int n = 0; n < 2; ++n)
#pragma unroll
        for (int j = 0; j < 4; ++j)
#pragma unroll
            for (int c = 0; c < 4; ++c) {
                const u32 v = cvt_pk_bf16(acc[c >> 1][xh][(c & 1) * 2][n][j], acc[c >> 1][xh][(c & 1) * 2 + 1][n][j]);
                O[n * 4 + j][c] = poison ? 0x7fc07fc0u : v;
            }
}

__global__ __launch_bounds__(512, 2) void fused_gemm_rs(Params p) {
    extern __shared__ __attribute__((aligned(16))) unsigned char shm[];
    LAS unsigned char* lds = (LAS unsigned char*)shm;
    const int lane = threadIdx.x & 63, wid = __builtin_amdgcn_readfirstlane(threadIdx.x >> 6), b = blockIdx.x;
    char* ws = p.ws[p.rank];
    const u32 cnt_v = ld_agent(ws + OFF_CNT + b * 64);
    const int xcd = b & 7, ix = b >> 3;
    u32 seq;
    if (ix < 4) {
        STAMP(63);
        seq = __builtin_amdgcn_readfirstlane(cnt_v) + 1u;
        comm_role(p, ws, lds, ix * 8 + xcd, seq, wid, lane);
    } else {
        STAMP(0);
        const int li = ix - 4, cg = li / 7, pmi = (xcd & 1) * 7 + li % 7;
        const int pn = MIX == 0 ? (xcd >> 1) * 4 + cg : (cg < 2 ? (xcd & 6) + cg : (MIX == 1 ? (xcd & 6) + 6 + cg : 12 - (xcd & 6) + cg));
        const int gi = (15 - pn) * 14 + pmi;
        LAS unsigned char* sc = lds + LDS_GEMM + wid * ENC_BYTES;
        enc_init(sc, lane);
        {
            const int t1 = PASS1(wid * NGEMM + gi);
            const int cls = t1 < 64 ? 0 : (t1 < 128 || (t1 >= 512 && t1 < 576) || (t1 >= 768 && t1 < 768 + 252)) ? 1 : 2;
            for (int i = cls == 0 ? 0 : cls == 1 ? DELAY1 : DELAY2; i > 0; --i) __builtin_amdgcn_s_sleep(8);
            int task = t1;
            for (int pass = 0; task < 4608; ++pass) {
                pack_task(p, ws, task, cnt_v, sc, lane);
                STAMP(pass < 2 ? 1 + pass : 3);
                task = gi < 112 ? 1792 + pass * 896 + (pass < 3 ? wid * 112 + gi : gi * 8 + wid) : 4608;
            }
        }
        seq = __builtin_amdgcn_readfirstlane(cnt_v) + 1u;
        const int dest = (p.rank + 1 + (pmi >> 1)) & 7, pm = dest * 2 + (pmi & 1);
        f32x4 acc[2][2][4][2];
        gemm_loop(ws, lds, ws + OFF_G + (size_t)pm * 32768, ws + OFF_G + (size_t)(16 + pn) * 32768,
                         (u32)(((lane & 4) ? 16 + pn : pm) * 128 + (lane & 3) * 4),
                  lane < 8 ? lane : 8 + (lane - 8) / 7 * 7 + ((lane - 8) % 7 + (ROT ? 7 - (pmi >> 1) : 0)) % 7, seq, wid, lane, acc);
        STAMP(4);
        {
            const bool poison = __builtin_amdgcn_readfirstlane(ld_agent(ws + OFF_TMO)) == seq;
            const int id0 = ((pn * 2 + (pmi & 1)) * 8 + wid) * 2;
            u32 rec = 0;
            char* ry = p.ws[dest] + OFF_RY + (size_t)(p.rank * 512 + id0) * 64 + lane * 8;
#pragma unroll
            for (int xh = 0; xh < 2; ++xh) {
                u32x4 O[8]; u32 L[16], H[16];
                acc_to_raw(acc, xh, poison, O);
#pragma unroll
                for (int d = 0; d < 4; ++d)
#pragma unroll
                    for (int q = 0; q < 4; ++q) { L[d * 4 + q] = perm(O[2 * d + 1][q], O[2 * d][q], 0x06040200u); H[d * 4 + q] = perm(O[2 * d + 1][q], O[2 * d][q], 0x07050301u); }
                rec = encode_store(L, H, ws + OFF_PY + (size_t)dest * (4 * MiB) + (size_t)(id0 + xh) * 8192, sc, lane,
                                   [&] { if (xh == 1) { WAIT_V(0); put_record(ry, seq, rec, lane, 8); } });
            }
            WAIT_V(0);
            put_record(ry + 64, seq, rec, lane, 8);
        }
        STAMP(5);
        for (bool first = pn < 8;; first = false) {
            __syncthreads();
            if (threadIdx.x == 0) {
                volatile LAS u32* t = (volatile LAS u32*)(lds + LDS_GEMM);
                t[0] = first ? (u32)(MIX ? xcd * 14 + li : xcd * 28 + li)
                             : 112u + __hip_atomic_fetch_add((GAS u32*)(ws + OFF_TICKET), 1u, __ATOMIC_RELAXED, __HIP_MEMORY_SCOPE_AGENT) - (seq - 1u) * (NGEMM + 16);
                t[1] = (u32)__builtin_amdgcn_s_memrealtime() / SUBTAU;
            }
            __syncthreads();
            const u32 tk = __builtin_amdgcn_readfirstlane(*(volatile LAS u32*)(lds + LDS_GEMM)), n0 = __builtin_amdgcn_readfirstlane(*(volatile LAS u32*)(lds + LDS_GEMM + 4));
            if (tk >= 128u) break;
            const int rq = tk & 3, cq = tk >> 2, opm = p.rank * 2 + (rq >> 1), opn = cq >> 1;
            gemm_piece(ws, lds, ws + OFF_G + (size_t)opm * 32768 + (rq & 1) * 16384, ws + OFF_G + (size_t)(16 + opn) * 32768 + (cq & 1) * 16384,
                       (u32)(opm * 128 + (rq & 1) * 8), (u32)((16 + opn) * 128 + (cq & 1) * 8), n0, seq, wid, lane, acc);
            STAMP(6);
            LAS f32x4* red = (LAS f32x4*)(lds + (wid & 3) * 16384) + lane;
            if (wid >= 4) {
#pragma unroll
                for (int i = 0; i < 16; ++i) red[i * 64] = acc[i >> 3][0][(i >> 1) & 3][i & 1];
            }
            __syncthreads();
            if (wid < 4) {
#pragma unroll
                for (int i = 0; i < 16; ++i) acc[i >> 3][0][(i >> 1) & 3][i & 1] += red[i * 64];
                const bool poison = __builtin_amdgcn_readfirstlane(ld_agent(ws + OFF_TMO)) == seq;
                u32x4 O[8]; acc_to_raw(acc, 0, poison, O);
                char* dst = p.out + (size_t)(rq * 128 + wid * 32) * 65536 + (p.rank * 4096 + cq * 128) * 2;
                const u32 vo = (u32)(lane >> 4) * 262144u + (u32)(lane & 15) * 16u;
                sgpr_settle();
#pragma unroll
                for (int q = 0; q < 8; ++q) st16_sys<0>(dst + (size_t)((q >> 2) * 16 + (q & 3)) * 65536, vo, O[q]);
            }
            STAMP(7);
        }
    }
    __syncthreads();
    if (threadIdx.x == 0) st_agent(ws + OFF_CNT + b * 64, seq);
}

extern "C" void kernel_launch(int rank, int nranks, ncclComm_t comm, hipStream_t stream,
                              void* const* d_in, const int* in_sizes, int n_in,
                              void* d_out, int out_size, const int* in_dtypes, int out_dtype,
                              void* const* d_out_all, void* const* d_ws_all, size_t ws_bytes) {
    if (nranks != NR || (unsigned)rank >= (unsigned)NR || n_in != 2 || ws_bytes < WS_NEED || in_sizes[0] != 4096 * 512 || in_sizes[1] != 512 * 32768 || out_size != 512 * 32768) {
        fprintf(stderr, "kernel_launch: unexpected shapes\n");
        return;
    }
    HIP_CHECK(hipFuncSetAttribute((const void*)fused_gemm_rs, hipFuncAttributeMaxDynamicSharedMemorySize, LDS_BYTES));
    Params p;
    memset(&p, 0, sizeof p);
    p.x = (const char*)d_in[0]; p.w = (const char*)d_in[1]; p.out = (char*)d_out;
    for (int i = 0; i < NR; ++i) p.ws[i] = (char*)d_ws_all[i];
    p.ws_tab = (char* const*)d_ws_all;
    p.rank = rank;
    int key[256];
    for (int i = 0; i < 256; ++i) {
        const int j = i >> 5, k = i & 31;
        key[i] = k >= 16 ? 112 * j + SKEW_NUM * (k - 16) - 1 : (k >= 14 ? 100000 + j : 112 * j + XSPREAD * (k >> 1) - (j == 0 ? 1000 : 0));
        int q = i;
        while (q > 0 && key[p.order[q - 1]] > key[i]) { p.order[q] = p.order[q - 1]; --q; }
        p.order[q] = (unsigned char)i;
    }
    for (int i = 0, n = 0; i < 256; ++i) if (p.order[i] & 16) p.worder[n++] = (unsigned char)((p.order[i] >> 5) * 16 + (p.order[i] & 15));
    hipLaunchKernelGGL(fused_gemm_rs, dim3(256), dim3(512), LDS_BYTES, stream, p);
    HIP_CHECK(hipGetLastError());
}
```

```cpp
#include <hip/hip_runtime.h>
#include <rccl/rccl.h>
#include <cstdio>
#include <cstdint>
#include <cstring>

#define HIP_CHECK(expr) do { hipError_t _e = (expr); if (_e != hipSuccess) { \
    fprintf(stderr, "HIP error %s at %s:%d\n", hipGetErrorString(_e), __FILE__, __LINE__); \
    return; } } while (0)

typedef unsigned u32;
typedef unsigned long long u64;
typedef u32 u32x4 __attribute__((ext_vector_type(4)));
typedef u32 u32x2 __attribute__((ext_vector_type(2)));
typedef short bf16x8 __attribute__((ext_vector_type(8)));
typedef float f32x4 __attribute__((ext_vector_type(4)));
typedef float f32x2 __attribute__((ext_vector_type(2)));
#define LAS __attribute__((address_space(3)))
#define GAS __attribute__((address_space(1)))

#define FORCE_RAW 0
#define GAUX 17
#define SUBTAU 36
#define XSPREAD 12
#define MIX 1
#define PASS1(v) (v)
#define HC 1
#define NPASS "1233222233333334"
#define NDIAG "0000111111112222"
#define SKEW_NUM 42

constexpr int NR = 8;
constexpr size_t MiB = 1ull << 20;
constexpr size_t OFF_CNT = 0;
constexpr size_t OFF_TMO = 16384;
constexpr size_t OFF_TICKET = 16384 + 256;
constexpr size_t OFF_YO = 16384 + 512;
constexpr size_t OFF_YT = 16384 + 2048;
constexpr size_t OFF_FG = 65536;
constexpr size_t OFF_FP = 512 * 1024;
constexpr size_t OFF_RIN = 1 * MiB;
constexpr size_t OFF_RY = 2 * MiB;
constexpr size_t OFF_PX = 4 * MiB;
constexpr size_t OFF_PW = 12 * MiB;
constexpr size_t OFF_PY = 76 * MiB;
constexpr size_t OFF_G = 140 * MiB;
constexpr size_t OFF_P = 236 * MiB;
constexpr size_t OFF_DBG = 268 * MiB;
constexpr size_t WS_NEED = 271 * MiB;
constexpr u64 DEADLINE = 40000000ull;
constexpr int LDS_GEMM = 131072, LDS_BYTES = LDS_GEMM + 8 * 1152;
constexpr int NCOMM = 32, NGEMM = 224;

#define DELAY1 0
#define DELAY2 4
struct Params {
    const char* x; const char* w; char* out;
    char* ws[NR];
    char* const* ws_tab;
    int rank; int pad;
    unsigned char order[256];
    unsigned char worder[128];
    int pk[16][4][2];
    int dq[16][4];
};

#define STAMP(i)

#define WAIT_V(n) asm volatile("s_waitcnt vmcnt(" #n ")" ::: "memory")
#define WAIT_P __builtin_amdgcn_s_waitcnt(0x0F70)
#define WAIT_L(n) asm volatile("s_waitcnt lgkmcnt(" #n ")" ::: "memory")
#define BAR __builtin_amdgcn_s_barrier()
#define SCHED __builtin_amdgcn_sched_barrier(0)

template <int OFF> __device__ __forceinline__ void st16_sys(const char* base, u32 voff, u32x4 v) {
    asm volatile("global_store_dwordx4 %0, %1, %2 offset:%3 sc0 sc1\n\ts_nop 1" :: "v"(voff), "v"(v), "s"(base), "i"(OFF) : "memory");
}
__device__ __forceinline__ void sgpr_settle() { asm volatile("s_nop 4" ::: "memory"); }
__device__ __forceinline__ u32 perm(u32 hi, u32 lo, u32 sel) { return __builtin_amdgcn_perm(hi, lo, sel); }
__device__ __forceinline__ u32 ld_agent(const char* p) { return __hip_atomic_load((GAS u32*)p, __ATOMIC_RELAXED, __HIP_MEMORY_SCOPE_AGENT); }
__device__ __forceinline__ void st_agent(char* p, u32 v) { __hip_atomic_store((GAS u32*)p, v, __ATOMIC_RELAXED, __HIP_MEMORY_SCOPE_AGENT); }
__device__ __forceinline__ u32 cvt_pk_bf16(float lo, float hi) { u32 r; asm("v_cvt_pk_bf16_f32 %0, %1, %2" : "=v"(r) : "v"(lo), "v"(hi)); return r; }

__device__ __forceinline__ void store_unit_linear(const char* dst, u32 voff, const u32x4 (&O)[8]) {
    const char* b = dst + 4096;
    sgpr_settle();
    st16_sys<-4096>(b, voff, O[0]); st16_sys<-3072>(b, voff, O[1]); st16_sys<-2048>(b, voff, O[2]); st16_sys<-1024>(b, voff, O[3]);
    st16_sys<0>(b, voff, O[4]); st16_sys<1024>(b, voff, O[5]); st16_sys<2048>(b, voff, O[6]); st16_sys<3072>(b, voff, O[7]);
}

constexpr u32 H_RAW = 1u << 16, H_DEAD = 2u << 16;
constexpr int ENC_BYTES = 1152;
__device__ __forceinline__ u32 bfi(u32 m, u32 a, u32 b) { return (a & m) | (b & ~m); }
__device__ __forceinline__ u32 scan32(u32 v) {
    v += (u32)__builtin_amdgcn_update_dpp(0, (int)v, 0x111, 0xf, 0xf, false);
    v += (u32)__builtin_amdgcn_update_dpp(0, (int)v, 0x112, 0xf, 0xf, false);
    v += (u32)__builtin_amdgcn_update_dpp(0, (int)v, 0x114, 0xf, 0xf, false);
    v += (u32)__builtin_amdgcn_update_dpp(0, (int)v, 0x118, 0xf, 0xf, false);
    v += (u32)__builtin_amdgcn_update_dpp(0, (int)v, 0x142, 0xa, 0xf, false);
    return v;
}
__device__ __forceinline__ void enc_init(LAS unsigned char* sc, int lane) {
    u32 sel = 0x0c0c0c0cu;
#pragma unroll
    for (int b = 3; b >= 0; --b) if (lane & (1 << b)) sel = (sel << 8) | (u32)b;
    if (lane < 16) ((LAS u32*)(sc + 1088))[lane] = sel;
}
template <class F>
__device__ __forceinline__ u32 encode_store(const u32x4 (&O)[8], const char* slot, LAS unsigned char* sc, int lane, F before_stores) {
    u32 d[16], S[16];
    u32 sum = 0;
#pragma unroll
    for (int t = 0; t < 16; ++t) {
        const u32 a = O[2 * (t >> 2)][t & 3], b = O[2 * (t >> 2) + 1][t & 3];
        const u32 ra = __builtin_amdgcn_alignbit(a, a, 31), rb = __builtin_amdgcn_alignbit(b, b, 31);
        d[t] = perm(rb, ra, 0x07050301u); S[t] = perm(rb, ra, 0x06040200u);
    }
#pragma unroll
    for (int t = 0; t < 16; t += 4) sum = __builtin_amdgcn_sad_u8(d[t], 0u, sum);
    sum = scan32(sum);
    const u32 tot = (u32)__builtin_amdgcn_readlane((int)sum, 31) + (u32)__builtin_amdgcn_readlane((int)sum, 63);
    const u32 T = min(max((tot >> 10) + 4u, 15u), 255u), T4 = T * 0x01010101u;
    *(LAS u32x4*)(sc + lane * 16) = (u32x4){0u, 0u, 0u, 0u};
    if (lane == 0) *(LAS u32x4*)(sc + 1024) = (u32x4){0u, 0u, 0u, 0u};
    u32 acc = 0;
#pragma unroll
    for (int t = 0; t < 16; ++t) { d[t] = T4 - d[t]; acc |= d[t]; }
    u32 rec = 0, head = T;
    bool raw = FORCE_RAW;
    if (__builtin_amdgcn_ballot_w64((acc & 0xf0f0f0f0u) != 0u) != 0ull) {
        u32 n = 0, sH = 0, sL = 0, sM = 0, sT = 0;
#pragma unroll
        for (int t = 0; t < 16; ++t) {
            const u32 e = T4 - d[t];
            const u32 w = ((T4 | 0x80808080u) - (e & 0x7f7f7f7fu)) ^ ((T4 ^ ~e) & 0x80808080u), mis = w & 0xf0f0f0f0u;
            d[t] = w & 0x0f0f0f0fu;
            if (mis) { ++n; sL = O[2 * (t >> 2)][t & 3]; sH = O[2 * (t >> 2) + 1][t & 3]; sM = mis; sT = t; }
        }
        const u32 bsh = (n ? (u32)__builtin_ctz(sM) : 0u) & 24u;
        const bool multi = n > 1u || (sM & ~(0xffu << bsh)) != 0u;
        u64 mask = __builtin_amdgcn_ballot_w64(n != 0u);
        const u32 cnt = (u32)__builtin_popcountll(mask);
        if (cnt > 7u || __builtin_amdgcn_ballot_w64(multi) != 0ull) raw = true;
        else {
            const u32 ent = ((u32)lane << 22) | (sT << 18) | (bsh << 13) | ((((bsh & 16u) ? sH : sL) >> ((bsh & 8u) * 2u)) & 0xffffu);
            for (u32 s = 1; mask; ++s, mask &= mask - 1) {
                const u32 e = (u32)__builtin_amdgcn_readlane((int)ent, __builtin_ctzll(mask));
                if ((u32)(lane & 7) == s) rec = e;
            }
            head |= cnt << 8;
        }
    }
    u32x4 Q; u32 cs[16], bo[16], bits = 0;
#pragma unroll
    for (int t = 0; t < 16; ++t) {
        const u32 sel = d[t] + (d[t] & 0x08080808u);
        const u32 r = perm(0xffffffaau, 0x5500ffffu, sel), r2 = perm(0xffffff00u, 0x0000ffffu, sel);
        Q[t >> 2] = (t & 3) ? bfi(0x03030303u << (2 * (t & 3)), r, Q[t >> 2]) : r;
        cs[t] = ((const LAS u32*)(size_t)__builtin_amdgcn_sad_u8(r2 & 0x20100804u, 0u, (u32)(size_t)sc))[272];
        bo[t] = bits;
        bits += (u32)__builtin_popcount(r2);
    }
    const u32 inc = scan32(bits);
    const u32 nb = max((u32)__builtin_amdgcn_readlane((int)inc, 31), (u32)__builtin_amdgcn_readlane((int)inc, 63)) >> 3;
    if (nb > 1024u) raw = true;
    before_stores();
    const u32 voff = (u32)lane * 16u;
    if (raw) {
        store_unit_linear(slot, voff, O);
        head = H_RAW;
    } else {
        const char* b = slot + 4096;
        sgpr_settle();
        st16_sys<-4096>(b, voff, (u32x4){S[0], S[1], S[2], S[3]}); st16_sys<-3072>(b, voff, (u32x4){S[4], S[5], S[6], S[7]});
        st16_sys<-2048>(b, voff, (u32x4){S[8], S[9], S[10], S[11]}); st16_sys<-1024>(b, voff, (u32x4){S[12], S[13], S[14], S[15]});
        st16_sys<0>(b, voff, Q);
        const u32 base = inc - bits + (lane & 32 ? 4u : 0u) + (u32)(size_t)sc * 8u;
#pragma unroll
        for (int t = 0; t < 16; ++t) {
            const u32 o = base + bo[t];
            const u64 x = (u64)perm(0u, d[t], cs[t]) << (o & 31u);
            LAS u32* q = (LAS u32*)(size_t)((o >> 3) & ~3u);
            __hip_atomic_fetch_or(q, (u32)x, __ATOMIC_RELAXED, __HIP_MEMORY_SCOPE_WAVEFRONT);
            __hip_atomic_fetch_or(q + 1, (u32)(x >> 32), __ATOMIC_RELAXED, __HIP_MEMORY_SCOPE_WAVEFRONT);
        }
        asm volatile("" ::: "memory");
        const u32 n16 = (nb + 15u) >> 4;
        const u32x4 lst = *(const LAS u32x4*)(sc + lane * 16);
        if ((u32)lane < n16) st16_sys<1024>(b, voff, lst);
        head |= n16 << 20;
    }
    if ((lane & 7) == 0) rec = head;
    return rec;
}
__device__ __forceinline__ void put_record(char* p, u32 seq, u32 w, int lane, int nl) {
    const u32 cnt = ((u32)__builtin_amdgcn_readfirstlane(w) >> 8) & 7u;
    if (lane < nl && (u32)(lane & 7) <= cnt) __hip_atomic_store((GAS u64*)p, ((u64)w << 32) | seq, __ATOMIC_RELAXED, __HIP_MEMORY_SCOPE_SYSTEM);
}

__device__ __forceinline__ void pack_task(const Params& p, char* ws, int task, u32 cnt_v, LAS unsigned char* sc, int lane) {
    int j, r;
    if (task < 512) { j = task >> 6; r = task & 63; }
    else if (task < 768) { j = (task - 512) >> 5; r = 64 + (task & 31); }
    else if (task >= 1792 && task < 2048) { j = (task - 1792) >> 5; r = 96 + (task & 31); }
    else { const int c = task - (task < 1792 ? 768 : 1024), bk = p.worder[c / 28], m = c % 28; j = bk >> 4; r = 128 + (m >> 2) * 64 + (bk & 15) * 4 + (m & 3); }
    u32x4 in[8];
    const u32 voff = (u32)lane * 16u;
    if (r < 64) {
        const int pm = r >> 2, un = r & 3;
        const char* src = p.x + (size_t)(pm * 256 + un * 64) * 1024 + j * 128 + (lane >> 3) * 1024;
        const int sw = (lane & 7) ^ (lane >> 4);
#pragma unroll
        for (int q = 0; q < 8; ++q) in[q] = *(const GAS u32x4*)(src + q * 8192 + ((sw ^ ((q & 1) << 2)) << 4));
        char* peer = p.ws_tab[(p.rank + 1 + (lane >> 3)) & 7];
        const u32 seq = __builtin_amdgcn_readfirstlane(cnt_v) + 1u;
        asm volatile("" : "+v"(peer));
        if (task < 64) STAMP(10);
        if ((pm >> 1) != p.rank) {
            const u32 rec = encode_store(in, ws + OFF_PX + (size_t)(seq & 1) * (4 * MiB) + (size_t)((j * 16 + pm) * 4 + un) * 8192, sc, lane, [] {});
            if (task < 64) STAMP(11);
            WAIT_V(0);
            if (task < 64) STAMP(12);
            put_record(peer + OFF_RIN + (size_t)(((seq & 1) * 8 + p.rank) * 1024 + (j * 32 + pm) * 4 + un) * 64 + (lane & 7) * 8, seq, rec, lane, 56);
        }
        store_unit_linear(ws + OFF_G + (size_t)(j * 32 + pm) * 32768 + un * 8192, voff, in);
        WAIT_V(0);
        if (lane == 0) st_agent(ws + OFF_FG + (j * 32 + pm) * 128 + un * 4, seq);
    } else {
        const int dd = (r - 64) >> 6, pn = (r >> 2) & 15, un = r & 3, dest = (p.rank + dd) & 7;
        const char* src = p.w + (size_t)(j * 64 + (un & 1) * 32 + (lane >> 4) * 8) * 65536 + (dest * 4096 + pn * 256 + (un >> 1) * 128) * 2 + (lane & 15) * 16;
#pragma unroll
        for (int q = 0; q < 8; ++q) in[q] = *(const GAS u32x4*)(src + (size_t)q * 65536);
        u32x4 O[8];
#pragma unroll
        for (int d = 0; d < 4; ++d)
#pragma unroll
            for (int q = 0; q < 4; ++q) { O[2 * d][q] = perm(in[2 * q + 1][d], in[2 * q][d], 0x05040100u); O[2 * d + 1][q] = perm(in[2 * q + 1][d], in[2 * q][d], 0x07060302u); }
        const u32 seq = __builtin_amdgcn_readfirstlane(cnt_v) + 1u;
        if (dd) {
            const u32 rec = encode_store(O, ws + OFF_PW + (size_t)((seq & 1) * 8 + dest) * (4 * MiB) + (size_t)((j * 16 + pn) * 4 + un) * 8192, sc, lane, [] {});
            WAIT_V(0);
            put_record(p.ws[dest] + OFF_RIN + (size_t)(((seq & 1) * 8 + p.rank) * 1024 + (j * 32 + 16 + pn) * 4 + un) * 64 + lane * 8, seq, rec, lane, 8);
        }
        const int blk = dd ? (64 + ((dd - 1) >> 1) * 8 + j) * 32 + ((dd - 1) & 1) * 16 + pn : j * 32 + 16 + pn;
        store_unit_linear(ws + OFF_G + (size_t)blk * 32768 + un * 8192, voff, O);
        WAIT_V(0);
        if (lane == 0) st_agent(ws + OFF_FG + blk * 128 + un * 4, seq);
    }
}

#define WIN 2
#define PACE 985
#define YLEAD 2
#define PFP 0
#define NB 2
#define TGRAN 4
#define SLACK 1
constexpr int NSLOT = 7, NDEC = 5, NDESC = 16, NIN = 960, LOOK = 8;
constexpr u32 FIN = 0x40000000u;
constexpr int L_SLOT = 0;
constexpr int L_DESC = 2 * NSLOT * 8192;
constexpr int L_CTL = L_DESC + 2 * NDESC * 64;
constexpr int L_ORD = L_CTL + 128;
static_assert(L_ORD + 512 <= LDS_BYTES, "lds");

template <int V> struct IC { static constexpr int v = V; };
__device__ __forceinline__ void wait_units(u32 n) {
    switch (n) {
        case 0: WAIT_V(0); break; case 1: WAIT_V(6); break; case 2: WAIT_V(12); break; case 3: WAIT_V(18); break; case 4: WAIT_V(24); break;
        case 5: WAIT_V(30); break; case 6: WAIT_V(36); break; default: WAIT_V(42); break;
    }
}
__device__ __forceinline__ void comm_role(const Params& p, char* ws, LAS unsigned char* lds, u32 s0, u32 seq, int wid, int lane) {
    constexpr u32 NY = 9 + 4 * HC;
    if (threadIdx.x < 64) ((LAS u32*)(lds + L_ORD))[threadIdx.x] = ((const u32*)p.order)[threadIdx.x];
    if (threadIdx.x >= 64 && threadIdx.x < 96) ((LAS u32*)(lds + L_CTL))[threadIdx.x - 64] = 0;
    __builtin_amdgcn_s_waitcnt(0x0F70);
    __syncthreads();
    volatile LAS u32* ctl = (volatile LAS u32*)(lds + L_CTL);
    volatile LAS u32* desc = (volatile LAS u32*)(lds + L_DESC);
    const u32 voff = (u32)lane * 16u;
    if (wid == 7) {
        const u32 st = lane >> 5, e = lane & 7, L = s0 + st, src = (p.rank + 1 + L % 7) & 7, j = L / 7;
        const u32 cin = j < 9 ? (NIN - j + 8) / 9 : 0;
        bool open = s0 < 63, dead = false;
        const char* rws = p.ws_tab[src];
        const u64 t0 = __builtin_amdgcn_s_memrealtime();
        volatile LAS u32* tk = (volatile LAS u32*)(lds + L_ORD + 256) + st * NDESC;
        GAS u32* yt = (GAS u32*)(ws + OFF_YT + (L % 7) * 128);
        if ((lane & 31) == 0) __hip_atomic_fetch_max(yt, seq << 10, __ATOMIC_RELAXED, __HIP_MEMORY_SCOPE_AGENT);
        if ((lane & 31) < LOOK) tk[(cin + (lane & 31)) % NDESC] = j + NY * (lane & 31);
        u32 x = (lane >> 3) & 3, drawn = cin + LOOK, cnt = (s0 == 62 && st) ? 0u : FIN, idle = 0;
        while (__builtin_amdgcn_ballot_w64(x < cnt) != 0ull) {
            if (!open) open = dead || ld_agent(ws + OFF_YO + (L % 7) * 128) - (seq - 1u) * 9u >= (9u * (j - 8u) + NY - 10u) / (NY - 9u);
            const u32 lim = ctl[st * 16 + 9] + LOOK, nd = (open && cnt == FIN && lim > drawn) ? lim - drawn : 0u;
            u32 tb = 0;
            if (drawn == cin + LOOK && __builtin_amdgcn_ballot_w64(nd != 0u) != 0ull) WAIT_V(0);
            if (nd && (lane & 31) == 0) tb = __hip_atomic_fetch_add(yt, nd, __ATOMIC_RELAXED, __HIP_MEMORY_SCOPE_AGENT);
            const bool act = open && x < cnt && x < min(lim, drawn);
            const u32 u = x < cin ? j + 9 * x : NIN + (tk[x % NDESC] & 511u);
            u32 id;
            if (u < NIN) {
                u32 bk = (lds + L_ORD)[u >> 2];
                if (!(bk & 16)) { const u32 i = bk & 15, pmi = i < 14 ? (i + 2 * (L % 7) + 2) % 14 : i; bk = (bk & ~15u) | ((p.rank * 2 + 2 + pmi) & 15); }
                id = bk * 4u + (u & 3);
            } else { const u32 y = u - NIN; id = (y & ~31u) | ((y & 15) << 1) | ((y >> 4) & 1); }
            const char* a = u >= NIN ? ws + OFF_RY + (size_t)(((seq & 1) * 8 + src) * 512 + id) * 64 : ws + OFF_RIN + (size_t)(((seq & 1) * 8 + src) * 1024 + id) * 64;
            u64 v = 0;
            u32 pf = 0;
            if (act) {
                pf = ld_agent(ws + OFF_FP + ((L % 7) * 256 + ((id >> 1) & 255u)) * 4);
                v = __hip_atomic_load((GAS u64*)(a + e * 8), __ATOMIC_RELAXED, __HIP_MEMORY_SCOPE_SYSTEM);
                asm volatile("" : "+v"(pf), "+v"(v));
            }
            const u32 hw = (u32)__builtin_amdgcn_ds_swizzle((int)(u32)(v >> 32), 0x0018);
            const u64 okm = __builtin_amdgcn_ballot_w64(((u32)v == seq || e > ((hw >> 8) & 7u)) && (u < NIN || pf >= seq));
            const bool adv = act && (dead || ((okm >> (lane & 56)) & 0xffull) == 0xffull);
            if (__builtin_amdgcn_ballot_w64(nd != 0u) != 0ull) {
                tb = (u32)__builtin_amdgcn_ds_swizzle((int)tb, 0) - (seq << 10) + LOOK * NY;
                if ((u32)(lane & 31) < nd) tk[(drawn + (lane & 31)) % NDESC] = tb + (lane & 31);
                if (nd && tb + nd >= 512u) cnt = drawn + (tb < 512u ? 512u - tb : 0u);
                drawn += nd;
            }
            if (__builtin_amdgcn_ballot_w64(adv) != 0ull) {
                if (adv) {
                    const size_t po = (size_t)((seq & 1) * 8 + p.rank) * (4 * MiB);
                    const char* s = u >= NIN ? rws + OFF_PY + po + (size_t)id * 8192
                        : rws + ((id & 64) ? OFF_PW + po : OFF_PX + (size_t)(seq & 1) * (4 * MiB)) + (size_t)(((id >> 7) * 16 + ((id >> 2) & 15)) * 4 + (id & 3)) * 8192;
                    const u32 en = (st * NDESC + x % NDESC) * 16, word = dead ? H_DEAD : (u32)(v >> 32);
                    const u64 sa = dead ? (u64)(ws + OFF_G) : (u64)s;
                    desc[en + e] = word;
                    if (e == 0) { desc[en + 8] = (u32)sa; desc[en + 9] = (u32)(sa >> 32); desc[en + 10] = word; desc[en + 11] = (u >= NIN ? 0x8000u : 0u) | id; }
                    x += 4;
                }
                u32 m = min(x, (u32)__builtin_amdgcn_update_dpp((int)x, (int)x, 0x128, 0xf, 0xf, false));
                m = min(m, (u32)__builtin_amdgcn_ds_swizzle((int)m, 0x401F));
                WAIT_L(0);
                if ((lane & 31) == 0) ctl[st * 16 + 8] = min(m, cnt);
            } else if ((++idle & 15u) == 0u && (ld_agent(ws + OFF_TMO) == seq || __builtin_amdgcn_s_memrealtime() - t0 > DEADLINE)) {
                st_agent(ws + OFF_TMO, seq); WAIT_V(0); dead = true;
            } else __builtin_amdgcn_s_sleep(1);
        }
        if ((lane & 31) == 0) ctl[st * 16 + 8] = cnt | FIN;
    } else if (wid >= NDEC) {
        const u32 me = wid - NDEC;
        volatile LAS u32* c = ctl + me * 16;
        u32 landed = 0, k = 0, slot = 0, pc = PACE;
        const u32 t0 = (u32)__builtin_amdgcn_s_memrealtime();
        u32 next = 0, now;
        auto clock = [&]() { return ((u32)__builtin_amdgcn_s_memrealtime() - t0) * 64u; };
        auto pace = [&](u32 n64) {
            while ((int)(now - next) < 0) now = clock();
            next += (n64 * pc) >> 4;
        };
        __builtin_amdgcn_s_setprio(3);
        for (;; ++k) {
            volatile LAS u32* rp = (lane & 16) ? desc + (me * NDESC + k % NDESC) * 16 + (lane & 15) : c + (lane & 15);
            u32 v;
            bool fin = false;
            while (true) {
                v = *rp;
                now = clock();
                const u32 pub = __builtin_amdgcn_readlane(v, 8);
                if ((pub & ~FIN) > k && (u32)__builtin_amdgcn_readlane(v, slot) + NSLOT > k) break;
                if (landed < k) {
                    wait_units(k - landed - 1);
                    ++landed;
                    if (lane == 0) c[10] = landed;
                } else if (pub == (k | FIN)) { fin = true; break; }
            }
            if (fin) break;
            const char* s = (const char*)(((u64)(u32)__builtin_amdgcn_readlane(v, 25) << 32) | (u32)__builtin_amdgcn_readlane(v, 24));
            const u32 head = __builtin_amdgcn_readlane(v, 26);
            if (__builtin_amdgcn_readlane(v, 27) & 0x8000) pc = PACE * NY / 9;
            if (s0 < 63 && k + YLEAD == (NIN - (s0 + me) / 7 + 8) / 9 && lane == 0) __hip_atomic_fetch_add((GAS u32*)(ws + OFF_YO + ((s0 + me) % 7) * 128), 1u, __ATOMIC_RELAXED, __HIP_MEMORY_SCOPE_AGENT);
            LAS unsigned char* d = lds + L_SLOT + (me * NSLOT + slot) * 8192;
            if ((int)(now - next) > (int)(SLACK * PACE / 4)) next = now - SLACK * PACE / 4;
            u32 n16 = (head >> 20) & 127u;
            if (head & (H_RAW | H_DEAD)) {
                pace(32);
                __builtin_amdgcn_global_load_lds((const GAS u32*)(s + 6144 + voff), (LAS u32*)(d + 6144), 16, 0, 17);
                __builtin_amdgcn_global_load_lds((const GAS u32*)(s + 6144 + voff), (LAS u32*)(d + 6144), 16, 1024, 17);
                n16 = 64; now = clock();
            }
            const u32 tail = ((n16 + TGRAN - 1) / TGRAN) * (TGRAN / 4);
            pace(NB == 1 ? 80 + tail : NB == 2 ? 32 + tail : 16 + tail);
            if (k < 16 && s0 < 63) STAMP(k);
            if ((u32)lane < max(n16, 1u)) __builtin_amdgcn_global_load_lds((const GAS u32*)(s + 4096 + voff), (LAS u32*)(d + 4096), 16, 1024, 17);
            __builtin_amdgcn_global_load_lds((const GAS u32*)(s + 4096 + voff), (LAS u32*)(d + 4096), 16, 0, 17);
            if (NB >= 3) { now = clock(); pace(NB == 3 ? 32 : 16); }
            __builtin_amdgcn_global_load_lds((const GAS u32*)(s + voff), (LAS u32*)d, 16, 0, 17);
            if (NB == 2 || NB >= 4) { now = clock(); pace(NB == 2 ? 48 : 16); }
            __builtin_amdgcn_global_load_lds((const GAS u32*)(s + voff), (LAS u32*)d, 16, 1024, 17);
            if (NB >= 3) { now = clock();
                pace(NB == 5 ? 16 : 32); }
            __builtin_amdgcn_global_load_lds((const GAS u32*)(s + voff), (LAS u32*)d, 16, 2048, 17);
            if (NB == 5) { now = clock(); pace(16); }
            __builtin_amdgcn_global_load_lds((const GAS u32*)(s + voff), (LAS u32*)d, 16, 3072, 17);
            asm volatile("s_waitcnt vmcnt(%0)" :: "i"(6 * WIN) : "memory");
            if (k + 1 > landed + WIN) landed = k + 1 - WIN;
            if (lane == 0) { c[9] = k; c[10] = landed; }
            if (++slot == NSLOT) slot = 0;
        }
        if (lane == 0) c[10] = k | FIN;
    } else {
        char* pend = nullptr;
        int k = 0;
        u32 st = wid & 1;
#define DT(y, i)
        u32 k03 = 0x03030303u, kfc = ~3u, khi = 0xffff0000u, k111 = 0x10101u;
        asm volatile("" : "+s"(k03), "+s"(kfc), "+s"(khi), "+s"(k111));
        const __amdgpu_buffer_rsrc_t prs = __builtin_amdgcn_make_buffer_rsrc(ws + OFF_P, 0, 28 * (int)MiB, 0x00020000);
        for (;;) {
            u32 e;
            {
                bool got = false;
                for (u32 idle = 0;;) {
                    st ^= 1;
                    const u32x2 lc = *(volatile LAS u32x2*)(lds + L_CTL + st * 64 + 40);
                    const u32 ld = __builtin_amdgcn_readfirstlane(lc[0]);
                    e = __builtin_amdgcn_readfirstlane(lc[1]);
                    if (e < (ld & ~FIN)) {
                        u32 o = e;
                        if (lane == 0) __hip_atomic_compare_exchange_strong((LAS u32*)(lds + L_CTL + st * 64 + 44), &o, e + 1, __ATOMIC_RELAXED, __ATOMIC_RELAXED, __HIP_MEMORY_SCOPE_WORKGROUP);
                        if (__builtin_amdgcn_readfirstlane(o) == e) { got = true; break; }
                        idle = 0;
                        continue;
                    }
                    if (ld & FIN) { if (++idle == 2) break; } else idle = 0;
                    if (pend) { WAIT_V(0); if (lane == 0) st_agent(pend, seq); pend = nullptr; }
                    else __builtin_amdgcn_s_sleep(1);
                }
                if (!got) break;
            }
            asm volatile("" ::: "memory");
            DT(0, -1);
            if ((k & 1) && s0 < 63) STAMP(k < 124 ? k >> 1 : 62);
            if (k == 1 && s0 >= 63) STAMP(61);
            const u32 en = (st * NDESC + e % NDESC) * 16, sn = e % NSLOT;
            const u32 ent = __builtin_amdgcn_readfirstlane(desc[en + 11]), id = ent & 0x3ffu, w = (s0 + st) % 7;
            const u32 rec = desc[en + (lane & 7)], head = __builtin_amdgcn_readfirstlane(rec);
            const LAS u32x4* sl = (const LAS u32x4*)(lds + L_SLOT + (st * NSLOT + sn) * 8192 + voff);
            u32x4 O[8];
            const u32 cnt = (head >> 8) & 7u;
            const u32 el = (rec >> 22) & 63u, et = (rec >> 18) & 15u, eb = (rec >> 16) & 3u, eq = (et >> 2) * 2u + (eb >> 1), eo = (et & 3u) * 4u + (eb & 1u) * 2u;
            ++k;
            auto unpack = [&](auto emit) __attribute__((always_inline)) {
            if (head & (H_RAW | H_DEAD)) {
#pragma unroll
                for (int q = 0; q < 8; ++q) O[q] = sl[q * 64];
                WAIT_L(0);
                if (lane == 0) ctl[st * 16 + sn] = e + 1;
                if (head & H_DEAD) {
#pragma unroll
                    for (int q = 0; q < 8; ++q) O[q] = (u32x4){0x7fc07fc0u, 0x7fc07fc0u, 0x7fc07fc0u, 0x7fc07fc0u};
                }
                emit(IC<0>{}); emit(IC<1>{}); emit(IC<2>{}); emit(IC<3>{});
            } else {
                u32 c[16], m[16], bo[16];
                u32x2 w[16];
                u32x4 S[4];
                const u32 T4 = (head & 0xffu) * 0x01010101u;
                const u32x4 Q = sl[4 * 64];
#pragma unroll
                for (int q = 0; q < 4; ++q) S[q] = sl[q * 64];
                u32 n = 0;
#pragma unroll
                for (int q = 0; q < 4; ++q) n += (u32)__builtin_popcount(Q[q] & (Q[q] >> 1) & 0x55555555u);
#pragma unroll
                for (int t = 0; t < 16; ++t) { c[t] = (Q[t >> 2] >> (2 * (t & 3))) & k03; m[t] = perm(0u, 0x01000000u, c[t]); }
                u32 bits = (min(scan32(n) - n, 1024u) + (u32)(L_SLOT + 5120) + (st * NSLOT + sn) * 8192u + (u32)(size_t)lds) * 8u + (lane & 32 ? 4u : 0u);
#define ALL16(stmt) do { _Pragma("unroll") for (int t = 0; t < 16; ++t) { stmt; } SCHED; } while (0)
                ALL16(bo[t] = bits; bits = __builtin_amdgcn_udot4(m[t], 0x08080808u, bits, false));
                u32 ad[16], x1[16];
                ALL16(ad[t] = bo[t] >> 3);
                ALL16(ad[t] &= kfc);
                ALL16(w[t] = ((u32x2){((const LAS u32*)(size_t)ad[t])[0], ((const LAS u32*)(size_t)ad[t])[1]}));
                ALL16(asm("v_mul_u32_u24 %0, %1, %2" : "=v"(m[t]) : "s"(k111), "v"(m[t])); x1[t] = perm(0u, 0x00101010u, c[t]));
                ALL16(x1[t] += m[t] << 8; c[t] = perm(0u, 0x0f040302u, c[t]));
                WAIT_L(0);
                if (lane == 0) ctl[st * 16 + sn] = e + 1;
                SCHED;
                auto fin = [&](auto dc) __attribute__((always_inline)) {
                    constexpr int d = decltype(dc)::v;
#pragma unroll
                    for (int t = 4 * d; t < 4 * d + 4; ++t) {
                        const u32 E = T4 - (perm(0u, __builtin_amdgcn_alignbit(w[t][1], w[t][0], bo[t]), x1[t]) & c[t]);
                        const u32 ra = perm(E, S[d][t & 3], 0x05010400u), rb = perm(E, S[d][t & 3], 0x07030602u);
                        O[2 * d][t & 3] = __builtin_amdgcn_alignbit(ra, ra, 1); O[2 * d + 1][t & 3] = __builtin_amdgcn_alignbit(rb, rb, 1);
                    }
                    emit(dc);
                    SCHED;
                };
                fin(IC<0>{}); fin(IC<1>{}); fin(IC<2>{}); fin(IC<3>{});
            }
            };
            char* flag = nullptr;
            if (!(ent & 0x8000u)) {
                const u32 blk = (8 + 7 * (id >> 7) + w) * 32 + ((id >> 2) & 31);
                char* dst = ws + OFF_G + (size_t)blk * 32768 + (id & 3) * 8192;
                DT(0, 0);
                unpack([&](auto dc) __attribute__((always_inline)) { constexpr int d = decltype(dc)::v;
                    st16_sys<2048 * d - 4096>(dst + 4096, voff, O[2 * d]); st16_sys<2048 * d - 3072>(dst + 4096, voff, O[2 * d + 1]); });
                DT(0, 1);
                if (cnt && lane >= 1 && (u32)lane <= cnt)
                    asm volatile("global_store_short %0, %1, %2 sc0 sc1" :: "v"(eq * 1024u + el * 16u + eo), "v"(rec), "s"(dst) : "memory");
                flag = ws + OFF_FG + blk * 128 + (id & 3) * 4;
                DT(0, 3);
            } else {
                u32x4 P[8];
                const u32 so = (w * 512 + id) * 8192;
#pragma unroll
                for (int q = 0; q < 8; ++q) P[q] = __builtin_amdgcn_raw_buffer_load_b128(prs, voff + (q & 3) * 1024, so + (q >> 2) * 4096, 17);
                u32 pe = __builtin_amdgcn_raw_buffer_load_b32(prs, eq * 1024u + el * 16u + (eo & 12u), so, 17);
                asm volatile("" ::: "memory");
                const u32 src = (p.rank + 1 + w) & 7;
                const u32 row0 = ((id >> 4) & 1) * 256 + (id & 1) * 128 + ((id >> 1) & 3) * 32, col0 = src * 4096 + ((id >> 5) & 15) * 256 + ((id >> 3) & 1) * 128;
                char* dst = p.out + (size_t)row0 * 65536 + col0 * 2;
                const u32 vo = (u32)(lane >> 4) * 262144u + (u32)(lane & 15) * 16u;
                DT(1, 0);
                unpack([](auto) {});
                WAIT_P;
                asm volatile("" : "+v"(pe));
                const u32 ye = cvt_pk_bf16(__uint_as_float(rec << 16) + __uint_as_float((eb & 1u) ? pe & 0xffff0000u : pe << 16), 0.f);
#pragma unroll
                for (int q = 0; q < 8; ++q) {
#pragma unroll
                    for (int i = 0; i < 4; ++i) {
                        const f32x2 y = (f32x2){__uint_as_float(O[q][i] << 16), __uint_as_float(O[q][i] & khi)} + (f32x2){__uint_as_float(P[q][i] << 16), __uint_as_float(P[q][i] & khi)};
                        O[q][i] = cvt_pk_bf16(y[0], y[1]);
                    }
                    st16_sys<0>(dst + (size_t)((q >> 2) * 16 + (q & 3)) * 65536, vo, O[q]);
                    SCHED;
                }
                DT(1, 1);
                if (cnt && lane >= 1 && (u32)lane <= cnt)
                    asm volatile("global_store_short %0, %1, %2 sc0 sc1" :: "v"(((eq >> 2) * 16u + (eq & 3u) + (el >> 4) * 4u) * 65536u + (el & 15u) * 16u + eo), "v"(ye), "s"(dst) : "memory");
                DT(1, 3);
            }
            if (pend) { WAIT_V(8); if (lane == 0) st_agent(pend, seq); }
            pend = flag;
        }
        if (pend) { WAIT_V(0); if (lane == 0) st_agent(pend, seq); }
        STAMP(62);
    }
}

__device__ __forceinline__ void spin_flags(char* ws, const char* fbase, u32 fvoff, u32 seq) {
    const u64 t0 = __builtin_amdgcn_s_memrealtime();
    while (__builtin_amdgcn_ballot_w64(ld_agent(fbase + fvoff) < seq) != 0ull) {
        if (ld_agent(ws + OFF_TMO) == seq || __builtin_amdgcn_s_memrealtime() - t0 > DEADLINE) { st_agent(ws + OFF_TMO, seq); break; }
        __builtin_amdgcn_s_sleep(4);
    }
    __builtin_amdgcn_s_waitcnt(0xC07F);
    WAIT_V(0);
}

__device__ __forceinline__ void gemm_loop(char* ws, LAS unsigned char* lds, const char* gX, const char* gW, u32 fvoff, int otab, int nstep, u32 seq, int wid, int lane, f32x4 (&acc)[2][2][4][2]) {
    constexpr size_t KSTEP = MiB, SX = 16384, SW = 16384;
    constexpr int FSTEP = 4096;
    const int wr = wid >> 2, wc = wid & 3, fr = lane & 15, fq = lane >> 4;
    const u32 voff = (u32)lane * 16u;
    const char* cX = gX + wid * 1024;
    const char* cW = gW + (wid >> 2) * 8192 + (wid & 3) * 1024;
    const char* fb = ws + OFF_FG;
    const u32 ldsw = (u32)wid * 1024u;
    const int woff = wr * 8192 + lane * 16;
    const int xoff0 = (wc * 32 + fr) * 128 + ((fq ^ (fr >> 1)) << 4), xoff1 = xoff0 ^ 64;
#define SA(b, h) (((b) * 2 + (h)) * 16384)
#define SB(b, h) ((4 + (b) * 2 + (h)) * 16384)
#define GLDS(src, dst) __builtin_amdgcn_global_load_lds((const GAS u32*)((src) + voff), (LAS u32*)(lds + (dst) + ldsw), 16, 0, GAUX)
#define STW(b, h, ptr) do { GLDS((ptr) + (h) * 4096, SA(b, h)); GLDS((ptr) + (h) * 4096 + SW, SA(b, h) + 8192); } while (0)
#define STX(b, h, ptr) do { GLDS((ptr) + (h) * SX, SB(b, h)); GLDS((ptr) + (h) * SX + 8192, SB(b, h) + 8192); } while (0)
#define LDW(b, h) do { _Pragma("unroll") for (int m = 0; m < 4; ++m) _Pragma("unroll") for (int k = 0; k < 2; ++k) Wt[m][k] = *(const LAS bf16x8*)(lds + SA(b, h) + woff + k * 4096 + m * 1024); } while (0)
#define LDX(dst, b, h) do { _Pragma("unroll") for (int n = 0; n < 2; ++n) { dst[n][0] = *(const LAS bf16x8*)(lds + SB(b, h) + xoff0 + n * 2048); dst[n][1] = *(const LAS bf16x8*)(lds + SB(b, h) + xoff1 + n * 2048); } } while (0)
#define MMA(h, xh, Xt) do { __builtin_amdgcn_s_setprio(1); _Pragma("unroll") for (int m = 0; m < 4; ++m) _Pragma("unroll") for (int n = 0; n < 2; ++n) _Pragma("unroll") for (int k = 0; k < 2; ++k) \
        acc[h][xh][m][n] = __builtin_amdgcn_mfma_f32_16x16x32_bf16(Xt[n][k], Wt[m][k], acc[h][xh][m][n], 0, 0, 0); __builtin_amdgcn_s_setprio(0); } while (0)
#define FLAG_LOAD(step) asm volatile("global_load_dword %0, %1, %2 sc1" : "+v"(fl) : "v"(fvoff), "s"(fb + (size_t)(step) * FSTEP) : "memory")
#define FLAG_CHECK(step) do { if (wid == 0 && __builtin_amdgcn_ballot_w64(fl < seq) != 0ull) spin_flags(ws, fb + (size_t)(step) * FSTEP, fvoff, seq); } while (0)
#pragma unroll
    for (int a = 0; a < 2; ++a)
#pragma unroll
        for (int b = 0; b < 2; ++b)
#pragma unroll
            for (int m = 0; m < 4; ++m)
#pragma unroll
                for (int n = 0; n < 2; ++n) acc[a][b][m][n] = (f32x4){0.f, 0.f, 0.f, 0.f};
    bf16x8 Wt[4][2], X0[2][2], X1[2][2];
    u32 fl = 0;
#define KSTAMP(i)
    if (wid == 0) { spin_flags(ws, fb, fvoff, seq); spin_flags(ws, fb + FSTEP, fvoff, seq); }
    WAIT_V(0); WAIT_L(0); BAR; asm volatile("" ::: "memory");
#define WAIT_VN(n) asm volatile("s_waitcnt vmcnt(" #n ")" : "+v"(fl) :: "memory")
    FLAG_LOAD(2);
    STX(0, 0, cX); STX(0, 1, cX); STW(0, 0, cW); STW(0, 1, cW);
    STX(1, 0, cX + KSTEP); STX(1, 1, cX + KSTEP); STW(1, 0, cW + KSTEP);
    WAIT_VN(8); BAR;
    if (wr == 1) BAR;
    for (int t = 0; t < nstep; t += 2) {
        const int t1 = __builtin_amdgcn_readlane(otab, t + 1), t2 = __builtin_amdgcn_readlane(otab, t + 2);
        const int t3 = __builtin_amdgcn_readlane(otab, t + 3), t4 = __builtin_amdgcn_readlane(otab, t + 4);
        const char* w1 = cW + (size_t)t1 * KSTEP;
        const char* x2 = cX + (size_t)t2 * KSTEP; const char* w2 = cW + (size_t)t2 * KSTEP;
        const char* x3 = cX + (size_t)t3 * KSTEP; const char* w3 = cW + (size_t)t3 * KSTEP;
        KSTAMP(t >> 1);
        FLAG_CHECK(t2); FLAG_LOAD(t3);
        LDX(X0, 0, 0); LDW(0, 0); LDX(X1, 0, 1); STW(1, 1, w1);
        WAIT_VN(9); WAIT_L(0); SCHED; BAR; SCHED; MMA(0, 0, X0); MMA(0, 1, X1); SCHED; BAR; SCHED;
        LDW(0, 1); STX(0, 0, x2); STX(0, 1, x2); STW(0, 0, w2);
        WAIT_VN(8); WAIT_L(0); SCHED; BAR; SCHED; MMA(1, 0, X0); MMA(1, 1, X1); SCHED; BAR; SCHED;
        FLAG_CHECK(t3); FLAG_LOAD(t4);
        LDX(X0, 1, 0); LDW(1, 0); LDX(X1, 1, 1); STW(0, 1, w2);
        WAIT_VN(9); WAIT_L(0); SCHED; BAR; SCHED; MMA(0, 0, X0); MMA(0, 1, X1); SCHED; BAR; SCHED;
        LDW(1, 1); STX(1, 0, x3); STX(1, 1, x3); STW(1, 0, w3);
        WAIT_VN(8); WAIT_L(0); SCHED; BAR; SCHED; MMA(1, 0, X0); MMA(1, 1, X1); SCHED; BAR; SCHED;
    }
#undef WAIT_VN
    KSTAMP(28);
    asm volatile("s_waitcnt vmcnt(0)" : "+v"(fl) :: "memory");
    if (wr == 0) BAR;
    BAR;
#undef SA
#undef SB
#undef GLDS
#undef STW
#undef STX
#undef LDW
#undef LDX
#undef MMA
#undef FLAG_LOAD
#undef FLAG_CHECK
#undef KSTAMP
}

__device__ __forceinline__ void gemm_piece(char* ws, LAS unsigned char* lds, const char* gX, const char* gW, u32 fx, u32 fw, u32 n0, u32 seq, int wid, int lane, f32x4 (&acc)[2][2][4][2]) {
    const int wr = wid >> 2, wc = wid & 3, fr = lane & 15, fq = lane >> 4;
    const u32 voff = (u32)lane * 16u;
    const char* cX = gX + wc * 4096;
    const char* cW = gW + wc * 4096;
    const int xoff0 = (wc * 32 + fr) * 128 + ((fq ^ (fr >> 1)) << 4), xoff1 = xoff0 ^ 64;
    auto stage = [&](int n) {
        const char* x = cX + (size_t)((n + n0) & 63) * MiB + voff; const char* w = cW + (size_t)((n + n0) & 63) * MiB + voff;
        LAS unsigned char* d = lds + (n & 3) * 32768 + wc * 4096;
        __builtin_amdgcn_global_load_lds((const GAS u32*)x, (LAS u32*)d, 16, 0, GAUX); __builtin_amdgcn_global_load_lds((const GAS u32*)x, (LAS u32*)d, 16, 1024, GAUX);
        __builtin_amdgcn_global_load_lds((const GAS u32*)x, (LAS u32*)d, 16, 2048, GAUX); __builtin_amdgcn_global_load_lds((const GAS u32*)x, (LAS u32*)d, 16, 3072, GAUX);
        __builtin_amdgcn_global_load_lds((const GAS u32*)w, (LAS u32*)(d + 16384), 16, 0, GAUX); __builtin_amdgcn_global_load_lds((const GAS u32*)w, (LAS u32*)(d + 16384), 16, 1024, GAUX);
        __builtin_amdgcn_global_load_lds((const GAS u32*)w, (LAS u32*)(d + 16384), 16, 2048, GAUX); __builtin_amdgcn_global_load_lds((const GAS u32*)w, (LAS u32*)(d + 16384), 16, 3072, GAUX);
    };
#pragma unroll
    for (int a = 0; a < 2; ++a)
#pragma unroll
        for (int m = 0; m < 4; ++m)
#pragma unroll
            for (int n = 0; n < 2; ++n) acc[a][0][m][n] = (f32x4){0.f, 0.f, 0.f, 0.f};
    if (wid < 4) spin_flags(ws, ws + OFF_FG, (u32)lane * 4096u + ((wid & 2) ? fw : fx) + (wid & 1) * 4, seq);
    WAIT_V(0); WAIT_L(0); BAR; asm volatile("" ::: "memory");
    if (wr == 1) { stage(0); stage(2); WAIT_V(8); BAR; } else stage(1);
    BAR;
    for (int n = wr; n < 64; n += 2) {
        bf16x8 Wt[8][2], Xt[2][2];
        const LAS unsigned char* bx = lds + (n & 3) * 32768;
#pragma unroll
        for (int i = 0; i < 2; ++i) { Xt[i][0] = *(const LAS bf16x8*)(bx + xoff0 + i * 2048); Xt[i][1] = *(const LAS bf16x8*)(bx + xoff1 + i * 2048); }
#pragma unroll
        for (int k = 0; k < 2; ++k)
#pragma unroll
            for (int c = 0; c < 8; ++c) Wt[c][k] = *(const LAS bf16x8*)(bx + 16384 + voff + k * 8192 + c * 1024);
        stage(n + 3);
        WAIT_V(8); WAIT_L(0); SCHED; BAR; SCHED;
        __builtin_amdgcn_s_setprio(1);
#pragma unroll
        for (int k = 0; k < 2; ++k)
#pragma unroll
            for (int c = 0; c < 8; ++c)
#pragma unroll
                for (int i = 0; i < 2; ++i) acc[c >> 2][0][c & 3][i] = __builtin_amdgcn_mfma_f32_16x16x32_bf16(Xt[i][k], Wt[c][k], acc[c >> 2][0][c & 3][i], 0, 0, 0);
        __builtin_amdgcn_s_setprio(0);
        SCHED; BAR; asm volatile("" ::: "memory");
    }
    WAIT_V(0);
    if (wr == 0) BAR;
    BAR; asm volatile("" ::: "memory");
}

__device__ __forceinline__ void acc_to_raw(const f32x4 (&acc)[2][2][4][2], int xh, bool poison, u32x4 (&O)[8]) {
#pragma unroll
    for (int n = 0; n < 2; ++n)
#pragma unroll
        for (int j = 0; j < 4; ++j)
#pragma unroll
            for (int c = 0; c < 4; ++c) {
                const u32 v = cvt_pk_bf16(acc[c >> 1][xh][(c & 1) * 2][n][j], acc[c >> 1][xh][(c & 1) * 2 + 1][n][j]);
                O[n * 4 + j][c] = poison ? 0x7fc07fc0u : v;
            }
}

__global__ __launch_bounds__(512, 2) void fused_gemm_rs(Params p) {
    extern __shared__ __attribute__((aligned(16))) unsigned char shm[];
    LAS unsigned char* lds = (LAS unsigned char*)shm;
    const int lane = threadIdx.x & 63, wid = __builtin_amdgcn_readfirstlane(threadIdx.x >> 6), b = blockIdx.x;
    char* ws = p.ws[p.rank];
    const u32 cnt_v = ld_agent(ws + OFF_CNT + b * 64);
    const int xcd = b & 7, ix = b >> 3;
    u32 seq;
    int s0 = b * 2;
    if (ix < 4) {
        STAMP(63);
        seq = __builtin_amdgcn_readfirstlane(cnt_v) + 1u;
    } else {
        STAMP(0);
        const int li = ix - 4, cg = li / 7, pmi = (xcd & 1) * 7 + li % 7;
        const int pn = MIX == 0 ? (xcd >> 1) * 4 + cg : (cg < 2 ? (xcd & 6) + cg : (MIX == 1 ? (xcd & 6) + 6 + cg : 12 - (xcd & 6) + cg));
        const int gi = (15 - pn) * 14 + pmi;
        LAS unsigned char* sc = lds + LDS_GEMM + wid * ENC_BYTES;
        enc_init(sc, lane);
        {
            const int t1 = PASS1(wid * NGEMM + gi);
            const int cls = t1 < 64 ? 0 : (t1 < 128 || (t1 >= 512 && t1 < 576) || (t1 >= 768 && t1 < 768 + 252)) ? 1 : 2;
            for (int i = cls == 0 ? 0 : cls == 1 ? DELAY1 : DELAY2; i > 0; --i) __builtin_amdgcn_s_sleep(8);
            int task = t1;
            for (int pass = 0; task < 4608; ++pass) {
                pack_task(p, ws, task, cnt_v, sc, lane);
                STAMP(pass < 2 ? 1 + pass : 3);
                task = pass < 4 ? p.pk[pn][pass][0] + wid * p.pk[pn][pass][1] + pmi : 4608;
            }
        }
        seq = __builtin_amdgcn_readfirstlane(cnt_v) + 1u;
        const int dest = (p.rank + 1 + (pmi >> 1)) & 7, pm = dest * 2 + (pmi & 1);
        f32x4 acc[2][2][4][2];
        for (int nd = p.dq[pn][0];; --nd) {
            const int q = p.dq[pn][nd], dw = pmi >> 1;
            const int bx = nd ? p.rank * 2 + (pmi & 1) : pm, bw = nd ? (64 + (dw >> 1) * 8) * 32 + (dw & 1) * 16 + q : 16 + pn;
            const int l8 = lane - 8;
            gemm_loop(ws, lds, ws + OFF_G + (size_t)bx * 32768, ws + OFF_G + (size_t)bw * 32768, (u32)(((lane & 4) ? bw : bx) * 128 + (lane & 3) * 4),
                      (nd || lane < 8 || lane >= 56) ? (lane & 7) : 8 + l8 / 6 * 7 + (dw + 13 - l8 % 6) % 7, nd ? 8 : 56, seq, wid, lane, acc);
            if (!nd) break;
            const bool poison = __builtin_amdgcn_readfirstlane(ld_agent(ws + OFF_TMO)) == seq;
            const int wv = (q * 2 + (pmi & 1)) * 8 + wid;
#pragma unroll
            for (int xh = 0; xh < 2; ++xh) { u32x4 O[8]; acc_to_raw(acc, xh, poison, O); store_unit_linear(ws + OFF_P + (size_t)(dw * 512 + wv * 2 + xh) * 8192, (u32)lane * 16u, O); }
            WAIT_V(0);
            if (lane == 0) st_agent(ws + OFF_FP + (dw * 256 + wv) * 4, seq);
            STAMP(8);
        }
        STAMP(4);
        {
            const bool poison = __builtin_amdgcn_readfirstlane(ld_agent(ws + OFF_TMO)) == seq;
            const int id0 = ((pn * 2 + (pmi & 1)) * 8 + wid) * 2;
            u32 rec = 0;
            char* ry = p.ws[dest] + OFF_RY + (size_t)(((seq & 1) * 8 + p.rank) * 512 + id0) * 64 + lane * 8;
#pragma unroll
            for (int xh = 0; xh < 2; ++xh) {
                u32x4 O[8];
                acc_to_raw(acc, xh, poison, O);
                rec = encode_store(O, ws + OFF_PY + (size_t)((seq & 1) * 8 + dest) * (4 * MiB) + (size_t)(id0 + xh) * 8192, sc, lane,
                                   [&] { if (xh == 1) { WAIT_V(0); put_record(ry, seq, rec, lane, 8); } });
            }
            WAIT_V(0);
            put_record(ry + 64, seq, rec, lane, 8);
        }
        STAMP(5);
        s0 = pn < HC ? 63 + (pn * 14 + pmi) * 2 : -1;
        for (bool first = pn < HC + 8; s0 < 0; first = false) {
            __syncthreads();
            if (threadIdx.x == 0) {
                volatile LAS u32* t = (volatile LAS u32*)(lds + LDS_GEMM);
                t[0] = first ? (u32)((pn - HC) * 14 + pmi)
                             : 112u + __hip_atomic_fetch_add((GAS u32*)(ws + OFF_TICKET), 1u, __ATOMIC_RELAXED, __HIP_MEMORY_SCOPE_AGENT) - (seq - 1u) * (NGEMM - 14 * HC + 16);
                t[1] = (u32)__builtin_amdgcn_s_memrealtime() / SUBTAU;
            }
            __syncthreads();
            const u32 tk = __builtin_amdgcn_readfirstlane(*(volatile LAS u32*)(lds + LDS_GEMM)), n0 = __builtin_amdgcn_readfirstlane(*(volatile LAS u32*)(lds + LDS_GEMM + 4));
            if (tk >= 128u) break;
            const int rq = tk & 3, cq = tk >> 2, opm = p.rank * 2 + (rq >> 1), opn = cq >> 1;
            gemm_piece(ws, lds, ws + OFF_G + (size_t)opm * 32768 + (rq & 1) * 16384, ws + OFF_G + (size_t)(16 + opn) * 32768 + (cq & 1) * 16384,
                       (u32)(opm * 128 + (rq & 1) * 8), (u32)((16 + opn) * 128 + (cq & 1) * 8), n0, seq, wid, lane, acc);
            STAMP(6);
            LAS f32x4* red = (LAS f32x4*)(lds + (wid & 3) * 16384) + lane;
            if (wid >= 4) {
#pragma unroll
                for (int i = 0; i < 16; ++i) red[i * 64] = acc[i >> 3][0][(i >> 1) & 3][i & 1];
            }
            __syncthreads();
            if (wid < 4) {
#pragma unroll
                for (int i = 0; i < 16; ++i) acc[i >> 3][0][(i >> 1) & 3][i & 1] += red[i * 64];
                const bool poison = __builtin_amdgcn_readfirstlane(ld_agent(ws + OFF_TMO)) == seq;
                u32x4 O[8]; acc_to_raw(acc, 0, poison, O);
                char* dst = p.out + (size_t)(rq * 128 + wid * 32) * 65536 + (p.rank * 4096 + cq * 128) * 2;
                const u32 vo = (u32)(lane >> 4) * 262144u + (u32)(lane & 15) * 16u;
                sgpr_settle();
#pragma unroll
                for (int q = 0; q < 8; ++q) st16_sys<0>(dst + (size_t)((q >> 2) * 16 + (q & 3)) * 65536, vo, O[q]);
            }
            STAMP(7);
        }
    }
    if (s0 >= 0) comm_role(p, ws, lds, s0, seq, wid, lane);
    __syncthreads();
    if (threadIdx.x == 0) st_agent(ws + OFF_CNT + b * 64, seq);
}

extern "C" void kernel_launch(int rank, int nranks, ncclComm_t comm, hipStream_t stream,
                              void* const* d_in, const int* in_sizes, int n_in,
                              void* d_out, int out_size, const int* in_dtypes, int out_dtype,
                              void* const* d_out_all, void* const* d_ws_all, size_t ws_bytes) {
    if (nranks != NR || (unsigned)rank >= (unsigned)NR || n_in != 2 || ws_bytes < WS_NEED || in_sizes[0] != 4096 * 512 || in_sizes[1] != 512 * 32768 || out_size != 512 * 32768) {
        fprintf(stderr, "kernel_launch: unexpected shapes\n");
        return;
    }
    HIP_CHECK(hipFuncSetAttribute((const void*)fused_gemm_rs, hipFuncAttributeMaxDynamicSharedMemorySize, LDS_BYTES));
    Params p;
    memset(&p, 0, sizeof p);
    p.x = (const char*)d_in[0]; p.w = (const char*)d_in[1]; p.out = (char*)d_out;
    for (int i = 0; i < NR; ++i) p.ws[i] = (char*)d_ws_all[i];
    p.ws_tab = (char* const*)d_ws_all;
    p.rank = rank;
    int key[256];
    for (int i = 0; i < 256; ++i) {
        const int j = i >> 5, k = i & 31;
        key[i] = k >= 16 ? 112 * j + SKEW_NUM * (k - 16) - 1 : (k >= 14 ? 100000 + j : k >= 12 ? 200000 : 112 * j + XSPREAD * (k >> 1) - (j == 0 ? 1000 : 0));
        int q = i;
        while (q > 0 && key[p.order[q - 1]] > key[i]) { p.order[q] = p.order[q - 1]; --q; }
        p.order[q] = (unsigned char)i;
    }
    for (int i = 0, n = 0; i < 256; ++i) if (p.order[i] & 16) p.worder[n++] = (unsigned char)((p.order[i] >> 5) * 16 + (p.order[i] & 15));
    const char* np = NPASS; const char* ndg = NDIAG; const bool pmaj = true;
    if (pmaj) for (int i = 38; i < 128; ++i) for (int q = i; q > 37 && ((p.worder[q - 1] & 15) * 8 + (p.worder[q - 1] >> 4)) > ((p.worder[q] & 15) * 8 + (p.worder[q] >> 4)); --q) { const unsigned char t = p.worder[q]; p.worder[q] = p.worder[q - 1]; p.worder[q - 1] = t; }
    int base = 1792;
    for (int ps = 0; ps < 4; ++ps) {
        int n = 0;
        for (int c = 15; c >= 0; --c) if (np[c] - '0' > ps + 1) n += 14;
        for (int c = 15, o = 0; c >= 0; --c) { const bool in = np[c] - '0' > ps + 1; p.pk[c][ps][0] = in ? base + o : 4608; p.pk[c][ps][1] = n; if (in) o += 14; }
        base += n * 8;
    }
    int nq = 0;
    for (int r = 1; r <= 3; ++r) for (int c = 0; c < 16; ++c) { const int n = ndg[c] - '0'; p.dq[c][0] = n; if (n >= r) p.dq[c][n + 1 - r] = nq++; }
    if (base < 4608 || nq != 16) { fprintf(stderr, "kernel_launch: bad schedule\n"); return; }
    hipLaunchKernelGGL(fused_gemm_rs, dim3(256), dim3(512), LDS_BYTES, stream, p);
    HIP_CHECK(hipGetLastError());
}
```

```cpp
#include <hip/hip_runtime.h>
#include <rccl/rccl.h>
#include <cstdio>
#include <cstdint>
#include <cstring>

#define HIP_CHECK(expr) do { hipError_t _e = (expr); if (_e != hipSuccess) { \
    fprintf(stderr, "HIP error %s at %s:%d\n", hipGetErrorString(_e), __FILE__, __LINE__); \
    return; } } while (0)

typedef unsigned u32;
typedef unsigned long long u64;
typedef u32 u32x4 __attribute__((ext_vector_type(4)));
typedef u32 u32x2 __attribute__((ext_vector_type(2)));
typedef short bf16x8 __attribute__((ext_vector_type(8)));
typedef float f32x4 __attribute__((ext_vector_type(4)));
typedef float f32x2 __attribute__((ext_vector_type(2)));
#define LAS __attribute__((address_space(3)))
#define GAS __attribute__((address_space(1)))

#define FORCE_RAW 0
#define GAUX 17
#define PSPLIT 1
#define NSTAT 4
#define XSPREAD 12
#define MIX 1
#define PASS1(v) (v)
#define HC 1
#define NPASS "1223122234443334"
#define NDIAG "0000111111112222"
#define NPUSH 9
#define SKEW_NUM 45

constexpr int NR = 8;
constexpr size_t MiB = 1ull << 20;
constexpr size_t OFF_CNT = 0;
constexpr size_t OFF_TMO = 16384;
constexpr size_t OFF_TICKET = 16384 + 256;
constexpr size_t OFF_YO = 16384 + 512;
constexpr size_t OFF_YT = 16384 + 2048;
constexpr size_t OFF_YQ = 16384 + 4096;
constexpr size_t OFF_PD = 16384 + 6144;
constexpr size_t OFF_FG = 65536;
constexpr size_t OFF_RIN = 1 * MiB;
constexpr size_t OFF_RY = 2 * MiB;
constexpr size_t OFF_PX = 4 * MiB;
constexpr size_t OFF_PW = 12 * MiB;
constexpr size_t OFF_PY = 76 * MiB;
constexpr size_t OFF_G = 140 * MiB;
constexpr size_t OFF_P = 243 * MiB;
constexpr size_t OFF_DBG = 272 * MiB;
constexpr size_t WS_NEED = 275 * MiB;
constexpr u64 DEADLINE = 40000000ull;
constexpr int LDS_GEMM = 131072, LDS_BYTES = 163840;
constexpr int NCOMM = 32, NGEMM = 224;

#define TS_A 120
#define TS_0 60
#define TS_P 220
#define TS_B 60
#define TS_D 320
struct Params {
    const char* x; const char* w; char* out; char* wsme;
    char* const* ws_tab;
    int rank; int pad[5];
    char* ws[NR];
    unsigned char order[256];
    unsigned char worder[128];
    int pk[16][4][2];
    int dq[16][4];
};

#define STAMP(i)

#define WAIT_V(n) asm volatile("s_waitcnt vmcnt(" #n ")" ::: "memory")
#define WAIT_P __builtin_amdgcn_s_waitcnt(0x0F70)
#define WAIT_L(n) asm volatile("s_waitcnt lgkmcnt(" #n ")" ::: "memory")
#define BAR __builtin_amdgcn_s_barrier()
#define SCHED __builtin_amdgcn_sched_barrier(0)

template <int OFF> __device__ __forceinline__ void st16_sys(const char* base, u32 voff, u32x4 v) {
    asm volatile("s_nop 4\n\tglobal_store_dwordx4 %0, %1, %2 offset:%3 sc0 sc1\n\ts_nop 1" :: "v"(voff), "v"(v), "s"(base), "i"(OFF) : "memory");
}
__device__ __forceinline__ const char* uni(const char* p) {
    return (const char*)(((u64)(u32)__builtin_amdgcn_readfirstlane((u32)((u64)p >> 32)) << 32) | (u32)__builtin_amdgcn_readfirstlane((u32)(u64)p));
}
__device__ __forceinline__ u32 perm(u32 hi, u32 lo, u32 sel) { return __builtin_amdgcn_perm(hi, lo, sel); }
__device__ __forceinline__ u32 ld_agent(const char* p) { return __hip_atomic_load((GAS u32*)p, __ATOMIC_RELAXED, __HIP_MEMORY_SCOPE_AGENT); }
__device__ __forceinline__ void st_agent(char* p, u32 v) { __hip_atomic_store((GAS u32*)p, v, __ATOMIC_RELAXED, __HIP_MEMORY_SCOPE_AGENT); }
__device__ __forceinline__ u32 gslot(u32 o, u32 seq) { return o + ((seq & 1u) && o - 8u < 7u ? 88u : 0u); }
__device__ __forceinline__ u32 cvt_pk_bf16(float lo, float hi) { u32 r; asm("v_cvt_pk_bf16_f32 %0, %1, %2" : "=v"(r) : "v"(lo), "v"(hi)); return r; }

__device__ __forceinline__ void store_unit_linear(const char* dst, u32 voff, const u32x4 (&O)[8]) {
    const char* b = uni(dst + 4096);
    st16_sys<-4096>(b, voff, O[0]); st16_sys<-3072>(b, voff, O[1]); st16_sys<-2048>(b, voff, O[2]); st16_sys<-1024>(b, voff, O[3]);
    st16_sys<0>(b, voff, O[4]); st16_sys<1024>(b, voff, O[5]); st16_sys<2048>(b, voff, O[6]); st16_sys<3072>(b, voff, O[7]);
}

constexpr u32 H_RAW = 1u << 11, H_DEAD = 1u << 12;
constexpr int ENC_BYTES = 1152;
__device__ __forceinline__ u32 bfi(u32 m, u32 a, u32 b) { return (a & m) | (b & ~m); }
__device__ __forceinline__ u32 scan32(u32 v) {
    v += (u32)__builtin_amdgcn_update_dpp(0, (int)v, 0x111, 0xf, 0xf, false);
    v += (u32)__builtin_amdgcn_update_dpp(0, (int)v, 0x112, 0xf, 0xf, false);
    v += (u32)__builtin_amdgcn_update_dpp(0, (int)v, 0x114, 0xf, 0xf, false);
    v += (u32)__builtin_amdgcn_update_dpp(0, (int)v, 0x118, 0xf, 0xf, false);
    v += (u32)__builtin_amdgcn_update_dpp(0, (int)v, 0x142, 0xa, 0xf, false);
    return v;
}
__device__ __forceinline__ void enc_init(LAS unsigned char* sc, int lane) {
    u32 sel = 0x0c0c0c0cu;
#pragma unroll
    for (int b = 3; b >= 0; --b) if (lane & (1 << b)) sel = (sel << 8) | (u32)b;
    if (lane < 16) ((LAS u32*)(sc + 1088))[lane] = sel;
}
template <class F>
__device__ __forceinline__ u32 encode_store(const u32x4 (&O)[8], const char* slot, LAS unsigned char* sc, int lane, F before_stores) {
    u32 d[16], S[16];
    u32 sum = 0;
#pragma unroll
    for (int t = 0; t < 16; ++t) {
        const u32 a = O[2 * (t >> 2)][t & 3], b = O[2 * (t >> 2) + 1][t & 3];
        const u32 ra = __builtin_amdgcn_alignbit(a, a, 31), rb = __builtin_amdgcn_alignbit(b, b, 31);
        d[t] = perm(rb, ra, 0x07050301u); S[t] = perm(rb, ra, 0x06040200u);
    }
#pragma unroll
    for (int t = 0; t < 16; t += 4) sum = __builtin_amdgcn_sad_u8(d[t], 0u, sum);
    sum = scan32(sum);
    const u32 tot = (u32)__builtin_amdgcn_readlane((int)sum, 31) + (u32)__builtin_amdgcn_readlane((int)sum, 63);
    const u32 T = min(max((tot >> 10) + 4u, 15u), 255u), T4 = T * 0x01010101u;
    *(LAS u32x4*)(sc + lane * 16) = (u32x4){0u, 0u, 0u, 0u};
    if (lane == 0) *(LAS u32x4*)(sc + 1024) = (u32x4){0u, 0u, 0u, 0u};
    u32 acc = 0;
#pragma unroll
    for (int t = 0; t < 16; ++t) { d[t] = T4 - d[t]; acc |= d[t]; }
    u32 rec = 0, head = T;
    bool raw = FORCE_RAW;
    if (__builtin_amdgcn_ballot_w64((acc & 0xf0f0f0f0u) != 0u) != 0ull) {
        u32 n = 0, sH = 0, sL = 0, sM = 0, sT = 0;
#pragma unroll
        for (int t = 0; t < 16; ++t) {
            const u32 e = T4 - d[t];
            const u32 w = ((T4 | 0x80808080u) - (e & 0x7f7f7f7fu)) ^ ((T4 ^ ~e) & 0x80808080u), mis = w & 0xf0f0f0f0u;
            d[t] = w & 0x0f0f0f0fu;
            if (mis) { ++n; sL = O[2 * (t >> 2)][t & 3]; sH = O[2 * (t >> 2) + 1][t & 3]; sM = mis; sT = t; }
        }
        const u32 bsh = (n ? (u32)__builtin_ctz(sM) : 0u) & 24u;
        const bool multi = n > 1u || (sM & ~(0xffu << bsh)) != 0u;
        u64 mask = __builtin_amdgcn_ballot_w64(n != 0u);
        const u32 cnt = (u32)__builtin_popcountll(mask);
        if (cnt > 7u || __builtin_amdgcn_ballot_w64(multi) != 0ull) raw = true;
        else {
            const u32 ent = ((u32)lane << 22) | (sT << 18) | (bsh << 13) | ((((bsh & 16u) ? sH : sL) >> ((bsh & 8u) * 2u)) & 0xffffu);
            for (u32 s = 1; mask; ++s, mask &= mask - 1) {
                const u32 e = (u32)__builtin_amdgcn_readlane((int)ent, __builtin_ctzll(mask));
                if ((u32)(lane & 7) == s) rec = e;
            }
            head |= cnt << 8;
        }
    }
    u32x4 Q; u32 cs[16], bo[16], bits = 0;
#pragma unroll
    for (int t = 0; t < 16; ++t) {
        const u32 sel = d[t] + (d[t] & 0x08080808u);
        const u32 r = perm(0xffffffaau, 0x5500ffffu, sel), r2 = perm(0xffffff00u, 0x0000ffffu, sel);
        Q[t >> 2] = (t & 3) ? bfi(0x03030303u << (2 * (t & 3)), r, Q[t >> 2]) : r;
        cs[t] = ((const LAS u32*)(size_t)__builtin_amdgcn_sad_u8(r2 & 0x20100804u, 0u, (u32)(size_t)sc))[272];
        bo[t] = bits;
        bits += (u32)__builtin_popcount(r2);
    }
    const u32 inc = scan32(bits);
    const u32 nb = max((u32)__builtin_amdgcn_readlane((int)inc, 31), (u32)__builtin_amdgcn_readlane((int)inc, 63)) >> 3;
    if (nb > 1024u) raw = true;
    before_stores();
    const u32 voff = (u32)lane * 16u;
    if (raw) {
        store_unit_linear(slot, voff, O);
        head = H_RAW;
    } else {
        const char* b = uni(slot + 4096);
        st16_sys<-4096>(b, voff, (u32x4){S[0], S[1], S[2], S[3]}); st16_sys<-3072>(b, voff, (u32x4){S[4], S[5], S[6], S[7]});
        st16_sys<-2048>(b, voff, (u32x4){S[8], S[9], S[10], S[11]}); st16_sys<-1024>(b, voff, (u32x4){S[12], S[13], S[14], S[15]});
        st16_sys<0>(b, voff, Q);
        const u32 base = inc - bits + (lane & 32 ? 4u : 0u) + (u32)(size_t)sc * 8u;
#pragma unroll
        for (int t = 0; t < 16; ++t) {
            const u32 o = base + bo[t];
            const u64 x = (u64)perm(0u, d[t], cs[t]) << (o & 31u);
            LAS u32* q = (LAS u32*)(size_t)((o >> 3) & ~3u);
            __hip_atomic_fetch_or(q, (u32)x, __ATOMIC_RELAXED, __HIP_MEMORY_SCOPE_WAVEFRONT);
            __hip_atomic_fetch_or(q + 1, (u32)(x >> 32), __ATOMIC_RELAXED, __HIP_MEMORY_SCOPE_WAVEFRONT);
        }
        asm volatile("" ::: "memory");
        const u32 n16 = (nb + 15u) >> 4;
        const u32x4 lst = *(const LAS u32x4*)(sc + lane * 16);
        if ((u32)lane < n16) st16_sys<1024>(b, voff, lst);
        head |= n16 << 13;
    }
    if ((lane & 7) == 0) rec = head;
    return rec;
}
__device__ __forceinline__ void put_record(char* p, u32 seq, u32 w, int lane, int nl) {
    const u32 cnt = ((u32)__builtin_amdgcn_readfirstlane(w) >> 8) & 7u;
    if (lane < nl && (u32)(lane & 7) <= cnt) __hip_atomic_store((GAS u64*)p, ((u64)w << 32) | seq, __ATOMIC_RELAXED, __HIP_MEMORY_SCOPE_SYSTEM);
}

__device__ __forceinline__ int task_code(const Params& p, int task) {
    if (task < 512) return (task >> 6) << 10 | (task & 63);
    if (task < 768) return ((task - 512) >> 5) << 10 | (64 + (task & 31));
    if (task >= 1792 && task < 2048) return ((task - 1792) >> 5) << 10 | (96 + (task & 31));
    const int c = task - (task < 1792 ? 768 : 1024), i = c / 28, m = c % 28, bk = (((const u32*)p.worder)[i >> 2] >> ((i & 3) * 8)) & 255;
    return (bk >> 4) << 10 | (128 + (m >> 2) * 64 + (bk & 15) * 4 + (m & 3));
}
__device__ __forceinline__ void task_load(const Params& p, int code, LAS unsigned char* buf, int lane) {
    const int j = code >> 10, r = code & 1023, un = r & 3;
    const char* s0; const char* s1; u32 stride;
    if (r < 64) {
        const char* src = p.x + (size_t)((r >> 2) * 256 + un * 64) * 1024 + j * 128 + (lane >> 3) * 1024;
        const int sw = (lane & 7) ^ (lane >> 4);
        s0 = src + (sw << 4); s1 = src + ((sw ^ 4) << 4); stride = 8192;
    } else {
        s0 = s1 = p.w + (size_t)(j * 64 + (un & 1) * 32 + (lane >> 4) * 8) * 65536 + (((p.rank + ((r - 64) >> 6)) & 7) * 4096 + ((r >> 2) & 15) * 256 + (un >> 1) * 128) * 2 + (lane & 15) * 16;
        stride = 65536;
    }
#pragma unroll
    for (int q = 0; q < 8; ++q) __builtin_amdgcn_global_load_lds((const GAS u32*)(((q & 1) ? s1 : s0) + q * stride), (LAS u32*)(buf + q * 1024), 16, 0, 0);
}
__device__ __forceinline__ u32 task_time(const Params& p, int task) {
    return task < 64 ? TS_0 : task < 512 ? TS_A + (task >> 6) * TS_B : task < 768 ? TS_A + ((task - 512) >> 5) * TS_B + TS_B / 2 : TS_A + (u32)(task - 768) * TS_D / 1024u;
}
__device__ __forceinline__ void sleep_until(u32 t) {
    for (int n = (int)(t - (u32)__builtin_amdgcn_s_memrealtime()); n > 11; n -= 23) __builtin_amdgcn_s_sleep(8);
}
template <class Next>
__device__ __forceinline__ void pack_tasks(const Params& p, char* ws, int task, bool urgent, Next next, u32 cnt_v, u32 tl, LAS unsigned char* buf, LAS unsigned char* sc, int lane) {
    const u32 voff = (u32)lane * 16u;
#define UT(i)
    int code = task_code(p, task);
    sleep_until(tl + task_time(p, task));
    task_load(p, code, buf, lane);
    char* peer = p.ws_tab[(p.rank + 1 + (lane >> 3)) & 7];
    asm volatile("" : "+v"(peer), "+v"(cnt_v) :: "memory");
    const u32 seq = __builtin_amdgcn_readfirstlane(cnt_v) + 1u;
    WAIT_P; WAIT_V(0);
    char* rp = nullptr; u32 rec = 0; int nl = 0;
    for (int pass = 0;; ++pass) {
        const int j = code >> 10, r = code & 1023, un = r & 3, dd = (r - 64) >> 6, b16 = (r >> 2) & 15;
        const bool isx = r < 64, enc = isx ? (b16 >> 1) != p.rank : dd != 0;
        u32x4 O[8];
        {
            u32x4 in[8];
#pragma unroll
            for (int q = 0; q < 8; ++q) in[q] = *(const LAS u32x4*)(buf + q * 1024 + voff);
            WAIT_L(0);
            if (isx) {
#pragma unroll
                for (int q = 0; q < 8; ++q) O[q] = in[q];
            } else {
#pragma unroll
                for (int d = 0; d < 4; ++d)
#pragma unroll
                    for (int q = 0; q < 4; ++q) { O[2 * d][q] = perm(in[2 * q + 1][d], in[2 * q][d], 0x05040100u); O[2 * d + 1][q] = perm(in[2 * q + 1][d], in[2 * q][d], 0x07060302u); }
            }
        }
        UT(1);
        const int blk = isx ? j * 32 + b16 : dd ? (64 + ((dd - 1) >> 1) * 8 + j) * 32 + ((dd - 1) & 1) * 16 + b16 : j * 32 + 16 + b16;
        bool late = false;
        auto side = [&]() {
            asm volatile("" : "+s"(pass));
            task = next(pass);
            if (task < 4608) {
                code = task_code(p, task);
                late = (u32)__builtin_amdgcn_s_memrealtime() - tl < task_time(p, task);
                if (!late) task_load(p, code, buf, lane);
            }
            store_unit_linear(ws + OFF_G + (size_t)blk * 32768 + un * 8192, voff, O);
        };
        auto settle = [&]() {
            WAIT_V(0);
            if (nl) { put_record(rp, seq, rec, lane, nl);
                nl = 0; }
        };
        if (!urgent) side();
        if (enc) {
            const int dest = (p.rank + dd) & 7;
            const char* slot = ws + (isx ? OFF_PX + (size_t)(seq & 1) * (4 * MiB) : OFF_PW + (size_t)((seq & 1) * 8 + dest) * (4 * MiB)) + (size_t)((j * 16 + b16) * 4 + un) * 8192;
            rec = encode_store(O, slot, sc, lane, [&] { UT(2); WAIT_P; if (!urgent) { settle(); if (lane == 0) st_agent(ws + OFF_FG + blk * 128 + un * 4, seq); } });
            rp = (isx ? peer : p.ws[dest]) + OFF_RIN + (size_t)(((seq & 1) * 8 + p.rank) * 1024 + (j * 32 + (isx ? 0 : 16) + b16) * 4 + un) * 64 + (lane & 7) * 8;
            nl = isx ? 56 : 8;
        }
        if (urgent) { UT(3); settle(); UT(4); side(); }
        if (urgent || !enc) { settle(); if (lane == 0) st_agent(ws + OFF_FG + blk * 128 + un * 4, seq); }
        urgent = false;
        STAMP(pass < 2 ? 1 + pass : 3);
        if (task >= 4608) { settle(); break; }
        if (late) {
            settle();
            sleep_until(tl + task_time(p, task));
            task_load(p, code, buf, lane);
            WAIT_P; WAIT_V(0);
        }
    }
}

__device__ __forceinline__ void push_task(const Params& p, int t, u32 cnt_v, u32 tl, int lane) {
    const int dd = t % 7, un = (t / 7) & 3, i = t / 28, dest = (p.rank + 1 + dd) & 7, w = 6 - dd;
    const int pm = (dest * 2 + 2 + (i + 2 * w + 2) % 14) & 15;
    const char* src = p.x + (size_t)(pm * 256 + un * 64) * 1024 + (lane >> 3) * 1024;
    const int sw = (lane & 7) ^ (lane >> 4);
    u32x4 in[8];
#pragma unroll
    for (int q = 0; q < 8; ++q) in[q] = *(const GAS u32x4*)(src + q * 8192 + ((sw ^ ((q & 1) << 2)) << 4));
    asm volatile("" : "+v"(cnt_v) :: "memory");
    const u32 seq = __builtin_amdgcn_readfirstlane(cnt_v) + 1u, blk = gslot(8 + w, seq) * 32 + pm;
    char* pw = p.ws[dest];
    WAIT_P;
    for (int n = (int)(tl + 110 + (t / 7) * TS_P / 16 - (u32)__builtin_amdgcn_s_memrealtime()); n > 2; n -= 6) __builtin_amdgcn_s_sleep(2);
    store_unit_linear(pw + OFF_G + (size_t)blk * 32768 + un * 8192, (u32)lane * 16u, in);
    WAIT_V(0);
    if (lane == 0) __hip_atomic_store((GAS u32*)(pw + OFF_FG + blk * 128 + un * 4), seq, __ATOMIC_RELAXED, __HIP_MEMORY_SCOPE_SYSTEM);
}

#define WIN 2
#define PACE 985
#define YLEAD 2
#define PFP 0
#define G1 24
#define G2 30
#define TGRAN 4
#define SLACK 1
constexpr int NSLOT = 7, NDEC = 5, NDESC = 16, NIN = 960 - 4 * NPUSH, LOOK = 8;
constexpr u32 FIN = 0x40000000u;
constexpr int L_SLOT = 0;
constexpr int L_DESC = 2 * NSLOT * 8192;
constexpr int L_CTL = L_DESC + 2 * NDESC * 64;
constexpr int L_ORD = L_CTL + 128;
static_assert(L_ORD + 512 <= LDS_BYTES, "lds");

template <int V> struct IC { static constexpr int v = V; };
__device__ __forceinline__ void wait_units(u32 n) {
    switch (n) {
        case 0: WAIT_V(0); break; case 1: WAIT_V(6); break; case 2: WAIT_V(12); break; case 3: WAIT_V(18); break; case 4: WAIT_V(24); break;
        case 5: WAIT_V(30); break; case 6: WAIT_V(36); break; default: WAIT_V(42); break;
    }
}
__device__ __forceinline__ void comm_role(const Params& p, char* ws, LAS unsigned char* lds, u32 s0, u32 seq, int wid, int lane, u32 tstart) {
    constexpr u32 NY = 9 + 4 * HC;
    if (threadIdx.x < 64) ((LAS u32*)(lds + L_ORD))[threadIdx.x] = ((const u32*)p.order)[threadIdx.x];
    if (threadIdx.x >= 64 && threadIdx.x < 96) ((LAS u32*)(lds + L_CTL))[threadIdx.x - 64] = 0;
    __builtin_amdgcn_s_waitcnt(0x0F70);
    __syncthreads();
    volatile LAS u32* ctl = (volatile LAS u32*)(lds + L_CTL);
    volatile LAS u32* desc = (volatile LAS u32*)(lds + L_DESC);
    const u32 voff = (u32)lane * 16u;
    if (wid == 7) {
        const u32 st = lane >> 5, e = lane & 7, L = s0 + st, src = (p.rank + 1 + L % 7) & 7, j = L / 7;
        const u32 cin = j < 9 ? (NIN - j + 8) / 9 : 0;
        bool open = s0 < 63, dead = false, pd = false;
        const char* rws = p.ws_tab[src];
        const u64 t0 = __builtin_amdgcn_s_memrealtime();
        volatile LAS u32* tk = (volatile LAS u32*)(lds + L_ORD + 256) + st * NDESC;
        GAS u32* yt = (GAS u32*)(ws + OFF_YT + (L % 7) * 128);
        if ((lane & 31) == 0) __hip_atomic_fetch_max(yt, seq << 10, __ATOMIC_RELAXED, __HIP_MEMORY_SCOPE_AGENT);
        if ((lane & 31) < LOOK) tk[(cin + (lane & 31)) % NDESC] = j + NY * (lane & 31);
        u32 x = (lane >> 3) & 3, drawn = cin + LOOK, cnt = (s0 == 62 && st) ? 0u : FIN, idle = 0;
        while (__builtin_amdgcn_ballot_w64(x < cnt) != 0ull) {
            if (!open) open = dead || ld_agent(ws + OFF_YO + (L % 7) * 128) - (seq - 1u) * 9u >= (9u * (j - 8u) + NY - 10u) / (NY - 9u);
            const u32 lim = ctl[st * 16 + 9] + LOOK, nd = (open && cnt == FIN && lim > drawn) ? lim - drawn : 0u;
            u32 tb = 0;
            if (drawn == cin + LOOK && __builtin_amdgcn_ballot_w64(nd != 0u) != 0ull) WAIT_V(0);
            if (nd && (lane & 31) == 0) tb = __hip_atomic_fetch_add(yt, nd, __ATOMIC_RELAXED, __HIP_MEMORY_SCOPE_AGENT);
            const bool act = open && x < cnt && x < min(lim, drawn);
            const u32 u = x < cin ? j + 9 * x : NIN + (tk[x % NDESC] & 511u);
            u32 id;
            if (u < NIN) {
                u32 bk = (lds + L_ORD)[u >> 2];
                if (!(bk & 16)) { const u32 i = bk & 15, pmi = i < 14 ? (i + 2 * (L % 7) + 2) % 14 : i; bk = (bk & ~15u) | ((p.rank * 2 + 2 + pmi) & 15); }
                id = bk * 4u + (u & 3);
            } else id = u - NIN;
            const char* a = u >= NIN ? ws + OFF_RY + (size_t)(((seq & 1) * 8 + src) * 512 + id) * 64 : ws + OFF_RIN + (size_t)(((seq & 1) * 8 + src) * 1024 + id) * 64;
            u64 v = 0;
            if (!pd && __builtin_amdgcn_ballot_w64(act && u >= NIN) != 0ull) pd = ld_agent(ws + OFF_PD) >= seq * (NGEMM * 8u);
            if (act) v = __hip_atomic_load((GAS u64*)(a + e * 8), __ATOMIC_RELAXED, __HIP_MEMORY_SCOPE_SYSTEM);
            const u32 hw = (u32)__builtin_amdgcn_ds_swizzle((int)(u32)(v >> 32), 0x0018);
            const u64 okm = __builtin_amdgcn_ballot_w64(((u32)v == seq || e > ((hw >> 8) & 7u)) && (u < NIN || pd));
            const bool adv = act && (dead || ((okm >> (lane & 56)) & 0xffull) == 0xffull);
            if (__builtin_amdgcn_ballot_w64(nd != 0u) != 0ull) {
                tb = (u32)__builtin_amdgcn_ds_swizzle((int)tb, 0) - (seq << 10) + LOOK * NY;
                if ((u32)(lane & 31) < nd) tk[(drawn + (lane & 31)) % NDESC] = tb + (lane & 31);
                if (nd && tb + nd >= 512u) cnt = drawn + (tb < 512u ? 512u - tb : 0u);
                drawn += nd;
            }
            if (__builtin_amdgcn_ballot_w64(adv) != 0ull) {
                if (adv) {
                    if (u >= NIN && !dead) id = (hw >> 20) & 511u;
                    const size_t po = (size_t)((seq & 1) * 8 + p.rank) * (4 * MiB);
                    const char* s = u >= NIN ? rws + OFF_PY + po + (size_t)id * 8192
                        : rws + ((id & 64) ? OFF_PW + po : OFF_PX + (size_t)(seq & 1) * (4 * MiB)) + (size_t)(((id >> 7) * 16 + ((id >> 2) & 15)) * 4 + (id & 3)) * 8192;
                    const u32 en = (st * NDESC + x % NDESC) * 16, word = dead ? H_DEAD : (u32)(v >> 32);
                    const u64 sa = dead ? (u64)(ws + OFF_G) : (u64)s;
                    desc[en + e] = word;
                    if (e == 0) { desc[en + 8] = (u32)sa; desc[en + 9] = (u32)(sa >> 32); desc[en + 10] = word; desc[en + 11] = (u >= NIN ? 0x8000u : 0u) | id; }
                    x += 4;
                }
                u32 m = min(x, (u32)__builtin_amdgcn_update_dpp((int)x, (int)x, 0x128, 0xf, 0xf, false));
                m = min(m, (u32)__builtin_amdgcn_ds_swizzle((int)m, 0x401F));
                WAIT_L(0);
                if ((lane & 31) == 0) ctl[st * 16 + 8] = min(m, cnt);
            } else if ((++idle & 15u) == 0u && (ld_agent(ws + OFF_TMO) == seq || __builtin_amdgcn_s_memrealtime() - t0 > DEADLINE)) {
                st_agent(ws + OFF_TMO, seq); WAIT_V(0); dead = true;
            } else __builtin_amdgcn_s_sleep(1);
        }
        if ((lane & 31) == 0) ctl[st * 16 + 8] = cnt | FIN;
    } else if (wid >= NDEC) {
        const u32 me = wid - NDEC;
        volatile LAS u32* c = ctl + me * 16;
        u32 landed = 0, k = 0, slot = 0, pc = PACE;
        const u32 t0 = (u32)__builtin_amdgcn_s_memrealtime();
        u32 next = 0, now;
        auto clock = [&]() { return ((u32)__builtin_amdgcn_s_memrealtime() - t0) * 64u; };
        auto pace = [&](u32 n64) {
            while ((int)(now - next) < 0) now = clock();
            next += (n64 * pc) >> 4;
        };
        __builtin_amdgcn_s_setprio(3);
        for (;; ++k) {
            volatile LAS u32* rp = (lane & 16) ? desc + (me * NDESC + k % NDESC) * 16 + (lane & 15) : c + (lane & 15);
            u32 v;
            bool fin = false;
            while (true) {
                v = *rp;
                now = clock();
                const u32 pub = __builtin_amdgcn_readlane(v, 8);
                if ((pub & ~FIN) > k && (u32)__builtin_amdgcn_readlane(v, slot) + NSLOT > k) break;
                if (landed < k) {
                    wait_units(k - landed - 1);
                    ++landed;
                    if (lane == 0) c[10] = landed;
                } else if (pub == (k | FIN)) { fin = true; break; }
            }
            if (fin) break;
            const char* s = (const char*)(((u64)(u32)__builtin_amdgcn_readlane(v, 25) << 32) | (u32)__builtin_amdgcn_readlane(v, 24));
            const u32 head = __builtin_amdgcn_readlane(v, 26);
            if (__builtin_amdgcn_readlane(v, 27) & 0x8000) pc = PACE * NY / 9;
            if (s0 < 63 && k + YLEAD == (NIN - (s0 + me) / 7 + 8) / 9 && lane == 0) __hip_atomic_fetch_add((GAS u32*)(ws + OFF_YO + ((s0 + me) % 7) * 128), 1u, __ATOMIC_RELAXED, __HIP_MEMORY_SCOPE_AGENT);
            LAS unsigned char* d = lds + L_SLOT + (me * NSLOT + slot) * 8192;
            if ((int)(now - next) > (int)(SLACK * PACE / 4)) next = now - SLACK * PACE / 4;
            u32 n16 = (head >> 13) & 127u;
            if (head & (H_RAW | H_DEAD)) {
                pace(32);
                __builtin_amdgcn_global_load_lds((const GAS u32*)(s + 6144 + voff), (LAS u32*)(d + 6144), 16, 0, 17);
                __builtin_amdgcn_global_load_lds((const GAS u32*)(s + 6144 + voff), (LAS u32*)(d + 6144), 16, 1024, 17);
                n16 = 64; now = clock();
            }
            const u32 tail = ((n16 + TGRAN - 1) / TGRAN) * (TGRAN / 4);
            pace(G1);
            if (k < 16 && s0 < 63) STAMP(k);
            if ((u32)lane < max(n16, 1u)) __builtin_amdgcn_global_load_lds((const GAS u32*)(s + 4096 + voff), (LAS u32*)(d + 4096), 16, 1024, 17);
            __builtin_amdgcn_global_load_lds((const GAS u32*)(s + 4096 + voff), (LAS u32*)(d + 4096), 16, 0, 17);
            now = clock(); pace(G2);
            __builtin_amdgcn_global_load_lds((const GAS u32*)(s + voff), (LAS u32*)d, 16, 0, 17);
            __builtin_amdgcn_global_load_lds((const GAS u32*)(s + voff), (LAS u32*)d, 16, 1024, 17);
            now = clock(); pace(80 + tail - G1 - G2);
            __builtin_amdgcn_global_load_lds((const GAS u32*)(s + voff), (LAS u32*)d, 16, 2048, 17);
            __builtin_amdgcn_global_load_lds((const GAS u32*)(s + voff), (LAS u32*)d, 16, 3072, 17);
            asm volatile("s_waitcnt vmcnt(%0)" :: "i"(6 * WIN) : "memory");
            if (k + 1 > landed + WIN) landed = k + 1 - WIN;
            if (lane == 0) { c[9] = k; c[10] = landed; }
            if (++slot == NSLOT) slot = 0;
        }
        if (lane == 0) c[10] = k | FIN;
    } else {
        char* pend = nullptr;
        int k = 0;
        u32 st = wid & 1;
#define DT(y, i)
        u32 k03 = 0x03030303u, kfc = ~3u, khi = 0xffff0000u;
        asm volatile("" : "+s"(k03), "+s"(kfc), "+s"(khi));
        const __amdgpu_buffer_rsrc_t prs = __builtin_amdgcn_make_buffer_rsrc(ws + OFF_P, 0, 28 * (int)MiB, 0x00020000);
        for (;;) {
            u32 e;
            {
                bool got = false;
                for (u32 idle = 0;;) {
                    st ^= 1;
                    const u32x2 lc = *(volatile LAS u32x2*)(lds + L_CTL + st * 64 + 40);
                    const u32 ld = __builtin_amdgcn_readfirstlane(lc[0]);
                    e = __builtin_amdgcn_readfirstlane(lc[1]);
                    if (e < (ld & ~FIN)) {
                        u32 o = e;
                        if (lane == 0) __hip_atomic_compare_exchange_strong((LAS u32*)(lds + L_CTL + st * 64 + 44), &o, e + 1, __ATOMIC_RELAXED, __ATOMIC_RELAXED, __HIP_MEMORY_SCOPE_WORKGROUP);
                        if (__builtin_amdgcn_readfirstlane(o) == e) { got = true; break; }
                        idle = 0;
                        continue;
                    }
                    if (ld & FIN) { if (++idle == 2) break; } else idle = 0;
                    if (pend) { WAIT_V(0); if (lane == 0) st_agent(pend, seq); pend = nullptr; }
                    else __builtin_amdgcn_s_sleep(1);
                }
                if (!got) break;
            }
            asm volatile("" ::: "memory");
            DT(0, -1);
            if ((k & 1) && s0 < 63) STAMP(k < 124 ? k >> 1 : 62);
            if (k == 1 && s0 >= 63) STAMP(61);
            const u32 en = (st * NDESC + e % NDESC) * 16, sn = e % NSLOT;
            const u32 ent = __builtin_amdgcn_readfirstlane(desc[en + 11]), id = ent & 0x3ffu, w = (s0 + st) % 7;
            const u32 rec = desc[en + (lane & 7)], head = __builtin_amdgcn_readfirstlane(rec);
            const LAS u32x4* sl = (const LAS u32x4*)(lds + L_SLOT + (st * NSLOT + sn) * 8192 + voff);
            u32x4 O[8];
            const u32 cnt = (head >> 8) & 7u;
            const u32 el = (rec >> 22) & 63u, et = (rec >> 18) & 15u, eb = (rec >> 16) & 3u, eq = (et >> 2) * 2u + (eb >> 1), eo = (et & 3u) * 4u + (eb & 1u) * 2u;
            ++k;
            auto unpack = [&](auto emit) __attribute__((always_inline)) {
            if (head & (H_RAW | H_DEAD)) {
#pragma unroll
                for (int q = 0; q < 8; ++q) O[q] = sl[q * 64];
                WAIT_L(0);
                if (lane == 0) ctl[st * 16 + sn] = e + 1;
                if (head & H_DEAD) {
#pragma unroll
                    for (int q = 0; q < 8; ++q) O[q] = (u32x4){0x7fc07fc0u, 0x7fc07fc0u, 0x7fc07fc0u, 0x7fc07fc0u};
                }
                emit(IC<0>{}); emit(IC<1>{}); emit(IC<2>{}); emit(IC<3>{});
            } else {
                u32 c[16], m[16], bo[16];
                u32x2 w[16];
                u32x4 S[4];
                const u32 T4 = (head & 0xffu) * 0x01010101u;
                const u32x4 Q = sl[4 * 64];
#pragma unroll
                for (int q = 0; q < 4; ++q) S[q] = sl[q * 64];
                u32 n = 0;
#pragma unroll
                for (int q = 0; q < 4; ++q) n += (u32)__builtin_popcount(Q[q] & (Q[q] >> 1) & 0x55555555u);
#pragma unroll
                for (int t = 0; t < 16; ++t) { c[t] = (Q[t >> 2] >> (2 * (t & 3))) & k03; m[t] = perm(0u, 0x01000000u, c[t]); }
                u32 bits = (min(scan32(n) - n, 1024u) + (u32)(L_SLOT + 5120) + (st * NSLOT + sn) * 8192u + (u32)(size_t)lds) * 8u + (lane & 32 ? 4u : 0u);
#define ALL16(stmt) do { _Pragma("unroll") for (int t = 0; t < 16; ++t) { stmt; } SCHED; } while (0)
                ALL16(bo[t] = bits; bits = __builtin_amdgcn_udot4(m[t], 0x08080808u, bits, false));
                u32 ad[16], x1[16];
                ALL16(ad[t] = bo[t] >> 3);
                ALL16(ad[t] &= kfc);
                ALL16(w[t] = ((u32x2){((const LAS u32*)(size_t)ad[t])[0], ((const LAS u32*)(size_t)ad[t])[1]}));
                ALL16(asm("v_mul_u32_u24 %0, 0x10101, %1" : "=v"(m[t]) : "v"(m[t])); x1[t] = perm(0u, 0x00101010u, c[t]));
                ALL16(x1[t] += m[t] << 8; c[t] = perm(0u, 0x0f040302u, c[t]));
                WAIT_L(0);
                if (lane == 0) ctl[st * 16 + sn] = e + 1;
                SCHED;
                auto fin = [&](auto dc) __attribute__((always_inline)) {
                    constexpr int d = decltype(dc)::v;
#pragma unroll
                    for (int t = 4 * d; t < 4 * d + 4; ++t) {
                        const u32 E = T4 - (perm(0u, __builtin_amdgcn_alignbit(w[t][1], w[t][0], bo[t]), x1[t]) & c[t]);
                        const u32 ra = perm(E, S[d][t & 3], 0x05010400u), rb = perm(E, S[d][t & 3], 0x07030602u);
                        O[2 * d][t & 3] = __builtin_amdgcn_alignbit(ra, ra, 1); O[2 * d + 1][t & 3] = __builtin_amdgcn_alignbit(rb, rb, 1);
                    }
                    emit(dc);
                    SCHED;
                };
                fin(IC<0>{}); fin(IC<1>{}); fin(IC<2>{}); fin(IC<3>{});
            }
            };
            char* flag = nullptr;
            if (!(ent & 0x8000u)) {
                const u32 blk = gslot(8 + 7 * (id >> 7) + w, seq) * 32 + ((id >> 2) & 31);
                char* dst = ws + OFF_G + (size_t)blk * 32768 + (id & 3) * 8192;
                DT(0, 0);
                unpack([&](auto dc) __attribute__((always_inline)) { constexpr int d = decltype(dc)::v;
                    st16_sys<2048 * d - 4096>(dst + 4096, voff, O[2 * d]); st16_sys<2048 * d - 3072>(dst + 4096, voff, O[2 * d + 1]); });
                DT(0, 1);
                if (cnt && lane >= 1 && (u32)lane <= cnt)
                    asm volatile("s_nop 4\n\tglobal_store_short %0, %1, %2 sc0 sc1" :: "v"(eq * 1024u + el * 16u + eo), "v"(rec), "s"(dst) : "memory");
                flag = ws + OFF_FG + blk * 128 + (id & 3) * 4;
                DT(0, 3);
            } else {
                u32x4 P[8];
                const u32 so = (w * 512 + id) * 8192;
#pragma unroll
                for (int q = 0; q < 8; ++q) P[q] = __builtin_amdgcn_raw_buffer_load_b128(prs, voff + (q & 3) * 1024, so + (q >> 2) * 4096, 17);
                u32 pe = __builtin_amdgcn_raw_buffer_load_b32(prs, eq * 1024u + el * 16u + (eo & 12u), so, 17);
                asm volatile("" ::: "memory");
                const u32 src = (p.rank + 1 + w) & 7;
                const u32 row0 = ((id >> 4) & 1) * 256 + (id & 1) * 128 + ((id >> 1) & 3) * 32, col0 = src * 4096 + ((id >> 5) & 15) * 256 + ((id >> 3) & 1) * 128;
                char* dst = p.out + (size_t)row0 * 65536 + col0 * 2;
                const u32 vo = (u32)(lane >> 4) * 262144u + (u32)(lane & 15) * 16u;
                DT(1, 0);
                unpack([](auto) {});
                WAIT_P;
                asm volatile("" : "+v"(pe));
                const u32 ye = cvt_pk_bf16(__uint_as_float(rec << 16) + __uint_as_float((eb & 1u) ? pe & 0xffff0000u : pe << 16), 0.f);
#pragma unroll
                for (int q = 0; q < 8; ++q) {
#pragma unroll
                    for (int i = 0; i < 4; ++i) {
                        const f32x2 y = (f32x2){__uint_as_float(O[q][i] << 16), __uint_as_float(O[q][i] & khi)} + (f32x2){__uint_as_float(P[q][i] << 16), __uint_as_float(P[q][i] & khi)};
                        O[q][i] = cvt_pk_bf16(y[0], y[1]);
                    }
                    st16_sys<0>(dst + (size_t)((q >> 2) * 16 + (q & 3)) * 65536, vo, O[q]);
                    SCHED;
                }
                DT(1, 1);
                if (cnt && lane >= 1 && (u32)lane <= cnt)
                    asm volatile("s_nop 4\n\tglobal_store_short %0, %1, %2 sc0 sc1" :: "v"(((eq >> 2) * 16u + (eq & 3u) + (el >> 4) * 4u) * 65536u + (el & 15u) * 16u + eo), "v"(ye), "s"(dst) : "memory");
                DT(1, 3);
            }
            if (pend) { WAIT_V(8); if (lane == 0) st_agent(pend, seq); }
            pend = flag;
        }
        if (pend) { WAIT_V(0); if (lane == 0) st_agent(pend, seq); }
        STAMP(62);
    }
}

__device__ __forceinline__ void spin_flags(char* ws, const char* fbase, u32 fvoff, u32 seq) {
    const u64 t0 = __builtin_amdgcn_s_memrealtime();
    while (__builtin_amdgcn_ballot_w64(__hip_atomic_load((GAS u32*)(fbase + fvoff), __ATOMIC_RELAXED, __HIP_MEMORY_SCOPE_SYSTEM) < seq) != 0ull) {
        if (ld_agent(ws + OFF_TMO) == seq || __builtin_amdgcn_s_memrealtime() - t0 > DEADLINE) { st_agent(ws + OFF_TMO, seq); break; }
        __builtin_amdgcn_s_sleep(4);
    }
    __builtin_amdgcn_s_waitcnt(0xC07F);
    WAIT_V(0);
}

__device__ __forceinline__ void gemm_loop(char* ws, LAS unsigned char* lds, const char* gX, const char* gW, u32 fvoff, int otab, int nstep, u32 seq, int wid, int lane, f32x4 (&acc)[2][2][4][2]) {
    constexpr size_t KSTEP = MiB, SX = 16384, SW = 16384;
    constexpr int FSTEP = 4096;
    const int wr = wid >> 2, wc = wid & 3, fr = lane & 15, fq = lane >> 4;
    const u32 voff = (u32)lane * 16u;
    const char* cX = uni(gX + wid * 1024);
    const char* cW = uni(gW + (wid >> 2) * 8192 + (wid & 3) * 1024);
    const char* fb = ws + OFF_FG;
    const u32 ldsw = (u32)wid * 1024u;
    const int woff = wr * 8192 + lane * 16;
    const int xoff0 = (wc * 32 + fr) * 128 + ((fq ^ (fr >> 1)) << 4), xoff1 = xoff0 ^ 64;
#define SA(b, h) (((b) * 2 + (h)) * 16384)
#define SB(b, h) ((4 + (b) * 2 + (h)) * 16384)
#define GLDS(src, dst) __builtin_amdgcn_global_load_lds((const GAS u32*)((src) + voff), (LAS u32*)(lds + (dst) + ldsw), 16, 0, GAUX)
#define STW(b, h, ptr) do { GLDS((ptr) + (h) * 4096, SA(b, h)); GLDS((ptr) + (h) * 4096 + SW, SA(b, h) + 8192); } while (0)
#define STX(b, h, ptr) do { GLDS((ptr) + (h) * SX, SB(b, h)); GLDS((ptr) + (h) * SX + 8192, SB(b, h) + 8192); } while (0)
#define LDW(b, h) do { _Pragma("unroll") for (int m = 0; m < 4; ++m) _Pragma("unroll") for (int k = 0; k < 2; ++k) Wt[m][k] = *(const LAS bf16x8*)(lds + SA(b, h) + woff + k * 4096 + m * 1024); } while (0)
#define LDX(dst, b, h) do { _Pragma("unroll") for (int n = 0; n < 2; ++n) { dst[n][0] = *(const LAS bf16x8*)(lds + SB(b, h) + xoff0 + n * 2048); dst[n][1] = *(const LAS bf16x8*)(lds + SB(b, h) + xoff1 + n * 2048); } } while (0)
#define MMA(h, xh, Xt) do { __builtin_amdgcn_s_setprio(1); _Pragma("unroll") for (int m = 0; m < 4; ++m) _Pragma("unroll") for (int n = 0; n < 2; ++n) _Pragma("unroll") for (int k = 0; k < 2; ++k) \
        acc[h][xh][m][n] = __builtin_amdgcn_mfma_f32_16x16x32_bf16(Xt[n][k], Wt[m][k], acc[h][xh][m][n], 0, 0, 0); __builtin_amdgcn_s_setprio(0); } while (0)
#define FLAG_LOAD(step) asm volatile("s_nop 4\n\tglobal_load_dword %0, %1, %2 sc0 sc1" : "+v"(fl) : "v"(fvoff), "s"(fb + (size_t)(step) * FSTEP) : "memory")
#define FLAG_CHECK(step) do { if (wid == 0 && __builtin_amdgcn_ballot_w64(fl < seq) != 0ull) spin_flags(ws, fb + (size_t)(step) * FSTEP, fvoff, seq); } while (0)
#pragma unroll
    for (int a = 0; a < 2; ++a)
#pragma unroll
        for (int b = 0; b < 2; ++b)
#pragma unroll
            for (int m = 0; m < 4; ++m)
#pragma unroll
                for (int n = 0; n < 2; ++n) acc[a][b][m][n] = (f32x4){0.f, 0.f, 0.f, 0.f};
    bf16x8 Wt[4][2], X0[2][2], X1[2][2];
    u32 fl = 0;
#define KSTAMP(i)
    if (wid == 0) { spin_flags(ws, fb, fvoff, seq); spin_flags(ws, fb + FSTEP, fvoff, seq); }
    WAIT_P; WAIT_V(0); WAIT_L(0); BAR; asm volatile("" ::: "memory");
#define WAIT_VN(n) asm volatile("s_waitcnt vmcnt(" #n ")" : "+v"(fl) :: "memory")
    FLAG_LOAD(2);
    STX(0, 0, cX); STX(0, 1, cX); STW(0, 0, cW); STW(0, 1, cW);
    STX(1, 0, cX + KSTEP); STX(1, 1, cX + KSTEP); STW(1, 0, cW + KSTEP);
    WAIT_VN(8); BAR;
    if (wr == 1) BAR;
    for (int t = 0; t < nstep; t += 2) {
        const int t1 = __builtin_amdgcn_readlane(otab, t + 1), t2 = __builtin_amdgcn_readlane(otab, t + 2);
        const int t3 = __builtin_amdgcn_readlane(otab, t + 3), t4 = __builtin_amdgcn_readlane(otab, t + 4);
        const char* w1 = cW + (size_t)t1 * KSTEP;
        const char* x2 = cX + (size_t)t2 * KSTEP; const char* w2 = cW + (size_t)t2 * KSTEP;
        const char* x3 = cX + (size_t)t3 * KSTEP; const char* w3 = cW + (size_t)t3 * KSTEP;
        KSTAMP(t >> 1);
        FLAG_CHECK(t2);
        LDX(X0, 0, 0); LDW(0, 0); LDX(X1, 0, 1); FLAG_LOAD(t3); STW(1, 1, w1);
        WAIT_VN(9); WAIT_L(0); SCHED; BAR; SCHED; MMA(0, 0, X0); MMA(0, 1, X1); SCHED; BAR; SCHED;
        LDW(0, 1); STX(0, 0, x2); STX(0, 1, x2); STW(0, 0, w2);
        WAIT_VN(8); WAIT_L(0); SCHED; BAR; SCHED; MMA(1, 0, X0); MMA(1, 1, X1); SCHED; BAR; SCHED;
        FLAG_CHECK(t3);
        LDX(X0, 1, 0); LDW(1, 0); LDX(X1, 1, 1); FLAG_LOAD(t4); STW(0, 1, w2);
        WAIT_VN(9); WAIT_L(0); SCHED; BAR; SCHED; MMA(0, 0, X0); MMA(0, 1, X1); SCHED; BAR; SCHED;
        LDW(1, 1); STX(1, 0, x3); STX(1, 1, x3); STW(1, 0, w3);
        WAIT_VN(8); WAIT_L(0); SCHED; BAR; SCHED; MMA(1, 0, X0); MMA(1, 1, X1); SCHED; BAR; SCHED;
    }
#undef WAIT_VN
    KSTAMP(28);
    asm volatile("s_waitcnt vmcnt(0)" : "+v"(fl) :: "memory");
    if (wr == 0) BAR;
    BAR;
#undef SA
#undef SB
#undef GLDS
#undef STW
#undef STX
#undef LDW
#undef LDX
#undef MMA
#undef FLAG_LOAD
#undef FLAG_CHECK
#undef KSTAMP
}

__device__ __forceinline__ void gemm_piece(char* ws, LAS unsigned char* lds, const char* gX, const char* gW, u32 fx, u32 fw, u32 seq, int wid, int lane, f32x4 (&acc)[2][2][4][2]) {
    const int wr = wid >> 2, wc = wid & 3, fr = lane & 15, fq = lane >> 4;
    const u32 voff = (u32)lane * 16u;
    const char* cX = uni(gX + wc * 4096);
    const char* cW = uni(gW + wc * 4096);
    const int xoff0 = (wc * 32 + fr) * 128 + ((fq ^ (fr >> 1)) << 4), xoff1 = xoff0 ^ 64;
#pragma unroll
    for (int a = 0; a < 2; ++a)
#pragma unroll
        for (int m = 0; m < 4; ++m)
#pragma unroll
            for (int n = 0; n < 2; ++n) acc[a][0][m][n] = (f32x4){0.f, 0.f, 0.f, 0.f};
    const u32 fo = ((wid & 2) ? fw : fx) + (wid & 1) * 4;
    if (wid < 4) spin_flags(ws, ws + OFF_FG, gslot(lane < 36 ? lane : 0, seq) * 4096u + fo, seq);
    WAIT_P; WAIT_V(0); WAIT_L(0); BAR; asm volatile("" ::: "memory");
#define PSRC(base, n) ((base) + (size_t)gslot((n) & 63, seq) * MiB + voff)
#define PLD(src, dst, off) __builtin_amdgcn_global_load_lds((const GAS u32*)(src), (LAS u32*)(dst), 16, off, GAUX)
#define PLD4(src, dst) do { PLD(src, dst, 0); PLD(src, dst, 1024); PLD(src, dst, 2048); PLD(src, dst, 3072); } while (0)
    LAS unsigned char* d0 = lds + wc * 4096;
    PLD4(PSRC(cX, wr), d0 + wr * 32768); PLD4(PSRC(cW, wr), d0 + wr * 32768 + 16384);
    PLD4(PSRC(cX, wr + 2), d0 + (wr + 2) * 32768); PLD4(PSRC(cW, wr + 2), d0 + (wr + 2) * 32768 + 16384);
    WAIT_V(8); BAR;
    if (wr == 1) BAR;
    int buf = wr;
#define PT(i)
    for (int n = wr; n < 64; n += 2) {
        bf16x8 Wt[8][2], Xt[2][2];
        const LAS unsigned char* bx = lds + buf * 32768;
        LAS unsigned char* dn = d0 + (buf ? buf - 1 : 4) * 32768;
        if (n == 30) spin_flags(ws, ws + OFF_FG, (u32)(lane < 36 ? 36 : lane) * 4096u + fo, seq);
#pragma unroll
        for (int i = 0; i < 2; ++i) { Xt[i][0] = *(const LAS bf16x8*)(bx + xoff0 + i * 2048); Xt[i][1] = *(const LAS bf16x8*)(bx + xoff1 + i * 2048); }
#pragma unroll
        for (int k = 0; k < 2; ++k)
#pragma unroll
            for (int c = 0; c < 8; ++c) Wt[c][k] = *(const LAS bf16x8*)(bx + 16384 + voff + k * 8192 + c * 1024);
        const char* xn = PSRC(cX, n + 4); const char* wn = PSRC(cW, n + 4);
        if (PSPLIT >= 1) PLD(xn, dn, 0);
        if (PSPLIT >= 2) PLD(xn, dn, 1024);
        WAIT_L(0); PT(0); SCHED; BAR; SCHED; PT(1);
        __builtin_amdgcn_s_setprio(1);
#pragma unroll
        for (int g = 0; g < 8; ++g) {
#pragma unroll
            for (int c = (g & 3) * 2; c < (g & 3) * 2 + 2; ++c)
#pragma unroll
                for (int i = 0; i < 2; ++i) acc[c >> 2][0][c & 3][i] = __builtin_amdgcn_mfma_f32_16x16x32_bf16(Xt[i][g >> 2], Wt[c][g >> 2], acc[c >> 2][0][c & 3][i], 0, 0, 0);
            SCHED;
            if (g == 0) PLD(wn, dn + 16384, 0); else if (g == 1) PLD(wn, dn + 16384, 1024); else if (g == 2) PLD(wn, dn + 16384, 2048); else if (g == 3) PLD(wn, dn + 16384, 3072);
            else if (g == 4) PLD(xn, dn, 3072); else if (g == 5) PLD(xn, dn, 2048); else if (g == 6) { if (PSPLIT < 2) PLD(xn, dn, 1024); } else if (PSPLIT < 1) PLD(xn, dn, 0);
            SCHED;
        }
        __builtin_amdgcn_s_setprio(0);
        PT(2); WAIT_V(8); PT(3); BAR; PT(4); asm volatile("" ::: "memory");
        buf = buf >= 3 ? buf - 3 : buf + 2;
    }
#undef PT
#undef PSRC
#undef PLD
#undef PLD4
    WAIT_V(0);
    if (wr == 0) BAR;
    BAR; asm volatile("" ::: "memory");
}

__device__ __forceinline__ void acc_to_raw(const f32x4 (&acc)[2][2][4][2], int xh, bool poison, u32x4 (&O)[8]) {
#pragma unroll
    for (int n = 0; n < 2; ++n)
#pragma unroll
        for (int j = 0; j < 4; ++j)
#pragma unroll
            for (int c = 0; c < 4; ++c) {
                const u32 v = cvt_pk_bf16(acc[c >> 1][xh][(c & 1) * 2][n][j], acc[c >> 1][xh][(c & 1) * 2 + 1][n][j]);
                O[n * 4 + j][c] = poison ? 0x7fc07fc0u : v;
            }
}

__global__ __launch_bounds__(512, 2) void fused_gemm_rs(Params p) {
    extern __shared__ __attribute__((aligned(16))) unsigned char shm[];
    LAS unsigned char* lds = (LAS unsigned char*)shm;
    const int lane = threadIdx.x & 63, wid = __builtin_amdgcn_readfirstlane(threadIdx.x >> 6), b = blockIdx.x;
    char* ws = p.wsme;
    const u32 cnt_v = ld_agent(ws + OFF_CNT + b * 64);
    const int xcd = b & 7, ix = b >> 3;
    u32 seq;
    int s0 = b * 2;
    if (ix < 4) {
        STAMP(63);
        seq = __builtin_amdgcn_readfirstlane(cnt_v) + 1u;
    } else {
        STAMP(0);
        const int li = ix - 4, cg = li / 7, pmi = (xcd & 1) * 7 + li % 7;
        const int pn = MIX == 0 ? (xcd >> 1) * 4 + cg : (cg < 2 ? (xcd & 6) + cg : (MIX == 1 ? (xcd & 6) + 6 + cg : 12 - (xcd & 6) + cg));
        const int gi = (15 - pn) * 14 + pmi;
        const u32 tl = (u32)__builtin_amdgcn_s_memrealtime();
        LAS unsigned char* sc = lds + LDS_GEMM + wid * ENC_BYTES;
        enc_init(sc, lane);
        {
            const int t1 = PASS1(wid * NGEMM + gi);
            const int cls = t1 < 64 ? 0 : (t1 < 128 || (t1 >= 512 && t1 < 576) || (t1 >= 768 && t1 < 768 + 252)) ? 1 : 2;
            if (wid >= 6 && (7 - wid) * NGEMM + gi < 28 * NPUSH) push_task(p, (7 - wid) * NGEMM + gi, cnt_v, tl, lane);
            pack_tasks(p, ws, t1, cls < 2, [&](int pass) { return pass < 4 ? p.pk[pn][pass][0] + wid * p.pk[pn][pass][1] + pmi : 4608; }, cnt_v, tl, lds + wid * 8192, sc, lane);
        }
        seq = __builtin_amdgcn_readfirstlane(cnt_v) + 1u;
        const int dest = (p.rank + 1 + (pmi >> 1)) & 7, pm = dest * 2 + (pmi & 1);
        f32x4 acc[2][2][4][2];
        for (int nd = p.dq[pn][0];; --nd) {
            const int q = p.dq[pn][nd], dw = pmi >> 1;
            const int bx = nd ? p.rank * 2 + (pmi & 1) : pm, bw = nd ? (64 + (dw >> 1) * 8) * 32 + (dw & 1) * 16 + q : 16 + pn;
            const int l8 = lane - 8;
            gemm_loop(ws, lds, ws + OFF_G + (size_t)bx * 32768, ws + OFF_G + (size_t)bw * 32768, (u32)(((lane & 4) ? bw : bx) * 128 + (lane & 3) * 4),
                      (nd || lane < 8 || lane >= 56) ? (lane & 7) : (int)gslot(8 + l8 / 6 * 7 + (dw + 13 - l8 % 6) % 7, seq), nd ? 8 : 56, seq, wid, lane, acc);
            if (!nd) break;
            const bool poison = __builtin_amdgcn_readfirstlane(ld_agent(ws + OFF_TMO)) == seq;
            const int wv = (q * 2 + (pmi & 1)) * 8 + wid;
#pragma unroll
            for (int xh = 0; xh < 2; ++xh) { u32x4 O[8]; acc_to_raw(acc, xh, poison, O); store_unit_linear(ws + OFF_P + (size_t)(dw * 512 + wv * 2 + xh) * 8192, (u32)lane * 16u, O); }
            WAIT_V(0);
            if (lane == 0) __hip_atomic_fetch_add((GAS u32*)(ws + OFF_PD), 1u, __ATOMIC_RELAXED, __HIP_MEMORY_SCOPE_AGENT);
            STAMP(8);
        }
        STAMP(4);
        {
            const bool poison = __builtin_amdgcn_readfirstlane(ld_agent(ws + OFF_TMO)) == seq;
            const int id0 = ((pn * 2 + (pmi & 1)) * 8 + wid) * 2;
            u32 rec = 0, pos[2];
            char* ry = p.ws[dest] + OFF_RY + (size_t)((seq & 1) * 8 + p.rank) * 512 * 64 + lane * 8;
#pragma unroll
            for (int xh = 0; xh < 2; ++xh) {
                pos[xh] = 0;
                char* yq = ws + OFF_YQ + dest * 128;
                asm volatile("" : "+v"(yq));
                if (lane == 0) pos[xh] = __hip_atomic_fetch_add((GAS u32*)yq, 1u, __ATOMIC_RELAXED, __HIP_MEMORY_SCOPE_AGENT);
                u32x4 O[8];
                acc_to_raw(acc, xh, poison, O);
                rec = encode_store(O, ws + OFF_PY + (size_t)((seq & 1) * 8 + dest) * (4 * MiB) + (size_t)(id0 + xh) * 8192, sc, lane,
                                   [&] { if (xh == 1) { WAIT_V(0); put_record(ry + (__builtin_amdgcn_readfirstlane(pos[0]) & 511u) * 64, seq, rec, lane, 8); } });
                if (lane == 0) rec |= (u32)(id0 + xh) << 20;
            }
            WAIT_V(0);
            put_record(ry + (__builtin_amdgcn_readfirstlane(pos[1]) & 511u) * 64, seq, rec, lane, 8);
        }
        STAMP(5);
        s0 = pn < HC ? 63 + (pn * 14 + pmi) * 2 : -1;
        for (bool first = pn < HC + NSTAT; s0 < 0; first = false) {
            __syncthreads();
            if (threadIdx.x == 0) {
                volatile LAS u32* t = (volatile LAS u32*)(lds + LDS_GEMM);
                t[0] = first ? (u32)((pn - HC) * 14 + pmi)
                             : 14u * NSTAT + __hip_atomic_fetch_add((GAS u32*)(ws + OFF_TICKET), 1u, __ATOMIC_RELAXED, __HIP_MEMORY_SCOPE_AGENT) - (seq - 1u) * (NGEMM - 14 * HC + 128 - 14 * NSTAT);
            }
            __syncthreads();
            const u32 tk = __builtin_amdgcn_readfirstlane(*(volatile LAS u32*)(lds + LDS_GEMM));
            if (tk >= 128u) break;
            const int rq = tk & 3, cq = tk >> 2, opm = p.rank * 2 + (rq >> 1), opn = cq >> 1;
            gemm_piece(ws, lds, ws + OFF_G + (size_t)opm * 32768 + (rq & 1) * 16384, ws + OFF_G + (size_t)(16 + opn) * 32768 + (cq & 1) * 16384,
                       (u32)(opm * 128 + (rq & 1) * 8), (u32)((16 + opn) * 128 + (cq & 1) * 8), seq, wid, lane, acc);
            STAMP(6);
            LAS f32x4* red = (LAS f32x4*)(lds + (wid & 3) * 16384) + lane;
            if (wid >= 4) {
#pragma unroll
                for (int i = 0; i < 16; ++i) red[i * 64] = acc[i >> 3][0][(i >> 1) & 3][i & 1];
            }
            __syncthreads();
            if (wid < 4) {
#pragma unroll
                for (int i = 0; i < 16; ++i) acc[i >> 3][0][(i >> 1) & 3][i & 1] += red[i * 64];
                const bool poison = __builtin_amdgcn_readfirstlane(ld_agent(ws + OFF_TMO)) == seq;
                u32x4 O[8]; acc_to_raw(acc, 0, poison, O);
                char* dst = p.out + (size_t)(rq * 128 + wid * 32) * 65536 + (p.rank * 4096 + cq * 128) * 2;
                const u32 vo = (u32)(lane >> 4) * 262144u + (u32)(lane & 15) * 16u;
#pragma unroll
                for (int q = 0; q < 8; ++q) st16_sys<0>(dst + (size_t)((q >> 2) * 16 + (q & 3)) * 65536, vo, O[q]);
            }
            STAMP(7);
        }
    }
    if (s0 >= 0) comm_role(p, ws, lds, s0, seq, wid, lane, 0);
    __syncthreads();
    if (threadIdx.x == 0) st_agent(ws + OFF_CNT + b * 64, seq);
}

extern "C" void kernel_launch(int rank, int nranks, ncclComm_t comm, hipStream_t stream,
                              void* const* d_in, const int* in_sizes, int n_in,
                              void* d_out, int out_size, const int* in_dtypes, int out_dtype,
                              void* const* d_out_all, void* const* d_ws_all, size_t ws_bytes) {
    if (nranks != NR || (unsigned)rank >= (unsigned)NR || n_in != 2 || ws_bytes < WS_NEED || in_sizes[0] != 4096 * 512 || in_sizes[1] != 512 * 32768 || out_size != 512 * 32768) {
        fprintf(stderr, "kernel_launch: unexpected shapes\n");
        return;
    }
    HIP_CHECK(hipFuncSetAttribute((const void*)fused_gemm_rs, hipFuncAttributeMaxDynamicSharedMemorySize, LDS_BYTES));
    Params p;
    memset(&p, 0, sizeof p);
    p.x = (const char*)d_in[0]; p.w = (const char*)d_in[1]; p.out = (char*)d_out;
    for (int i = 0; i < NR; ++i) p.ws[i] = (char*)d_ws_all[i];
    p.ws_tab = (char* const*)d_ws_all;
    p.rank = rank; p.wsme = p.ws[rank];
    int key[256];
    for (int i = 0; i < 256; ++i) {
        const int j = i >> 5, k = i & 31;
        key[i] = k >= 16 ? 112 * j + SKEW_NUM * (k - 16) - 1 : (k >= 14 ? 100000 + j : (k >= 12 || (j == 0 && k < NPUSH)) ? 200000 : 112 * j + XSPREAD * (k >> 1) - (j == 0 ? 1000 : 0));
        int q = i;
        while (q > 0 && key[p.order[q - 1]] > key[i]) { p.order[q] = p.order[q - 1]; --q; }
        p.order[q] = (unsigned char)i;
    }
    for (int i = 0, n = 0; i < 256; ++i) if (p.order[i] & 16) p.worder[n++] = (unsigned char)((p.order[i] >> 5) * 16 + (p.order[i] & 15));
    const char* np = NPASS; const char* ndg = NDIAG; const bool pmaj = true;
    if (pmaj) for (int i = 38; i < 128; ++i) for (int q = i; q > 37 && ((p.worder[q - 1] & 15) * 8 + (p.worder[q - 1] >> 4)) > ((p.worder[q] & 15) * 8 + (p.worder[q] >> 4)); --q) { const unsigned char t = p.worder[q]; p.worder[q] = p.worder[q - 1]; p.worder[q - 1] = t; }
    int base = 1792;
    for (int ps = 0; ps < 4; ++ps) {
        int n = 0;
        for (int c = 15; c >= 0; --c) if (np[c] - '0' > ps + 1) n += 14;
        for (int c = 15, o = 0; c >= 0; --c) { const bool in = np[c] - '0' > ps + 1; p.pk[c][ps][0] = in ? base + o : 4608; p.pk[c][ps][1] = n; if (in) o += 14; }
        base += n * 8;
    }
    int nq = 0;
    for (int r = 1; r <= 3; ++r) for (int c = 0; c < 16; ++c) { const int n = ndg[c] - '0'; p.dq[c][0] = n; if (n >= r) p.dq[c][n + 1 - r] = nq++; }
    if (base < 4608 || nq != 16) { fprintf(stderr, "kernel_launch: bad schedule\n"); return; }
    hipLaunchKernelGGL(fused_gemm_rs, dim3(256), dim3(512), LDS_BYTES, stream, p);
    HIP_CHECK(hipGetLastError());
}
```

```cpp
#include <hip/hip_runtime.h>
#include <rccl/rccl.h>
#include <cstdio>
#include <cstdint>
#include <cstring>

#define HIP_CHECK(expr) do { hipError_t _e = (expr); if (_e != hipSuccess) { \
    fprintf(stderr, "HIP error %s at %s:%d\n", hipGetErrorString(_e), __FILE__, __LINE__); \
    return; } } while (0)

typedef unsigned u32;
typedef unsigned long long u64;
typedef u32 u32x4 __attribute__((ext_vector_type(4)));
typedef u32 u32x2 __attribute__((ext_vector_type(2)));
typedef short bf16x8 __attribute__((ext_vector_type(8)));
typedef float f32x4 __attribute__((ext_vector_type(4)));
typedef float f32x2 __attribute__((ext_vector_type(2)));
#define LAS __attribute__((address_space(3)))
#define GAS __attribute__((address_space(1)))

#define FORCE_RAW 0
#define GAUX 17
#define PSPLIT 1
#define NSTAT 4
#define XSPREAD 12
#define HC 1
#define NPASS (OWNJOB ? "1000000011110000" : "1122000011110000")
#define NDIAG "0000111111112222"
#define NPUSH 9
#define LEVEL3 1
#define XOWN 100000
#define LSLEEP 1
#define LPRIO 0
#define OWNJOB 0
constexpr int NPLAIN = OWNJOB ? 512 : 1024, K0 = OWNJOB ? 8 : 0, NPD = OWNJOB ? 2048 : 1792;
#define PUSHCOLS 0x0f0eu
#define SKEW_NUM 47

constexpr int NR = 8;
constexpr size_t MiB = 1ull << 20;
constexpr size_t OFF_CNT = 0;
constexpr size_t OFF_TMO = 16384;
constexpr size_t OFF_TICKET = 16384 + 256;
constexpr size_t OFF_YO = 16384 + 512;
constexpr size_t OFF_YT = 16384 + 2048;
constexpr size_t OFF_YQ = 16384 + 4096;
constexpr size_t OFF_PD = 16384 + 6144;
constexpr size_t OFF_FG = 65536;
constexpr size_t OFF_RIN = 1 * MiB;
constexpr size_t OFF_RY = 2 * MiB;
constexpr size_t OFF_XF = 3 * MiB;
constexpr size_t OFF_PX = 4 * MiB;
constexpr size_t OFF_PW = 12 * MiB;
constexpr size_t OFF_PY = 76 * MiB;
constexpr size_t OFF_G = 140 * MiB;
constexpr size_t OFF_P = 211 * MiB;
constexpr size_t OFF_DBG = 243 * MiB;
constexpr size_t WS_NEED = 246 * MiB;
constexpr u64 DEADLINE = 40000000ull;
constexpr int LDS_GEMM = 131072, LDS_BYTES = 163840;
constexpr int NCOMM = 32, NGEMM = 224;

#define TS_A 120
#define TS_0 60
#define TS_P 220
#define TS_B 60
#define TS_D 40
struct Params {
    const char* x; const char* w; char* out; char* wsme;
    char* const* ws_tab;
    int rank; int pstride[2]; int pad[3];
    char* wsr[NR];
    u32 sched[16][2];
    unsigned char order[256];
};

#define STAMP(i)

#define WAIT_V(n) asm volatile("s_waitcnt vmcnt(" #n ")" ::: "memory")
#define WAIT_P __builtin_amdgcn_s_waitcnt(0x0F70)
#define WAIT_L(n) asm volatile("s_waitcnt lgkmcnt(" #n ")" ::: "memory")
#define BAR __builtin_amdgcn_s_barrier()
#define SCHED __builtin_amdgcn_sched_barrier(0)

template <int OFF> __device__ __forceinline__ void st16_sys(const char* base, u32 voff, u32x4 v) {
    asm volatile("s_nop 4\n\tglobal_store_dwordx4 %0, %1, %2 offset:%3 sc0 sc1\n\ts_nop 1" :: "v"(voff), "v"(v), "s"(base), "i"(OFF) : "memory");
}
__device__ __forceinline__ const char* uni(const char* p) {
    return (const char*)(((u64)(u32)__builtin_amdgcn_readfirstlane((u32)((u64)p >> 32)) << 32) | (u32)__builtin_amdgcn_readfirstlane((u32)(u64)p));
}
__device__ __forceinline__ u32 perm(u32 hi, u32 lo, u32 sel) { return __builtin_amdgcn_perm(hi, lo, sel); }
__device__ __forceinline__ u32 ld_agent(const char* p) { return __hip_atomic_load((GAS u32*)p, __ATOMIC_RELAXED, __HIP_MEMORY_SCOPE_AGENT); }
__device__ __forceinline__ void st_agent(char* p, u32 v) { __hip_atomic_store((GAS u32*)p, v, __ATOMIC_RELAXED, __HIP_MEMORY_SCOPE_AGENT); }
__device__ __forceinline__ u32 gslot(u32 o, u32 seq) { return o + ((seq & 1u) && o - 8u < 7u ? 56u : 0u); }
typedef __bf16 bf16x2 __attribute__((ext_vector_type(2)));
__device__ __forceinline__ u32 cvt_pk_bf16(float lo, float hi) { return __builtin_bit_cast(u32, __builtin_convertvector((f32x2){lo, hi}, bf16x2)); }

__device__ __forceinline__ void store_unit_linear(const char* dst, u32 voff, const u32x4 (&O)[8]) {
    const char* b = uni(dst + 4096);
    st16_sys<-4096>(b, voff, O[0]); st16_sys<-3072>(b, voff, O[1]); st16_sys<-2048>(b, voff, O[2]); st16_sys<-1024>(b, voff, O[3]);
    st16_sys<0>(b, voff, O[4]); st16_sys<1024>(b, voff, O[5]); st16_sys<2048>(b, voff, O[6]); st16_sys<3072>(b, voff, O[7]);
}

constexpr u32 H_RAW = 1u << 11, H_DEAD = 1u << 12, H_L3 = 1u << 17;
constexpr int ENC_BYTES = 1152;
__device__ __forceinline__ u32 bfi(u32 m, u32 a, u32 b) { return (a & m) | (b & ~m); }
__device__ __forceinline__ u32 scan32(u32 v) {
    v += (u32)__builtin_amdgcn_update_dpp(0, (int)v, 0x111, 0xf, 0xf, false);
    v += (u32)__builtin_amdgcn_update_dpp(0, (int)v, 0x112, 0xf, 0xf, false);
    v += (u32)__builtin_amdgcn_update_dpp(0, (int)v, 0x114, 0xf, 0xf, false);
    v += (u32)__builtin_amdgcn_update_dpp(0, (int)v, 0x118, 0xf, 0xf, false);
    v += (u32)__builtin_amdgcn_update_dpp(0, (int)v, 0x142, 0xa, 0xf, false);
    return v;
}
__device__ __forceinline__ void enc_init(LAS unsigned char* sc, int lane) {
    u32 sel = 0x0c0c0c0cu;
#pragma unroll
    for (int b = 3; b >= 0; --b) if (lane & (1 << b)) sel = (sel << 8) | (u32)b;
    if (lane < 16) ((LAS u32*)(sc + 1088))[lane] = sel;
}
template <class F>
__device__ __forceinline__ u32 encode_store(const u32x4 (&O)[8], const char* slot, LAS unsigned char* sc, int lane, F before_stores) {
    u32 d[16], S[16];
    u32 sum = 0;
#pragma unroll
    for (int t = 0; t < 16; ++t) {
        const u32 a = O[2 * (t >> 2)][t & 3], b = O[2 * (t >> 2) + 1][t & 3];
        const u32 ra = __builtin_amdgcn_alignbit(a, a, 31), rb = __builtin_amdgcn_alignbit(b, b, 31);
        d[t] = perm(rb, ra, 0x07050301u); S[t] = perm(rb, ra, 0x06040200u);
    }
#pragma unroll
    for (int t = 0; t < 16; t += 4) sum = __builtin_amdgcn_sad_u8(d[t], 0u, sum);
    sum = scan32(sum);
    const u32 tot = (u32)__builtin_amdgcn_readlane((int)sum, 31) + (u32)__builtin_amdgcn_readlane((int)sum, 63);
    const u32 T = min(max((tot >> 10) + 4u, 15u), 244u), T4 = T * 0x01010101u;
    *(LAS u32x4*)(sc + lane * 16) = (u32x4){0u, 0u, 0u, 0u};
    if (lane == 0) *(LAS u32x4*)(sc + 1024) = (u32x4){0u, 0u, 0u, 0u};
    u32 acc = 0;
#pragma unroll
    for (int t = 0; t < 16; ++t) { d[t] = T4 - d[t]; acc |= d[t]; }
    u32 rec = 0, head = T;
    bool raw = FORCE_RAW;
    u64 xmask = 0;
    u32 xt = 0, xb = 0, xe = 0;
    if (__builtin_amdgcn_ballot_w64((acc & 0xf0f0f0f0u) != 0u) != 0ull) {
        u32 n = 0, sM = 0;
#pragma unroll
        for (int t = 0; t < 16; ++t) {
            const u32 e = T4 - d[t];
            const u32 w = ((T4 | 0x80808080u) - (e & 0x7f7f7f7fu)) ^ ((T4 ^ ~e) & 0x80808080u);
            const u32 mis = (((w >> 4) & 0x0f0f0f0fu) + 0x0f0f0f0fu) & 0x10101010u;
            d[t] = (w & 0x0f0f0f0fu) | (mis >> 1);
            if (mis) { ++n; xe = e; sM = mis; xt = t; }
        }
        xb = (n ? (u32)__builtin_ctz(sM) : 0u) & 24u;
        xmask = __builtin_amdgcn_ballot_w64(n != 0u);
        if (__builtin_popcountll(xmask) > 7 || __builtin_amdgcn_ballot_w64(n > 1u || (sM & (sM - 1u)) != 0u) != 0ull) raw = true;
    }
    u32x4 Q; u32 cs[16], bo[16], bits = 0;
#pragma unroll
    for (int t = 0; t < 16; ++t) {
        const u32 sel = d[t] + (d[t] & 0x08080808u);
        const u32 r = perm(0xffffffaau, 0x5500ffffu, sel), r2 = perm(0xffffff00u, 0x0000ffffu, sel);
        Q[t >> 2] = (t & 3) ? bfi(0x03030303u << (2 * (t & 3)), r, Q[t >> 2]) : r;
        cs[t] = ((const LAS u32*)(size_t)__builtin_amdgcn_sad_u8(r2 & 0x20100804u, 0u, (u32)(size_t)sc))[272];
        bo[t] = bits;
        bits += (u32)__builtin_popcount(r2);
    }
    const u32 inc = scan32(bits);
    const u32 nb = max((u32)__builtin_amdgcn_readlane((int)inc, 31), (u32)__builtin_amdgcn_readlane((int)inc, 63)) >> 3;
    if (nb > 960u) raw = true;
    if (xmask && !raw) {
        u32 pos = (inc - bits) >> 3;
#pragma unroll
        for (int i = 0; i < 4; ++i) {
            const u32 f = 2u * (xt & 3u), m = (u32)i < (xt >> 2) ? ~0u : (u32)i > (xt >> 2) ? 0u : ((1u << f) - 1u) * 0x01010101u | ((0x01010101u << f) & ((1u << xb) - 1u));
            pos += (u32)__builtin_popcount(Q[i] & (Q[i] >> 1) & 0x55555555u & m);
        }
        const u32 ent = ((u32)(lane & 32) * 32u + pos) << 8 | ((xe >> xb) & 255u);
        head |= (u32)__builtin_popcountll(xmask) << 8;
        for (u32 s = 1; xmask; ++s, xmask &= xmask - 1) {
            const u32 e = (u32)__builtin_amdgcn_readlane((int)ent, __builtin_ctzll(xmask));
            if ((u32)(lane & 7) == s) rec = e;
        }
    }
    before_stores();
    const u32 voff = (u32)lane * 16u;
    if (raw) {
        store_unit_linear(slot, voff, O);
        head = H_RAW;
    } else {
        const char* b = uni(slot + 4096);
        st16_sys<-4096>(b, voff, (u32x4){S[0], S[1], S[2], S[3]}); st16_sys<-3072>(b, voff, (u32x4){S[4], S[5], S[6], S[7]});
        st16_sys<-2048>(b, voff, (u32x4){S[8], S[9], S[10], S[11]}); st16_sys<-1024>(b, voff, (u32x4){S[12], S[13], S[14], S[15]});
        st16_sys<0>(b, voff, Q);
        const u32 base = inc - bits + (lane & 32 ? 4u : 0u) + (u32)(size_t)sc * 8u;
#pragma unroll
        for (int t = 0; t < 16; ++t) {
            const u32 o = base + bo[t];
            const u64 x = (u64)perm(0u, d[t] ^ 0x05050505u, cs[t]) << (o & 31u);
            LAS u32* q = (LAS u32*)(size_t)((o >> 3) & ~3u);
            __hip_atomic_fetch_or(q, (u32)x, __ATOMIC_RELAXED, __HIP_MEMORY_SCOPE_WAVEFRONT);
            __hip_atomic_fetch_or(q + 1, (u32)(x >> 32), __ATOMIC_RELAXED, __HIP_MEMORY_SCOPE_WAVEFRONT);
        }
        asm volatile("" ::: "memory");
        u32 nl = (nb + 127u) >> 7, from = 0;
        const u32 cnt = (head >> 8) & 7u, rem = nb & 127u;
        if (rem && rem <= 28u - 4u * cnt) {
            --nl;
            const u32 i = (u32)(lane & 7) - cnt - 1u, ov = (rem + 3u) >> 2;
            if (i < ov) rec = *(const LAS u32*)(sc + nl * 128u + i * 4u);
            head |= ov << 29;
        }
        if (LEVEL3 && nl == 4u) {
            const u32x2 v = *(const LAS u32x2*)(sc + lane * 8);
            const u32 g[4] = {v[0] & 0x0f0f0f0fu, v[1] & 0x0f0f0f0fu, (v[0] >> 4) & 0x0f0f0f0fu, (v[1] >> 4) & 0x0f0f0f0fu};
            u32 q2 = 0, c2[4], b2[4], bits2 = 0;
#pragma unroll
            for (int t = 0; t < 4; ++t) {
                const u32 sel = g[t] + (g[t] & 0x08080808u);
                const u32 r = perm(0xffffffaau, 0x55ffff00u, sel), r2 = perm(0xffffff00u, 0x00ffff00u, sel);
                q2 = t ? bfi(0x03030303u << (2 * t), r, q2) : r;
                c2[t] = ((const LAS u32*)(size_t)__builtin_amdgcn_sad_u8(r2 & 0x20100804u, 0u, (u32)(size_t)sc))[272];
                b2[t] = bits2;
                bits2 += (u32)__builtin_popcount(r2);
            }
            const u32 inc2 = scan32(bits2);
            if (max((u32)__builtin_amdgcn_readlane((int)inc2, 31), (u32)__builtin_amdgcn_readlane((int)inc2, 63)) <= 1024u) {
                const u32 base2 = inc2 - bits2 + (lane & 32 ? 4u : 0u) + ((u32)(size_t)sc + 896u) * 8u;
                *(LAS u32*)(sc + 640 + lane * 4) = q2;
#pragma unroll
                for (int t = 0; t < 4; ++t) {
                    const u32 o = base2 + b2[t];
                    const u64 x = (u64)perm(0u, g[t], c2[t]) << (o & 31u);
                    LAS u32* q = (LAS u32*)(size_t)((o >> 3) & ~3u);
                    __hip_atomic_fetch_or(q, (u32)x, __ATOMIC_RELAXED, __HIP_MEMORY_SCOPE_WAVEFRONT);
                    __hip_atomic_fetch_or(q + 1, (u32)(x >> 32), __ATOMIC_RELAXED, __HIP_MEMORY_SCOPE_WAVEFRONT);
                }
                asm volatile("" ::: "memory");
                nl = 3; from = 640; head |= H_L3;
            }
        }
        if ((u32)lane < nl * 8u) st16_sys<1024>(b, voff, *(const LAS u32x4*)(sc + from + lane * 16));
        head |= nl << 13;
    }
    if ((lane & 7) == 0) rec = head;
    return rec;
}
__device__ __forceinline__ void put_record(char* p, u32 seq, u32 w, int lane, int nl) {
    const u32 hd = __builtin_amdgcn_readfirstlane(w), cnt = ((hd >> 8) & 7u) + (hd >> 29);
    if (lane < nl && (u32)(lane & 7) <= cnt) __hip_atomic_store((GAS u64*)p, ((u64)w << 32) | seq, __ATOMIC_RELAXED, __HIP_MEMORY_SCOPE_SYSTEM);
}

__device__ __forceinline__ int x_task(int t, int rank) {
    if (t >= 512) return ((t >> 5) & 7) << 10 | (64 + ((t >> 8) & 1) * 32 + (t & 31));
    if (t - 56u < 64u) return ((t - 56) >> 3) << 10 | (rank * 8 + (t & 7));
    const int v = t < 56 ? t : t - 64;
    return (v / 56) << 10 | ((rank * 8 + 8 + v % 56) & 63);
}
__device__ __forceinline__ void task_load(const Params& p, int code, LAS unsigned char* buf, int lane) {
    const int j = code >> 10, r = code & 1023, un = r & 3;
    const char* s0; const char* s1; u32 stride;
    if (r < 64) {
        const char* src = p.x + (size_t)((r >> 2) * 256 + un * 64) * 1024 + j * 128 + (lane >> 3) * 1024;
        const int sw = (lane & 7) ^ (lane >> 4);
        s0 = src + (sw << 4); s1 = src + ((sw ^ 4) << 4); stride = 8192;
    } else {
        s0 = s1 = p.w + (size_t)(j * 64 + (un & 1) * 32 + (lane >> 4) * 8) * 65536 + (((p.rank + ((r - 64) >> 6)) & 7) * 4096 + ((r >> 2) & 15) * 256 + (un >> 1) * 128) * 2 + (lane & 15) * 16;
        stride = 65536;
    }
#pragma unroll
    for (int q = 0; q < 8; ++q) __builtin_amdgcn_global_load_lds((const GAS u32*)(((q & 1) ? s1 : s0) + q * stride), (LAS u32*)(buf + q * 1024), 16, 0, 0);
}
__device__ __forceinline__ u32 task_time(const Params& p, int code) {
    const u32 j = code >> 10, r = code & 1023, c = (r >> 2) & 15;
    return r < 64 ? ((c >> 1) == (u32)p.rank ? TS_A : j ? TS_A + j * TS_B : TS_0) : (!OWNJOB && r < 128) ? TS_A + j * TS_B + TS_B / 2 + (c >> 3) * 8 * TS_B : TS_A + c * TS_D + (j & 3) * (TS_D / 4);
}
__device__ __forceinline__ void sleep_until(u32 t) {
    for (int n = (int)(t - (u32)__builtin_amdgcn_s_memrealtime()); n > 11; n -= 23) __builtin_amdgcn_s_sleep(8);
}
template <class Next>
__device__ __forceinline__ void pack_tasks(const Params& p, char* ws, char* wd, Next next, u32 cnt_v, u32 tl, LAS unsigned char* lds, LAS unsigned char* sc, int wid, int lane) {
    const u32 voff = (u32)lane * 16u;
    int code = __builtin_amdgcn_readfirstlane(next(0));
    if (code < 0) return;
    auto place = [&](int c) { return lds + ((c & 1023) >= (OWNJOB ? 64 : 128) ? (c >> 10) * 2 + (c & 1) : wid + 8) * 8192; };
#define UT(i)
    sleep_until(tl + task_time(p, code));
    task_load(p, code, place(code), lane);
    char* peer = p.ws_tab[(p.rank + 1 + (lane >> 3)) & 7];
    asm volatile("" : "+v"(peer), "+v"(cnt_v) :: "memory");
    const u32 seq = __builtin_amdgcn_readfirstlane(cnt_v) + 1u;
    WAIT_P; WAIT_V(0);
    char* rp = nullptr; u32 rec = 0; int nl = 0;
    for (int pass = 0;; ++pass) {
        const int j = code >> 10, r = code & 1023, un = r & 3, dd = (r - 64) >> 6, b16 = (r >> 2) & 15;
        const bool isx = r < 64, enc = isx ? (b16 >> 1) != p.rank : dd != 0;
        const bool urgent = j < 2 && (isx || b16 < 4);
        LAS unsigned char* buf = place(code);
        u32x4 O[8];
        {
            u32x4 in[8];
#pragma unroll
            for (int q = 0; q < 8; ++q) in[q] = *(const LAS u32x4*)(buf + q * 1024 + voff);
            WAIT_L(0);
            if (isx) {
#pragma unroll
                for (int q = 0; q < 8; ++q) O[q] = in[q];
            } else {
#pragma unroll
                for (int d = 0; d < 4; ++d)
#pragma unroll
                    for (int q = 0; q < 4; ++q) { O[2 * d][q] = perm(in[2 * q + 1][d], in[2 * q][d], 0x05040100u); O[2 * d + 1][q] = perm(in[2 * q + 1][d], in[2 * q][d], 0x07060302u); }
            }
        }
        UT(1);
        char* flag = nullptr;
        const int blk = j * 32 + (r >> 2);
        const bool rawlast = urgent && isx && enc;
        if (rawlast) {
        } else if (r < 128) {
            store_unit_linear(ws + OFF_G + (size_t)blk * 32768 + un * 8192, voff, O);
            flag = ws + OFF_FG + blk * 128 + un * 4;
            if (isx && !enc) {
#pragma unroll
                for (int q = 0; q < 8; ++q) O[q] = *(const LAS u32x4*)(buf + ((q & 3) * 16 + (lane & 15)) * 128 + ((((q >> 2) * 4 + (lane >> 4)) ^ ((lane & 15) >> 1)) << 4));
                WAIT_L(0);
                store_unit_linear(ws + OFF_XF + ((r & 7) * 8 + j) * 8192, voff, O);
            }
        }
        if (r >= (OWNJOB ? 64 : 128)) {
#pragma unroll
            for (int q = 0; q < 8; ++q) *(LAS u32x4*)(buf + q * 1024 + voff) = O[q];
        }
        bool late = urgent;
        code = __builtin_amdgcn_readfirstlane(next(pass + 1));
        if (code >= 0 && !late) {
            late = (u32)__builtin_amdgcn_s_memrealtime() - tl < task_time(p, code);
            if (!late) task_load(p, code, place(code), lane);
        }
        auto settle = [&]() {
            WAIT_V(0);
            if (nl) { put_record(rp, seq, rec, lane, nl); nl = 0; }
            if (flag) { if (lane == 0) st_agent(flag, seq); flag = nullptr; }
        };
        if (enc) {
            const int dest = (p.rank + dd) & 7;
            const char* slot = ws + (isx ? OFF_PX + (size_t)(seq & 1) * (4 * MiB) : OFF_PW + (size_t)((seq & 1) * 8 + dest) * (4 * MiB)) + (size_t)((j * 16 + b16) * 4 + un) * 8192;
            rec = encode_store(O, slot, sc, lane, [&] { UT(2); WAIT_P; settle(); });
            rp = (isx ? peer : wd) + OFF_RIN + (size_t)(((seq & 1) * 8 + p.rank) * 1024 + (j * 32 + (isx ? 0 : 16) + b16) * 4 + un) * 64 + (lane & 7) * 8;
            nl = isx ? 56 : 8;
        }
        UT(3);
        if (!enc || late || code < 0) settle();
        UT(4);
        if (rawlast) { store_unit_linear(ws + OFF_G + (size_t)blk * 32768 + un * 8192, voff, O); flag = ws + OFF_FG + blk * 128 + un * 4; settle(); }
        STAMP(pass < 2 ? 1 + pass : 3);
        if (code < 0) break;
        if (late) {
            sleep_until(tl + task_time(p, code));
            task_load(p, code, place(code), lane);
            WAIT_P; WAIT_V(0);
        }
    }
}

__device__ __forceinline__ void push_task(const Params& p, int t, u32 cnt_v, u32 tl, int lane) {
    const int dd = t % 7, un = (t / 7) & 3, i = t / 28, dest = (p.rank + 1 + dd) & 7, w = 6 - dd;
    const int pm = (dest * 2 + 2 + (i + 2 * w + 2) % 14) & 15;
    const char* src = p.x + (size_t)(pm * 256 + un * 64) * 1024 + (lane >> 3) * 1024;
    const int sw = (lane & 7) ^ (lane >> 4);
    u32x4 in[8];
#pragma unroll
    for (int q = 0; q < 8; ++q) in[q] = *(const GAS u32x4*)(src + q * 8192 + ((sw ^ ((q & 1) << 2)) << 4));
    asm volatile("" : "+v"(cnt_v) :: "memory");
    const u32 seq = __builtin_amdgcn_readfirstlane(cnt_v) + 1u, blk = gslot(8 + w, seq) * 32 + pm;
    char* pw = p.wsr[dd];
    WAIT_P;
    for (int n = (int)(tl + 110 + (t / 7) * TS_P / 16 - (u32)__builtin_amdgcn_s_memrealtime()); n > 2; n -= 6) __builtin_amdgcn_s_sleep(2);
    store_unit_linear(pw + OFF_G + (size_t)blk * 32768 + un * 8192, (u32)lane * 16u, in);
    WAIT_V(0);
    if (lane == 0) __hip_atomic_store((GAS u32*)(pw + OFF_FG + blk * 128 + un * 4), seq, __ATOMIC_RELAXED, __HIP_MEMORY_SCOPE_SYSTEM);
}

#define WIN 2
#define PACE 978
#define YLEAD 2
#define PFP 0
#define G1 24
#define G2 30
#define TGRAN 8
#define SLACK 1
constexpr int NSLOT = 8, NDEC = 5, NDESC = 16, NIN = 960 - 4 * NPUSH, LOOK = 8;
constexpr u32 FIN = 0x40000000u;
constexpr int L_SLOT = 0;
constexpr int L_DESC = 2 * NSLOT * 8192;
constexpr int L_CTL = L_DESC + 2 * NDESC * 64;
constexpr int L_ORD = L_CTL + 128;
static_assert(L_ORD + 512 <= LDS_BYTES, "lds");

template <int V> struct IC { static constexpr int v = V; };
__device__ __forceinline__ void wait_units(u32 n) {
    switch (n) {
        case 0: WAIT_V(0); break; case 1: WAIT_V(6); break; case 2: WAIT_V(12); break; case 3: WAIT_V(18); break; case 4: WAIT_V(24); break;
        case 5: WAIT_V(30); break; case 6: WAIT_V(36); break; default: WAIT_V(42); break;
    }
}
__device__ __forceinline__ void comm_role(const Params& p, char* ws, LAS unsigned char* lds, u32 s0, u32 seq, int wid, int lane, u32 tstart) {
    constexpr u32 NY = 9 + 4 * HC;
    if (s0 >= 63) __syncthreads();
    if (threadIdx.x < 64) ((LAS u32*)(lds + L_ORD))[threadIdx.x] = ((const u32*)p.order)[threadIdx.x];
    if (threadIdx.x >= 64 && threadIdx.x < 96) ((LAS u32*)(lds + L_CTL))[threadIdx.x - 64] = 0;
    __builtin_amdgcn_s_waitcnt(0x0F70);
    __syncthreads();
    volatile LAS u32* ctl = (volatile LAS u32*)(lds + L_CTL);
    volatile LAS u32* desc = (volatile LAS u32*)(lds + L_DESC);
    const u32 voff = (u32)lane * 16u;
    if (wid == 7) {
        const u32 st = lane >> 5, e = lane & 7, L = s0 + st, src = (p.rank + 1 + L % 7) & 7, j = L / 7;
        const u32 cin = j < 9 ? (NIN - j + 8) / 9 : 0;
        bool open = s0 < 63, dead = false, pd = false;
        const char* rws = p.ws_tab[src];
        const u64 t0 = __builtin_amdgcn_s_memrealtime();
        volatile LAS u32* tk = (volatile LAS u32*)(lds + L_ORD + 256) + st * NDESC;
        GAS u32* yt = (GAS u32*)(ws + OFF_YT + (L % 7) * 128);
        if ((lane & 31) == 0) __hip_atomic_fetch_max(yt, seq << 10, __ATOMIC_RELAXED, __HIP_MEMORY_SCOPE_AGENT);
        if ((lane & 31) < LOOK) tk[(cin + (lane & 31)) % NDESC] = j + NY * (lane & 31);
        u32 x = (lane >> 3) & 3, drawn = cin + LOOK, cnt = (s0 == 62 && st) ? 0u : FIN, idle = 0;
        while (__builtin_amdgcn_ballot_w64(x < cnt) != 0ull) {
            if (!open) open = dead || ld_agent(ws + OFF_YO + (L % 7) * 128) - (seq - 1u) * 9u >= (9u * (j - 8u) + NY - 10u) / (NY - 9u);
            const u32 lim = ctl[st * 16 + 9] + LOOK, nd = (open && cnt == FIN && lim > drawn) ? lim - drawn : 0u;
            u32 tb = 0;
            if (drawn == cin + LOOK && __builtin_amdgcn_ballot_w64(nd != 0u) != 0ull) WAIT_V(0);
            if (nd && (lane & 31) == 0) tb = __hip_atomic_fetch_add(yt, nd, __ATOMIC_RELAXED, __HIP_MEMORY_SCOPE_AGENT);
            const bool act = open && x < cnt && x < min(lim, drawn);
            const u32 u = x < cin ? j + 9 * x : NIN + (tk[x % NDESC] & 511u);
            u32 id;
            if (u < NIN) {
                u32 bk = (lds + L_ORD)[u >> 2];
                if (!(bk & 16)) { const u32 i = bk & 15, pmi = i < 14 ? (i + 2 * (L % 7) + 2) % 14 : i; bk = (bk & ~15u) | ((p.rank * 2 + 2 + pmi) & 15); }
                id = bk * 4u + (u & 3);
            } else id = u - NIN;
            const char* a = u >= NIN ? ws + OFF_RY + (size_t)(((seq & 1) * 8 + src) * 512 + id) * 64 : ws + OFF_RIN + (size_t)(((seq & 1) * 8 + src) * 1024 + id) * 64;
            u64 v = 0;
            if (!pd && __builtin_amdgcn_ballot_w64(act && u >= NIN) != 0ull) pd = ld_agent(ws + OFF_PD) >= seq * NPD;
            if (act) v = __hip_atomic_load((GAS u64*)(a + e * 8), __ATOMIC_RELAXED, __HIP_MEMORY_SCOPE_SYSTEM);
            const u32 hw = (u32)__builtin_amdgcn_ds_swizzle((int)(u32)(v >> 32), 0x0018);
            const u64 okm = __builtin_amdgcn_ballot_w64(((u32)v == seq || e > ((hw >> 8) & 7u) + (hw >> 29)) && (u < NIN || pd));
            const bool adv = act && (dead || ((okm >> (lane & 56)) & 0xffull) == 0xffull);
            if (__builtin_amdgcn_ballot_w64(nd != 0u) != 0ull) {
                tb = (u32)__builtin_amdgcn_ds_swizzle((int)tb, 0) - (seq << 10) + LOOK * NY;
                if ((u32)(lane & 31) < nd) tk[(drawn + (lane & 31)) % NDESC] = tb + (lane & 31);
                if (nd && tb + nd >= 512u) cnt = drawn + (tb < 512u ? 512u - tb : 0u);
                drawn += nd;
            }
            if (__builtin_amdgcn_ballot_w64(adv) != 0ull) {
                if (adv) {
                    if (u >= NIN && !dead) id = (hw >> 20) & 511u;
                    const size_t po = (size_t)((seq & 1) * 8 + p.rank) * (4 * MiB);
                    const char* s = u >= NIN ? rws + OFF_PY + po + (size_t)id * 8192
                        : rws + ((id & 64) ? OFF_PW + po : OFF_PX + (size_t)(seq & 1) * (4 * MiB)) + (size_t)(((id >> 7) * 16 + ((id >> 2) & 15)) * 4 + (id & 3)) * 8192;
                    const u32 en = (st * NDESC + x % NDESC) * 16, word = dead ? H_DEAD : (u32)(v >> 32);
                    const u64 sa = dead ? (u64)(ws + OFF_G) : (u64)s;
                    desc[en + e] = word;
                    if (e == 0) { desc[en + 8] = (u32)sa; desc[en + 9] = (u32)(sa >> 32); desc[en + 10] = word; desc[en + 11] = (u >= NIN ? 0x8000u : 0u) | id; }
                    if (e == 1) {
                        const u32 blk = gslot(8 + 7 * (id >> 7) + L % 7, seq) * 32 + ((id >> 2) & 31);
                        const u32 row0 = ((id >> 4) & 1) * 256 + (id & 1) * 128 + ((id >> 1) & 3) * 32, col0 = src * 4096 + ((id >> 5) & 15) * 256 + ((id >> 3) & 1) * 128;
                        const u64 da = u >= NIN ? (u64)(p.out + (size_t)row0 * 65536 + col0 * 2) : (u64)(ws + OFF_G + (size_t)blk * 32768 + (id & 3) * 8192 + 4096);
                        desc[en + 12] = (u32)da; desc[en + 13] = (u32)(da >> 32);
                        desc[en + 14] = u >= NIN ? ((L % 7) * 512 + id) * 8192 : (u32)OFF_FG + blk * 128 + (id & 3) * 4;
                    }
                    x += 4;
                }
                u32 m = min(x, (u32)__builtin_amdgcn_update_dpp((int)x, (int)x, 0x128, 0xf, 0xf, false));
                m = min(m, (u32)__builtin_amdgcn_ds_swizzle((int)m, 0x401F));
                WAIT_L(0);
                if ((lane & 31) == 0) ctl[st * 16 + 8] = min(m, cnt);
            } else if ((++idle & 15u) == 0u && (ld_agent(ws + OFF_TMO) == seq || __builtin_amdgcn_s_memrealtime() - t0 > DEADLINE)) {
                st_agent(ws + OFF_TMO, seq); WAIT_V(0); dead = true;
            } else __builtin_amdgcn_s_sleep(1);
        }
        if ((lane & 31) == 0) ctl[st * 16 + 8] = cnt | FIN;
    } else if (wid >= NDEC) {
        const u32 me = wid - NDEC;
        volatile LAS u32* c = ctl + me * 16;
        u32 landed = 0, k = 0, slot = 0, pc = PACE;
        const u32 t0 = (u32)__builtin_amdgcn_s_memrealtime();
        u32 next = 0, now;
        auto clock = [&]() { return ((u32)__builtin_amdgcn_s_memrealtime() - t0) * 64u; };
        auto pace = [&](u32 n64) {
            if (LSLEEP && (int)(next - now) > 1100) { __builtin_amdgcn_s_sleep(2); if ((int)(next - now) > 1600) __builtin_amdgcn_s_sleep(2); now = clock(); }
            if (LPRIO) __builtin_amdgcn_s_setprio(0);
            while ((int)(now - next) < 0) now = clock();
            if (LPRIO) __builtin_amdgcn_s_setprio(3);
            next += (n64 * pc) >> 4;
        };
        __builtin_amdgcn_s_setprio(3);
        for (;; ++k) {
            volatile LAS u32* rp = (lane & 16) ? desc + (me * NDESC + k % NDESC) * 16 + (lane & 15) : c + (lane & 15);
            u32 v;
            bool fin = false;
            while (true) {
                v = *rp;
                now = clock();
                const u32 pub = __builtin_amdgcn_readlane(v, 8);
                if ((pub & ~FIN) > k && (u32)__builtin_amdgcn_readlane(v, slot) + NSLOT > k) break;
                if (landed < k) {
                    wait_units(k - landed - 1);
                    ++landed;
                    if (lane == 0) c[10] = landed;
                } else if (pub == (k | FIN)) { fin = true; break; }
            }
            if (fin) break;
            const char* s = (const char*)(((u64)(u32)__builtin_amdgcn_readlane(v, 25) << 32) | (u32)__builtin_amdgcn_readlane(v, 24));
            const u32 head = __builtin_amdgcn_readlane(v, 26);
            if (__builtin_amdgcn_readlane(v, 27) & 0x8000) pc = PACE * NY / 9;
            if (s0 < 63 && k + YLEAD == (NIN - (s0 + me) / 7 + 8) / 9 && lane == 0) __hip_atomic_fetch_add((GAS u32*)(ws + OFF_YO + ((s0 + me) % 7) * 128), 1u, __ATOMIC_RELAXED, __HIP_MEMORY_SCOPE_AGENT);
            LAS unsigned char* d = lds + L_SLOT + (me * NSLOT + slot) * 8192;
            if ((int)(now - next) > (int)(SLACK * PACE / 4)) next = now - SLACK * PACE / 4;
            u32 n16 = ((head >> 13) & 15u) * 8u;
            if (head & (H_RAW | H_DEAD)) {
                pace(32);
                __builtin_amdgcn_global_load_lds((const GAS u32*)(s + 6144 + voff), (LAS u32*)(d + 6144), 16, 0, 17);
                __builtin_amdgcn_global_load_lds((const GAS u32*)(s + 6144 + voff), (LAS u32*)(d + 6144), 16, 1024, 17);
                n16 = 64; now = clock();
            }
            const u32 tail = ((n16 + TGRAN - 1) / TGRAN) * (TGRAN / 4);
            pace(G1);
            if (k < 16 && s0 < 63) STAMP(k);
            if ((u32)lane < max(n16, 1u)) __builtin_amdgcn_global_load_lds((const GAS u32*)(s + 4096 + voff), (LAS u32*)(d + 4096), 16, 1024, 17);
            __builtin_amdgcn_global_load_lds((const GAS u32*)(s + 4096 + voff), (LAS u32*)(d + 4096), 16, 0, 17);
            now = clock(); pace(G2);
            __builtin_amdgcn_global_load_lds((const GAS u32*)(s + voff), (LAS u32*)d, 16, 0, 17);
            __builtin_amdgcn_global_load_lds((const GAS u32*)(s + voff), (LAS u32*)d, 16, 1024, 17);
            now = clock(); pace(80 + tail - G1 - G2);
            __builtin_amdgcn_global_load_lds((const GAS u32*)(s + voff), (LAS u32*)d, 16, 2048, 17);
            __builtin_amdgcn_global_load_lds((const GAS u32*)(s + voff), (LAS u32*)d, 16, 3072, 17);
            asm volatile("s_waitcnt vmcnt(%0)" :: "i"(6 * WIN) : "memory");
            if (k + 1 > landed + WIN) landed = k + 1 - WIN;
            if (lane == 0) { c[9] = k; c[10] = landed; }
            if (++slot == NSLOT) slot = 0;
        }
        if (lane == 0) c[10] = k | FIN;
    } else {
        u32 pend = 0;
        int k = 0;
#define DT(y, i)
        u32 k03 = 0x03030303u, kfc = ~3u, khi = 0xffff0000u;
        asm volatile("" : "+s"(k03), "+s"(kfc), "+s"(khi));
        const __amdgpu_buffer_rsrc_t prs = __builtin_amdgcn_make_buffer_rsrc(ws + OFF_P, 0, 28 * (int)MiB, 0x00020000);
        for (;;) {
            const u32 cv = ctl[lane & 31];
            const u32 l0 = __builtin_amdgcn_readlane(cv, 10), e0 = __builtin_amdgcn_readlane(cv, 11), l1 = __builtin_amdgcn_readlane(cv, 26), e1 = __builtin_amdgcn_readlane(cv, 27);
            const u32 a0 = (l0 & ~FIN) - e0, a1 = (l1 & ~FIN) - e1;
            if ((a0 | a1) == 0u) {
                if (l0 & l1 & FIN) break;
                if (pend) { WAIT_V(0); if (lane == 0) st_agent(ws + pend, seq); pend = 0; }
                else __builtin_amdgcn_s_sleep(1);
                DT(1, 3);
                continue;
            }
            const u32 st = a1 > a0 || (a1 == a0 && (wid & 1)), e = st ? e1 : e0, sn = e % NSLOT;
            const u32 sb = (u32)(size_t)lds + L_SLOT + (st * NSLOT + sn) * 8192u;
            const LAS u32x4* sl = (const LAS u32x4*)(size_t)(sb + voff);
            asm volatile("" ::: "memory");
            u32 got = e;
            if (lane == 0) __hip_atomic_compare_exchange_strong((LAS u32*)(lds + L_CTL + st * 64 + 44), &got, e + 1, __ATOMIC_RELAXED, __ATOMIC_RELAXED, __HIP_MEMORY_SCOPE_WORKGROUP);
            u32 rec = desc[(st * NDESC + e % NDESC) * 16 + (lane & 15)], q2 = *(const LAS u32*)(size_t)(sb + 5120u + lane * 4u);
            u32x4 Q = sl[4 * 64], S[4];
#pragma unroll
            for (int q = 0; q < 4; ++q) S[q] = sl[q * 64];
            asm volatile("" : "+v"(got), "+v"(rec), "+v"(q2), "+v"(Q) :: "memory");
            if (__builtin_amdgcn_readfirstlane(got) != e) continue;
            DT(1, 4);
            if ((k & 1) && s0 < 63) STAMP(k < 124 ? k >> 1 : 62);
            if (k == 1 && s0 >= 63) STAMP(61);
            const u32 ent = __builtin_amdgcn_readlane(rec, 11), aux = __builtin_amdgcn_readlane(rec, 14), head = __builtin_amdgcn_readfirstlane(rec);
            char* dst = (char*)(((u64)(u32)__builtin_amdgcn_readlane(rec, 13) << 32) | (u32)__builtin_amdgcn_readlane(rec, 12));
            u32x4 O[8];
            const u32 cnt = (head >> 8) & 7u;
            ++k;
            auto unpack = [&](auto emit) __attribute__((always_inline)) {
            if (head & (H_RAW | H_DEAD)) {
#pragma unroll
                for (int q = 0; q < 8; ++q) O[q] = q < 4 ? S[q] : q == 4 ? Q : sl[q * 64];
                WAIT_L(0);
                if (lane == 0) ctl[st * 16 + sn] = e + 1;
                if (head & H_DEAD) {
#pragma unroll
                    for (int q = 0; q < 8; ++q) O[q] = (u32x4){0x7fc07fc0u, 0x7fc07fc0u, 0x7fc07fc0u, 0x7fc07fc0u};
                }
                emit(IC<0>{}); emit(IC<1>{}); emit(IC<2>{}); emit(IC<3>{}); emit(IC<4>{}); emit(IC<5>{}); emit(IC<6>{}); emit(IC<7>{});
            } else {
                u32 c[16], m[16], bo[16];
                u32x2 w[16];
                const u32 T4 = (head & 0xffu) * 0x01010101u;
                auto expo = [&](u32 v) { return T4 - ((v & 0x0f0f0f0fu) ^ 0x05050505u); };
                const u32 oi = (u32)lane - cnt - 1u;
                u32 n = (u32)__builtin_popcount(q2 & (q2 >> 1) & 0x55555555u) << 16;
#pragma unroll
                for (int q = 0; q < 4; ++q) n += (u32)__builtin_popcount(Q[q] & (Q[q] >> 1) & 0x55555555u);
                n = scan32(n) - n;
#define ALL16(stmt) do { _Pragma("unroll") for (int t = 0; t < 16; ++t) { stmt; } SCHED; } while (0)
                u32 c2[4], m2[4], b2[4];
                u32x2 w2[4];
                {
                    u32 bits2 = (min(n >> 16, 128u) + sb + 5376u) * 8u + (lane & 32 ? 4u : 0u);
#pragma unroll
                    for (int t = 0; t < 4; ++t) {
                        c2[t] = (q2 >> (2 * t)) & k03; m2[t] = perm(0u, 0x01000000u, c2[t]);
                        b2[t] = bits2;
                        const LAS u32* q = (const LAS u32*)(size_t)((bits2 >> 3) & kfc);
                        w2[t] = (u32x2){q[0], q[1]};
                        bits2 = __builtin_amdgcn_udot4(m2[t], 0x08080808u, bits2, false);
                    }
                    SCHED;
                }
                ALL16(c[t] = (Q[t >> 2] >> (2 * (t & 3))) & k03; m[t] = perm(0u, 0x01000000u, c[t]));
                u32 pos = min(n & 0xffffu, 1024u - 64u) + sb + (lane & 32 ? 7168u : 6144u);
                u32 ad[16], x1[16];
                ALL16(bo[t] = pos; ad[t] = pos & kfc; pos = __builtin_amdgcn_udot4(m[t], 0x01010101u, pos, false));
                if (head & H_L3) {
                    u32 g[4];
#pragma unroll
                    for (int t = 0; t < 4; ++t) {
                        asm("v_mul_u32_u24 %0, 0x10101, %1" : "=v"(m2[t]) : "v"(m2[t]));
                        g[t] = T4 - ((perm(0u, __builtin_amdgcn_alignbit(w2[t][1], w2[t][0], b2[t]), (m2[t] << 8) + perm(0u, 0x00101010u, c2[t])) & perm(0u, 0x0f040300u, c2[t])) ^ 0x05050505u);
                    }
                    *(LAS u32x2*)(size_t)(sb + 6144u + lane * 8u) = (u32x2){g[0], g[1]};
                    *(LAS u32x2*)(size_t)(sb + 7168u + lane * 8u) = (u32x2){g[2], g[3]};
                    if (head >> 29) {
                        if (oi < (head >> 29)) { *(LAS u32*)(size_t)(sb + 6656u + oi * 4u) = expo(rec); *(LAS u32*)(size_t)(sb + 7680u + oi * 4u) = expo(rec >> 4); }
                    }
                } else {
                    if (oi < (head >> 29)) *(LAS u32*)(size_t)(sb + 5120u + ((head >> 13) & 7u) * 128u + oi * 4u) = rec;
                    asm volatile("" ::: "memory");
                    const u32x4 v = sl[5 * 64];
                    *(LAS u32x4*)(size_t)(sb + 6144u + voff) = (u32x4){expo(v[0]), expo(v[1]), expo(v[2]), expo(v[3])};
                    *(LAS u32x4*)(size_t)(sb + 7168u + voff) = (u32x4){expo(v[0] >> 4), expo(v[1] >> 4), expo(v[2] >> 4), expo(v[3] >> 4)};
                }
                if (cnt) { if ((u32)lane - 1u < cnt) *(LAS unsigned char*)(size_t)(sb + 6144u + ((rec >> 8) & 2047u)) = (unsigned char)rec; }
                asm volatile("" ::: "memory");
                DT(1, 0);
                ALL16(w[t] = ((u32x2){((const LAS u32*)(size_t)ad[t])[0], ((const LAS u32*)(size_t)ad[t])[1]}));
                const u32 zl = ~(T4 - 0x00040302u) & 0x00ffffffu;
                ALL16(asm("v_mul_u32_u24 %0, 0x10101, %1" : "=v"(m[t]) : "v"(m[t])); c[t] = perm(0u, zl, c[t]));
                ALL16(x1[t] = c[t] + (m[t] << 8));
                WAIT_L(0);
                if (lane == 0) ctl[st * 16 + sn] = e + 1;
                DT(1, 1);
                SCHED;
                auto fin = [&](auto hc) __attribute__((always_inline)) {
                    constexpr int h = decltype(hc)::v, d = h >> 1;
#pragma unroll
                    for (int t = 2 * h; t < 2 * h + 2; ++t) {
                        const u32 E = perm(0u, __builtin_amdgcn_alignbyte(w[t][1], w[t][0], bo[t]), x1[t]) & ~c[t];
                        const u32 ra = perm(E, S[d][t & 3], 0x05010400u), rb = perm(E, S[d][t & 3], 0x07030602u);
                        O[2 * d][t & 3] = __builtin_amdgcn_alignbit(ra, ra, 1); O[2 * d + 1][t & 3] = __builtin_amdgcn_alignbit(rb, rb, 1);
                    }
                    if constexpr (h > 0) emit(IC<h - 1>{});
                    SCHED;
                };
                fin(IC<0>{}); fin(IC<1>{}); fin(IC<2>{}); fin(IC<3>{}); fin(IC<4>{}); fin(IC<5>{}); fin(IC<6>{}); fin(IC<7>{});
                emit(IC<7>{});
            }
            };
            u32 flag = 0;
            if (!(ent & 0x8000u)) {
                DT(0, 0);
                unpack([&](auto dc) __attribute__((always_inline)) { constexpr int d = decltype(dc)::v;
                    st16_sys<1024 * d - 4096>(dst, voff, O[d]); });
                DT(0, 1);
                flag = aux;
                DT(0, 3);
            } else {
                u32x4 P[8];
                const u32 so = aux;
#pragma unroll
                for (int q = 0; q < 8; ++q) P[q] = __builtin_amdgcn_raw_buffer_load_b128(prs, voff + (q & 3) * 1024, so + (q >> 2) * 4096, 17);
                asm volatile("" ::: "memory");
                const u32 vo = (u32)(lane >> 4) * 262144u + (u32)(lane & 15) * 16u;
                unpack([](auto) {});
                WAIT_P;
#pragma unroll
                for (int q = 0; q < 8; ++q) {
#pragma unroll
                    for (int i = 0; i < 4; ++i) {
                        const f32x2 y = (f32x2){__uint_as_float(O[q][i] << 16), __uint_as_float(O[q][i] & khi)} + (f32x2){__uint_as_float(P[q][i] << 16), __uint_as_float(P[q][i] & khi)};
                        O[q][i] = cvt_pk_bf16(y[0], y[1]);
                    }
                    st16_sys<0>(dst + (size_t)((q >> 2) * 16 + (q & 3)) * 65536, vo, O[q]);
                    SCHED;
                }
                DT(1, 2);
            }
            if (pend) { WAIT_V(8); if (lane == 0) st_agent(ws + pend, seq); }
            pend = flag;
            DT(0, 2);
        }
        if (pend) { WAIT_V(0); if (lane == 0) st_agent(ws + pend, seq); }
        STAMP(62);
    }
}

__device__ __forceinline__ void spin_flags(char* ws, const char* fbase, u32 fvoff, u32 want, u32 seq) {
    const u64 t0 = __builtin_amdgcn_s_memrealtime();
    while (__builtin_amdgcn_ballot_w64(__hip_atomic_load((GAS u32*)(fbase + fvoff), __ATOMIC_RELAXED, __HIP_MEMORY_SCOPE_SYSTEM) < want) != 0ull) {
        if (ld_agent(ws + OFF_TMO) == seq || __builtin_amdgcn_s_memrealtime() - t0 > DEADLINE) { st_agent(ws + OFF_TMO, seq); break; }
        __builtin_amdgcn_s_sleep(4);
    }
    __builtin_amdgcn_s_waitcnt(0xC07F);
    WAIT_V(0);
}
__device__ __forceinline__ void spin_flags(char* ws, const char* fbase, u32 fvoff, u32 seq) { spin_flags(ws, fbase, fvoff, seq, seq); }

__device__ __forceinline__ void gemm_loop(char* ws, LAS unsigned char* lds, const char* gX, const char* gW, u32 fvoff, int otab, int nstep, u32 seq, int wid, int lane, f32x4 (&acc)[2][2][4][2]) {
    constexpr size_t KSTEP = MiB, SX = 16384, SW = 16384;
    constexpr int FSTEP = 4096;
    const int wr = wid >> 2, wc = wid & 3, fr = lane & 15, fq = lane >> 4;
    const u32 voff = (u32)lane * 16u;
    const char* cX = uni(gX + wid * 1024);
    const char* cW = uni(gW + (wid >> 2) * 8192 + (wid & 3) * 1024);
    const char* fb = ws + OFF_FG;
    const u32 ldsw = (u32)wid * 1024u;
    const int woff = wr * 8192 + lane * 16;
    const int xoff0 = (wc * 32 + fr) * 128 + ((fq ^ (fr >> 1)) << 4), xoff1 = xoff0 ^ 64;
#define SA(b, h) (((b) * 2 + (h)) * 16384)
#define SB(b, h) ((4 + (b) * 2 + (h)) * 16384)
#define GLDS(src, dst) __builtin_amdgcn_global_load_lds((const GAS u32*)((src) + voff), (LAS u32*)(lds + (dst) + ldsw), 16, 0, GAUX)
#define STW(b, h, ptr) do { GLDS((ptr) + (h) * 4096, SA(b, h)); GLDS((ptr) + (h) * 4096 + SW, SA(b, h) + 8192); } while (0)
#define STX(b, h, ptr) do { GLDS((ptr) + (h) * SX, SB(b, h)); GLDS((ptr) + (h) * SX + 8192, SB(b, h) + 8192); } while (0)
#define LDW(b, h) do { _Pragma("unroll") for (int m = 0; m < 4; ++m) _Pragma("unroll") for (int k = 0; k < 2; ++k) Wt[m][k] = *(const LAS bf16x8*)(lds + SA(b, h) + woff + k * 4096 + m * 1024); } while (0)
#define LDX(dst, b, h) do { _Pragma("unroll") for (int n = 0; n < 2; ++n) { dst[n][0] = *(const LAS bf16x8*)(lds + SB(b, h) + xoff0 + n * 2048); dst[n][1] = *(const LAS bf16x8*)(lds + SB(b, h) + xoff1 + n * 2048); } } while (0)
#define MMA(h, xh, Xt) do { __builtin_amdgcn_s_setprio(1); _Pragma("unroll") for (int m = 0; m < 4; ++m) _Pragma("unroll") for (int n = 0; n < 2; ++n) _Pragma("unroll") for (int k = 0; k < 2; ++k) \
        acc[h][xh][m][n] = __builtin_amdgcn_mfma_f32_16x16x32_bf16(Xt[n][k], Wt[m][k], acc[h][xh][m][n], 0, 0, 0); __builtin_amdgcn_s_setprio(0); } while (0)
#define FLAG_LOAD(step) asm volatile("s_nop 4\n\tglobal_load_dword %0, %1, %2 sc0 sc1" : "+v"(fl) : "v"(fvoff), "s"(fb + (size_t)(step) * FSTEP) : "memory")
#define FLAG_CHECK(step) do { if (wid == 0 && __builtin_amdgcn_ballot_w64(fl < seq) != 0ull) spin_flags(ws, fb + (size_t)(step) * FSTEP, fvoff, seq); } while (0)
#pragma unroll
    for (int a = 0; a < 2; ++a)
#pragma unroll
        for (int b = 0; b < 2; ++b)
#pragma unroll
            for (int m = 0; m < 4; ++m)
#pragma unroll
                for (int n = 0; n < 2; ++n) acc[a][b][m][n] = (f32x4){0.f, 0.f, 0.f, 0.f};
    bf16x8 Wt[4][2], X0[2][2], X1[2][2];
    u32 fl = 0;
#define KSTAMP(i)
    if (wid == 0) { spin_flags(ws, fb, fvoff, seq); spin_flags(ws, fb + FSTEP, fvoff, seq); }
    WAIT_P; WAIT_V(0); WAIT_L(0); BAR; asm volatile("" ::: "memory");
#define WAIT_VN(n) asm volatile("s_waitcnt vmcnt(" #n ")" : "+v"(fl) :: "memory")
    FLAG_LOAD(2);
    STX(0, 0, cX); STX(0, 1, cX); STW(0, 0, cW); STW(0, 1, cW);
    STX(1, 0, cX + KSTEP); STX(1, 1, cX + KSTEP); STW(1, 0, cW + KSTEP);
    WAIT_VN(8); BAR;
    if (wr == 1) BAR;
    for (int t = 0; t < nstep; t += 2) {
        const int t1 = __builtin_amdgcn_readlane(otab, t + 1), t2 = __builtin_amdgcn_readlane(otab, t + 2);
        const int t3 = __builtin_amdgcn_readlane(otab, t + 3), t4 = __builtin_amdgcn_readlane(otab, t + 4);
        const char* w1 = cW + (size_t)t1 * KSTEP;
        const char* x2 = cX + (size_t)t2 * KSTEP; const char* w2 = cW + (size_t)t2 * KSTEP;
        const char* x3 = cX + (size_t)t3 * KSTEP; const char* w3 = cW + (size_t)t3 * KSTEP;
        KSTAMP(t >> 1);
        FLAG_CHECK(t2);
        LDX(X0, 0, 0); LDW(0, 0); LDX(X1, 0, 1); FLAG_LOAD(t3); STW(1, 1, w1);
        WAIT_VN(9); WAIT_L(0); SCHED; BAR; SCHED; MMA(0, 0, X0); MMA(0, 1, X1); SCHED; BAR; SCHED;
        LDW(0, 1); STX(0, 0, x2); STX(0, 1, x2); STW(0, 0, w2);
        WAIT_VN(8); WAIT_L(0); SCHED; BAR; SCHED; MMA(1, 0, X0); MMA(1, 1, X1); SCHED; BAR; SCHED;
        FLAG_CHECK(t3);
        LDX(X0, 1, 0); LDW(1, 0); LDX(X1, 1, 1); FLAG_LOAD(t4); STW(0, 1, w2);
        WAIT_VN(9); WAIT_L(0); SCHED; BAR; SCHED; MMA(0, 0, X0); MMA(0, 1, X1); SCHED; BAR; SCHED;
        LDW(1, 1); STX(1, 0, x3); STX(1, 1, x3); STW(1, 0, w3);
        WAIT_VN(8); WAIT_L(0); SCHED; BAR; SCHED; MMA(1, 0, X0); MMA(1, 1, X1); SCHED; BAR; SCHED;
    }
#undef WAIT_VN
    KSTAMP(28);
    asm volatile("s_waitcnt vmcnt(0)" : "+v"(fl) :: "memory");
    if (wr == 0) BAR;
    BAR;
#undef SA
#undef SB
#undef GLDS
#undef STW
#undef STX
#undef LDW
#undef LDX
#undef MMA
#undef FLAG_LOAD
#undef FLAG_CHECK
#undef KSTAMP
}

__device__ __forceinline__ void gemm_piece(char* ws, LAS unsigned char* lds, const char* gX, const char* gW, u32 fx, u32 fw, u32 pu, u32 seq, int wid, int lane, f32x4 (&acc)[2][2][4][2]) {
    const int wr = wid >> 2, wc = wid & 3, fr = lane & 15, fq = lane >> 4;
    const u32 voff = (u32)lane * 16u;
    const char* cX = uni(gX + wc * 4096);
    const char* cW = uni(gW + wc * 4096);
    const int xoff0 = (wc * 32 + fr) * 128 + ((fq ^ (fr >> 1)) << 4), xoff1 = xoff0 ^ 64;
#pragma unroll
    for (int a = 0; a < 2; ++a)
#pragma unroll
        for (int m = 0; m < 4; ++m)
#pragma unroll
            for (int n = 0; n < 2; ++n) acc[a][0][m][n] = (f32x4){0.f, 0.f, 0.f, 0.f};
    const u32 fo = ((wid & 2) ? fw : fx) + (wid & 1) * 4;
    if (wid < 4) spin_flags(ws, ws + OFF_FG, gslot(lane < 36 ? lane : 0, seq) * 4096u + fo, seq);
    else if (OWNJOB) {
        spin_flags(ws, ws + OFF_PD, 0u, seq * NPD, seq);
        const __amdgpu_buffer_rsrc_t pr = __builtin_amdgcn_make_buffer_rsrc(ws + OFF_P + (size_t)(pu + 2 * wc) * 8192, 0, 8192, 0x00020000);
#pragma unroll
        for (int q = 0; q < 8; ++q) {
            const u32x4 v = __builtin_amdgcn_raw_buffer_load_b128(pr, voff, q * 1024, 17);
#pragma unroll
            for (int c = 0; c < 4; ++c) { acc[c >> 1][0][(c & 1) * 2][q >> 2][q & 3] = __uint_as_float(v[c] << 16); acc[c >> 1][0][(c & 1) * 2 + 1][q >> 2][q & 3] = __uint_as_float(v[c] & 0xffff0000u); }
        }
    }
    WAIT_P; WAIT_V(0); WAIT_L(0); BAR; asm volatile("" ::: "memory");
#define PSRC(base, n) ((base) + (size_t)gslot((n) & 63, seq) * MiB + voff)
#define PLD(src, dst, off) __builtin_amdgcn_global_load_lds((const GAS u32*)(src), (LAS u32*)(dst), 16, off, GAUX)
#define PLD4(src, dst) do { PLD(src, dst, 0); PLD(src, dst, 1024); PLD(src, dst, 2048); PLD(src, dst, 3072); } while (0)
    LAS unsigned char* d0 = lds + wc * 4096;
    PLD4(PSRC(cX, wr + K0), d0 + wr * 32768); PLD4(PSRC(cW, wr + K0), d0 + wr * 32768 + 16384);
    PLD4(PSRC(cX, wr + K0 + 2), d0 + (wr + 2) * 32768); PLD4(PSRC(cW, wr + K0 + 2), d0 + (wr + 2) * 32768 + 16384);
    WAIT_V(8); BAR;
    if (wr == 1) BAR;
    int buf = wr;
#define PT(i)
    for (int n = wr + K0; n < 64; n += 2) {
        bf16x8 Wt[8][2], Xt[2][2];
        const LAS unsigned char* bx = lds + buf * 32768;
        LAS unsigned char* dn = d0 + (buf ? buf - 1 : 4) * 32768;
        if (n == 30) spin_flags(ws, ws + OFF_FG, (u32)(lane < 36 ? 36 : lane) * 4096u + fo, seq);
#pragma unroll
        for (int i = 0; i < 2; ++i) { Xt[i][0] = *(const LAS bf16x8*)(bx + xoff0 + i * 2048); Xt[i][1] = *(const LAS bf16x8*)(bx + xoff1 + i * 2048); }
#pragma unroll
        for (int k = 0; k < 2; ++k)
#pragma unroll
            for (int c = 0; c < 8; ++c) Wt[c][k] = *(const LAS bf16x8*)(bx + 16384 + voff + k * 8192 + c * 1024);
        const char* xn = PSRC(cX, n + 4); const char* wn = PSRC(cW, n + 4);
        if (PSPLIT >= 1) PLD(xn, dn, 0);
        if (PSPLIT >= 2) PLD(xn, dn, 1024);
        WAIT_L(0); PT(0); SCHED; BAR; SCHED; PT(1);
        __builtin_amdgcn_s_setprio(1);
#pragma unroll
        for (int g = 0; g < 8; ++g) {
#pragma unroll
            for (int c = (g & 3) * 2; c < (g & 3) * 2 + 2; ++c)
#pragma unroll
                for (int i = 0; i < 2; ++i) acc[c >> 2][0][c & 3][i] = __builtin_amdgcn_mfma_f32_16x16x32_bf16(Xt[i][g >> 2], Wt[c][g >> 2], acc[c >> 2][0][c & 3][i], 0, 0, 0);
            SCHED;
            if (g == 0) PLD(wn, dn + 16384, 0); else if (g == 1) PLD(wn, dn + 16384, 1024); else if (g == 2) PLD(wn, dn + 16384, 2048); else if (g == 3) PLD(wn, dn + 16384, 3072);
            else if (g == 4) PLD(xn, dn, 3072); else if (g == 5) PLD(xn, dn, 2048); else if (g == 6) { if (PSPLIT < 2) PLD(xn, dn, 1024); } else if (PSPLIT < 1) PLD(xn, dn, 0);
            SCHED;
        }
        __builtin_amdgcn_s_setprio(0);
        PT(2); WAIT_V(8); PT(3); BAR; PT(4); asm volatile("" ::: "memory");
        buf = buf >= 3 ? buf - 3 : buf + 2;
    }
#undef PT
#undef PSRC
#undef PLD
#undef PLD4
    WAIT_V(0);
    if (wr == 0) BAR;
    BAR; asm volatile("" ::: "memory");
}

__device__ __forceinline__ void gemm_local(const Params& p, char* ws, const LAS unsigned char* lds, u32 seq, int wid, int lane, f32x4 (&acc)[2][2][4][2]) {
    const __amdgpu_buffer_rsrc_t xr = __builtin_amdgcn_make_buffer_rsrc(ws + OFF_XF + wid * 65536, 0, 65536, 0x00020000);
    auto ldx = [&](int u, int i) { return __builtin_bit_cast(bf16x8, __builtin_amdgcn_raw_buffer_load_b128(xr, lane * 16 + i * 1024, u * 4096, 17)); };
    if (wid == 0) spin_flags(ws, ws + OFF_FG, (u32)((lane & 7) * 4096 + (p.rank * 2 + (lane >> 5)) * 128 + ((lane >> 3) & 3) * 4), seq);
    __syncthreads();
#pragma unroll
    for (int i = 0; i < 32; ++i) acc[i >> 4][(i >> 3) & 1][(i >> 1) & 3][i & 1] = (f32x4){0.f, 0.f, 0.f, 0.f};
    bf16x8 Xa[4], Xb[4];
#pragma unroll
    for (int i = 0; i < 4; ++i) Xa[i] = ldx(0, i);
    auto step = [&](int u, const bf16x8 (&X)[4]) __attribute__((always_inline)) {
        bf16x8 w[8];
#pragma unroll
        for (int c = 0; c < 8; ++c) w[c] = *(const LAS bf16x8*)(lds + u * 8192 + c * 1024 + lane * 16);
        SCHED;
#pragma unroll
        for (int c = 0; c < 8; ++c)
#pragma unroll
            for (int i = 0; i < 4; ++i) acc[c >> 2][i >> 1][c & 3][i & 1] = __builtin_amdgcn_mfma_f32_16x16x32_bf16(X[i], w[c], acc[c >> 2][i >> 1][c & 3][i & 1], 0, 0, 0);
        SCHED;
    };
#pragma unroll 1
    for (int u = 0; u < 16; u += 2) {
#pragma unroll
        for (int i = 0; i < 4; ++i) Xb[i] = ldx(u + 1, i);
        step(u, Xa);
#pragma unroll
        for (int i = 0; i < 4; ++i) Xa[i] = ldx((u + 2) & 15, i);
        step(u + 1, Xb);
    }
    WAIT_P;
}

__device__ __forceinline__ void acc_to_raw(const f32x4 (&acc)[2][2][4][2], int xh, bool poison, u32x4 (&O)[8]) {
#pragma unroll
    for (int n = 0; n < 2; ++n)
#pragma unroll
        for (int j = 0; j < 4; ++j)
#pragma unroll
            for (int c = 0; c < 4; ++c) {
                const u32 v = cvt_pk_bf16(acc[c >> 1][xh][(c & 1) * 2][n][j], acc[c >> 1][xh][(c & 1) * 2 + 1][n][j]);
                O[n * 4 + j][c] = poison ? 0x7fc07fc0u : v;
            }
}

__global__ __launch_bounds__(512, 2) void fused_gemm_rs(Params p) {
    extern __shared__ __attribute__((aligned(16))) unsigned char shm[];
    LAS unsigned char* lds = (LAS unsigned char*)shm;
    const int lane = threadIdx.x & 63, wid = __builtin_amdgcn_readfirstlane(threadIdx.x >> 6), b = blockIdx.x;
    const int xcd = b & 7, ix = b >> 3;
    const int li = ix - 4, cg = li / 7, pmi = ((xcd & 1) * 7 + li % 7) & 15, pn = ((xcd & 6) + cg + (cg < 2 ? 0 : 6)) & 15;
    char* ws = p.wsme;
    char* wd = p.wsr[pmi >> 1];
    u32 sc0 = p.sched[pn][0], sc1 = p.sched[pn][1];
    asm volatile("" : "+s"(ws), "+s"(wd), "+s"(sc0), "+s"(sc1));
    const u32 cnt_v = ld_agent(ws + OFF_CNT + b * 64);
    u32 seq;
    int s0 = b * 2;
    if (ix < 4) {
        STAMP(63);
        seq = __builtin_amdgcn_readfirstlane(cnt_v) + 1u;
    } else {
        STAMP(0);
        const u32 tl = (u32)__builtin_amdgcn_s_memrealtime();
        LAS unsigned char* sc = lds + LDS_GEMM + wid * ENC_BYTES;
        enc_init(sc, lane);
        const int dest = (p.rank + 1 + (pmi >> 1)) & 7, pm = dest * 2 + (pmi & 1), dw = pmi >> 1;
        f32x4 acc[2][2][4][2];
        {
            const int pc = __builtin_popcount(PUSHCOLS & ((1u << pn) - 1u)), pt = (7 - wid) * 14 * __builtin_popcount(PUSHCOLS) + pc * 14 + pmi;
            if (wid >= 5 && ((PUSHCOLS >> pn) & 1u) && pt < 28 * NPUSH) push_task(p, pt, cnt_v, tl, lane);
        }
        const int oj = (3 - pn) * 14 + pmi, own = OWNJOB && (unsigned)oj < 32u, nd = own ? 1 : sc1 & 255, jw = own ? 7 : dw;
        for (int it = 0; it == 0 || it < nd; ++it) {
            const int q = own ? oj >> 1 : (sc1 >> (8 * it + 8)) & 255;
            int np = 0, t0 = 0, t1 = 0;
            if (it == 0) { t0 = (sc0 & 0xffff) + wid * p.pstride[0] + pmi; t1 = (sc0 >> 16) + wid * p.pstride[1] + pmi; np = (t0 < NPLAIN) + (t1 < NPLAIN); }
            pack_tasks(p, ws, wd, [&](int i) {
                           return i < np ? x_task(i ? t1 : t0, p.rank) : (nd && i < np + 2) ? (((i - np) * 4 + (wid >> 1)) << 10 | (64 + ((jw + 1) & 7) * 64 + q * 4 + (pmi & 1) * 2 + (wid & 1))) : -1; },
                       cnt_v, tl, lds, sc, wid, lane);
            if (!nd) break;
            seq = __builtin_amdgcn_readfirstlane(cnt_v) + 1u;
            gemm_local(p, ws, lds, seq, wid, lane, acc);
            const bool poison = __builtin_amdgcn_readfirstlane(ld_agent(ws + OFF_TMO)) == seq;
#pragma unroll
            for (int xh = 0; xh < 2; ++xh) {
                u32x4 O[8]; acc_to_raw(acc, xh, poison, O);
                store_unit_linear(ws + OFF_P + (size_t)(jw * 512 + (((q * 2 + (wid >> 2)) * 8 + (pmi & 1) * 4 + (wid & 1) * 2 + xh) * 2 + ((wid >> 1) & 1))) * 8192, (u32)lane * 16u, O);
            }
            WAIT_V(0);
            if (lane == 0) __hip_atomic_fetch_add((GAS u32*)(ws + OFF_PD), 1u, __ATOMIC_RELAXED, __HIP_MEMORY_SCOPE_AGENT);
            __syncthreads();
            STAMP(8);
        }
        seq = __builtin_amdgcn_readfirstlane(cnt_v) + 1u;
        {
            const int l8 = lane - 8;
            gemm_loop(ws, lds, ws + OFF_G + (size_t)pm * 32768, ws + OFF_G + (size_t)(16 + pn) * 32768, (u32)(((lane & 4) ? 16 + pn : pm) * 128 + (lane & 3) * 4),
                      (lane < 8 || lane >= 56) ? (lane & 7) : (int)gslot(8 + l8 / 6 * 7 + (dw + 13 - l8 % 6) % 7, seq), 56, seq, wid, lane, acc);
        }
        STAMP(4);
        {
            const bool poison = __builtin_amdgcn_readfirstlane(ld_agent(ws + OFF_TMO)) == seq;
            const int id0 = ((pn * 2 + (pmi & 1)) * 8 + wid) * 2;
            u32 rec = 0, pos[2];
            char* ry = wd + OFF_RY + (size_t)((seq & 1) * 8 + p.rank) * 512 * 64 + lane * 8;
#pragma unroll
            for (int xh = 0; xh < 2; ++xh) {
                pos[xh] = 0;
                char* yq = ws + OFF_YQ + dest * 128;
                asm volatile("" : "+v"(yq));
                if (lane == 0) pos[xh] = __hip_atomic_fetch_add((GAS u32*)yq, 1u, __ATOMIC_RELAXED, __HIP_MEMORY_SCOPE_AGENT);
                u32x4 O[8];
                acc_to_raw(acc, xh, poison, O);
                rec = encode_store(O, ws + OFF_PY + (size_t)((seq & 1) * 8 + dest) * (4 * MiB) + (size_t)(id0 + xh) * 8192, sc, lane,
                                   [&] { if (xh == 1) { WAIT_V(0); put_record(ry + (__builtin_amdgcn_readfirstlane(pos[0]) & 511u) * 64, seq, rec, lane, 8); } });
                if (lane == 0) rec |= (u32)(id0 + xh) << 20;
            }
            WAIT_V(0);
            put_record(ry + (__builtin_amdgcn_readfirstlane(pos[1]) & 511u) * 64, seq, rec, lane, 8);
        }
        STAMP(5);
        s0 = pn < HC ? 63 + (pn * 14 + pmi) * 2 : -1;
        for (bool first = pn < HC + NSTAT; s0 < 0; first = false) {
            __syncthreads();
            if (threadIdx.x == 0) {
                volatile LAS u32* t = (volatile LAS u32*)(lds + LDS_GEMM);
                t[0] = first ? (u32)((pn - HC) * 14 + pmi)
                             : 14u * NSTAT + __hip_atomic_fetch_add((GAS u32*)(ws + OFF_TICKET), 1u, __ATOMIC_RELAXED, __HIP_MEMORY_SCOPE_AGENT) - (seq - 1u) * (NGEMM - 14 * HC + 128 - 14 * NSTAT);
            }
            __syncthreads();
            const u32 tk = __builtin_amdgcn_readfirstlane(*(volatile LAS u32*)(lds + LDS_GEMM));
            if (tk >= 128u) break;
            const int rq = tk & 3, cq = tk >> 2, opm = p.rank * 2 + (rq >> 1), opn = cq >> 1;
            gemm_piece(ws, lds, ws + OFF_G + (size_t)opm * 32768 + (rq & 1) * 16384, ws + OFF_G + (size_t)(16 + opn) * 32768 + (cq & 1) * 16384,
                       (u32)(opm * 128 + (rq & 1) * 8), (u32)((16 + opn) * 128 + (cq & 1) * 8),
                       (u32)(7 * 512 + ((opn * 2 + (rq >> 1)) * 8 + (cq & 1) * 4) * 2 + (rq & 1)), seq, wid, lane, acc);
            STAMP(6);
            LAS f32x4* red = (LAS f32x4*)(lds + (wid & 3) * 16384) + lane;
            if (wid >= 4) {
#pragma unroll
                for (int i = 0; i < 16; ++i) red[i * 64] = acc[i >> 3][0][(i >> 1) & 3][i & 1];
            }
            __syncthreads();
            if (wid < 4) {
#pragma unroll
                for (int i = 0; i < 16; ++i) acc[i >> 3][0][(i >> 1) & 3][i & 1] += red[i * 64];
                const bool poison = __builtin_amdgcn_readfirstlane(ld_agent(ws + OFF_TMO)) == seq;
                u32x4 O[8]; acc_to_raw(acc, 0, poison, O);
                char* dst = p.out + (size_t)(rq * 128 + wid * 32) * 65536 + (p.rank * 4096 + cq * 128) * 2;
                const u32 vo = (u32)(lane >> 4) * 262144u + (u32)(lane & 15) * 16u;
#pragma unroll
                for (int q = 0; q < 8; ++q) st16_sys<0>(dst + (size_t)((q >> 2) * 16 + (q & 3)) * 65536, vo, O[q]);
            }
            STAMP(7);
        }
    }
    if (s0 >= 0) comm_role(p, ws, lds, s0, seq, wid, lane, 0);
    __syncthreads();
    if (threadIdx.x == 0) st_agent(ws + OFF_CNT + b * 64, seq);
}

extern "C" void kernel_launch(int rank, int nranks, ncclComm_t comm, hipStream_t stream,
                              void* const* d_in, const int* in_sizes, int n_in,
                              void* d_out, int out_size, const int* in_dtypes, int out_dtype,
                              void* const* d_out_all, void* const* d_ws_all, size_t ws_bytes) {
    if (nranks != NR || (unsigned)rank >= (unsigned)NR || n_in != 2 || ws_bytes < WS_NEED || in_sizes[0] != 4096 * 512 || in_sizes[1] != 512 * 32768 || out_size != 512 * 32768) {
        fprintf(stderr, "kernel_launch: unexpected shapes\n");
        return;
    }
    HIP_CHECK(hipFuncSetAttribute((const void*)fused_gemm_rs, hipFuncAttributeMaxDynamicSharedMemorySize, LDS_BYTES));
    Params p;
    memset(&p, 0, sizeof p);
    p.x = (const char*)d_in[0]; p.w = (const char*)d_in[1]; p.out = (char*)d_out;
    for (int i = 0; i < NR; ++i) p.wsr[i] = (char*)d_ws_all[(rank + 1 + i) % NR];
    p.ws_tab = (char* const*)d_ws_all;
    p.rank = rank; p.wsme = (char*)d_ws_all[rank];
    int key[256];
    for (int i = 0; i < 256; ++i) {
        const int j = i >> 5, k = i & 31;
        key[i] = k >= 16 ? 112 * j + SKEW_NUM * (k - 16) - 1 : (k >= 14 ? XOWN + j : (k >= 12 || (j == 0 && k < NPUSH)) ? 200000 : 112 * j + XSPREAD * (k >> 1) - (j == 0 ? 1000 : 0));
        int q = i;
        while (q > 0 && key[p.order[q - 1]] > key[i]) { p.order[q] = p.order[q - 1]; --q; }
        p.order[q] = (unsigned char)i;
    }
    const char* np = NPASS; const char* ndg = NDIAG;
    int base = 0;
    for (int ps = 0; ps < 2; ++ps) {
        int n = 0;
        for (int c = 0; c < 16; ++c) if (np[c] - '0' > ps) n += 14;
        for (int c = 0, o = 0; c < 16; ++c) { const bool in = np[c] - '0' > ps; p.sched[c][0] |= (u32)(in ? base + o : NPLAIN) << (16 * ps); if (in) o += 14; }
        p.pstride[ps] = n;
        base += n * 8;
    }
    int nq = 0;
    for (int r = 1; r <= 3; ++r) for (int c = 0; c < 16; ++c) { const int n = ndg[c] - '0'; p.sched[c][1] |= (u32)n; if (n >= r) p.sched[c][1] |= (u32)nq++ << (8 * r); }
    if (base < NPLAIN || nq != 16) { fprintf(stderr, "kernel_launch: bad schedule\n"); return; }
    hipLaunchKernelGGL(fused_gemm_rs, dim3(256), dim3(512), LDS_BYTES, stream, p);
    HIP_CHECK(hipGetLastError());
}
```

```cpp
#include <hip/hip_runtime.h>
#include <rccl/rccl.h>
#include <cstdio>
#include <cstdint>
#include <cstring>

#define HIP_CHECK(expr) do { hipError_t _e = (expr); if (_e != hipSuccess) { \
    fprintf(stderr, "HIP error %s at %s:%d\n", hipGetErrorString(_e), __FILE__, __LINE__); \
    return; } } while (0)

typedef unsigned u32;
typedef unsigned long long u64;
typedef u32 u32x4 __attribute__((ext_vector_type(4)));
typedef u32 u32x2 __attribute__((ext_vector_type(2)));
typedef short bf16x8 __attribute__((ext_vector_type(8)));
typedef float f32x4 __attribute__((ext_vector_type(4)));
typedef float f32x2 __attribute__((ext_vector_type(2)));
#define LAS __attribute__((address_space(3)))
#define GAS __attribute__((address_space(1)))

constexpr int GAUX = 17;
constexpr int HC = 1;
constexpr int NSTAT = 4;
constexpr int NPUSH = 9;
constexpr int NPLAIN = 1024;
constexpr int SKEW_NUM = 47, XSPREAD = 12;
static const char NPASS[] = "1120000111111000";
static const char NDIAG[] = "0001111111111222";
constexpr u32 TS_A = 120, TS_B = 60, TS_D = 24, TS_Q = 150, TS_P = 240;

constexpr int NR = 8;
constexpr size_t MiB = 1ull << 20;
constexpr size_t OFF_CNT = 0;
constexpr size_t OFF_TMO = 16384;
constexpr size_t OFF_TICKET = 16384 + 256;
constexpr size_t OFF_YO = 16384 + 512;
constexpr size_t OFF_YT = 16384 + 2048;
constexpr size_t OFF_YQ = 16384 + 4096;
constexpr size_t OFF_PD = 16384 + 6144;
constexpr size_t OFF_FG = 65536;
constexpr size_t OFF_RIN = 1 * MiB;
constexpr size_t OFF_RY = 2 * MiB;
constexpr size_t OFF_XF = 3 * MiB;
constexpr size_t OFF_PX = 4 * MiB;
constexpr size_t OFF_PW = 12 * MiB;
constexpr size_t OFF_PY = 76 * MiB;
constexpr size_t OFF_G = 140 * MiB;
constexpr size_t OFF_P = 211 * MiB;
constexpr size_t WS_NEED = 239 * MiB;
constexpr u64 DEADLINE = 40000000ull;
constexpr int LDS_GEMM = 131072, LDS_BYTES = 163840;
constexpr int NGEMM = 224;

struct Params {
    const char* x; const char* w; char* out; char* wsme;
    char* const* ws_tab;
    int rank; int pstride[2]; int pad[3];
    char* wsr[NR];
    u32 sched[16][2];
    unsigned char order[256];
};

#define WAIT_V(n) asm volatile("s_waitcnt vmcnt(" #n ")" ::: "memory")
#define WAIT_P __builtin_amdgcn_s_waitcnt(0x0F70)
#define WAIT_L(n) asm volatile("s_waitcnt lgkmcnt(" #n ")" ::: "memory")
#define BAR __builtin_amdgcn_s_barrier()
#define SCHED __builtin_amdgcn_sched_barrier(0)

template <int OFF> __device__ __forceinline__ void st16_sys(const char* base, u32 voff, u32x4 v) {
    asm volatile("s_nop 4\n\tglobal_store_dwordx4 %0, %1, %2 offset:%3 sc0 sc1\n\ts_nop 1" :: "v"(voff), "v"(v), "s"(base), "i"(OFF) : "memory");
}
__device__ __forceinline__ const char* uni(const char* p) {
    return (const char*)(((u64)(u32)__builtin_amdgcn_readfirstlane((u32)((u64)p >> 32)) << 32) | (u32)__builtin_amdgcn_readfirstlane((u32)(u64)p));
}
__device__ __forceinline__ u32 perm(u32 hi, u32 lo, u32 sel) { return __builtin_amdgcn_perm(hi, lo, sel); }
__device__ __forceinline__ u32 ld_agent(const char* p) { return __hip_atomic_load((GAS u32*)p, __ATOMIC_RELAXED, __HIP_MEMORY_SCOPE_AGENT); }
__device__ __forceinline__ void st_agent(char* p, u32 v) { __hip_atomic_store((GAS u32*)p, v, __ATOMIC_RELAXED, __HIP_MEMORY_SCOPE_AGENT); }
__device__ __forceinline__ u32 gslot(u32 o, u32 seq) { return o + ((seq & 1u) && o - 8u < 7u ? 56u : 0u); }
typedef __bf16 bf16x2 __attribute__((ext_vector_type(2)));
__device__ __forceinline__ u32 cvt_pk_bf16(float lo, float hi) { return __builtin_bit_cast(u32, __builtin_convertvector((f32x2){lo, hi}, bf16x2)); }

__device__ __forceinline__ void store_unit_linear(const char* dst, u32 voff, const u32x4 (&O)[8]) {
    const char* b = uni(dst + 4096);
    st16_sys<-4096>(b, voff, O[0]); st16_sys<-3072>(b, voff, O[1]); st16_sys<-2048>(b, voff, O[2]); st16_sys<-1024>(b, voff, O[3]);
    st16_sys<0>(b, voff, O[4]); st16_sys<1024>(b, voff, O[5]); st16_sys<2048>(b, voff, O[6]); st16_sys<3072>(b, voff, O[7]);
}

constexpr u32 H_RAW = 1u << 11, H_DEAD = 1u << 12, H_L3 = 1u << 17;
constexpr int ENC_BYTES = 1152;
__device__ __forceinline__ u32 bfi(u32 m, u32 a, u32 b) { return (a & m) | (b & ~m); }
__device__ __forceinline__ u32 scan32(u32 v) {
    v += (u32)__builtin_amdgcn_update_dpp(0, (int)v, 0x111, 0xf, 0xf, false);
    v += (u32)__builtin_amdgcn_update_dpp(0, (int)v, 0x112, 0xf, 0xf, false);
    v += (u32)__builtin_amdgcn_update_dpp(0, (int)v, 0x114, 0xf, 0xf, false);
    v += (u32)__builtin_amdgcn_update_dpp(0, (int)v, 0x118, 0xf, 0xf, false);
    v += (u32)__builtin_amdgcn_update_dpp(0, (int)v, 0x142, 0xa, 0xf, false);
    return v;
}
__device__ __forceinline__ void enc_init(LAS unsigned char* sc, int lane) {
    u32 sel = 0x0c0c0c0cu;
#pragma unroll
    for (int b = 3; b >= 0; --b) if (lane & (1 << b)) sel = (sel << 8) | (u32)b;
    if (lane < 16) ((LAS u32*)(sc + 1088))[lane] = sel;
}
template <class F>
__device__ __forceinline__ u32 encode_store(const u32x4 (&O)[8], const char* slot, LAS unsigned char* sc, int lane, F before_stores) {
    u32 d[16], S[16];
    u32 sum = 0;
#pragma unroll
    for (int t = 0; t < 16; ++t) {
        const u32 a = O[2 * (t >> 2)][t & 3], b = O[2 * (t >> 2) + 1][t & 3];
        const u32 ra = __builtin_amdgcn_alignbit(a, a, 31), rb = __builtin_amdgcn_alignbit(b, b, 31);
        d[t] = perm(rb, ra, 0x07050301u); S[t] = perm(rb, ra, 0x06040200u);
    }
#pragma unroll
    for (int t = 0; t < 16; t += 4) sum = __builtin_amdgcn_sad_u8(d[t], 0u, sum);
    sum = scan32(sum);
    const u32 tot = (u32)__builtin_amdgcn_readlane((int)sum, 31) + (u32)__builtin_amdgcn_readlane((int)sum, 63);
    const u32 T = min(max((tot >> 10) + 4u, 15u), 244u), T4 = T * 0x01010101u;
    *(LAS u32x4*)(sc + lane * 16) = (u32x4){0u, 0u, 0u, 0u};
    if (lane == 0) *(LAS u32x4*)(sc + 1024) = (u32x4){0u, 0u, 0u, 0u};
    u32 acc = 0;
#pragma unroll
    for (int t = 0; t < 16; ++t) { d[t] = T4 - d[t]; acc |= d[t]; }
    u32 rec = 0, head = T;
    bool raw = false;
    u64 xmask = 0;
    u32 xt = 0, xb = 0, xe = 0;
    if (__builtin_amdgcn_ballot_w64((acc & 0xf0f0f0f0u) != 0u) != 0ull) {
        u32 n = 0, sM = 0;
#pragma unroll
        for (int t = 0; t < 16; ++t) {
            const u32 e = T4 - d[t];
            const u32 w = ((T4 | 0x80808080u) - (e & 0x7f7f7f7fu)) ^ ((T4 ^ ~e) & 0x80808080u);
            const u32 mis = (((w >> 4) & 0x0f0f0f0fu) + 0x0f0f0f0fu) & 0x10101010u;
            d[t] = (w & 0x0f0f0f0fu) | (mis >> 1);
            if (mis) { ++n; xe = e; sM = mis; xt = t; }
        }
        xb = (n ? (u32)__builtin_ctz(sM) : 0u) & 24u;
        xmask = __builtin_amdgcn_ballot_w64(n != 0u);
        if (__builtin_popcountll(xmask) > 7 || __builtin_amdgcn_ballot_w64(n > 1u || (sM & (sM - 1u)) != 0u) != 0ull) raw = true;
    }
    u32x4 Q; u32 cs[16], bo[16], bits = 0;
#pragma unroll
    for (int t = 0; t < 16; ++t) {
        const u32 sel = d[t] + (d[t] & 0x08080808u);
        const u32 r = perm(0xffffffaau, 0x5500ffffu, sel), r2 = perm(0xffffff00u, 0x0000ffffu, sel);
        Q[t >> 2] = (t & 3) ? bfi(0x03030303u << (2 * (t & 3)), r, Q[t >> 2]) : r;
        cs[t] = ((const LAS u32*)(size_t)__builtin_amdgcn_sad_u8(r2 & 0x20100804u, 0u, (u32)(size_t)sc))[272];
        bo[t] = bits;
        bits += (u32)__builtin_popcount(r2);
    }
    const u32 inc = scan32(bits);
    const u32 nb = max((u32)__builtin_amdgcn_readlane((int)inc, 31), (u32)__builtin_amdgcn_readlane((int)inc, 63)) >> 3;
    if (nb > 960u) raw = true;
    if (xmask && !raw) {
        u32 pos = (inc - bits) >> 3;
#pragma unroll
        for (int i = 0; i < 4; ++i) {
            const u32 f = 2u * (xt & 3u), m = (u32)i < (xt >> 2) ? ~0u : (u32)i > (xt >> 2) ? 0u : ((1u << f) - 1u) * 0x01010101u | ((0x01010101u << f) & ((1u << xb) - 1u));
            pos += (u32)__builtin_popcount(Q[i] & (Q[i] >> 1) & 0x55555555u & m);
        }
        const u32 ent = ((u32)(lane & 32) * 32u + pos) << 8 | ((xe >> xb) & 255u);
        head |= (u32)__builtin_popcountll(xmask) << 8;
        for (u32 s = 1; xmask; ++s, xmask &= xmask - 1) {
            const u32 e = (u32)__builtin_amdgcn_readlane((int)ent, __builtin_ctzll(xmask));
            if ((u32)(lane & 7) == s) rec = e;
        }
    }
    before_stores();
    const u32 voff = (u32)lane * 16u;
    if (raw) {
        store_unit_linear(slot, voff, O);
        head = H_RAW;
    } else {
        const char* b = uni(slot + 4096);
        st16_sys<-4096>(b, voff, (u32x4){S[0], S[1], S[2], S[3]}); st16_sys<-3072>(b, voff, (u32x4){S[4], S[5], S[6], S[7]});
        st16_sys<-2048>(b, voff, (u32x4){S[8], S[9], S[10], S[11]}); st16_sys<-1024>(b, voff, (u32x4){S[12], S[13], S[14], S[15]});
        st16_sys<0>(b, voff, Q);
        const u32 base = inc - bits + (lane & 32 ? 4u : 0u) + (u32)(size_t)sc * 8u;
#pragma unroll
        for (int t = 0; t < 16; ++t) {
            const u32 o = base + bo[t];
            const u64 x = (u64)perm(0u, d[t] ^ 0x05050505u, cs[t]) << (o & 31u);
            LAS u32* q = (LAS u32*)(size_t)((o >> 3) & ~3u);
            __hip_atomic_fetch_or(q, (u32)x, __ATOMIC_RELAXED, __HIP_MEMORY_SCOPE_WAVEFRONT);
            __hip_atomic_fetch_or(q + 1, (u32)(x >> 32), __ATOMIC_RELAXED, __HIP_MEMORY_SCOPE_WAVEFRONT);
        }
        asm volatile("" ::: "memory");
        u32 nl = (nb + 127u) >> 7, from = 0;
        const u32 cnt = (head >> 8) & 7u, rem = nb & 127u;
        if (rem && rem <= 28u - 4u * cnt) {
            --nl;
            const u32 i = (u32)(lane & 7) - cnt - 1u, ov = (rem + 3u) >> 2;
            if (i < ov) rec = *(const LAS u32*)(sc + nl * 128u + i * 4u);
            head |= ov << 29;
        }
        if (nl == 4u) {
            const u32x2 v = *(const LAS u32x2*)(sc + lane * 8);
            const u32 g[4] = {v[0] & 0x0f0f0f0fu, v[1] & 0x0f0f0f0fu, (v[0] >> 4) & 0x0f0f0f0fu, (v[1] >> 4) & 0x0f0f0f0fu};
            u32 q2 = 0, c2[4], b2[4], bits2 = 0;
#pragma unroll
            for (int t = 0; t < 4; ++t) {
                const u32 sel = g[t] + (g[t] & 0x08080808u);
                const u32 r = perm(0xffffffaau, 0x55ffff00u, sel), r2 = perm(0xffffff00u, 0x00ffff00u, sel);
                q2 = t ? bfi(0x03030303u << (2 * t), r, q2) : r;
                c2[t] = ((const LAS u32*)(size_t)__builtin_amdgcn_sad_u8(r2 & 0x20100804u, 0u, (u32)(size_t)sc))[272];
                b2[t] = bits2;
                bits2 += (u32)__builtin_popcount(r2);
            }
            const u32 inc2 = scan32(bits2);
            if (max((u32)__builtin_amdgcn_readlane((int)inc2, 31), (u32)__builtin_amdgcn_readlane((int)inc2, 63)) <= 1024u) {
                const u32 base2 = inc2 - bits2 + (lane & 32 ? 4u : 0u) + ((u32)(size_t)sc + 896u) * 8u;
                *(LAS u32*)(sc + 640 + lane * 4) = q2;
#pragma unroll
                for (int t = 0; t < 4; ++t) {
                    const u32 o = base2 + b2[t];
                    const u64 x = (u64)perm(0u, g[t], c2[t]) << (o & 31u);
                    LAS u32* q = (LAS u32*)(size_t)((o >> 3) & ~3u);
                    __hip_atomic_fetch_or(q, (u32)x, __ATOMIC_RELAXED, __HIP_MEMORY_SCOPE_WAVEFRONT);
                    __hip_atomic_fetch_or(q + 1, (u32)(x >> 32), __ATOMIC_RELAXED, __HIP_MEMORY_SCOPE_WAVEFRONT);
                }
                asm volatile("" ::: "memory");
                nl = 3; from = 640; head |= H_L3;
            }
        }
        if ((u32)lane < nl * 8u) st16_sys<1024>(b, voff, *(const LAS u32x4*)(sc + from + lane * 16));
        head |= nl << 13;
    }
    if ((lane & 7) == 0) rec = head;
    return rec;
}
__device__ __forceinline__ void put_record(char* p, u32 seq, u32 w, int lane, int nl) {
    const u32 hd = __builtin_amdgcn_readfirstlane(w), cnt = ((hd >> 8) & 7u) + (hd >> 29);
    if (lane < nl && (u32)(lane & 7) <= cnt) __hip_atomic_store((GAS u64*)p, ((u64)w << 32) | seq, __ATOMIC_RELAXED, __HIP_MEMORY_SCOPE_SYSTEM);
}

__device__ __forceinline__ int x_task(int t, int rank) {
    if (t >= 512) return ((t >> 5) & 7) << 10 | (64 + ((t >> 8) & 1) * 32 + (t & 31));
    if (t - 112u < 64u) return ((t - 112) >> 3) << 10 | (rank * 8 + (t & 7));
    const int v = t < 112 ? t : t - 64;
    return (v / 56) << 10 | ((rank * 8 + 8 + v % 56) & 63);
}
__device__ __forceinline__ void task_load(const Params& p, int code, LAS unsigned char* buf, int lane) {
    const int j = code >> 10, r = code & 1023, un = r & 3;
    const char* s0; const char* s1; u32 stride;
    if (r < 64) {
        const char* src = p.x + (size_t)((r >> 2) * 256 + un * 64) * 1024 + j * 128 + (lane >> 3) * 1024;
        const int sw = (lane & 7) ^ (lane >> 4);
        s0 = src + (sw << 4); s1 = src + ((sw ^ 4) << 4); stride = 8192;
    } else {
        s0 = s1 = p.w + (size_t)(j * 64 + (un & 1) * 32 + (lane >> 4) * 8) * 65536 + (((p.rank + ((r - 64) >> 6)) & 7) * 4096 + ((r >> 2) & 15) * 256 + (un >> 1) * 128) * 2 + (lane & 15) * 16;
        stride = 65536;
    }
#pragma unroll
    for (int q = 0; q < 8; ++q) __builtin_amdgcn_global_load_lds((const GAS u32*)(((q & 1) ? s1 : s0) + q * stride), (LAS u32*)(buf + q * 1024), 16, 0, 0);
}
__device__ __forceinline__ u32 task_time(const Params& p, int code) {
    const u32 j = code >> 10, r = code & 1023, c = (r >> 2) & 15;
    if (r < 64) return (c >> 1) == (u32)p.rank ? TS_A : j < 2 ? 0u : TS_A + j * TS_B;
    if (r < 128) return TS_A + j * TS_B + TS_B / 2 + (c >> 3) * 8 * TS_B;
    return c < 4 ? (j < 2 ? 0u : TS_A) : TS_A + c * TS_D + (j & 3) * (TS_D / 4);
}
__device__ __forceinline__ void sleep_until(u32 t) {
    for (int n = (int)(t - (u32)__builtin_amdgcn_s_memrealtime()); n > 11; n -= 23) __builtin_amdgcn_s_sleep(8);
}
template <class Next>
__device__ __forceinline__ void pack_tasks(const Params& p, char* ws, char* wd, Next next, u32 cnt_v, u32 tl, LAS unsigned char* lds, LAS unsigned char* sc, int wid, int lane) {
    const u32 voff = (u32)lane * 16u;
    int code = __builtin_amdgcn_readfirstlane(next(0));
    if (code < 0) return;
    auto place = [&](int c) { return lds + ((c & 1023) >= 128 ? (c >> 10) * 2 + (c & 1) : wid + 8) * 8192; };
    sleep_until(tl + task_time(p, code));
    task_load(p, code, place(code), lane);
    char* peer = p.ws_tab[(p.rank + 1 + (lane >> 3)) & 7];
    asm volatile("" : "+v"(peer), "+v"(cnt_v) :: "memory");
    const u32 seq = __builtin_amdgcn_readfirstlane(cnt_v) + 1u;
    WAIT_P; WAIT_V(0);
    char* rp = nullptr; u32 rec = 0; int nl = 0;
    for (int pass = 0;; ++pass) {
        const int j = code >> 10, r = code & 1023, un = r & 3, dd = (r - 64) >> 6, b16 = (r >> 2) & 15;
        const bool isx = r < 64, enc = isx ? (b16 >> 1) != p.rank : dd != 0;
        const bool urgent = j < 2 && (isx || b16 < 4);
        LAS unsigned char* buf = place(code);
        u32x4 O[8];
        {
            u32x4 in[8];
#pragma unroll
            for (int q = 0; q < 8; ++q) in[q] = *(const LAS u32x4*)(buf + q * 1024 + voff);
            WAIT_L(0);
            if (isx) {
#pragma unroll
                for (int q = 0; q < 8; ++q) O[q] = in[q];
            } else {
#pragma unroll
                for (int d = 0; d < 4; ++d)
#pragma unroll
                    for (int q = 0; q < 4; ++q) { O[2 * d][q] = perm(in[2 * q + 1][d], in[2 * q][d], 0x05040100u); O[2 * d + 1][q] = perm(in[2 * q + 1][d], in[2 * q][d], 0x07060302u); }
            }
        }
        char* flag = nullptr;
        const int blk = j * 32 + (r >> 2);
        const bool rawlast = urgent && isx && enc;
        if (rawlast) {
        } else if (r < 128) {
            store_unit_linear(ws + OFF_G + (size_t)blk * 32768 + un * 8192, voff, O);
            flag = ws + OFF_FG + blk * 128 + un * 4;
            if (isx && !enc) {
#pragma unroll
                for (int q = 0; q < 8; ++q) O[q] = *(const LAS u32x4*)(buf + ((q & 3) * 16 + (lane & 15)) * 128 + ((((q >> 2) * 4 + (lane >> 4)) ^ ((lane & 15) >> 1)) << 4));
                WAIT_L(0);
                store_unit_linear(ws + OFF_XF + ((r & 7) * 8 + j) * 8192, voff, O);
            }
        }
        if (r >= 128) {
#pragma unroll
            for (int q = 0; q < 8; ++q) *(LAS u32x4*)(buf + q * 1024 + voff) = O[q];
        }
        bool late = urgent;
        code = __builtin_amdgcn_readfirstlane(next(pass + 1));
        if (code >= 0 && !late) {
            late = (u32)__builtin_amdgcn_s_memrealtime() - tl < task_time(p, code);
            if (!late) task_load(p, code, place(code), lane);
        }
        auto settle = [&]() {
            WAIT_V(0);
            if (nl) { put_record(rp, seq, rec, lane, nl); nl = 0; }
            if (flag) { if (lane == 0) st_agent(flag, seq); flag = nullptr; }
        };
        if (enc) {
            const int dest = (p.rank + dd) & 7;
            const char* slot = ws + (isx ? OFF_PX + (size_t)(seq & 1) * (4 * MiB) : OFF_PW + (size_t)((seq & 1) * 8 + dest) * (4 * MiB)) + (size_t)((j * 16 + b16) * 4 + un) * 8192;
            rec = encode_store(O, slot, sc, lane, [&] { WAIT_P; settle(); });
            rp = (isx ? peer : wd) + OFF_RIN + (size_t)(((seq & 1) * 8 + p.rank) * 1024 + (j * 32 + (isx ? 0 : 16) + b16) * 4 + un) * 64 + (lane & 7) * 8;
            nl = isx ? 56 : 8;
        }
        if (!enc || late || code < 0) settle();
        if (rawlast) { store_unit_linear(ws + OFF_G + (size_t)blk * 32768 + un * 8192, voff, O); flag = ws + OFF_FG + blk * 128 + un * 4; settle(); }
        if (code < 0) break;
        if (late) {
            sleep_until(tl + task_time(p, code));
            task_load(p, code, place(code), lane);
            WAIT_P; WAIT_V(0);
        }
    }
}

__device__ __forceinline__ void push_task(const Params& p, int t, u32 seq, u32 tl, int lane) {
    const int dd = t % 7, un = (t / 7) & 3, i = t / 28, dest = (p.rank + 1 + dd) & 7, w = 6 - dd;
    const int pm = (dest * 2 + 2 + (i + 2 * w + 2) % 14) & 15;
    const char* src = p.x + (size_t)(pm * 256 + un * 64) * 1024 + (lane >> 3) * 1024;
    const int sw = (lane & 7) ^ (lane >> 4);
    u32x4 in[8];
#pragma unroll
    for (int q = 0; q < 8; ++q) in[q] = *(const GAS u32x4*)(src + q * 8192 + ((sw ^ ((q & 1) << 2)) << 4));
    const u32 blk = gslot(8 + w, seq) * 32 + pm;
    char* pw = p.wsr[dd];
    WAIT_P;
    for (int n = (int)(tl + TS_Q + (t / 7) * TS_P / 16 - (u32)__builtin_amdgcn_s_memrealtime()); n > 2; n -= 6) __builtin_amdgcn_s_sleep(2);
    store_unit_linear(pw + OFF_G + (size_t)blk * 32768 + un * 8192, (u32)lane * 16u, in);
    WAIT_V(0);
    if (lane == 0) __hip_atomic_store((GAS u32*)(pw + OFF_FG + blk * 128 + un * 4), seq, __ATOMIC_RELAXED, __HIP_MEMORY_SCOPE_SYSTEM);
}

#define WIN 2
#define PACE 978
#define YLEAD 2
#define G1 24
#define G2 30
#define TGRAN 8
#define SLACK 1
constexpr int NSLOT = 8, NDEC = 5, NDESC = 16, NIN = 960 - 4 * NPUSH, LOOK = 8;
constexpr u32 FIN = 0x40000000u;
constexpr int L_SLOT = 0;
constexpr int L_DESC = 2 * NSLOT * 8192;
constexpr int L_CTL = L_DESC + 2 * NDESC * 64;
constexpr int L_ORD = L_CTL + 128;
static_assert(L_ORD + 512 <= LDS_BYTES, "lds");

template <int V> struct IC { static constexpr int v = V; };
__device__ __forceinline__ void wait_units(u32 n) {
    switch (n) {
        case 0: WAIT_V(0); break; case 1: WAIT_V(6); break; case 2: WAIT_V(12); break; case 3: WAIT_V(18); break; case 4: WAIT_V(24); break;
        case 5: WAIT_V(30); break; case 6: WAIT_V(36); break; default: WAIT_V(42); break;
    }
}
__device__ __forceinline__ void comm_role(const Params& p, char* ws, LAS unsigned char* lds, u32 s0, u32 seq, int wid, int lane, u32 tstart) {
    constexpr u32 NY = 9 + 4 * HC;
    if (s0 >= 63) __syncthreads();
    if (s0 < 63 && threadIdx.x < 64) ((LAS u32*)(lds + L_ORD))[threadIdx.x] = ((const u32*)p.order)[threadIdx.x];
    if (threadIdx.x >= 64 && threadIdx.x < 96) ((LAS u32*)(lds + L_CTL))[threadIdx.x - 64] = 0;
    __builtin_amdgcn_s_waitcnt(0x0F70);
    __syncthreads();
    volatile LAS u32* ctl = (volatile LAS u32*)(lds + L_CTL);
    volatile LAS u32* desc = (volatile LAS u32*)(lds + L_DESC);
    const u32 voff = (u32)lane * 16u;
    if (wid == 7) {
        const u32 st = lane >> 5, e = lane & 7, L = s0 + st, src = (p.rank + 1 + L % 7) & 7, j = L / 7;
        const u32 cin = j < 9 ? (NIN - j + 8) / 9 : 0;
        bool open = s0 < 63, dead = false, pd = false;
        const char* rws = p.ws_tab[src];
        const u64 t0 = __builtin_amdgcn_s_memrealtime();
        volatile LAS u32* tk = (volatile LAS u32*)(lds + L_ORD + 256) + st * NDESC;
        GAS u32* yt = (GAS u32*)(ws + OFF_YT + (L % 7) * 128);
        if ((lane & 31) == 0) __hip_atomic_fetch_max(yt, seq << 10, __ATOMIC_RELAXED, __HIP_MEMORY_SCOPE_AGENT);
        if ((lane & 31) < LOOK) tk[(cin + (lane & 31)) % NDESC] = j + NY * (lane & 31);
        u32 x = (lane >> 3) & 3, drawn = cin + LOOK, cnt = (s0 == 62 && st) ? 0u : FIN, idle = 0;
        while (__builtin_amdgcn_ballot_w64(x < cnt) != 0ull) {
            u32 yo = 0, pv = 0;
            if (!open) yo = ld_agent(ws + OFF_YO + (L % 7) * 128);
            const u32 lim = ctl[st * 16 + 9] + LOOK, nd = (open && cnt == FIN && lim > drawn) ? lim - drawn : 0u;
            u32 tb = 0;
            if (drawn == cin + LOOK && __builtin_amdgcn_ballot_w64(nd != 0u) != 0ull) WAIT_V(0);
            if (nd && (lane & 31) == 0) tb = __hip_atomic_fetch_add(yt, nd, __ATOMIC_RELAXED, __HIP_MEMORY_SCOPE_AGENT);
            const bool act = x < cnt && x < min(lim, drawn);
            const u32 u = x < cin ? j + 9 * x : NIN + (tk[x % NDESC] & 511u);
            u32 id;
            if (u < NIN) {
                u32 bk = (lds + L_ORD)[u >> 2];
                if (!(bk & 16)) { const u32 i = bk & 15, pmi = i < 14 ? (i + 2 * (L % 7) + 2) % 14 : i; bk = (bk & ~15u) | ((p.rank * 2 + 2 + pmi) & 15); }
                id = bk * 4u + (u & 3);
            } else id = u - NIN;
            const char* a = u >= NIN ? ws + OFF_RY + (size_t)(((seq & 1) * 8 + src) * 512 + id) * 64 : ws + OFF_RIN + (size_t)(((seq & 1) * 8 + src) * 1024 + id) * 64;
            u64 v = 0;
            if (!pd && lim + LOOK > cin) pv = ld_agent(ws + OFF_PD);
            if (act) v = __hip_atomic_load((GAS u64*)(a + e * 8), __ATOMIC_RELAXED, __HIP_MEMORY_SCOPE_SYSTEM);
            asm volatile("" : "+v"(yo), "+v"(pv), "+v"(v));
            open = open || dead || yo - (seq - 1u) * 9u - (9u * (j - 8u) + NY - 10u) / (NY - 9u) < 10u;
            pd = pd || pv >= seq * (NGEMM * 8u);
            const u32 hw = (u32)__builtin_amdgcn_ds_swizzle((int)(u32)(v >> 32), 0x0018);
            const u64 okm = __builtin_amdgcn_ballot_w64(((u32)v == seq || e > ((hw >> 8) & 7u) + (hw >> 29)) && (u < NIN || pd));
            const bool adv = act && open && (dead || ((okm >> (lane & 56)) & 0xffull) == 0xffull);
            if (__builtin_amdgcn_ballot_w64(nd != 0u) != 0ull) {
                tb = (u32)__builtin_amdgcn_ds_swizzle((int)tb, 0) - (seq << 10) + LOOK * NY;
                if ((u32)(lane & 31) < nd) tk[(drawn + (lane & 31)) % NDESC] = tb + (lane & 31);
                if (nd && tb + nd >= 512u) cnt = drawn + (tb < 512u ? 512u - tb : 0u);
                drawn += nd;
            }
            if (__builtin_amdgcn_ballot_w64(adv) != 0ull) {
                if (adv) {
                    if (u >= NIN && !dead) id = (hw >> 20) & 511u;
                    const size_t po = (size_t)((seq & 1) * 8 + p.rank) * (4 * MiB);
                    const char* s = u >= NIN ? rws + OFF_PY + po + (size_t)id * 8192
                        : rws + ((id & 64) ? OFF_PW + po : OFF_PX + (size_t)(seq & 1) * (4 * MiB)) + (size_t)(((id >> 7) * 16 + ((id >> 2) & 15)) * 4 + (id & 3)) * 8192;
                    const u32 en = (st * NDESC + x % NDESC) * 16, word = dead ? H_DEAD : (u32)(v >> 32);
                    const u64 sa = dead ? (u64)(ws + OFF_G) : (u64)s;
                    desc[en + e] = word;
                    if (e == 0) { desc[en + 8] = (u32)sa; desc[en + 9] = (u32)(sa >> 32); desc[en + 10] = word; desc[en + 11] = (u >= NIN ? 0x8000u : 0u) | id; }
                    if (e == 1) {
                        const u32 blk = gslot(8 + 7 * (id >> 7) + L % 7, seq) * 32 + ((id >> 2) & 31);
                        const u32 row0 = ((id >> 4) & 1) * 256 + (id & 1) * 128 + ((id >> 1) & 3) * 32, col0 = src * 4096 + ((id >> 5) & 15) * 256 + ((id >> 3) & 1) * 128;
                        const u64 da = u >= NIN ? (u64)(p.out + (size_t)row0 * 65536 + col0 * 2) : (u64)(ws + OFF_G + (size_t)blk * 32768 + (id & 3) * 8192 + 4096);
                        desc[en + 12] = (u32)da; desc[en + 13] = (u32)(da >> 32);
                        desc[en + 14] = u >= NIN ? ((L % 7) * 512 + id) * 8192 : (u32)OFF_FG + blk * 128 + (id & 3) * 4;
                    }
                    x += 4;
                }
                u32 m = min(x, (u32)__builtin_amdgcn_update_dpp((int)x, (int)x, 0x128, 0xf, 0xf, false));
                m = min(m, (u32)__builtin_amdgcn_ds_swizzle((int)m, 0x401F));
                WAIT_L(0);
                if ((lane & 31) == 0) ctl[st * 16 + 8] = min(m, cnt);
            } else if ((++idle & 15u) == 0u && (ld_agent(ws + OFF_TMO) == seq || __builtin_amdgcn_s_memrealtime() - t0 > DEADLINE)) {
                st_agent(ws + OFF_TMO, seq); WAIT_V(0); dead = true;
            } else __builtin_amdgcn_s_sleep(1);
        }
        if ((lane & 31) == 0) ctl[st * 16 + 8] = cnt | FIN;
    } else if (wid >= NDEC) {
        const u32 me = wid - NDEC;
        volatile LAS u32* c = ctl + me * 16;
        u32 landed = 0, k = 0, slot = 0, pc = PACE;
        const u32 t0 = (u32)__builtin_amdgcn_s_memrealtime();
        u32 next = 0, now;
        auto clock = [&]() { return ((u32)__builtin_amdgcn_s_memrealtime() - t0) * 64u; };
        auto pace = [&](u32 n64) {
            if ((int)(next - now) > 1100) { __builtin_amdgcn_s_sleep(2); if ((int)(next - now) > 1600) __builtin_amdgcn_s_sleep(2); now = clock(); }
            while ((int)(now - next) < 0) now = clock();
            next += (n64 * pc) >> 4;
        };
        __builtin_amdgcn_s_setprio(3);
        for (;; ++k) {
            volatile LAS u32* rp = (lane & 16) ? desc + (me * NDESC + k % NDESC) * 16 + (lane & 15) : c + (lane & 15);
            u32 v;
            bool fin = false;
            while (true) {
                v = *rp;
                now = clock();
                const u32 pub = __builtin_amdgcn_readlane(v, 8);
                if ((pub & ~FIN) > k && (u32)__builtin_amdgcn_readlane(v, slot) + NSLOT > k) break;
                if (landed < k) {
                    wait_units(k - landed - 1);
                    ++landed;
                    if (lane == 0) c[10] = landed;
                } else if (pub == (k | FIN)) { fin = true; break; }
            }
            if (fin) break;
            const char* s = (const char*)(((u64)(u32)__builtin_amdgcn_readlane(v, 25) << 32) | (u32)__builtin_amdgcn_readlane(v, 24));
            const u32 head = __builtin_amdgcn_readlane(v, 26);
            if (__builtin_amdgcn_readlane(v, 27) & 0x8000) pc = PACE * NY / 9;
            if (s0 < 63 && k + YLEAD == (NIN - (s0 + me) / 7 + 8) / 9 && lane == 0) __hip_atomic_fetch_add((GAS u32*)(ws + OFF_YO + ((s0 + me) % 7) * 128), 1u, __ATOMIC_RELAXED, __HIP_MEMORY_SCOPE_AGENT);
            LAS unsigned char* d = lds + L_SLOT + (me * NSLOT + slot) * 8192;
            if (next + SLACK * PACE / 4 - now > (1u << 20)) next = now - SLACK * PACE / 4;
            u32 n16 = ((head >> 13) & 15u) * 8u;
            if (head & (H_RAW | H_DEAD)) {
                pace(32);
                __builtin_amdgcn_global_load_lds((const GAS u32*)(s + 6144 + voff), (LAS u32*)(d + 6144), 16, 0, 17);
                __builtin_amdgcn_global_load_lds((const GAS u32*)(s + 6144 + voff), (LAS u32*)(d + 6144), 16, 1024, 17);
                n16 = 64; now = clock();
            }
            const u32 tail = ((n16 + TGRAN - 1) / TGRAN) * (TGRAN / 4);
            pace(G1);
            if ((u32)lane < max(n16, 1u)) __builtin_amdgcn_global_load_lds((const GAS u32*)(s + 4096 + voff), (LAS u32*)(d + 4096), 16, 1024, 17);
            __builtin_amdgcn_global_load_lds((const GAS u32*)(s + 4096 + voff), (LAS u32*)(d + 4096), 16, 0, 17);
            now = clock(); pace(G2);
            __builtin_amdgcn_global_load_lds((const GAS u32*)(s + voff), (LAS u32*)d, 16, 0, 17);
            __builtin_amdgcn_global_load_lds((const GAS u32*)(s + voff), (LAS u32*)d, 16, 1024, 17);
            now = clock(); pace(80 + tail - G1 - G2);
            __builtin_amdgcn_global_load_lds((const GAS u32*)(s + voff), (LAS u32*)d, 16, 2048, 17);
            __builtin_amdgcn_global_load_lds((const GAS u32*)(s + voff), (LAS u32*)d, 16, 3072, 17);
            asm volatile("s_waitcnt vmcnt(%0)" :: "i"(6 * WIN) : "memory");
            if (k + 1 > landed + WIN) landed = k + 1 - WIN;
            if (lane == 0) { c[9] = k; c[10] = landed; }
            if (++slot == NSLOT) slot = 0;
        }
        if (lane == 0) c[10] = k | FIN;
    } else {
        if (s0 < 63) for (int t = wid * 32 + s0 / 2; t < 28 * NPUSH; t += 32 * NDEC) push_task(p, t, seq, tstart, lane);
        u32 pend = 0;
        int k = 0;
        u32 k03 = 0x03030303u, kfc = ~3u, khi = 0xffff0000u;
        asm volatile("" : "+s"(k03), "+s"(kfc), "+s"(khi));
        const __amdgpu_buffer_rsrc_t prs = __builtin_amdgcn_make_buffer_rsrc(ws + OFF_P, 0, 28 * (int)MiB, 0x00020000);
        for (;;) {
            const u32 cv = ctl[lane & 31];
            const u32 l0 = __builtin_amdgcn_readlane(cv, 10), e0 = __builtin_amdgcn_readlane(cv, 11), l1 = __builtin_amdgcn_readlane(cv, 26), e1 = __builtin_amdgcn_readlane(cv, 27);
            const u32 a0 = (l0 & ~FIN) - e0, a1 = (l1 & ~FIN) - e1;
            if ((a0 | a1) == 0u) {
                if (l0 & l1 & FIN) break;
                if (pend) { WAIT_V(0); if (lane == 0) st_agent(ws + pend, seq); pend = 0; }
                else __builtin_amdgcn_s_sleep(1);
                continue;
            }
            const u32 st = a1 > a0 || (a1 == a0 && (wid & 1)), e = st ? e1 : e0, sn = e % NSLOT;
            const u32 sb = (u32)(size_t)lds + L_SLOT + (st * NSLOT + sn) * 8192u;
            const LAS u32x4* sl = (const LAS u32x4*)(size_t)(sb + voff);
            asm volatile("" ::: "memory");
            u32 got = e;
            if (lane == 0) __hip_atomic_compare_exchange_strong((LAS u32*)(lds + L_CTL + st * 64 + 44), &got, e + 1, __ATOMIC_RELAXED, __ATOMIC_RELAXED, __HIP_MEMORY_SCOPE_WORKGROUP);
            u32 rec = desc[(st * NDESC + e % NDESC) * 16 + (lane & 15)], q2 = *(const LAS u32*)(size_t)(sb + 5120u + lane * 4u);
            u32x4 Q = sl[4 * 64], S[4];
#pragma unroll
            for (int q = 0; q < 4; ++q) S[q] = sl[q * 64];
            asm volatile("" : "+v"(got), "+v"(rec), "+v"(q2), "+v"(Q) :: "memory");
            if (__builtin_amdgcn_readfirstlane(got) != e) continue;
            const u32 ent = __builtin_amdgcn_readlane(rec, 11), aux = __builtin_amdgcn_readlane(rec, 14), head = __builtin_amdgcn_readfirstlane(rec);
            char* dst = (char*)(((u64)(u32)__builtin_amdgcn_readlane(rec, 13) << 32) | (u32)__builtin_amdgcn_readlane(rec, 12));
            u32x4 O[8];
            const u32 cnt = (head >> 8) & 7u;
            ++k;
            auto unpack = [&](auto emit) __attribute__((always_inline)) {
            if (head & (H_RAW | H_DEAD)) {
#pragma unroll
                for (int q = 0; q < 8; ++q) O[q] = q < 4 ? S[q] : q == 4 ? Q : sl[q * 64];
                WAIT_L(0);
                if (lane == 0) ctl[st * 16 + sn] = e + 1;
                if (head & H_DEAD) {
#pragma unroll
                    for (int q = 0; q < 8; ++q) O[q] = (u32x4){0x7fc07fc0u, 0x7fc07fc0u, 0x7fc07fc0u, 0x7fc07fc0u};
                }
                emit(IC<0>{}); emit(IC<1>{}); emit(IC<2>{}); emit(IC<3>{}); emit(IC<4>{}); emit(IC<5>{}); emit(IC<6>{}); emit(IC<7>{});
            } else {
                u32 c[16], m[16], bo[16];
                u32x2 w[16];
                const u32 T4 = (head & 0xffu) * 0x01010101u;
                auto expo = [&](u32 v) { return T4 - ((v & 0x0f0f0f0fu) ^ 0x05050505u); };
                const u32 oi = (u32)lane - cnt - 1u;
                u32 n = (u32)__builtin_popcount(q2 & (q2 >> 1) & 0x55555555u) << 16;
#pragma unroll
                for (int q = 0; q < 4; ++q) n += (u32)__builtin_popcount(Q[q] & (Q[q] >> 1) & 0x55555555u);
                n = scan32(n) - n;
#define ALL16(stmt) do { _Pragma("unroll") for (int t = 0; t < 16; ++t) { stmt; } SCHED; } while (0)
                u32 c2[4], m2[4], b2[4];
                u32x2 w2[4];
                {
                    u32 bits2 = (min(n >> 16, 128u) + sb + 5376u) * 8u + (lane & 32 ? 4u : 0u);
#pragma unroll
                    for (int t = 0; t < 4; ++t) {
                        c2[t] = (q2 >> (2 * t)) & k03; m2[t] = perm(0u, 0x01000000u, c2[t]);
                        b2[t] = bits2;
                        const LAS u32* q = (const LAS u32*)(size_t)((bits2 >> 3) & kfc);
                        w2[t] = (u32x2){q[0], q[1]};
                        bits2 = __builtin_amdgcn_udot4(m2[t], 0x08080808u, bits2, false);
                    }
                    SCHED;
                }
                ALL16(c[t] = (Q[t >> 2] >> (2 * (t & 3))) & k03; m[t] = perm(0u, 0x01000000u, c[t]));
                u32 pos = min(n & 0xffffu, 1024u - 64u) + sb + (lane & 32 ? 7168u : 6144u);
                u32 ad[16], x1[16];
                ALL16(bo[t] = pos; ad[t] = pos & kfc; pos = __builtin_amdgcn_udot4(m[t], 0x01010101u, pos, false));
                if (head & H_L3) {
                    u32 g[4];
#pragma unroll
                    for (int t = 0; t < 4; ++t) {
                        asm("v_mul_u32_u24 %0, 0x10101, %1" : "=v"(m2[t]) : "v"(m2[t]));
                        g[t] = T4 - ((perm(0u, __builtin_amdgcn_alignbit(w2[t][1], w2[t][0], b2[t]), (m2[t] << 8) + perm(0u, 0x00101010u, c2[t])) & perm(0u, 0x0f040300u, c2[t])) ^ 0x05050505u);
                    }
                    *(LAS u32x2*)(size_t)(sb + 6144u + lane * 8u) = (u32x2){g[0], g[1]};
                    *(LAS u32x2*)(size_t)(sb + 7168u + lane * 8u) = (u32x2){g[2], g[3]};
                    if (head >> 29) {
                        if (oi < (head >> 29)) { *(LAS u32*)(size_t)(sb + 6656u + oi * 4u) = expo(rec); *(LAS u32*)(size_t)(sb + 7680u + oi * 4u) = expo(rec >> 4); }
                    }
                } else {
                    if (oi < (head >> 29)) *(LAS u32*)(size_t)(sb + 5120u + ((head >> 13) & 7u) * 128u + oi * 4u) = rec;
                    asm volatile("" ::: "memory");
                    const u32x4 v = sl[5 * 64];
                    *(LAS u32x4*)(size_t)(sb + 6144u + voff) = (u32x4){expo(v[0]), expo(v[1]), expo(v[2]), expo(v[3])};
                    *(LAS u32x4*)(size_t)(sb + 7168u + voff) = (u32x4){expo(v[0] >> 4), expo(v[1] >> 4), expo(v[2] >> 4), expo(v[3] >> 4)};
                }
                if (cnt) { if ((u32)lane - 1u < cnt) *(LAS unsigned char*)(size_t)(sb + 6144u + ((rec >> 8) & 2047u)) = (unsigned char)rec; }
                asm volatile("" ::: "memory");
                ALL16(w[t] = ((u32x2){((const LAS u32*)(size_t)ad[t])[0], ((const LAS u32*)(size_t)ad[t])[1]}));
                const u32 zl = ~(T4 - 0x00040302u) & 0x00ffffffu;
                ALL16(asm("v_mul_u32_u24 %0, 0x10101, %1" : "=v"(m[t]) : "v"(m[t])); c[t] = perm(0u, zl, c[t]));
                ALL16(x1[t] = c[t] + (m[t] << 8));
                WAIT_L(0);
                if (lane == 0) ctl[st * 16 + sn] = e + 1;
                SCHED;
                auto fin = [&](auto hc) __attribute__((always_inline)) {
                    constexpr int h = decltype(hc)::v, d = h >> 1;
#pragma unroll
                    for (int t = 2 * h; t < 2 * h + 2; ++t) {
                        const u32 E = perm(0u, __builtin_amdgcn_alignbyte(w[t][1], w[t][0], bo[t]), x1[t]) & ~c[t];
                        const u32 ra = perm(E, S[d][t & 3], 0x05010400u), rb = perm(E, S[d][t & 3], 0x07030602u);
                        O[2 * d][t & 3] = __builtin_amdgcn_alignbit(ra, ra, 1); O[2 * d + 1][t & 3] = __builtin_amdgcn_alignbit(rb, rb, 1);
                    }
                    if constexpr (h > 0) emit(IC<h - 1>{});
                    SCHED;
                };
                fin(IC<0>{}); fin(IC<1>{}); fin(IC<2>{}); fin(IC<3>{}); fin(IC<4>{}); fin(IC<5>{}); fin(IC<6>{}); fin(IC<7>{});
                emit(IC<7>{});
            }
            };
            u32 flag = 0;
            if (!(ent & 0x8000u)) {
                unpack([&](auto dc) __attribute__((always_inline)) { constexpr int d = decltype(dc)::v;
                    st16_sys<1024 * d - 4096>(dst, voff, O[d]); });
                flag = aux;
            } else {
                u32x4 P[8];
                const u32 so = aux;
#pragma unroll
                for (int q = 0; q < 8; ++q) P[q] = __builtin_amdgcn_raw_buffer_load_b128(prs, voff + (q & 3) * 1024, so + (q >> 2) * 4096, 17);
                asm volatile("" ::: "memory");
                const u32 vo = (u32)(lane >> 4) * 262144u + (u32)(lane & 15) * 16u;
                unpack([](auto) {});
                WAIT_P;
#pragma unroll
                for (int q = 0; q < 8; ++q) {
#pragma unroll
                    for (int i = 0; i < 4; ++i) {
                        const f32x2 y = (f32x2){__uint_as_float(O[q][i] << 16), __uint_as_float(O[q][i] & khi)} + (f32x2){__uint_as_float(P[q][i] << 16), __uint_as_float(P[q][i] & khi)};
                        O[q][i] = cvt_pk_bf16(y[0], y[1]);
                    }
                    st16_sys<0>(dst + (size_t)((q >> 2) * 16 + (q & 3)) * 65536, vo, O[q]);
                    SCHED;
                }
            }
            if (pend) { WAIT_V(8); if (lane == 0) st_agent(ws + pend, seq); }
            pend = flag;
        }
        if (pend) { WAIT_V(0); if (lane == 0) st_agent(ws + pend, seq); }
    }
}

__device__ __forceinline__ void spin_flags(char* ws, const char* fbase, u32 fvoff, u32 seq) {
    const u64 t0 = __builtin_amdgcn_s_memrealtime();
    while (__builtin_amdgcn_ballot_w64(__hip_atomic_load((GAS u32*)(fbase + fvoff), __ATOMIC_RELAXED, __HIP_MEMORY_SCOPE_SYSTEM) < seq) != 0ull) {
        if (ld_agent(ws + OFF_TMO) == seq || __builtin_amdgcn_s_memrealtime() - t0 > DEADLINE) { st_agent(ws + OFF_TMO, seq); break; }
        __builtin_amdgcn_s_sleep(4);
    }
    __builtin_amdgcn_s_waitcnt(0xC07F);
    WAIT_V(0);
}

__device__ __forceinline__ void gemm_loop(char* ws, LAS unsigned char* lds, const char* gX, const char* gW, u32 fvoff, int otab, int nstep, u32 seq, int wid, int lane, f32x4 (&acc)[2][2][4][2]) {
    constexpr size_t KSTEP = MiB, SX = 16384, SW = 16384;
    constexpr int FSTEP = 4096;
    const int wr = wid >> 2, wc = wid & 3, fr = lane & 15, fq = lane >> 4;
    const u32 voff = (u32)lane * 16u;
    const char* cX = uni(gX + wid * 1024);
    const char* cW = uni(gW + (wid >> 2) * 8192 + (wid & 3) * 1024);
    const char* fb = ws + OFF_FG;
    const u32 ldsw = (u32)wid * 1024u;
    const int woff = wr * 8192 + lane * 16;
    const int xoff0 = (wc * 32 + fr) * 128 + ((fq ^ (fr >> 1)) << 4), xoff1 = xoff0 ^ 64;
    const LAS unsigned char* xb0 = lds + 65536 + xoff0; const LAS unsigned char* xb1 = lds + 65536 + xoff1;
    asm("" : "+v"(xb0), "+v"(xb1));
#define SA(b, h) (((b) * 2 + (h)) * 16384)
#define SB(b, h) ((4 + (b) * 2 + (h)) * 16384)
#define GLDS(src, dst) __builtin_amdgcn_global_load_lds((const GAS u32*)((src) + voff), (LAS u32*)(lds + (dst) + ldsw), 16, 0, GAUX)
#define STW(b, h, ptr) do { GLDS((ptr) + (h) * 4096, SA(b, h)); GLDS((ptr) + (h) * 4096 + SW, SA(b, h) + 8192); } while (0)
#define STX(b, h, ptr) do { GLDS((ptr) + (h) * SX, SB(b, h)); GLDS((ptr) + (h) * SX + 8192, SB(b, h) + 8192); } while (0)
#define LDW(b, h) do { _Pragma("unroll") for (int m = 0; m < 4; ++m) _Pragma("unroll") for (int k = 0; k < 2; ++k) Wt[m][k] = *(const LAS bf16x8*)(lds + SA(b, h) + woff + k * 4096 + m * 1024); } while (0)
#define LDX(dst, b, h) do { _Pragma("unroll") for (int n = 0; n < 2; ++n) { dst[n][0] = *(const LAS bf16x8*)(xb0 + SA(b, h) + n * 2048); dst[n][1] = *(const LAS bf16x8*)(xb1 + SA(b, h) + n * 2048); } } while (0)
#define MMA(h, xh, Xt) do { _Pragma("unroll") for (int m = 0; m < 4; ++m) _Pragma("unroll") for (int n = 0; n < 2; ++n) _Pragma("unroll") for (int k = 0; k < 2; ++k) \
        acc[h][xh][m][n] = __builtin_amdgcn_mfma_f32_16x16x32_bf16(Xt[n][k], Wt[m][k], acc[h][xh][m][n], 0, 0, 0); } while (0)
#define FLAG_LOAD(step) asm volatile("s_nop 4\n\tglobal_load_dword %0, %1, %2 sc0 sc1" : "+v"(fl) : "v"(fvoff), "s"(fb + (size_t)(step) * FSTEP) : "memory")
#define FLAG_CHECK(step) do { if (wid == 0 && __builtin_amdgcn_ballot_w64(fl < seq) != 0ull) spin_flags(ws, fb + (size_t)(step) * FSTEP, fvoff, seq); } while (0)
#pragma unroll
    for (int a = 0; a < 2; ++a)
#pragma unroll
        for (int b = 0; b < 2; ++b)
#pragma unroll
            for (int m = 0; m < 4; ++m)
#pragma unroll
                for (int n = 0; n < 2; ++n) acc[a][b][m][n] = (f32x4){0.f, 0.f, 0.f, 0.f};
    bf16x8 Wt[4][2], X0[2][2], X1[2][2];
    u32 fl = 0;
    if (wid == 0) { spin_flags(ws, fb, fvoff, seq); spin_flags(ws, fb + FSTEP, fvoff, seq); }
    WAIT_P; WAIT_V(0); WAIT_L(0); BAR; asm volatile("" ::: "memory");
#define PRIO(n) __builtin_amdgcn_s_setprio(n)
#define WAIT_VN(n) asm volatile("s_waitcnt vmcnt(" #n ")" : "+v"(fl) :: "memory")
    FLAG_LOAD(2);
    STX(0, 0, cX); STX(0, 1, cX); STW(0, 0, cW); STW(0, 1, cW);
    STX(1, 0, cX + KSTEP); STX(1, 1, cX + KSTEP); STW(1, 0, cW + KSTEP);
    WAIT_VN(8); BAR;
    if (wr == 1) BAR;
    for (int t = 0; t < nstep; t += 2) {
        const int t1 = __builtin_amdgcn_readlane(otab, t + 1), t2 = __builtin_amdgcn_readlane(otab, t + 2);
        const int t3 = __builtin_amdgcn_readlane(otab, t + 3), t4 = __builtin_amdgcn_readlane(otab, t + 4);
        const char* w1 = cW + (size_t)t1 * KSTEP;
        const char* x2 = cX + (size_t)t2 * KSTEP; const char* w2 = cW + (size_t)t2 * KSTEP;
        const char* x3 = cX + (size_t)t3 * KSTEP; const char* w3 = cW + (size_t)t3 * KSTEP;
        FLAG_CHECK(t2);
        LDX(X0, 0, 0); LDW(0, 0); LDX(X1, 0, 1); FLAG_LOAD(t3); STW(1, 1, w1);
        WAIT_VN(9); WAIT_L(0); SCHED; BAR; PRIO(1); SCHED; MMA(0, 0, X0); MMA(0, 1, X1); PRIO(0); SCHED; BAR; SCHED;
        LDW(0, 1); STX(0, 0, x2); STX(0, 1, x2); STW(0, 0, w2);
        WAIT_VN(8); WAIT_L(0); SCHED; BAR; PRIO(1); SCHED; MMA(1, 0, X0); MMA(1, 1, X1); PRIO(0); SCHED; BAR; SCHED;
        FLAG_CHECK(t3);
        LDX(X0, 1, 0); LDW(1, 0); LDX(X1, 1, 1); FLAG_LOAD(t4); STW(0, 1, w2);
        WAIT_VN(9); WAIT_L(0); SCHED; BAR; PRIO(1); SCHED; MMA(0, 0, X0); MMA(0, 1, X1); PRIO(0); SCHED; BAR; SCHED;
        LDW(1, 1); STX(1, 0, x3); STX(1, 1, x3); STW(1, 0, w3);
        WAIT_VN(8); WAIT_L(0); SCHED; BAR; PRIO(1); SCHED; MMA(1, 0, X0); MMA(1, 1, X1); PRIO(0); SCHED; BAR; SCHED;
    }
#undef WAIT_VN
#undef PRIO
    asm volatile("s_waitcnt vmcnt(0)" : "+v"(fl) :: "memory");
    if (wr == 0) BAR;
    BAR;
#undef SA
#undef SB
#undef GLDS
#undef STW
#undef STX
#undef LDW
#undef LDX
#undef MMA
#undef FLAG_LOAD
#undef FLAG_CHECK
}

__device__ __forceinline__ void gemm_piece(char* ws, LAS unsigned char* lds, const char* gX, const char* gW, u32 fx, u32 fw, u32 seq, int wid, int lane, f32x4 (&acc)[2][2][4][2]) {
    const int wr = wid >> 2, wc = wid & 3, fr = lane & 15, fq = lane >> 4;
    const u32 voff = (u32)lane * 16u;
    const char* cX = uni(gX + wc * 4096);
    const char* cW = uni(gW + wc * 4096);
    const int xoff0 = (wc * 32 + fr) * 128 + ((fq ^ (fr >> 1)) << 4), xoff1 = xoff0 ^ 64;
#pragma unroll
    for (int a = 0; a < 2; ++a)
#pragma unroll
        for (int m = 0; m < 4; ++m)
#pragma unroll
            for (int n = 0; n < 2; ++n) acc[a][0][m][n] = (f32x4){0.f, 0.f, 0.f, 0.f};
    const u32 fo = ((wid & 2) ? fw : fx) + (wid & 1) * 4;
    if (wid < 4) spin_flags(ws, ws + OFF_FG, gslot(lane < 36 ? lane : 0, seq) * 4096u + fo, seq);
    WAIT_P; WAIT_V(0); WAIT_L(0); BAR; asm volatile("" ::: "memory");
#define PSRC(base, n) ((base) + (size_t)gslot((n) & 63, seq) * MiB + voff)
#define PLD(src, dst, off) __builtin_amdgcn_global_load_lds((const GAS u32*)(src), (LAS u32*)(dst), 16, off, GAUX)
#define PLD4(src, dst) do { PLD(src, dst, 0); PLD(src, dst, 1024); PLD(src, dst, 2048); PLD(src, dst, 3072); } while (0)
    LAS unsigned char* d0 = lds + wc * 4096;
    PLD4(PSRC(cX, wr), d0 + wr * 32768); PLD4(PSRC(cW, wr), d0 + wr * 32768 + 16384);
    PLD4(PSRC(cX, wr + 2), d0 + (wr + 2) * 32768); PLD4(PSRC(cW, wr + 2), d0 + (wr + 2) * 32768 + 16384);
    WAIT_V(8); BAR;
    if (wr == 1) BAR;
    int buf = wr;
    for (int n = wr; n < 64; n += 2) {
        bf16x8 Wt[8][2], Xt[2][2];
        const LAS unsigned char* bx = lds + buf * 32768;
        LAS unsigned char* dn = d0 + (buf ? buf - 1 : 4) * 32768;
        if (n == 30) spin_flags(ws, ws + OFF_FG, (u32)(lane < 36 ? 36 : lane) * 4096u + fo, seq);
#pragma unroll
        for (int i = 0; i < 2; ++i) { Xt[i][0] = *(const LAS bf16x8*)(bx + xoff0 + i * 2048); Xt[i][1] = *(const LAS bf16x8*)(bx + xoff1 + i * 2048); }
#pragma unroll
        for (int k = 0; k < 2; ++k)
#pragma unroll
            for (int c = 0; c < 8; ++c) Wt[c][k] = *(const LAS bf16x8*)(bx + 16384 + voff + k * 8192 + c * 1024);
        const char* xn = PSRC(cX, n + 4); const char* wn = PSRC(cW, n + 4);
        PLD(xn, dn, 0);
        WAIT_L(0); SCHED; BAR; SCHED;
        __builtin_amdgcn_s_setprio(1);
#pragma unroll
        for (int g = 0; g < 8; ++g) {
#pragma unroll
            for (int c = (g & 3) * 2; c < (g & 3) * 2 + 2; ++c)
#pragma unroll
                for (int i = 0; i < 2; ++i) acc[c >> 2][0][c & 3][i] = __builtin_amdgcn_mfma_f32_16x16x32_bf16(Xt[i][g >> 2], Wt[c][g >> 2], acc[c >> 2][0][c & 3][i], 0, 0, 0);
            SCHED;
            if (g == 0) PLD(wn, dn + 16384, 0); else if (g == 1) PLD(wn, dn + 16384, 1024); else if (g == 2) PLD(wn, dn + 16384, 2048); else if (g == 3) PLD(wn, dn + 16384, 3072);
            else if (g == 4) PLD(xn, dn, 3072); else if (g == 5) PLD(xn, dn, 2048); else if (g == 6) PLD(xn, dn, 1024);
            SCHED;
        }
        __builtin_amdgcn_s_setprio(0);
        WAIT_V(8); BAR; asm volatile("" ::: "memory");
        buf = buf >= 3 ? buf - 3 : buf + 2;
    }
#undef PSRC
#undef PLD
#undef PLD4
    WAIT_V(0);
    if (wr == 0) BAR;
    BAR; asm volatile("" ::: "memory");
}

__device__ __forceinline__ void gemm_local(const Params& p, char* ws, const LAS unsigned char* lds, u32 seq, int wid, int lane, f32x4 (&acc)[2][2][4][2]) {
    const __amdgpu_buffer_rsrc_t xr = __builtin_amdgcn_make_buffer_rsrc(ws + OFF_XF + wid * 65536, 0, 65536, 0x00020000);
    auto ldx = [&](int u, int i) { return __builtin_bit_cast(bf16x8, __builtin_amdgcn_raw_buffer_load_b128(xr, lane * 16 + i * 1024, u * 4096, 17)); };
    if (wid == 0) spin_flags(ws, ws + OFF_FG, (u32)((lane & 7) * 4096 + (p.rank * 2 + (lane >> 5)) * 128 + ((lane >> 3) & 3) * 4), seq);
    __syncthreads();
#pragma unroll
    for (int i = 0; i < 32; ++i) acc[i >> 4][(i >> 3) & 1][(i >> 1) & 3][i & 1] = (f32x4){0.f, 0.f, 0.f, 0.f};
    bf16x8 Xa[4], Xb[4];
#pragma unroll
    for (int i = 0; i < 4; ++i) Xa[i] = ldx(0, i);
    auto step = [&](int u, const bf16x8 (&X)[4]) __attribute__((always_inline)) {
        bf16x8 w[8];
#pragma unroll
        for (int c = 0; c < 8; ++c) w[c] = *(const LAS bf16x8*)(lds + u * 8192 + c * 1024 + lane * 16);
        SCHED;
#pragma unroll
        for (int c = 0; c < 8; ++c)
#pragma unroll
            for (int i = 0; i < 4; ++i) acc[c >> 2][i >> 1][c & 3][i & 1] = __builtin_amdgcn_mfma_f32_16x16x32_bf16(X[i], w[c], acc[c >> 2][i >> 1][c & 3][i & 1], 0, 0, 0);
        SCHED;
    };
#pragma unroll 1
    for (int u = 0; u < 16; u += 2) {
#pragma unroll
        for (int i = 0; i < 4; ++i) Xb[i] = ldx(u + 1, i);
        step(u, Xa);
#pragma unroll
        for (int i = 0; i < 4; ++i) Xa[i] = ldx((u + 2) & 15, i);
        step(u + 1, Xb);
    }
    WAIT_P;
}

__device__ __forceinline__ void acc_to_raw(const f32x4 (&acc)[2][2][4][2], int xh, bool poison, u32x4 (&O)[8]) {
#pragma unroll
    for (int n = 0; n < 2; ++n)
#pragma unroll
        for (int j = 0; j < 4; ++j)
#pragma unroll
            for (int c = 0; c < 4; ++c) O[n * 4 + j][c] = cvt_pk_bf16(acc[c >> 1][xh][(c & 1) * 2][n][j], acc[c >> 1][xh][(c & 1) * 2 + 1][n][j]);
    if (poison) {
        asm volatile("" ::: "memory");
#pragma unroll
        for (int q = 0; q < 8; ++q) O[q] = (u32x4){0x7fc07fc0u, 0x7fc07fc0u, 0x7fc07fc0u, 0x7fc07fc0u};
    }
}

__global__ __launch_bounds__(512, 2) void fused_gemm_rs(Params p) {
    extern __shared__ __attribute__((aligned(16))) unsigned char shm[];
    LAS unsigned char* lds = (LAS unsigned char*)shm;
    const int lane = threadIdx.x & 63, wid = __builtin_amdgcn_readfirstlane(threadIdx.x >> 6), b = blockIdx.x;
    const int xcd = b & 7, ix = b >> 3;
    const int li = ix - 4, cg = li / 7, pmi = ((xcd & 1) * 7 + li % 7) & 15, pn = ((xcd & 6) + cg + (cg < 2 ? 0 : 6)) & 15;
    char* ws = p.wsme;
    char* wd = p.wsr[pmi >> 1];
    u32 sc0 = p.sched[pn][0], sc1 = p.sched[pn][1];
    asm volatile("" : "+s"(ws), "+s"(wd), "+s"(sc0), "+s"(sc1));
    const u32 cnt_v = ld_agent(ws + OFF_CNT + b * 64);
    const u64 t_start = __builtin_amdgcn_s_memrealtime();
    u32 seq;
    int s0 = b * 2;
    if (ix < 4) {
        seq = __builtin_amdgcn_readfirstlane(cnt_v) + 1u;
    } else {
        const u32 tl = (u32)t_start;
        LAS unsigned char* sc = lds + LDS_GEMM + wid * ENC_BYTES;
        enc_init(sc, lane);
        const int dest = (p.rank + 1 + (pmi >> 1)) & 7, pm = dest * 2 + (pmi & 1), dw = pmi >> 1;
        f32x4 acc[2][2][4][2];
        auto local = [&](int q) __attribute__((always_inline)) {
            gemm_local(p, ws, lds, seq, wid, lane, acc);
            const bool poison = __builtin_amdgcn_readfirstlane(ld_agent(ws + OFF_TMO)) == seq;
#pragma unroll
            for (int xh = 0; xh < 2; ++xh) {
                u32x4 O[8]; acc_to_raw(acc, xh, poison, O);
                store_unit_linear(ws + OFF_P + (size_t)(dw * 512 + (((q * 2 + (wid >> 2)) * 8 + (pmi & 1) * 4 + (wid & 1) * 2 + xh) * 2 + ((wid >> 1) & 1))) * 8192, (u32)lane * 16u, O);
            }
            WAIT_V(0);
            if (lane == 0) __hip_atomic_fetch_add((GAS u32*)(ws + OFF_PD), 1u, __ATOMIC_RELAXED, __HIP_MEMORY_SCOPE_AGENT);
        };
        auto wcode = [&](int q, int i) { return (i * 4 + (wid >> 1)) << 10 | (128 + dw * 64 + q * 4 + (pmi & 1) * 2 + (wid & 1)); };
        const int nd = sc1 & 255;
        for (int it = 0; it == 0 || it < nd; ++it) {
            const int q = (sc1 >> (8 * it + 8)) & 255;
            int np = 0, t0 = 0, t1 = 0;
            if (it == 0) { t0 = (sc0 & 0xffff) + wid * p.pstride[0] + pmi; t1 = (sc0 >> 16) + wid * p.pstride[1] + pmi; np = (t0 < NPLAIN) + (t1 < NPLAIN); }
            pack_tasks(p, ws, wd, [&](int i) { return i < np ? x_task(i ? t1 : t0, p.rank) : (nd && i < np + 2) ? wcode(q, i - np) : -1; }, cnt_v, tl, lds, sc, wid, lane);
            if (!nd) break;
            seq = __builtin_amdgcn_readfirstlane(cnt_v) + 1u;
            local(q);
            __syncthreads();
        }
        seq = __builtin_amdgcn_readfirstlane(cnt_v) + 1u;
        {
            const int l8 = lane - 8;
            gemm_loop(ws, lds, ws + OFF_G + (size_t)pm * 32768, ws + OFF_G + (size_t)(16 + pn) * 32768, (u32)(((lane & 4) ? 16 + pn : pm) * 128 + (lane & 3) * 4),
                      (lane < 8 || lane >= 56) ? (lane & 7) : (int)gslot(8 + l8 / 6 * 7 + (dw + 13 - l8 % 6) % 7, seq), 56, seq, wid, lane, acc);
        }
        {
            const bool poison = __builtin_amdgcn_readfirstlane(ld_agent(ws + OFF_TMO)) == seq;
            const int id0 = ((pn * 2 + (pmi & 1)) * 8 + wid) * 2;
            u32 rec = 0, pos[2];
            char* ry = wd + OFF_RY + (size_t)((seq & 1) * 8 + p.rank) * 512 * 64 + lane * 8;
#pragma unroll
            for (int xh = 0; xh < 2; ++xh) {
                pos[xh] = 0;
                char* yq = ws + OFF_YQ + dest * 128;
                asm volatile("" : "+v"(yq));
                if (lane == 0) pos[xh] = __hip_atomic_fetch_add((GAS u32*)yq, 1u, __ATOMIC_RELAXED, __HIP_MEMORY_SCOPE_AGENT);
                u32x4 O[8];
                acc_to_raw(acc, xh, poison, O);
                rec = encode_store(O, ws + OFF_PY + (size_t)((seq & 1) * 8 + dest) * (4 * MiB) + (size_t)(id0 + xh) * 8192, sc, lane,
                                   [&] { if (xh == 1) { WAIT_V(0); put_record(ry + (__builtin_amdgcn_readfirstlane(pos[0]) & 511u) * 64, seq, rec, lane, 8); } });
                if (lane == 0) rec |= (u32)(id0 + xh) << 20;
            }
            WAIT_V(0);
            put_record(ry + (__builtin_amdgcn_readfirstlane(pos[1]) & 511u) * 64, seq, rec, lane, 8);
        }
        s0 = pn < HC ? 63 + (pn * 14 + pmi) * 2 : -1;
        for (bool first = pn < HC + NSTAT; s0 < 0; first = false) {
            __syncthreads();
            if (threadIdx.x == 0) {
                volatile LAS u32* t = (volatile LAS u32*)(lds + LDS_GEMM);
                t[0] = first ? (u32)((pn - HC) * 14 + pmi)
                             : 14u * NSTAT + __hip_atomic_fetch_add((GAS u32*)(ws + OFF_TICKET), 1u, __ATOMIC_RELAXED, __HIP_MEMORY_SCOPE_AGENT) - (seq - 1u) * (NGEMM - 14 * HC + 128 - 14 * NSTAT);
            }
            __syncthreads();
            const u32 tk = __builtin_amdgcn_readfirstlane(*(volatile LAS u32*)(lds + LDS_GEMM));
            if (tk >= 128u) break;
            const int rq = tk & 3, cq = tk >> 2, opm = p.rank * 2 + (rq >> 1), opn = cq >> 1;
            gemm_piece(ws, lds, ws + OFF_G + (size_t)opm * 32768 + (rq & 1) * 16384, ws + OFF_G + (size_t)(16 + opn) * 32768 + (cq & 1) * 16384,
                       (u32)(opm * 128 + (rq & 1) * 8), (u32)((16 + opn) * 128 + (cq & 1) * 8), seq, wid, lane, acc);
            LAS f32x4* red = (LAS f32x4*)(lds + (wid & 3) * 16384) + lane;
            if (wid >= 4) {
#pragma unroll
                for (int i = 0; i < 16; ++i) red[i * 64] = acc[i >> 3][0][(i >> 1) & 3][i & 1];
            }
            __syncthreads();
            if (wid < 4) {
#pragma unroll
                for (int i = 0; i < 16; ++i) acc[i >> 3][0][(i >> 1) & 3][i & 1] += red[i * 64];
                const bool poison = __builtin_amdgcn_readfirstlane(ld_agent(ws + OFF_TMO)) == seq;
                u32x4 O[8]; acc_to_raw(acc, 0, poison, O);
                char* dst = p.out + (size_t)(rq * 128 + wid * 32) * 65536 + (p.rank * 4096 + cq * 128) * 2;
                const u32 vo = (u32)(lane >> 4) * 262144u + (u32)(lane & 15) * 16u;
#pragma unroll
                for (int q = 0; q < 8; ++q) st16_sys<0>(dst + (size_t)((q >> 2) * 16 + (q & 3)) * 65536, vo, O[q]);
            }
        }
    }
    if (s0 >= 0) comm_role(p, ws, lds, s0, seq, wid, lane, (u32)t_start);
    __syncthreads();
    if (threadIdx.x == 0) st_agent(ws + OFF_CNT + b * 64, seq);
}

extern "C" void kernel_launch(int rank, int nranks, ncclComm_t comm, hipStream_t stream,
                              void* const* d_in, const int* in_sizes, int n_in,
                              void* d_out, int out_size, const int* in_dtypes, int out_dtype,
                              void* const* d_out_all, void* const* d_ws_all, size_t ws_bytes) {
    if (nranks != NR || (unsigned)rank >= (unsigned)NR || n_in != 2 || ws_bytes < WS_NEED || in_sizes[0] != 4096 * 512 || in_sizes[1] != 512 * 32768 || out_size != 512 * 32768) {
        fprintf(stderr, "kernel_launch: unexpected shapes\n");
        return;
    }
    HIP_CHECK(hipFuncSetAttribute((const void*)fused_gemm_rs, hipFuncAttributeMaxDynamicSharedMemorySize, LDS_BYTES));
    Params p;
    memset(&p, 0, sizeof p);
    p.x = (const char*)d_in[0]; p.w = (const char*)d_in[1]; p.out = (char*)d_out;
    for (int i = 0; i < NR; ++i) p.wsr[i] = (char*)d_ws_all[(rank + 1 + i) % NR];
    p.ws_tab = (char* const*)d_ws_all;
    p.rank = rank; p.wsme = (char*)d_ws_all[rank];
    int key[256];
    for (int i = 0; i < 256; ++i) {
        const int j = i >> 5, k = i & 31;
        key[i] = k >= 16 ? 112 * j + SKEW_NUM * (k - 16) - 1 : (k >= 14 ? 100000 + j : (k >= 12 || (j == 0 && k < NPUSH)) ? 200000 : 112 * j + XSPREAD * (k >> 1) - (j == 0 ? 1000 : 0));
        int q = i;
        while (q > 0 && key[p.order[q - 1]] > key[i]) { p.order[q] = p.order[q - 1]; --q; }
        p.order[q] = (unsigned char)i;
    }
    const char* np = NPASS; const char* ndg = NDIAG;
    int base = 0;
    for (int ps = 0; ps < 2; ++ps) {
        int n = 0;
        for (int c = 0; c < 16; ++c) if (np[c] - '0' > ps) n += 14;
        for (int c = 0, o = 0; c < 16; ++c) { const bool in = np[c] - '0' > ps; p.sched[c][0] |= (u32)(in ? base + o : NPLAIN) << (16 * ps); if (in) o += 14; }
        p.pstride[ps] = n;
        base += n * 8;
    }
    int nq = 0;
    for (int r = 1; r <= 3; ++r) for (int i = 0; i < 16; ++i) { const int c = r == 1 ? i : 15 - i, n = ndg[c] - '0'; p.sched[c][1] |= (u32)n; if (n >= r) p.sched[c][1] |= (u32)nq++ << (8 * r); }
    if (base < NPLAIN || nq != 16) { fprintf(stderr, "kernel_launch: bad schedule\n"); return; }
    hipLaunchKernelGGL(fused_gemm_rs, dim3(256), dim3(512), LDS_BYTES, stream, p);
    HIP_CHECK(hipGetLastError());
}
```

```cpp
#include <hip/hip_runtime.h>
#include <rccl/rccl.h>
#include <cstdio>
#include <cstdint>
#include <cstring>

#define HIP_CHECK(expr) do { hipError_t _e = (expr); if (_e != hipSuccess) { \
    fprintf(stderr, "HIP error %s at %s:%d\n", hipGetErrorString(_e), __FILE__, __LINE__); \
    return; } } while (0)

typedef unsigned u32;
typedef unsigned long long u64;
typedef u32 u32x4 __attribute__((ext_vector_type(4)));
typedef u32 u32x2 __attribute__((ext_vector_type(2)));
typedef short bf16x8 __attribute__((ext_vector_type(8)));
typedef float f32x4 __attribute__((ext_vector_type(4)));
typedef float f32x2 __attribute__((ext_vector_type(2)));
#define LAS __attribute__((address_space(3)))
#define GAS __attribute__((address_space(1)))

constexpr int GAUX = 17;
constexpr int HC = 1;
constexpr int NSTAT = 4;
constexpr int NHALF = 30;
constexpr int NPUSH = 9;
constexpr int NPLAIN = 1024;
constexpr int SKEW_NUM = 47, XSPREAD = 12;
static const char NPASS[] = "1120000111111000";
static const char NDIAG[] = "0001111111111222";
constexpr u32 TS_A = 120, TS_B = 60, TS_D = 24, TS_Q = 110, TS_P = 240;

constexpr int NR = 8;
constexpr size_t MiB = 1ull << 20;
constexpr size_t OFF_CNT = 0;
constexpr size_t OFF_TMO = 16384;
constexpr size_t OFF_TICKET = 16384 + 256;
constexpr size_t OFF_YO = 16384 + 512;
constexpr size_t OFF_YT = 16384 + 2048;
constexpr size_t OFF_YQ = 16384 + 4096;
constexpr size_t OFF_PD = 16384 + 6144;
constexpr size_t OFF_HF = 32768;
constexpr size_t OFF_FG = 65536;
constexpr size_t OFF_RIN = 1 * MiB;
constexpr size_t OFF_RY = 2 * MiB;
constexpr size_t OFF_XF = 3 * MiB;
constexpr size_t OFF_PX = 4 * MiB;
constexpr size_t OFF_PW = 12 * MiB;
constexpr size_t OFF_PY = 76 * MiB;
constexpr size_t OFF_G = 140 * MiB;
constexpr size_t OFF_P = 211 * MiB;
constexpr size_t OFF_H = 239 * MiB;
constexpr size_t WS_NEED = 243 * MiB;
constexpr u64 DEADLINE = 40000000ull;
constexpr int LDS_GEMM = 131072, LDS_BYTES = 163840;
constexpr int NGEMM = 224;

struct Params {
    const char* x; const char* w; char* out; char* wsme;
    char* const* ws_tab;
    int rank; int pstride[2]; int pad[3];
    char* wsr[NR];
    u32 sched[16][2];
    unsigned char order[256];
};

#define WAIT_V(n) asm volatile("s_waitcnt vmcnt(" #n ")" ::: "memory")
#define WAIT_P __builtin_amdgcn_s_waitcnt(0x0F70)
#define WAIT_L(n) asm volatile("s_waitcnt lgkmcnt(" #n ")" ::: "memory")
#define BAR do { asm volatile("" ::: "memory"); __builtin_amdgcn_s_barrier(); asm volatile("" ::: "memory"); } while (0)
#define SCHED __builtin_amdgcn_sched_barrier(0)

template <int OFF> __device__ __forceinline__ void st16_sys(const char* base, u32 voff, u32x4 v) {
    asm volatile("s_nop 4\n\tglobal_store_dwordx4 %0, %1, %2 offset:%3 sc0 sc1\n\ts_nop 1" :: "v"(voff), "v"(v), "s"(base), "i"(OFF) : "memory");
}
__device__ __forceinline__ const char* uni(const char* p) {
    return (const char*)(((u64)(u32)__builtin_amdgcn_readfirstlane((u32)((u64)p >> 32)) << 32) | (u32)__builtin_amdgcn_readfirstlane((u32)(u64)p));
}
__device__ __forceinline__ u32 perm(u32 hi, u32 lo, u32 sel) { return __builtin_amdgcn_perm(hi, lo, sel); }
__device__ __forceinline__ u32 ld_agent(const char* p) { return __hip_atomic_load((GAS u32*)p, __ATOMIC_RELAXED, __HIP_MEMORY_SCOPE_AGENT); }
__device__ __forceinline__ void st_agent(char* p, u32 v) { __hip_atomic_store((GAS u32*)p, v, __ATOMIC_RELAXED, __HIP_MEMORY_SCOPE_AGENT); }
__device__ __forceinline__ u32 gslot(u32 o, u32 seq) { return o + ((seq & 1u) && o - 8u < 7u ? 56u : 0u); }
typedef __bf16 bf16x2 __attribute__((ext_vector_type(2)));
__device__ __forceinline__ u32 cvt_pk_bf16(float lo, float hi) { return __builtin_bit_cast(u32, __builtin_convertvector((f32x2){lo, hi}, bf16x2)); }

__device__ __forceinline__ void store_unit_linear(const char* dst, u32 voff, const u32x4 (&O)[8]) {
    const char* b = uni(dst + 4096);
    st16_sys<-4096>(b, voff, O[0]); st16_sys<-3072>(b, voff, O[1]); st16_sys<-2048>(b, voff, O[2]); st16_sys<-1024>(b, voff, O[3]);
    st16_sys<0>(b, voff, O[4]); st16_sys<1024>(b, voff, O[5]); st16_sys<2048>(b, voff, O[6]); st16_sys<3072>(b, voff, O[7]);
}

constexpr u32 H_RAW = 1u << 11, H_DEAD = 1u << 12, H_L3 = 1u << 17;
constexpr int ENC_BYTES = 1152;
__device__ __forceinline__ u32 bfi(u32 m, u32 a, u32 b) { return __builtin_amdgcn_bitop3_b32(m, a, b, 0xca); }
__device__ __forceinline__ u32 scan32(u32 v) {
    v += (u32)__builtin_amdgcn_update_dpp(0, (int)v, 0x111, 0xf, 0xf, false);
    v += (u32)__builtin_amdgcn_update_dpp(0, (int)v, 0x112, 0xf, 0xf, false);
    v += (u32)__builtin_amdgcn_update_dpp(0, (int)v, 0x114, 0xf, 0xf, false);
    v += (u32)__builtin_amdgcn_update_dpp(0, (int)v, 0x118, 0xf, 0xf, false);
    v += (u32)__builtin_amdgcn_update_dpp(0, (int)v, 0x142, 0xa, 0xf, false);
    return v;
}
__device__ __forceinline__ void enc_init(LAS unsigned char* sc, int lane) {
    u32 sel = 0x0c0c0c0cu;
#pragma unroll
    for (int b = 3; b >= 0; --b) if (lane & (1 << b)) sel = (sel << 8) | (u32)b;
    if (lane < 16) ((LAS u32*)(sc + 1088))[lane] = sel;
}
template <class F>
__device__ __forceinline__ u32 encode_store(const u32x4 (&O)[8], const char* slot, LAS unsigned char* sc, int lane, u32& T, bool again, F before_stores) {
    u32 d[16], S[16];
    u32 sum = 0;
#pragma unroll
    for (int t = 0; t < 16; ++t) {
        const u32 a = O[2 * (t >> 2)][t & 3], b = O[2 * (t >> 2) + 1][t & 3];
        const u32 ra = __builtin_amdgcn_alignbit(a, a, 31), rb = __builtin_amdgcn_alignbit(b, b, 31);
        d[t] = perm(rb, ra, 0x07050301u); S[t] = perm(rb, ra, 0x06040200u);
    }
    if (T == 0u) {
#pragma unroll
        for (int t = 0; t < 16; t += 4) sum = __builtin_amdgcn_sad_u8(d[t], 0u, sum);
        sum = scan32(sum);
        const u32 tot = (u32)__builtin_amdgcn_readlane((int)sum, 31) + (u32)__builtin_amdgcn_readlane((int)sum, 63);
        T = min(max((tot >> 10) + 4u, 15u), 244u);
    }
    const u32 T4 = T * 0x01010101u;
    *(LAS u32x4*)(sc + lane * 16) = (u32x4){0u, 0u, 0u, 0u};
    if (lane == 0) *(LAS u32x4*)(sc + 1024) = (u32x4){0u, 0u, 0u, 0u};
    u32 acc = 0;
#pragma unroll
    for (int t = 0; t < 16; ++t) { d[t] = T4 - d[t]; acc |= d[t]; }
    u32 rec = 0, head = T;
    bool raw = false;
    u64 xmask = 0;
    u32 xt = 0, xb = 0, xe = 0;
    if (__builtin_amdgcn_ballot_w64((acc & 0xf0f0f0f0u) != 0u) != 0ull) {
        u32 n = 0, sM = 0;
#pragma unroll
        for (int t = 0; t < 16; ++t) {
            if (__builtin_amdgcn_ballot_w64((d[t] & 0xf0f0f0f0u) != 0u) == 0ull) continue;
            asm volatile("" ::: "memory");
            const u32 e = T4 - d[t];
            const u32 w = ((T4 | 0x80808080u) - (e & 0x7f7f7f7fu)) ^ ((T4 ^ ~e) & 0x80808080u);
            const u32 mis = (((w >> 4) & 0x0f0f0f0fu) + 0x0f0f0f0fu) & 0x10101010u;
            d[t] = (w & 0x0f0f0f0fu) | (mis >> 1);
            if (mis) { ++n; xe = e; sM = mis; xt = t; }
        }
        xb = (n ? (u32)__builtin_ctz(sM) : 0u) & 24u;
        xmask = __builtin_amdgcn_ballot_w64(n != 0u);
        if (__builtin_popcountll(xmask) > 7 || __builtin_amdgcn_ballot_w64(n > 1u || (sM & (sM - 1u)) != 0u) != 0ull) raw = true;
    }
    u32x4 Q; u32 cs[16], bo[16], bits = 0;
#pragma unroll
    for (int t = 0; t < 16; ++t) {
        const u32 sel = d[t] + (d[t] & 0x08080808u);
        const u32 r = perm(0xffffffaau, 0x5500ffffu, sel), r2 = perm(0xffffff00u, 0x0000ffffu, sel);
        Q[t >> 2] = (t & 3) ? bfi(0x03030303u << (2 * (t & 3)), r, Q[t >> 2]) : r;
        cs[t] = ((const LAS u32*)(size_t)__builtin_amdgcn_sad_u8(r2 & 0x20100804u, 0u, (u32)(size_t)sc))[272];
        bo[t] = bits;
        bits += (u32)__builtin_popcount(r2);
    }
    const u32 inc = scan32(bits);
    const u32 nb = max((u32)__builtin_amdgcn_readlane((int)inc, 31), (u32)__builtin_amdgcn_readlane((int)inc, 63)) >> 3;
    if (nb > 960u) raw = true;
    if (xmask && !raw) {
        u32 pos = (inc - bits) >> 3;
#pragma unroll
        for (int i = 0; i < 4; ++i) {
            const u32 f = 2u * (xt & 3u), m = (u32)i < (xt >> 2) ? ~0u : (u32)i > (xt >> 2) ? 0u : ((1u << f) - 1u) * 0x01010101u | ((0x01010101u << f) & ((1u << xb) - 1u));
            pos += (u32)__builtin_popcount(Q[i] & (Q[i] >> 1) & 0x55555555u & m);
        }
        const u32 ent = ((u32)(lane & 32) * 32u + pos) << 8 | ((xe >> xb) & 255u);
        head |= (u32)__builtin_popcountll(xmask) << 8;
        for (u32 s = 1; xmask; ++s, xmask &= xmask - 1) {
            const u32 e = (u32)__builtin_amdgcn_readlane((int)ent, __builtin_ctzll(xmask));
            if ((u32)(lane & 7) == s) rec = e;
        }
    }
    const u32 voff = (u32)lane * 16u;
    if (before_stores()) {
        const u32x4 q = {0x7fc07fc0u, 0x7fc07fc0u, 0x7fc07fc0u, 0x7fc07fc0u}, N[8] = {q, q, q, q, q, q, q, q};
        store_unit_linear(slot, voff, N);
        head = H_RAW;
    } else if (raw) {
        store_unit_linear(slot, voff, O);
        head = H_RAW;
    } else {
        const char* b = uni(slot + 4096);
        st16_sys<-4096>(b, voff, (u32x4){S[0], S[1], S[2], S[3]}); st16_sys<-3072>(b, voff, (u32x4){S[4], S[5], S[6], S[7]});
        st16_sys<-2048>(b, voff, (u32x4){S[8], S[9], S[10], S[11]}); st16_sys<-1024>(b, voff, (u32x4){S[12], S[13], S[14], S[15]});
        st16_sys<0>(b, voff, Q);
        const u32 base = inc - bits + (lane & 32 ? 4u : 0u) + (u32)(size_t)sc * 8u;
#pragma unroll
        for (int t = 0; t < 16; ++t) {
            const u32 o = base + bo[t];
            const u64 x = (u64)perm(0u, d[t] ^ 0x05050505u, cs[t]) << (o & 31u);
            LAS u32* q = (LAS u32*)(size_t)((o >> 3) & ~3u);
            __hip_atomic_fetch_or(q, (u32)x, __ATOMIC_RELAXED, __HIP_MEMORY_SCOPE_WAVEFRONT);
            __hip_atomic_fetch_or(q + 1, (u32)(x >> 32), __ATOMIC_RELAXED, __HIP_MEMORY_SCOPE_WAVEFRONT);
        }
        asm volatile("" ::: "memory");
        u32 nl = (nb + 127u) >> 7, from = 0;
        const u32 cnt = (head >> 8) & 7u, rem = nb & 127u;
        if (rem && rem <= 28u - 4u * cnt) {
            --nl;
            const u32 i = (u32)(lane & 7) - cnt - 1u, ov = (rem + 3u) >> 2;
            if (i < ov) rec = *(const LAS u32*)(sc + nl * 128u + i * 4u);
            head |= ov << 29;
        }
        if (again && nl == 4u) {
            const u32x2 v = *(const LAS u32x2*)(sc + lane * 8);
            const u32 g[4] = {v[0] & 0x0f0f0f0fu, v[1] & 0x0f0f0f0fu, (v[0] >> 4) & 0x0f0f0f0fu, (v[1] >> 4) & 0x0f0f0f0fu};
            u32 q2 = 0, c2[4], b2[4], bits2 = 0;
#pragma unroll
            for (int t = 0; t < 4; ++t) {
                const u32 sel = g[t] + (g[t] & 0x08080808u);
                const u32 r = perm(0xffffffaau, 0x55ffff00u, sel), r2 = perm(0xffffff00u, 0x00ffff00u, sel);
                q2 = t ? bfi(0x03030303u << (2 * t), r, q2) : r;
                c2[t] = ((const LAS u32*)(size_t)__builtin_amdgcn_sad_u8(r2 & 0x20100804u, 0u, (u32)(size_t)sc))[272];
                b2[t] = bits2;
                bits2 += (u32)__builtin_popcount(r2);
            }
            const u32 inc2 = scan32(bits2);
            if (max((u32)__builtin_amdgcn_readlane((int)inc2, 31), (u32)__builtin_amdgcn_readlane((int)inc2, 63)) <= 1024u) {
                const u32 base2 = inc2 - bits2 + (lane & 32 ? 4u : 0u) + ((u32)(size_t)sc + 896u) * 8u;
                *(LAS u32*)(sc + 640 + lane * 4) = q2;
#pragma unroll
                for (int t = 0; t < 4; ++t) {
                    const u32 o = base2 + b2[t];
                    const u64 x = (u64)perm(0u, g[t], c2[t]) << (o & 31u);
                    LAS u32* q = (LAS u32*)(size_t)((o >> 3) & ~3u);
                    __hip_atomic_fetch_or(q, (u32)x, __ATOMIC_RELAXED, __HIP_MEMORY_SCOPE_WAVEFRONT);
                    __hip_atomic_fetch_or(q + 1, (u32)(x >> 32), __ATOMIC_RELAXED, __HIP_MEMORY_SCOPE_WAVEFRONT);
                }
                asm volatile("" ::: "memory");
                nl = 3; from = 640; head |= H_L3;
            }
        }
        if ((u32)lane < nl * 8u) st16_sys<1024>(b, voff, *(const LAS u32x4*)(sc + from + lane * 16));
        head |= nl << 13;
    }
    if ((lane & 7) == 0) rec = head;
    return rec;
}
__device__ __forceinline__ void put_record(char* p, u32 seq, u32 w, int lane, int nl) {
    const u32 hd = __builtin_amdgcn_readfirstlane(w), cnt = ((hd >> 8) & 7u) + (hd >> 29);
    if (lane < nl && (u32)(lane & 7) <= cnt) __hip_atomic_store((GAS u64*)p, ((u64)w << 32) | seq, __ATOMIC_RELAXED, __HIP_MEMORY_SCOPE_SYSTEM);
}

__device__ __forceinline__ int x_task(int t, int rank) {
    if (t >= 512) return ((t >> 5) & 7) << 10 | (64 + ((t >> 8) & 1) * 32 + (t & 31));
    if (t - 112u < 64u) return ((t - 112) >> 3) << 10 | (rank * 8 + (t & 7));
    const int v = t < 112 ? t : t - 64;
    return (v / 56) << 10 | ((rank * 8 + 8 + v % 56) & 63);
}
__device__ __forceinline__ void task_load(const Params& p, int code, LAS unsigned char* buf, int lane) {
    const int j = code >> 10, r = code & 1023, un = r & 3;
    const char* s0; const char* s1; u32 stride;
    if (r < 64) {
        const char* src = p.x + (size_t)((r >> 2) * 256 + un * 64) * 1024 + j * 128 + (lane >> 3) * 1024;
        const int sw = (lane & 7) ^ (lane >> 4);
        s0 = src + (sw << 4); s1 = src + ((sw ^ 4) << 4); stride = 8192;
    } else {
        s0 = s1 = p.w + (size_t)(j * 64 + (un & 1) * 32 + (lane >> 4) * 8) * 65536 + (((p.rank + ((r - 64) >> 6)) & 7) * 4096 + ((r >> 2) & 15) * 256 + (un >> 1) * 128) * 2 + (lane & 15) * 16;
        stride = 65536;
    }
#pragma unroll
    for (int q = 0; q < 8; ++q) __builtin_amdgcn_global_load_lds((const GAS u32*)(((q & 1) ? s1 : s0) + q * stride), (LAS u32*)(buf + q * 1024), 16, 0, 0);
}
__device__ __forceinline__ u32 task_time(const Params& p, int code) {
    const u32 j = code >> 10, r = code & 1023, c = (r >> 2) & 15;
    if (r < 64) return (c >> 1) == (u32)p.rank ? TS_A : j < 2 ? 0u : TS_A + j * TS_B;
    if (r < 128) return TS_A + j * TS_B + TS_B / 2 + (c >> 3) * 8 * TS_B;
    return c < 4 ? (j < 2 ? 0u : TS_A) : TS_A + c * TS_D + (j & 3) * (TS_D / 4);
}
__device__ __forceinline__ void sleep_until(u32 t) {
    for (int n = (int)(t - (u32)__builtin_amdgcn_s_memrealtime()); n > 11; n -= 23) __builtin_amdgcn_s_sleep(8);
}
template <class Next>
__device__ __forceinline__ void pack_tasks(const Params& p, char* ws, char* wd, Next next, u32 cnt_v, u32 tl, LAS unsigned char* lds, LAS unsigned char* sc, int wid, int lane) {
    const u32 voff = (u32)lane * 16u;
    int code = __builtin_amdgcn_readfirstlane(next(0));
    if (code < 0) return;
    auto place = [&](int c) { return lds + ((c & 1023) >= 128 ? (c >> 10) * 2 + (c & 1) : wid + 8) * 8192; };
    sleep_until(tl + task_time(p, code));
    task_load(p, code, place(code), lane);
    char* peer = p.ws_tab[(p.rank + 1 + (lane >> 3)) & 7];
    asm volatile("" : "+v"(peer), "+v"(cnt_v) :: "memory");
    const u32 seq = __builtin_amdgcn_readfirstlane(cnt_v) + 1u;
    WAIT_P; WAIT_V(0);
    char* rp = nullptr; u32 rec = 0; int nl = 0;
    for (int pass = 0;; ++pass) {
        const int j = code >> 10, r = code & 1023, un = r & 3, dd = (r - 64) >> 6, b16 = (r >> 2) & 15;
        const bool isx = r < 64, enc = isx ? (b16 >> 1) != p.rank : dd != 0;
        const bool urgent = j < 2 && (isx || b16 < 4);
        LAS unsigned char* buf = place(code);
        u32x4 O[8];
        {
            u32x4 in[8];
#pragma unroll
            for (int q = 0; q < 8; ++q) in[q] = *(const LAS u32x4*)(buf + q * 1024 + voff);
            WAIT_L(0);
            if (isx) {
#pragma unroll
                for (int q = 0; q < 8; ++q) O[q] = in[q];
            } else {
#pragma unroll
                for (int d = 0; d < 4; ++d)
#pragma unroll
                    for (int q = 0; q < 4; ++q) { O[2 * d][q] = perm(in[2 * q + 1][d], in[2 * q][d], 0x05040100u); O[2 * d + 1][q] = perm(in[2 * q + 1][d], in[2 * q][d], 0x07060302u); }
            }
        }
        char* flag = nullptr;
        const int blk = j * 32 + (r >> 2);
        const bool rawlast = urgent && isx && enc;
        if (!rawlast && r < 128) {
            store_unit_linear(ws + OFF_G + (size_t)blk * 32768 + un * 8192, voff, O);
            flag = ws + OFF_FG + blk * 128 + un * 4;
            if (isx && !enc) {
#pragma unroll
                for (int q = 0; q < 8; ++q) O[q] = *(const LAS u32x4*)(buf + ((q & 3) * 16 + (lane & 15)) * 128 + ((((q >> 2) * 4 + (lane >> 4)) ^ ((lane & 15) >> 1)) << 4));
                WAIT_L(0);
                store_unit_linear(ws + OFF_XF + ((r & 7) * 8 + j) * 8192, voff, O);
            }
        }
        if (r >= 128) {
#pragma unroll
            for (int q = 0; q < 8; ++q) *(LAS u32x4*)(buf + q * 1024 + voff) = O[q];
        }
        bool late = urgent;
        code = __builtin_amdgcn_readfirstlane(next(pass + 1));
        if (code >= 0 && !late) {
            late = (u32)__builtin_amdgcn_s_memrealtime() - tl < task_time(p, code);
            if (!late) task_load(p, code, place(code), lane);
        }
        auto settle = [&]() {
            WAIT_V(0);
            if (nl) { put_record(rp, seq, rec, lane, nl); nl = 0; }
            if (flag) { if (lane == 0) st_agent(flag, seq); flag = nullptr; }
        };
        if (enc) {
            const int dest = (p.rank + dd) & 7;
            const char* slot = ws + (isx ? OFF_PX + (size_t)(seq & 1) * (4 * MiB) : OFF_PW + (size_t)((seq & 1) * 8 + dest) * (4 * MiB)) + (size_t)((j * 16 + b16) * 4 + un) * 8192;
            u32 T = 0;
            rec = encode_store(O, slot, sc, lane, T, !(urgent && j == 0), [&] { WAIT_P; settle(); return false; });
            rp = (isx ? peer : wd) + OFF_RIN + (size_t)(((seq & 1) * 8 + p.rank) * 1024 + (j * 32 + (isx ? 0 : 16) + b16) * 4 + un) * 64 + (lane & 7) * 8;
            nl = isx ? 56 : 8;
        }
        if (!enc || late || code < 0) settle();
        if (rawlast) { store_unit_linear(ws + OFF_G + (size_t)blk * 32768 + un * 8192, voff, O); flag = ws + OFF_FG + blk * 128 + un * 4; settle(); }
        if (code < 0) break;
        if (late) {
            sleep_until(tl + task_time(p, code));
            task_load(p, code, place(code), lane);
            WAIT_P; WAIT_V(0);
        }
    }
}

__device__ __forceinline__ void push_task(const Params& p, int t, u32 cnt_v, u32 tl, int lane) {
    const int dd = t % 7, un = (t / 7) & 3, i = t / 28, dest = (p.rank + 1 + dd) & 7, w = 6 - dd;
    const int pm = (dest * 2 + 2 + (i + 2 * w + 2) % 14) & 15;
    const char* src = p.x + (size_t)(pm * 256 + un * 64) * 1024 + (lane >> 3) * 1024;
    const int sw = (lane & 7) ^ (lane >> 4);
    u32x4 in[8];
#pragma unroll
    for (int q = 0; q < 8; ++q) in[q] = *(const GAS u32x4*)(src + q * 8192 + ((sw ^ ((q & 1) << 2)) << 4));
    WAIT_P; SCHED;
    const u32 seq = __builtin_amdgcn_readfirstlane(cnt_v) + 1u, blk = gslot(8 + w, seq) * 32 + pm;
    char* pw = p.wsr[dd];
    for (int n = (int)(tl + TS_Q + (t / 7) * TS_P / 16 - (u32)__builtin_amdgcn_s_memrealtime()); n > 2; n -= 6) __builtin_amdgcn_s_sleep(2);
    store_unit_linear(pw + OFF_G + (size_t)blk * 32768 + un * 8192, (u32)lane * 16u, in);
    WAIT_V(0);
    if (lane == 0) __hip_atomic_store((GAS u32*)(pw + OFF_FG + blk * 128 + un * 4), seq, __ATOMIC_RELAXED, __HIP_MEMORY_SCOPE_SYSTEM);
}

constexpr u32 WIN = 2;
constexpr u32 PACE = 978;
constexpr u32 PACE_E = 1022, KE = 26;
constexpr u32 SLACK = 1;
constexpr u32 G1 = 24, G2 = 30;
constexpr u32 YLEAD = 2;
constexpr int NSLOT = 8, NDEC = 5, NDESC = 16, NIN = 960 - 4 * NPUSH, LOOK = 8;
constexpr u32 FIN = 0x40000000u;
constexpr int L_SLOT = 0;
constexpr int L_DESC = 2 * NSLOT * 8192;
constexpr int L_CTL = L_DESC + 2 * NDESC * 64;
constexpr int L_ORD = L_CTL + 128;
static_assert(L_ORD + 512 <= LDS_BYTES, "lds");

template <int V> struct IC { static constexpr int v = V; };
__device__ __forceinline__ void wait_units(u32 n) {
    switch (n) {
        case 0: WAIT_V(0); break; case 1: WAIT_V(6); break; case 2: WAIT_V(12); break; case 3: WAIT_V(18); break; case 4: WAIT_V(24); break;
        case 5: WAIT_V(30); break; case 6: WAIT_V(36); break; default: WAIT_V(42); break;
    }
}
__device__ __forceinline__ void comm_role(const Params& p, char* ws, LAS unsigned char* lds, u32 s0, u32 cnt_v, int wid, int lane, u32 tstart) {
    constexpr u32 NY = 9 + 4 * HC;
    if (s0 >= 63) __syncthreads();
    if (threadIdx.x < 32) ((LAS u32*)(lds + L_CTL))[threadIdx.x] = 0;
    WAIT_L(0); BAR;
    volatile LAS u32* ctl = (volatile LAS u32*)(lds + L_CTL);
    volatile LAS u32* desc = (volatile LAS u32*)(lds + L_DESC);
    const u32 voff = (u32)lane * 16u;
    if (wid == 7) {
        const char* rws = p.ws_tab[(p.rank + 1 + (s0 + (lane >> 5)) % 7) & 7];
        if (s0 < 63) ((LAS u32*)(lds + L_ORD))[lane] = ((const u32*)p.order)[lane];
        WAIT_P;
        const u32 seq = __builtin_amdgcn_readfirstlane(cnt_v) + 1u;
        const u32 st = lane >> 5, e = lane & 7, L = s0 + st, src = (p.rank + 1 + L % 7) & 7, j = L / 7;
        const u32 cin = j < 9 ? (NIN - j + 8) / 9 : 0;
        bool open = s0 < 63, dead = false, pd = false;
        const u64 t0 = __builtin_amdgcn_s_memrealtime();
        volatile LAS u32* tk = (volatile LAS u32*)(lds + L_ORD + 256) + st * 32;
        GAS u32* yt = (GAS u32*)(ws + OFF_YT + (L % 7) * 128);
        if ((lane & 31) == 0) __hip_atomic_fetch_max(yt, seq << 10, __ATOMIC_RELAXED, __HIP_MEMORY_SCOPE_AGENT);
        if ((lane & 31) < LOOK) tk[(cin + (lane & 31)) % 32] = j + NY * (lane & 31);
        u32 x = (lane >> 3) & 3, drawn = cin + LOOK, cnt = (s0 == 62 && st) ? 0u : FIN, idle = 0, pub = 0, xmin = 0;
        while (__builtin_amdgcn_ballot_w64(x < cnt) != 0ull) {
            u32 yo = 0, pv = 0;
            if (!open) yo = ld_agent(ws + OFF_YO + (L % 7) * 128);
            const u32 lim = ctl[st * 16 + 9] + LOOK, nd = (open && cnt == FIN && lim > drawn) ? lim - drawn : 0u;
            u32 tb = 0;
            if (drawn == cin + LOOK && __builtin_amdgcn_ballot_w64(nd != 0u) != 0ull) WAIT_V(0);
            if (nd && (lane & 31) == 0) tb = __hip_atomic_fetch_add(yt, nd, __ATOMIC_RELAXED, __HIP_MEMORY_SCOPE_AGENT);
            const bool act = x < cnt && x < min(lim, drawn) && x < xmin + 12u;
            const u32 u = x < cin ? j + 9 * x : NIN + (tk[x % 32] & 511u);
            u32 id;
            if (u < NIN) {
                u32 bk = (lds + L_ORD)[u >> 2];
                if (!(bk & 16)) { const u32 i = bk & 15, pmi = i < 14 ? (i + 2 * (L % 7) + 2) % 14 : i; bk = (bk & ~15u) | ((p.rank * 2 + 2 + pmi) & 15); }
                id = bk * 4u + (u & 3);
            } else id = u - NIN;
            const char* a = u >= NIN ? ws + OFF_RY + (size_t)(((seq & 1) * 8 + src) * 512 + id) * 64 : ws + OFF_RIN + (size_t)(((seq & 1) * 8 + src) * 1024 + id) * 64;
            u64 v = 0;
            if (!pd && lim + LOOK > cin) pv = ld_agent(ws + OFF_PD);
            if (act) v = __hip_atomic_load((GAS u64*)(a + e * 8), __ATOMIC_RELAXED, __HIP_MEMORY_SCOPE_SYSTEM);
            asm volatile("" : "+v"(yo), "+v"(pv), "+v"(v));
            open = open || dead || yo - (seq - 1u) * 9u - (9u * (j - 8u) + NY - 10u) / (NY - 9u) < 10u;
            pd = pd || pv >= seq * (NGEMM * 8u);
            const u32 hw = (u32)__builtin_amdgcn_ds_swizzle((int)(u32)(v >> 32), 0x0018);
            const u64 okm = __builtin_amdgcn_ballot_w64(((u32)v == seq || e > ((hw >> 8) & 7u) + (hw >> 29)) && (u < NIN || pd));
            const bool adv = act && open && (dead || ((okm >> (lane & 56)) & 0xffull) == 0xffull);
            const u32 am = (u32)(__builtin_amdgcn_ballot_w64(adv && e == 0) >> (lane & 32)), pe = pub + (u32)__builtin_popcount(am & ((1u << (lane & 24)) - 1u));
            pub += (u32)__builtin_popcount(am);
            if (__builtin_amdgcn_ballot_w64(nd != 0u) != 0ull) {
                tb = (u32)__builtin_amdgcn_ds_swizzle((int)tb, 0) - (seq << 10) + LOOK * NY;
                if ((u32)(lane & 31) < nd) tk[(drawn + (lane & 31)) % 32] = tb + (lane & 31);
                if (nd && tb + nd >= 512u) cnt = drawn + (tb < 512u ? 512u - tb : 0u);
                drawn += nd;
            }
            if (__builtin_amdgcn_ballot_w64(adv) != 0ull) {
                if (adv) {
                    if (u >= NIN && !dead) id = (hw >> 20) & 511u;
                    const size_t po = (size_t)((seq & 1) * 8 + p.rank) * (4 * MiB);
                    const char* s = u >= NIN ? rws + OFF_PY + po + (size_t)id * 8192
                        : rws + ((id & 64) ? OFF_PW + po : OFF_PX + (size_t)(seq & 1) * (4 * MiB)) + (size_t)(((id >> 7) * 16 + ((id >> 2) & 15)) * 4 + (id & 3)) * 8192;
                    const u32 en = (st * NDESC + pe % NDESC) * 16, word = dead ? H_DEAD : (u32)(v >> 32);
                    const u64 sa = dead ? (u64)(ws + OFF_G) : (u64)s;
                    desc[en + e] = word;
                    if (e == 0) { desc[en + 8] = (u32)sa; desc[en + 9] = (u32)(sa >> 32); desc[en + 10] = word; desc[en + 11] = (u >= NIN ? 0x8000u : 0u) | id; }
                    if (e == 1) {
                        const u32 blk = gslot(8 + 7 * (id >> 7) + L % 7, seq) * 32 + ((id >> 2) & 31);
                        const u32 row0 = ((id >> 4) & 1) * 256 + (id & 1) * 128 + ((id >> 1) & 3) * 32, col0 = src * 4096 + ((id >> 5) & 15) * 256 + ((id >> 3) & 1) * 128;
                        const u64 da = u >= NIN ? (u64)(p.out + (size_t)row0 * 65536 + col0 * 2) : (u64)(ws + OFF_G + (size_t)blk * 32768 + (id & 3) * 8192 + 4096);
                        desc[en + 12] = (u32)da; desc[en + 13] = (u32)(da >> 32);
                        desc[en + 14] = u >= NIN ? ((L % 7) * 512 + id) * 8192 : (u32)OFF_FG + blk * 128 + (id & 3) * 4;
                    }
                    x += 4;
                }
                u32 m = min(x, (u32)__builtin_amdgcn_update_dpp((int)x, (int)x, 0x128, 0xf, 0xf, false));
                m = min(m, (u32)__builtin_amdgcn_ds_swizzle((int)m, 0x401F));
                WAIT_L(0);
                xmin = m;
                if ((lane & 31) == 0) ctl[st * 16 + 8] = pub;
            } else if ((++idle & 15u) == 0u && (ld_agent(ws + OFF_TMO) == seq || __builtin_amdgcn_s_memrealtime() - t0 > DEADLINE)) {
                st_agent(ws + OFF_TMO, seq); WAIT_V(0); dead = true;
            } else __builtin_amdgcn_s_sleep(1);
        }
        if ((lane & 31) == 0) ctl[st * 16 + 8] = cnt | FIN;
    } else if (wid >= NDEC) {
        WAIT_P;
        const u32 me = wid - NDEC;
        volatile LAS u32* c = ctl + me * 16;
        u32 landed = 0, k = 0, slot = 0, pc = PACE_E;
        const u32 t0 = (u32)__builtin_amdgcn_s_memrealtime();
        u32 next = 0, now;
        auto clock = [&]() { return ((u32)__builtin_amdgcn_s_memrealtime() - t0) * 64u; };
        auto pace = [&](u32 n64) {
            if ((int)(next - now) > 1100) { __builtin_amdgcn_s_sleep(2); if ((int)(next - now) > 1600) __builtin_amdgcn_s_sleep(2); now = clock(); }
            while ((int)(now - next) < 0) now = clock();
            next += (n64 * pc) >> 4;
        };
        __builtin_amdgcn_s_setprio(3);
        for (;; ++k) {
            volatile LAS u32* rp = (lane & 16) ? desc + (me * NDESC + k % NDESC) * 16 + (lane & 15) : c + (lane & 15);
            u32 v;
            bool fin = false;
            while (true) {
                v = *rp;
                now = clock();
                const u32 pub = __builtin_amdgcn_readlane(v, 8);
                if ((pub & ~FIN) > k && (u32)__builtin_amdgcn_readlane(v, slot) + NSLOT > k) break;
                if (landed < k) {
                    wait_units(k - landed - 1);
                    ++landed;
                    if (lane == 0) c[10] = landed;
                } else if (pub == (k | FIN)) { fin = true; break; }
            }
            if (fin) break;
            const char* s = (const char*)(((u64)(u32)__builtin_amdgcn_readlane(v, 25) << 32) | (u32)__builtin_amdgcn_readlane(v, 24));
            const u32 head = __builtin_amdgcn_readlane(v, 26);
            if (k == KE) pc = PACE;
            if (__builtin_amdgcn_readlane(v, 27) & 0x8000) pc = PACE * NY / 9;
            if (s0 < 63 && k + YLEAD == (NIN - (s0 + me) / 7 + 8) / 9 && lane == 0) __hip_atomic_fetch_add((GAS u32*)(ws + OFF_YO + ((s0 + me) % 7) * 128), 1u, __ATOMIC_RELAXED, __HIP_MEMORY_SCOPE_AGENT);
            LAS unsigned char* d = lds + L_SLOT + (me * NSLOT + slot) * 8192;
            if (next + SLACK * PACE / 4 - now > (1u << 20)) next = now - SLACK * PACE / 4;
            u32 n16 = ((head >> 13) & 15u) * 8u;
            if (head & (H_RAW | H_DEAD)) {
                pace(32);
                __builtin_amdgcn_global_load_lds((const GAS u32*)(s + 6144 + voff), (LAS u32*)(d + 6144), 16, 0, 17);
                __builtin_amdgcn_global_load_lds((const GAS u32*)(s + 6144 + voff), (LAS u32*)(d + 6144), 16, 1024, 17);
                n16 = 64; now = clock();
            }
            const u32 tail = n16 / 4;
            pace(G1);
            if ((u32)lane < max(n16, 1u)) __builtin_amdgcn_global_load_lds((const GAS u32*)(s + 4096 + voff), (LAS u32*)(d + 4096), 16, 1024, 17);
            __builtin_amdgcn_global_load_lds((const GAS u32*)(s + 4096 + voff), (LAS u32*)(d + 4096), 16, 0, 17);
            now = clock(); pace(G2);
            __builtin_amdgcn_global_load_lds((const GAS u32*)(s + voff), (LAS u32*)d, 16, 0, 17);
            __builtin_amdgcn_global_load_lds((const GAS u32*)(s + voff), (LAS u32*)d, 16, 1024, 17);
            now = clock(); pace(80 + tail - G1 - G2);
            __builtin_amdgcn_global_load_lds((const GAS u32*)(s + voff), (LAS u32*)d, 16, 2048, 17);
            __builtin_amdgcn_global_load_lds((const GAS u32*)(s + voff), (LAS u32*)d, 16, 3072, 17);
            asm volatile("s_waitcnt vmcnt(%0)" :: "i"(6 * WIN) : "memory");
            if (k + 1 > landed + WIN) landed = k + 1 - WIN;
            if (lane == 0) { c[9] = k; c[10] = landed; }
            if (++slot == NSLOT) slot = 0;
        }
        if (lane == 0) c[10] = k | FIN;
    } else {
        if (s0 < 63) for (int t = wid * 32 + s0 / 2; t < 28 * NPUSH; t += 32 * NDEC) push_task(p, t, cnt_v, tstart, lane);
        WAIT_P;
        const u32 seq = __builtin_amdgcn_readfirstlane(cnt_v) + 1u;
        u32 pend = 0;
        u32 k03 = 0x03030303u, kfc = ~3u, khi = 0xffff0000u;
        asm volatile("" : "+s"(k03), "+s"(kfc), "+s"(khi));
        const __amdgpu_buffer_rsrc_t prs = __builtin_amdgcn_make_buffer_rsrc(ws + OFF_P, 0, 28 * (int)MiB, 0x00020000);
        for (;;) {
            const u32 cv = ctl[lane & 31];
            const u32 l0 = __builtin_amdgcn_readlane(cv, 10), e0 = __builtin_amdgcn_readlane(cv, 11), l1 = __builtin_amdgcn_readlane(cv, 26), e1 = __builtin_amdgcn_readlane(cv, 27);
            const u32 a0 = (l0 & ~FIN) - e0, a1 = (l1 & ~FIN) - e1;
            if ((a0 | a1) == 0u) {
                if (l0 & l1 & FIN) break;
                if (pend) { WAIT_V(0); if (lane == 0) st_agent(ws + pend, seq); pend = 0; }
                else __builtin_amdgcn_s_sleep(1);
                continue;
            }
            const u32 st = a1 > a0 || (a1 == a0 && (wid & 1)), e = st ? e1 : e0, sn = e % NSLOT;
            const u32 sb = (u32)(size_t)lds + L_SLOT + (st * NSLOT + sn) * 8192u;
            const LAS u32x4* sl = (const LAS u32x4*)(size_t)(sb + voff);
            asm volatile("" ::: "memory");
            u32 got = e;
            if (lane == 0) __hip_atomic_compare_exchange_strong((LAS u32*)(lds + L_CTL + st * 64 + 44), &got, e + 1, __ATOMIC_RELAXED, __ATOMIC_RELAXED, __HIP_MEMORY_SCOPE_WORKGROUP);
            u32 rec = desc[(st * NDESC + e % NDESC) * 16 + (lane & 15)], q2 = *(const LAS u32*)(size_t)(sb + 5120u + lane * 4u);
            u32x4 Q = sl[4 * 64], S[4];
#pragma unroll
            for (int q = 0; q < 4; ++q) S[q] = sl[q * 64];
            asm volatile("" : "+v"(got), "+v"(rec), "+v"(q2), "+v"(Q) :: "memory");
            if (__builtin_amdgcn_readfirstlane(got) != e) continue;
            const u32 ent = __builtin_amdgcn_readlane(rec, 11), aux = __builtin_amdgcn_readlane(rec, 14), head = __builtin_amdgcn_readfirstlane(rec);
            char* dst = (char*)(((u64)(u32)__builtin_amdgcn_readlane(rec, 13) << 32) | (u32)__builtin_amdgcn_readlane(rec, 12));
            u32x4 O[8];
            const u32 cnt = (head >> 8) & 7u;
            auto unpack = [&](auto emit) __attribute__((always_inline)) {
            if (head & (H_RAW | H_DEAD)) {
#pragma unroll
                for (int q = 0; q < 8; ++q) O[q] = q < 4 ? S[q] : q == 4 ? Q : sl[q * 64];
                WAIT_L(0);
                if (lane == 0) ctl[st * 16 + sn] = e + 1;
                if (head & H_DEAD) {
#pragma unroll
                    for (int q = 0; q < 8; ++q) O[q] = (u32x4){0x7fc07fc0u, 0x7fc07fc0u, 0x7fc07fc0u, 0x7fc07fc0u};
                }
                emit(IC<0>{}); emit(IC<1>{}); emit(IC<2>{}); emit(IC<3>{}); emit(IC<4>{}); emit(IC<5>{}); emit(IC<6>{}); emit(IC<7>{});
            } else {
                u32 c[16], m[16], bo[16];
                u32x2 w[16];
                const u32 T4 = (head & 0xffu) * 0x01010101u;
                auto expo = [&](u32 v) { return T4 - ((v & 0x0f0f0f0fu) ^ 0x05050505u); };
                const u32 oi = (u32)lane - cnt - 1u;
                u32 n = (u32)__builtin_popcount(q2 & (q2 >> 1) & 0x55555555u) << 16;
#pragma unroll
                for (int q = 0; q < 4; ++q) n += (u32)__builtin_popcount(Q[q] & (Q[q] >> 1) & 0x55555555u);
                n = scan32(n) - n;
#define ALL16(stmt) do { _Pragma("unroll") for (int t = 0; t < 16; ++t) { stmt; } SCHED; } while (0)
                u32 c2[4], m2[4], b2[4];
                u32x2 w2[4];
                {
                    u32 bits2 = (min(n >> 16, 128u) + sb + 5376u) * 8u + (lane & 32 ? 4u : 0u);
#pragma unroll
                    for (int t = 0; t < 4; ++t) {
                        c2[t] = (q2 >> (2 * t)) & k03; m2[t] = perm(0u, 0x01000000u, c2[t]);
                        b2[t] = bits2;
                        const LAS u32* q = (const LAS u32*)(size_t)((bits2 >> 3) & kfc);
                        w2[t] = (u32x2){q[0], q[1]};
                        bits2 = __builtin_amdgcn_udot4(m2[t], 0x08080808u, bits2, false);
                    }
                    SCHED;
                }
                ALL16(c[t] = (Q[t >> 2] >> (2 * (t & 3))) & k03; m[t] = perm(0u, 0x01000000u, c[t]));
                u32 pos = min(n & 0xffffu, 1024u - 64u) + sb + (lane & 32 ? 7168u : 6144u);
                u32 ad[16], x1[16];
                ALL16(bo[t] = pos; ad[t] = pos & kfc; pos = __builtin_amdgcn_udot4(m[t], 0x01010101u, pos, false));
                if (head & H_L3) {
                    u32 g[4];
#pragma unroll
                    for (int t = 0; t < 4; ++t) {
                        asm("v_mul_u32_u24 %0, 0x10101, %1" : "=v"(m2[t]) : "v"(m2[t]));
                        g[t] = T4 - ((perm(0u, __builtin_amdgcn_alignbit(w2[t][1], w2[t][0], b2[t]), (m2[t] << 8) + perm(0u, 0x00101010u, c2[t])) & perm(0u, 0x0f040300u, c2[t])) ^ 0x05050505u);
                    }
                    *(LAS u32x2*)(size_t)(sb + 6144u + lane * 8u) = (u32x2){g[0], g[1]};
                    *(LAS u32x2*)(size_t)(sb + 7168u + lane * 8u) = (u32x2){g[2], g[3]};
                    if (head >> 29) {
                        if (oi < (head >> 29)) { *(LAS u32*)(size_t)(sb + 6656u + oi * 4u) = expo(rec); *(LAS u32*)(size_t)(sb + 7680u + oi * 4u) = expo(rec >> 4); }
                    }
                } else {
                    if (oi < (head >> 29)) *(LAS u32*)(size_t)(sb + 5120u + ((head >> 13) & 7u) * 128u + oi * 4u) = rec;
                    asm volatile("" ::: "memory");
                    const u32x4 v = sl[5 * 64];
                    *(LAS u32x4*)(size_t)(sb + 6144u + voff) = (u32x4){expo(v[0]), expo(v[1]), expo(v[2]), expo(v[3])};
                    *(LAS u32x4*)(size_t)(sb + 7168u + voff) = (u32x4){expo(v[0] >> 4), expo(v[1] >> 4), expo(v[2] >> 4), expo(v[3] >> 4)};
                }
                if (cnt) { if ((u32)lane - 1u < cnt) *(LAS unsigned char*)(size_t)(sb + 6144u + ((rec >> 8) & 2047u)) = (unsigned char)rec; }
                asm volatile("" ::: "memory");
                ALL16(w[t] = ((u32x2){((const LAS u32*)(size_t)ad[t])[0], ((const LAS u32*)(size_t)ad[t])[1]}));
                const u32 zl = ~(T4 - 0x00040302u) & 0x00ffffffu;
                ALL16(asm("v_mul_u32_u24 %0, 0x10101, %1" : "=v"(m[t]) : "v"(m[t])); c[t] = perm(0u, zl, c[t]));
                ALL16(x1[t] = c[t] + (m[t] << 8));
                WAIT_L(0);
                if (lane == 0) ctl[st * 16 + sn] = e + 1;
                SCHED;
                auto fin = [&](auto hc) __attribute__((always_inline)) {
                    constexpr int h = decltype(hc)::v, d = h >> 1;
#pragma unroll
                    for (int t = 2 * h; t < 2 * h + 2; ++t) {
                        const u32 E = perm(0u, __builtin_amdgcn_alignbyte(w[t][1], w[t][0], bo[t]), x1[t]) & ~c[t];
                        const u32 ra = perm(E, S[d][t & 3], 0x05010400u), rb = perm(E, S[d][t & 3], 0x07030602u);
                        O[2 * d][t & 3] = __builtin_amdgcn_alignbit(ra, ra, 1); O[2 * d + 1][t & 3] = __builtin_amdgcn_alignbit(rb, rb, 1);
                    }
                    if constexpr (h > 0) emit(IC<h - 1>{});
                    SCHED;
                };
                fin(IC<0>{}); fin(IC<1>{}); fin(IC<2>{}); fin(IC<3>{}); fin(IC<4>{}); fin(IC<5>{}); fin(IC<6>{}); fin(IC<7>{});
                emit(IC<7>{});
            }
            };
            u32 flag = 0;
            if (!(ent & 0x8000u)) {
                unpack([&](auto dc) __attribute__((always_inline)) { constexpr int d = decltype(dc)::v;
                    st16_sys<1024 * d - 4096>(dst, voff, O[d]); });
                flag = aux;
            } else {
                u32x4 P[8];
                const u32 so = aux;
#pragma unroll
                for (int q = 0; q < 8; ++q) P[q] = __builtin_amdgcn_raw_buffer_load_b128(prs, voff + (q & 3) * 1024, so + (q >> 2) * 4096, 17);
                asm volatile("" ::: "memory");
                const u32 vo = (u32)(lane >> 4) * 262144u + (u32)(lane & 15) * 16u;
                unpack([](auto) {});
                WAIT_P;
#pragma unroll
                for (int q = 0; q < 8; ++q) {
#pragma unroll
                    for (int i = 0; i < 4; ++i) {
                        const f32x2 y = (f32x2){__uint_as_float(O[q][i] << 16), __uint_as_float(O[q][i] & khi)} + (f32x2){__uint_as_float(P[q][i] << 16), __uint_as_float(P[q][i] & khi)};
                        O[q][i] = cvt_pk_bf16(y[0], y[1]);
                    }
                    st16_sys<0>(dst + (size_t)((q >> 2) * 16 + (q & 3)) * 65536, vo, O[q]);
                    SCHED;
                }
            }
            if (pend) { WAIT_V(8); if (lane == 0) st_agent(ws + pend, seq); }
            pend = flag;
        }
        if (pend) { WAIT_V(0); if (lane == 0) st_agent(ws + pend, seq); }
    }
}

__device__ __forceinline__ void spin_flags(char* ws, const char* fbase, u32 fvoff, u32 seq) {
    const u64 t0 = __builtin_amdgcn_s_memrealtime();
    while (__builtin_amdgcn_ballot_w64(__hip_atomic_load((GAS u32*)(fbase + fvoff), __ATOMIC_RELAXED, __HIP_MEMORY_SCOPE_SYSTEM) < seq) != 0ull) {
        if (ld_agent(ws + OFF_TMO) == seq || __builtin_amdgcn_s_memrealtime() - t0 > DEADLINE) { st_agent(ws + OFF_TMO, seq); break; }
        __builtin_amdgcn_s_sleep(4);
    }
    __builtin_amdgcn_s_waitcnt(0xC07F);
    WAIT_V(0);
}

__device__ __forceinline__ void gemm_loop(char* ws, LAS unsigned char* lds, const char* gX, const char* gW, u32 fvoff, int otab, int nstep, u32 seq, int wid, int lane, f32x4 (&acc)[2][2][4][2]) {
    constexpr size_t KSTEP = MiB, SX = 16384, SW = 16384;
    constexpr int FSTEP = 4096;
    const int wr = wid >> 2, wc = wid & 3, fr = lane & 15, fq = lane >> 4;
    const u32 voff = (u32)lane * 16u;
    const char* cX = uni(gX + wid * 1024);
    const char* cW = uni(gW + (wid >> 2) * 8192 + (wid & 3) * 1024);
    const char* fb = ws + OFF_FG;
    const u32 ldsw = (u32)wid * 1024u;
    const int woff = wr * 8192 + lane * 16;
    const int xoff0 = (wc * 32 + fr) * 128 + ((fq ^ (fr >> 1)) << 4), xoff1 = xoff0 ^ 64;
    const LAS unsigned char* xb0 = lds + 65536 + xoff0; const LAS unsigned char* xb1 = lds + 65536 + xoff1;
    asm("" : "+v"(xb0), "+v"(xb1));
#define SA(b, h) (((b) * 2 + (h)) * 16384)
#define SB(b, h) ((4 + (b) * 2 + (h)) * 16384)
#define GLDS(src, dst) __builtin_amdgcn_global_load_lds((const GAS u32*)((src) + voff), (LAS u32*)(lds + (dst) + ldsw), 16, 0, GAUX)
#define STW(b, h, ptr) do { GLDS((ptr) + (h) * 4096, SA(b, h)); GLDS((ptr) + (h) * 4096 + SW, SA(b, h) + 8192); } while (0)
#define STX(b, h, ptr) do { GLDS((ptr) + (h) * SX, SB(b, h)); GLDS((ptr) + (h) * SX + 8192, SB(b, h) + 8192); } while (0)
#define LDW(b, h) do { _Pragma("unroll") for (int m = 0; m < 4; ++m) _Pragma("unroll") for (int k = 0; k < 2; ++k) Wt[m][k] = *(const LAS bf16x8*)(lds + SA(b, h) + woff + k * 4096 + m * 1024); } while (0)
#define LDX(dst, b, h) do { _Pragma("unroll") for (int n = 0; n < 2; ++n) { dst[n][0] = *(const LAS bf16x8*)(xb0 + SA(b, h) + n * 2048); dst[n][1] = *(const LAS bf16x8*)(xb1 + SA(b, h) + n * 2048); } } while (0)
#define MMA(h, xh, Xt) do { _Pragma("unroll") for (int m = 0; m < 4; ++m) _Pragma("unroll") for (int n = 0; n < 2; ++n) _Pragma("unroll") for (int k = 0; k < 2; ++k) \
        acc[h][xh][m][n] = __builtin_amdgcn_mfma_f32_16x16x32_bf16(Xt[n][k], Wt[m][k], acc[h][xh][m][n], 0, 0, 0); } while (0)
#define FLAG_LOAD(step) asm volatile("s_nop 4\n\tglobal_load_dword %0, %1, %2 sc0 sc1" : "+v"(fl) : "v"(fvoff), "s"(fb + (size_t)(step) * FSTEP) : "memory")
#define FLAG_CHECK(step) do { if (wid == 0 && __builtin_amdgcn_ballot_w64(fl < seq) != 0ull) spin_flags(ws, fb + (size_t)(step) * FSTEP, fvoff, seq); } while (0)
#pragma unroll
    for (int a = 0; a < 2; ++a)
#pragma unroll
        for (int b = 0; b < 2; ++b)
#pragma unroll
            for (int m = 0; m < 4; ++m)
#pragma unroll
                for (int n = 0; n < 2; ++n) acc[a][b][m][n] = (f32x4){0.f, 0.f, 0.f, 0.f};
    bf16x8 Wt[4][2], X0[2][2], X1[2][2];
    u32 fl = 0;
    if (wid == 0) { spin_flags(ws, fb, fvoff, seq); spin_flags(ws, fb + FSTEP, fvoff, seq); }
    WAIT_P; WAIT_V(0); WAIT_L(0); BAR; asm volatile("" ::: "memory");
#define PRIO(n) __builtin_amdgcn_s_setprio(n)
#define WAIT_VN(n) asm volatile("s_waitcnt vmcnt(" #n ")" : "+v"(fl) :: "memory")
    FLAG_LOAD(2);
    STX(0, 0, cX); STX(0, 1, cX); STW(0, 0, cW); STW(0, 1, cW);
    STX(1, 0, cX + KSTEP); STX(1, 1, cX + KSTEP); STW(1, 0, cW + KSTEP);
    WAIT_VN(8); BAR;
    if (wr == 1) BAR;
    for (int t = 0; t < nstep; t += 2) {
        const int t1 = __builtin_amdgcn_readlane(otab, t + 1), t2 = __builtin_amdgcn_readlane(otab, t + 2);
        const int t3 = __builtin_amdgcn_readlane(otab, t + 3), t4 = __builtin_amdgcn_readlane(otab, t + 4);
        const char* w1 = cW + (size_t)t1 * KSTEP;
        const char* x2 = cX + (size_t)t2 * KSTEP; const char* w2 = cW + (size_t)t2 * KSTEP;
        const char* x3 = cX + (size_t)t3 * KSTEP; const char* w3 = cW + (size_t)t3 * KSTEP;
        FLAG_CHECK(t2);
        LDX(X0, 0, 0); LDW(0, 0); LDX(X1, 0, 1); FLAG_LOAD(t3); STW(1, 1, w1);
        WAIT_VN(9); WAIT_L(0); SCHED; BAR; PRIO(1); SCHED; MMA(0, 0, X0); MMA(0, 1, X1); PRIO(0); SCHED; BAR; SCHED;
        LDW(0, 1); STX(0, 0, x2); STX(0, 1, x2); STW(0, 0, w2);
        WAIT_VN(8); WAIT_L(0); SCHED; BAR; PRIO(1); SCHED; MMA(1, 0, X0); MMA(1, 1, X1); PRIO(0); SCHED; BAR; SCHED;
        FLAG_CHECK(t3);
        LDX(X0, 1, 0); LDW(1, 0); LDX(X1, 1, 1); FLAG_LOAD(t4); STW(0, 1, w2);
        WAIT_VN(9); WAIT_L(0); SCHED; BAR; PRIO(1); SCHED; MMA(0, 0, X0); MMA(0, 1, X1); PRIO(0); SCHED; BAR; SCHED;
        LDW(1, 1); STX(1, 0, x3); STX(1, 1, x3); STW(1, 0, w3);
        WAIT_VN(8); WAIT_L(0); SCHED; BAR; PRIO(1); SCHED; MMA(1, 0, X0); MMA(1, 1, X1); PRIO(0); SCHED; BAR; SCHED;
    }
#undef WAIT_VN
#undef PRIO
    asm volatile("s_waitcnt vmcnt(0)" : "+v"(fl) :: "memory");
    if (wr == 0) BAR;
    BAR;
#undef SA
#undef SB
#undef GLDS
#undef STW
#undef STX
#undef LDW
#undef LDX
#undef MMA
#undef FLAG_LOAD
#undef FLAG_CHECK
}

__device__ __forceinline__ void gemm_piece(char* ws, LAS unsigned char* lds, const char* gX, const char* gW, u32 fx, u32 fw, int n0, int n1, u32 seq, int wid, int lane, f32x4 (&acc)[2][2][4][2]) {
    const int wr = wid >> 2, wc = wid & 3, fr = lane & 15, fq = lane >> 4;
    const u32 voff = (u32)lane * 16u;
    const char* cX = uni(gX + wc * 4096);
    const char* cW = uni(gW + wc * 4096);
    const int xoff0 = (wc * 32 + fr) * 128 + ((fq ^ (fr >> 1)) << 4), xoff1 = xoff0 ^ 64;
#pragma unroll
    for (int a = 0; a < 2; ++a)
#pragma unroll
        for (int m = 0; m < 4; ++m)
#pragma unroll
            for (int n = 0; n < 2; ++n) acc[a][0][m][n] = (f32x4){0.f, 0.f, 0.f, 0.f};
    const u32 fo = ((wid & 2) ? fw : fx) + (wid & 1) * 4;
    if (wid < 4) spin_flags(ws, ws + OFF_FG, gslot(min(n0 + min(lane, 35), n1 - 1), seq) * 4096u + fo, seq);
    WAIT_P; WAIT_V(0); WAIT_L(0); BAR; asm volatile("" ::: "memory");
#define PSRC(base, n) ((base) + (size_t)gslot(min((n), n1 - 1), seq) * MiB + voff)
#define PLD(src, dst, off) __builtin_amdgcn_global_load_lds((const GAS u32*)(src), (LAS u32*)(dst), 16, off, GAUX)
#define PLD4(src, dst) do { PLD(src, dst, 0); PLD(src, dst, 1024); PLD(src, dst, 2048); PLD(src, dst, 3072); } while (0)
    LAS unsigned char* d0 = lds + wc * 4096;
    PLD4(PSRC(cX, n0 + wr), d0 + wr * 32768); PLD4(PSRC(cW, n0 + wr), d0 + wr * 32768 + 16384);
    PLD4(PSRC(cX, n0 + wr + 2), d0 + (wr + 2) * 32768); PLD4(PSRC(cW, n0 + wr + 2), d0 + (wr + 2) * 32768 + 16384);
    WAIT_V(8); BAR;
    if (wr == 1) BAR;
    int buf = wr;
    for (int n = n0 + wr; n < n1; n += 2) {
        bf16x8 Wt[8][2], Xt[2][2];
        const LAS unsigned char* bx = lds + buf * 32768;
        LAS unsigned char* dn = d0 + (buf ? buf - 1 : 4) * 32768;
        if (n == 30 && n1 == 64) spin_flags(ws, ws + OFF_FG, (u32)(lane < 36 ? 36 : lane) * 4096u + fo, seq);
#pragma unroll
        for (int i = 0; i < 2; ++i) { Xt[i][0] = *(const LAS bf16x8*)(bx + xoff0 + i * 2048); Xt[i][1] = *(const LAS bf16x8*)(bx + xoff1 + i * 2048); }
#pragma unroll
        for (int k = 0; k < 2; ++k)
#pragma unroll
            for (int c = 0; c < 8; ++c) Wt[c][k] = *(const LAS bf16x8*)(bx + 16384 + voff + k * 8192 + c * 1024);
        const char* xn = PSRC(cX, n + 4); const char* wn = PSRC(cW, n + 4);
        PLD(xn, dn, 0);
        WAIT_L(0); SCHED; BAR; SCHED;
        __builtin_amdgcn_s_setprio(1);
#pragma unroll
        for (int g = 0; g < 8; ++g) {
#pragma unroll
            for (int c = (g & 3) * 2; c < (g & 3) * 2 + 2; ++c)
#pragma unroll
                for (int i = 0; i < 2; ++i) acc[c >> 2][0][c & 3][i] = __builtin_amdgcn_mfma_f32_16x16x32_bf16(Xt[i][g >> 2], Wt[c][g >> 2], acc[c >> 2][0][c & 3][i], 0, 0, 0);
            SCHED;
            if (g == 0) PLD(wn, dn + 16384, 0); else if (g == 1) PLD(wn, dn + 16384, 1024); else if (g == 2) PLD(wn, dn + 16384, 2048); else if (g == 3) PLD(wn, dn + 16384, 3072);
            else if (g == 4) PLD(xn, dn, 3072); else if (g == 5) PLD(xn, dn, 2048); else if (g == 6) PLD(xn, dn, 1024);
            SCHED;
        }
        __builtin_amdgcn_s_setprio(0);
        WAIT_V(8); BAR; asm volatile("" ::: "memory");
        buf = buf >= 3 ? buf - 3 : buf + 2;
    }
#undef PSRC
#undef PLD
#undef PLD4
    WAIT_V(0);
    if (wr == 0) BAR;
    BAR; asm volatile("" ::: "memory");
}

__device__ __forceinline__ bool gemm_local(const Params& p, char* ws, const LAS unsigned char* lds, u32 seq, int wid, int lane, f32x4 (&acc)[2][2][4][2]) {
    const __amdgpu_buffer_rsrc_t xr = __builtin_amdgcn_make_buffer_rsrc(ws + OFF_XF + wid * 65536, 0, 65536, 0x00020000);
    auto ldx = [&](int u, int i) { return __builtin_bit_cast(bf16x8, __builtin_amdgcn_raw_buffer_load_b128(xr, lane * 16 + i * 1024, u * 4096, 17)); };
    if (wid == 0) spin_flags(ws, ws + OFF_FG, (u32)((lane & 7) * 4096 + (p.rank * 2 + (lane >> 5)) * 128 + ((lane >> 3) & 3) * 4), seq);
    __syncthreads();
    u32 tmo = ld_agent(ws + OFF_TMO);
#pragma unroll
    for (int i = 0; i < 32; ++i) acc[i >> 4][(i >> 3) & 1][(i >> 1) & 3][i & 1] = (f32x4){0.f, 0.f, 0.f, 0.f};
    bf16x8 Xa[4], Xb[4];
#pragma unroll
    for (int i = 0; i < 4; ++i) Xa[i] = ldx(0, i);
    auto step = [&](int u, const bf16x8 (&X)[4]) __attribute__((always_inline)) {
        bf16x8 w[8];
#pragma unroll
        for (int c = 0; c < 8; ++c) w[c] = *(const LAS bf16x8*)(lds + u * 8192 + c * 1024 + lane * 16);
        SCHED;
#pragma unroll
        for (int c = 0; c < 8; ++c)
#pragma unroll
            for (int i = 0; i < 4; ++i) acc[c >> 2][i >> 1][c & 3][i & 1] = __builtin_amdgcn_mfma_f32_16x16x32_bf16(X[i], w[c], acc[c >> 2][i >> 1][c & 3][i & 1], 0, 0, 0);
        SCHED;
    };
#pragma unroll 1
    for (int u = 0; u < 16; u += 2) {
#pragma unroll
        for (int i = 0; i < 4; ++i) Xb[i] = ldx(u + 1, i);
        step(u, Xa);
#pragma unroll
        for (int i = 0; i < 4; ++i) Xa[i] = ldx((u + 2) & 15, i);
        step(u + 1, Xb);
    }
    WAIT_P;
    asm volatile("" : "+v"(tmo));
    return __builtin_amdgcn_readfirstlane(tmo) == seq;
}

__device__ __forceinline__ void acc_to_raw(const f32x4 (&acc)[2][2][4][2], int xh, bool poison, u32x4 (&O)[8]) {
#pragma unroll
    for (int n = 0; n < 2; ++n)
#pragma unroll
        for (int j = 0; j < 4; ++j)
#pragma unroll
            for (int c = 0; c < 4; ++c) O[n * 4 + j][c] = cvt_pk_bf16(acc[c >> 1][xh][(c & 1) * 2][n][j], acc[c >> 1][xh][(c & 1) * 2 + 1][n][j]);
    if (poison) {
        asm volatile("" ::: "memory");
#pragma unroll
        for (int q = 0; q < 8; ++q) O[q] = (u32x4){0x7fc07fc0u, 0x7fc07fc0u, 0x7fc07fc0u, 0x7fc07fc0u};
    }
}

__global__ __launch_bounds__(512, 2) void fused_gemm_rs(Params p) {
    extern __shared__ __attribute__((aligned(16))) unsigned char shm[];
    LAS unsigned char* lds = (LAS unsigned char*)shm;
    const int lane = threadIdx.x & 63, wid = __builtin_amdgcn_readfirstlane(threadIdx.x >> 6), b = blockIdx.x;
    const int xcd = b & 7, ix = b >> 3;
    const int li = ix - 4, cg = li / 7, pmi = ((xcd & 1) * 7 + li % 7) & 15, pn = ((xcd & 6) + cg + (cg < 2 ? 0 : 6)) & 15;
    char* ws = p.wsme;
    char* wd = p.wsr[pmi >> 1];
    u32 sc0 = p.sched[pn][0], sc1 = p.sched[pn][1];
    asm volatile("" : "+s"(ws), "+s"(wd), "+s"(sc0), "+s"(sc1));
    const u32 cnt_v = ld_agent(ws + OFF_CNT + b * 64);
    const u64 t_start = __builtin_amdgcn_s_memrealtime();
    u32 seq;
    int s0 = b * 2;
    if (ix >= 4) {
        const u32 tl = (u32)t_start;
        LAS unsigned char* sc = lds + LDS_GEMM + wid * ENC_BYTES;
        enc_init(sc, lane);
        const int dest = (p.rank + 1 + (pmi >> 1)) & 7, pm = dest * 2 + (pmi & 1), dw = pmi >> 1;
        f32x4 acc[2][2][4][2];
        auto local = [&](int q) __attribute__((always_inline)) {
            if (pn >= 8 && pn <= 12) for (int i = (li % 7) * 2; i > 0; --i) __builtin_amdgcn_s_sleep(8);
            const bool poison = gemm_local(p, ws, lds, seq, wid, lane, acc);
#pragma unroll
            for (int xh = 0; xh < 2; ++xh) {
                u32x4 O[8]; acc_to_raw(acc, xh, poison, O);
                store_unit_linear(ws + OFF_P + (size_t)(dw * 512 + (((q * 2 + (wid >> 2)) * 8 + (pmi & 1) * 4 + (wid & 1) * 2 + xh) * 2 + ((wid >> 1) & 1))) * 8192, (u32)lane * 16u, O);
            }
            WAIT_V(0);
            if (lane == 0) __hip_atomic_fetch_add((GAS u32*)(ws + OFF_PD), 1u, __ATOMIC_RELAXED, __HIP_MEMORY_SCOPE_AGENT);
        };
        auto wcode = [&](int q, int i) { return (i * 4 + (wid >> 1)) << 10 | (128 + dw * 64 + q * 4 + (pmi & 1) * 2 + (wid & 1)); };
        const int nd = sc1 & 255;
        for (int it = 0; it == 0 || it < nd; ++it) {
            const int q = (sc1 >> (8 * it + 8)) & 255;
            int np = 0, t0 = 0, t1 = 0;
            if (it == 0) { t0 = (sc0 & 0xffff) + wid * p.pstride[0] + pmi; t1 = (sc0 >> 16) + wid * p.pstride[1] + pmi; np = (t0 < NPLAIN) + (t1 < NPLAIN); }
            pack_tasks(p, ws, wd, [&](int i) { return i < np ? x_task(i ? t1 : t0, p.rank) : (nd && i < np + 2) ? wcode(q, i - np) : -1; }, cnt_v, tl, lds, sc, wid, lane);
            if (!nd) break;
            seq = __builtin_amdgcn_readfirstlane(cnt_v) + 1u;
            local(q);
            __syncthreads();
        }
        seq = __builtin_amdgcn_readfirstlane(cnt_v) + 1u;
        {
            const int l8 = lane - 8;
            gemm_loop(ws, lds, ws + OFF_G + (size_t)pm * 32768, ws + OFF_G + (size_t)(16 + pn) * 32768, (u32)(((lane & 4) ? 16 + pn : pm) * 128 + (lane & 3) * 4),
                      (lane < 8 || lane >= 56) ? (lane & 7) : (int)gslot(8 + l8 / 6 * 7 + (dw + 13 - l8 % 6) % 7, seq), 56, seq, wid, lane, acc);
        }
        {
            u32 tmo = ld_agent(ws + OFF_TMO);
            const int id0 = ((pn * 2 + (pmi & 1)) * 8 + wid) * 2;
            u32 rec = 0, pos[2], T = 0;
            char* ry = wd + OFF_RY + (size_t)((seq & 1) * 8 + p.rank) * 512 * 64 + lane * 8;
#pragma unroll
            for (int xh = 0; xh < 2; ++xh) {
                pos[xh] = 0;
                char* yq = ws + OFF_YQ + dest * 128;
                asm volatile("" : "+v"(yq));
                if (lane == 0) pos[xh] = __hip_atomic_fetch_add((GAS u32*)yq, 1u, __ATOMIC_RELAXED, __HIP_MEMORY_SCOPE_AGENT);
                u32x4 O[8];
                acc_to_raw(acc, xh, false, O);
                rec = encode_store(O, ws + OFF_PY + (size_t)((seq & 1) * 8 + dest) * (4 * MiB) + (size_t)(id0 + xh) * 8192, sc, lane, T, true,
                                   [&] { if (xh == 1) { WAIT_V(0); put_record(ry + (__builtin_amdgcn_readfirstlane(pos[0]) & 511u) * 64, seq, rec, lane, 8); }
                                         asm volatile("" : "+v"(tmo)); return __builtin_amdgcn_readfirstlane(tmo) == seq; });
                if (lane == 0) rec |= (u32)(id0 + xh) << 20;
            }
            WAIT_V(0);
            put_record(ry + (__builtin_amdgcn_readfirstlane(pos[1]) & 511u) * 64, seq, rec, lane, 8);
        }
        s0 = pn < HC ? 63 + (pn * 14 + pmi) * 2 : -1;
        for (bool first = pn < HC + NSTAT; s0 < 0; first = false) {
            __syncthreads();
            if (threadIdx.x == 0) {
                volatile LAS u32* t = (volatile LAS u32*)(lds + LDS_GEMM);
                t[0] = first ? (u32)((pn - HC) * 14 + pmi)
                             : 14u * NSTAT + __hip_atomic_fetch_add((GAS u32*)(ws + OFF_TICKET), 1u, __ATOMIC_RELAXED, __HIP_MEMORY_SCOPE_AGENT) - (seq - 1u) * (NGEMM - 14 * HC + 128 + NHALF - 14 * NSTAT);
            }
            __syncthreads();
            u32 tk = __builtin_amdgcn_readfirstlane(*(volatile LAS u32*)(lds + LDS_GEMM));
            if (tk >= 128u + NHALF) break;
            const int half = tk >= 128u - NHALF ? (tk - (128u - NHALF)) & 1 : -1;
            if (half >= 0) tk = (tk + 128u - NHALF) >> 1;
            const int rq = tk & 3, cq = tk >> 2, opm = p.rank * 2 + (rq >> 1), opn = cq >> 1;
            gemm_piece(ws, lds, ws + OFF_G + (size_t)opm * 32768 + (rq & 1) * 16384, ws + OFF_G + (size_t)(16 + opn) * 32768 + (cq & 1) * 16384,
                       (u32)(opm * 128 + (rq & 1) * 8), (u32)((16 + opn) * 128 + (cq & 1) * 8), half == 1 ? 32 : 0, half == 0 ? 32 : 64, seq, wid, lane, acc);
            LAS f32x4* red = (LAS f32x4*)(lds + (wid & 3) * 16384) + lane;
            if (wid >= 4) {
#pragma unroll
                for (int i = 0; i < 16; ++i) red[i * 64] = acc[i >> 3][0][(i >> 1) & 3][i & 1];
            }
            __syncthreads();
            if (wid < 4) {
#pragma unroll
                for (int i = 0; i < 16; ++i) acc[i >> 3][0][(i >> 1) & 3][i & 1] += red[i * 64];
                char* hs = ws + OFF_H + (size_t)((tk & 63) * 4 + wid) * 16384;
                if (half == 0) {
#pragma unroll
                    for (int h = 0; h < 2; ++h) { u32x4 O[8];
#pragma unroll
                        for (int i = 0; i < 8; ++i) O[i] = __builtin_bit_cast(u32x4, acc[h][0][i >> 1][i & 1]);
                        store_unit_linear(hs + h * 8192, (u32)lane * 16u, O); }
                    WAIT_V(0);
                    if (lane == 0) st_agent(ws + OFF_HF + tk * 128 + wid * 4, seq);
                    continue;
                }
                if (half == 1) {
                    spin_flags(ws, ws + OFF_HF + tk * 128 + wid * 4, 0u, seq);
                    const __amdgpu_buffer_rsrc_t hr = __builtin_amdgcn_make_buffer_rsrc(hs, 0, 16384, 0x00020000);
#pragma unroll
                    for (int i = 0; i < 16; ++i) acc[i >> 3][0][(i >> 1) & 3][i & 1] += __builtin_bit_cast(f32x4, __builtin_amdgcn_raw_buffer_load_b128(hr, lane * 16, i * 1024, 17));
                }
                const bool poison = __builtin_amdgcn_readfirstlane(ld_agent(ws + OFF_TMO)) == seq;
                u32x4 O[8]; acc_to_raw(acc, 0, poison, O);
                char* dst = p.out + (size_t)(rq * 128 + wid * 32) * 65536 + (p.rank * 4096 + cq * 128) * 2;
                const u32 vo = (u32)(lane >> 4) * 262144u + (u32)(lane & 15) * 16u;
#pragma unroll
                for (int q = 0; q < 8; ++q) st16_sys<0>(dst + (size_t)((q >> 2) * 16 + (q & 3)) * 65536, vo, O[q]);
            }
        }
    }
    if (s0 >= 0) comm_role(p, ws, lds, s0, cnt_v, wid, lane, (u32)t_start);
    __syncthreads();
    seq = __builtin_amdgcn_readfirstlane(cnt_v) + 1u;
    if (threadIdx.x == 0) st_agent(ws + OFF_CNT + b * 64, seq);
}

extern "C" void kernel_launch(int rank, int nranks, ncclComm_t comm, hipStream_t stream,
                              void* const* d_in, const int* in_sizes, int n_in,
                              void* d_out, int out_size, const int* in_dtypes, int out_dtype,
                              void* const* d_out_all, void* const* d_ws_all, size_t ws_bytes) {
    if (nranks != NR || (unsigned)rank >= (unsigned)NR || n_in != 2 || ws_bytes < WS_NEED || in_sizes[0] != 4096 * 512 || in_sizes[1] != 512 * 32768 || out_size != 512 * 32768) {
        fprintf(stderr, "kernel_launch: unexpected shapes\n");
        return;
    }
    HIP_CHECK(hipFuncSetAttribute((const void*)fused_gemm_rs, hipFuncAttributeMaxDynamicSharedMemorySize, LDS_BYTES));
    Params p;
    memset(&p, 0, sizeof p);
    p.x = (const char*)d_in[0]; p.w = (const char*)d_in[1]; p.out = (char*)d_out;
    for (int i = 0; i < NR; ++i) p.wsr[i] = (char*)d_ws_all[(rank + 1 + i) % NR];
    p.ws_tab = (char* const*)d_ws_all;
    p.rank = rank; p.wsme = (char*)d_ws_all[rank];
    int key[256];
    for (int i = 0; i < 256; ++i) {
        const int j = i >> 5, k = i & 31;
        key[i] = k >= 16 ? 112 * j + SKEW_NUM * (k - 16) - 1 : (k >= 14 ? 100000 + j : (k >= 12 || (j == 0 && k < NPUSH)) ? 200000 : 112 * j + XSPREAD * (k >> 1) - (j == 0 ? 1000 : 0));
        int q = i;
        while (q > 0 && key[p.order[q - 1]] > key[i]) { p.order[q] = p.order[q - 1]; --q; }
        p.order[q] = (unsigned char)i;
    }
    const char* np = NPASS; const char* ndg = NDIAG;
    int base = 0;
    for (int ps = 0; ps < 2; ++ps) {
        int n = 0;
        for (int c = 0; c < 16; ++c) if (np[c] - '0' > ps) n += 14;
        for (int c = 0, o = 0; c < 16; ++c) { const bool in = np[c] - '0' > ps; p.sched[c][0] |= (u32)(in ? base + o : NPLAIN) << (16 * ps); if (in) o += 14; }
        p.pstride[ps] = n;
        base += n * 8;
    }
    int nq = 0;
    for (int r = 1; r <= 3; ++r) for (int i = 0; i < 16; ++i) { const int c = r == 1 ? i : 15 - i, n = ndg[c] - '0'; p.sched[c][1] |= (u32)n; if (n >= r) p.sched[c][1] |= (u32)nq++ << (8 * r); }
    if (base < NPLAIN || nq != 16) { fprintf(stderr, "kernel_launch: bad schedule\n"); return; }
    hipLaunchKernelGGL(fused_gemm_rs, dim3(256), dim3(512), LDS_BYTES, stream, p);
    HIP_CHECK(hipGetLastError());
}
```
